```python
import numpy as np

kernel_name = "gpu_ag_gemm_m8192_k8192_n8192_f32_silu"


ABSMAX_RTOL = 0.001

COMPUTE_DTYPE = "float32"
EPILOGUE = "silu"
IN_QUANT = None
OUT_QUANT = None
IN_DTYPES = {"x": "float32", "w_mat": "float32"}
WORLD_SIZE = 8

def setup_inputs():

    x = np.random.randn(8192, 8192).astype(np.float32)
    w_mat = (np.random.randn(8192, 8192) / np.sqrt(8192)).astype(np.float32)
    return {"x": x, "w_mat": w_mat}

def _epilogue(y):
    return (y / (1.0 + np.exp(-np.clip(y, -60.0, 60.0)))).astype(np.float32)

def reference(x, w_mat):
    y = (x @ w_mat).astype(np.float32)
    return _epilogue(y)
```

```cpp
#include <hip/hip_runtime.h>
#include <rccl/rccl.h>
#include <cstdio>
#include <cstdint>

#define HIP_CHECK(expr) do { hipError_t _e = (expr); if (_e != hipSuccess) { \
    fprintf(stderr, "HIP error %s at %s:%d\n", hipGetErrorString(_e), __FILE__, __LINE__); \
    return; } } while (0)

namespace {

typedef _Float16 half8 __attribute__((ext_vector_type(8)));
typedef float f32x4 __attribute__((ext_vector_type(4)));
typedef unsigned u32x4 __attribute__((ext_vector_type(4)));
typedef unsigned u32x2 __attribute__((ext_vector_type(2)));
#define LAS __attribute__((address_space(3)))
#define GAS __attribute__((address_space(1)))

constexpr int NR = 8;
constexpr int KDIM = 8192;
constexpr int NSH = 1024;
constexpr int KSTEPS = 128;
constexpr unsigned STEP_B = 16384;
constexpr size_t PANEL_B = (size_t)KSTEPS * STEP_B;
constexpr int PA_AWAY = 24, PA_OWNAWAY = 32, PB = 34, PB_FOREIGN = 42, NPANEL = 50;

constexpr size_t OFF_CALL = 0;
constexpr size_t OFF_TMO = 4096;
constexpr size_t OFF_DONE = 4096 + 256;
constexpr size_t OFF_FLAG = 8192;
constexpr size_t OFF_DBG = 65536;
constexpr size_t OFF_BUF = 2u << 20;
constexpr size_t WS_NEED = OFF_BUF + 2 * NPANEL * PANEL_B;

constexpr int NWG = 256, NTHREADS = 512;
constexpr int GSLOTS = 24;
constexpr int NGEMM = GSLOTS * 8;
constexpr int NCOMM = NWG - NGEMM;
#define NHOME 24
#define NCROSS (NCOMM - NHOME)
constexpr unsigned DONE_PER_CALL = 240 * 4;
constexpr unsigned long long DEADLINE = 100000000ull;

constexpr int NSTAGE = 4;
constexpr unsigned STAGE_B = 2 * STEP_B;
constexpr unsigned LDS_MB = NSTAGE * STAGE_B;
constexpr unsigned LDS_TMO = LDS_MB + 3 * 4 * 512;
constexpr unsigned LDS_BYTES = LDS_TMO + 64;

struct Params {
  const float* x;
  const float* w;
  float* out[NR];
  char* ws[NR];
  int rank;
  int pad;
};

#define STAMP(idx) do {} while (0)

__device__ __forceinline__ void store_sys(void* ptr, u32x4 v) {
  asm volatile("global_store_dwordx4 %0, %1, off sc0 sc1\n\ts_nop 1" :: "v"(ptr), "v"(v) : "memory");
}
__device__ __forceinline__ void wait_vm0() { asm volatile("s_waitcnt vmcnt(0)" ::: "memory"); }
__device__ __forceinline__ void wg_barrier() {
  asm volatile("" ::: "memory");
  __builtin_amdgcn_s_barrier();
  asm volatile("" ::: "memory");
}
__device__ __forceinline__ u32x4 cvt8(f32x4 a, f32x4 b) {
  half8 h;
  h[0] = (_Float16)a[0]; h[1] = (_Float16)a[1]; h[2] = (_Float16)a[2]; h[3] = (_Float16)a[3];
  h[4] = (_Float16)b[0]; h[5] = (_Float16)b[1]; h[6] = (_Float16)b[2]; h[7] = (_Float16)b[3];
  return __builtin_bit_cast(u32x4, h);
}
__device__ __forceinline__ f32x4 ld4(const float* ptr) { return *(const GAS f32x4*)ptr; }

template <int ND, int MODE>
__device__ __forceinline__ void x_stream(const Params& p, int first, int stride, int count, unsigned call,
                                         int a, const int (&dst)[ND]) {
  const int lane = threadIdx.x & 63, wave = threadIdx.x >> 6;
  const size_t par = OFF_BUF + (size_t)(call & 1) * NPANEL * PANEL_B;
  auto decode = [&](int u, int& tile, int& ks, int& panel) {
    if (MODE == 2) { tile = u & 7; ks = u >> 3; panel = tile < 6 ? a * 6 + tile : PA_OWNAWAY + tile - 6; }
    else if (MODE == 0) { tile = u >> 7; ks = u & 127; panel = a * 6 + tile; }
    else { tile = 6 + (u >> 7); ks = u & 127; panel = PA_AWAY + a * 2 + (u >> 7); }
  };
  const float* src0 = p.x + (size_t)(wave * 16 + (lane & 15)) * KDIM + (lane >> 4) * 4;
  f32x4 v[4];
  int u = first;
  if (u >= count) return;
  {
    int tile, ks, panel; decode(u, tile, ks, panel);
    const float* s = src0 + (size_t)tile * 128 * KDIM + ks * 64;
#pragma unroll
    for (int i = 0; i < 4; ++i) v[i] = ld4(s + i * 16);
  }
  for (; u < count; u += stride) {
    int tile, ks, panel; decode(u, tile, ks, panel);
    const u32x4 h0 = cvt8(v[0], v[1]), h1 = cvt8(v[2], v[3]);
    {
      const int un = u + stride < count ? u + stride : u;
      int t2, k2, p2; decode(un, t2, k2, p2);
      const float* s = src0 + (size_t)t2 * 128 * KDIM + k2 * 64;
#pragma unroll
      for (int i = 0; i < 4; ++i) v[i] = ld4(s + i * 16);
    }
    const size_t off = par + (size_t)panel * PANEL_B + (size_t)ks * STEP_B + wave * 2048 + lane * 16;
#pragma unroll
    for (int d = 0; d < ND; ++d) {
      char* b = p.ws[dst[d] & 7] + off;
      store_sys(b, h0);
      store_sys(b + 1024, h1);
    }
    wait_vm0();
    wg_barrier();
    if (threadIdx.x == 0) {
#pragma unroll
      for (int d = 0; d < ND; ++d)
        __hip_atomic_store((unsigned*)(p.ws[dst[d] & 7] + OFF_FLAG) + panel * KSTEPS + ks, call,
                           __ATOMIC_RELAXED, __HIP_MEMORY_SCOPE_SYSTEM);
    }
  }
}

template <bool LOCAL>
__device__ __forceinline__ void w_stream(const Params& p, int first, int stride, unsigned call, int rank) {
  const int lane = threadIdx.x & 63, wave = threadIdx.x >> 6;
  const int count = KSTEPS * 4;
  const size_t par = OFF_BUF + (size_t)(call & 1) * NPANEL * PANEL_B;
  const float* src0 = p.w + (size_t)((wave & 1) * 32 + (lane >> 4) * 4) * NSH + (wave >> 1) * 64 + (lane & 15) * 4;
  f32x4 v[8];
  int u = first;
  if (u >= count) return;
  {
    const float* s = src0 + (size_t)(u >> 2) * 64 * NSH + (u & 3) * 256;
#pragma unroll
    for (int j = 0; j < 8; ++j) v[j] = ld4(s + ((j & 3) + (j >> 2) * 16) * NSH);
  }
  for (; u < count; u += stride) {
    const int ks = u >> 2, q = u & 3;
    u32x4 h[4];
#pragma unroll
    for (int f = 0; f < 4; ++f)
      h[f] = cvt8((f32x4){v[0][f], v[1][f], v[2][f], v[3][f]}, (f32x4){v[4][f], v[5][f], v[6][f], v[7][f]});
    {
      const int un = u + stride < count ? u + stride : u;
      const float* s = src0 + (size_t)(un >> 2) * 64 * NSH + (un & 3) * 256;
#pragma unroll
      for (int j = 0; j < 8; ++j) v[j] = ld4(s + ((j & 3) + (j >> 2) * 16) * NSH);
    }
    const int dst = LOCAL ? rank : (((rank & 4) ^ 4) | q);
    const int panel0 = LOCAL ? PB + q * 2 : PB_FOREIGN + (rank & 3) * 2;
    char* wsd = p.ws[dst & 7];
    char* b = wsd + par + (size_t)(panel0 + (wave >> 2)) * PANEL_B + (size_t)ks * STEP_B + (wave & 3) * 4096 + lane * 16;
#pragma unroll
    for (int f = 0; f < 4; ++f) store_sys(b + f * 1024, h[f]);
    wait_vm0();
    wg_barrier();
    if (threadIdx.x < 2)
      __hip_atomic_store((unsigned*)(wsd + OFF_FLAG) + (panel0 + threadIdx.x) * KSTEPS + ks, call,
                         __ATOMIC_RELAXED, __HIP_MEMORY_SCOPE_SYSTEM);
  }
}

__device__ __forceinline__ bool get_job(int xcd, int s, int j, int a, int& pa, int& pb) {
  if (xcd < 4) {
    pa = ((a + 1 + s % 3) & 3) * 6 + 2 * j + (xcd >> 1);
    pb = PB + (xcd & 1) * 8 + s / 3;
    return true;
  }
  if (j == 0) {
    pa = a * 6 + (xcd & 1) * 3 + s % 3;
    pb = PB + (xcd < 6 ? 8 : 0) + s / 3;
    return true;
  }
  if (xcd == 4) {
    const int b = s & 3, q = s >> 2;
    pa = PA_AWAY + b * 2 + (j - 1);
    pb = PB_FOREIGN + ((b + 1 + (q >> 1)) & 3) * 2 + (q & 1);
    return true;
  }
  if (s >= 16) return false;
  if (xcd < 7) {
    pa = PA_AWAY + (s & 3) * 2 + (j - 1);
    pb = PB + (xcd - 5) * 4 + (s >> 2);
    return true;
  }
  if (j == 1) {
    pa = PA_OWNAWAY + (s & 1);
    pb = PB + (s >> 1);
    return true;
  }
  return false;
}

__device__ __forceinline__ float silu(float y) {
  const float c = fminf(fmaxf(y, -60.0f), 60.0f);
  return y * __builtin_amdgcn_rcpf(1.0f + __builtin_amdgcn_exp2f(c * -1.44269504088896341f));
}

#define GLDS_BITS " sc0 sc1"

__global__ void __launch_bounds__(NTHREADS, 2) ag_gemm_silu(const Params p) {
  __shared__ __attribute__((aligned(16))) char smem[LDS_BYTES];
  const int rank = p.rank & 7, a = rank & 3;
  const int lane = threadIdx.x & 63;
  const int wave = __builtin_amdgcn_readfirstlane(threadIdx.x >> 6);
  const int xcd = blockIdx.x & 7, slot = blockIdx.x >> 3;
  char* const ws = p.ws[rank];
  unsigned* const tmo = (unsigned*)(ws + OFF_TMO);

  unsigned* const cnt = (unsigned*)(ws + OFF_CALL) + blockIdx.x;
  const unsigned call = __builtin_amdgcn_readfirstlane(
      __hip_atomic_load(cnt, __ATOMIC_RELAXED, __HIP_MEMORY_SCOPE_AGENT)) + 1u;
  if (wave == 0) STAMP(0);
  for (unsigned i = threadIdx.x; i < (LDS_BYTES - LDS_MB) / 4; i += NTHREADS) ((LAS unsigned*)(smem + LDS_MB))[i] = 0u;
  asm volatile("s_waitcnt vmcnt(0) lgkmcnt(0)" ::: "memory");
  wg_barrier();
  if (threadIdx.x == 0) __hip_atomic_store(cnt, call, __ATOMIC_RELAXED, __HIP_MEMORY_SCOPE_AGENT);

  if (slot >= GSLOTS) {
    const int c = blockIdx.x - NGEMM;
    if (c < NHOME) {
      const int g = rank & 4;
      const int dst[3] = {g | ((a + 1) & 3), g | ((a + 2) & 3), g | ((a + 3) & 3)};
      x_stream<3, 0>(p, c, NHOME, 6 * KSTEPS, call, a, dst);
    } else if (c < NHOME + NCROSS) {
      w_stream<false>(p, c - NHOME, NCROSS, call, rank);
      if (wave == 0) STAMP(1);
      const int g = (rank & 4) ^ 4;
      const int dst[4] = {g | a, g | ((a + 1) & 3), g | ((a + 2) & 3), g | ((a + 3) & 3)};
      x_stream<4, 1>(p, c - NHOME, NCROSS, 2 * KSTEPS, call, a, dst);
    }
    if (wave == 0) STAMP(2);
    return;
  }

  w_stream<true>(p, blockIdx.x, NGEMM, call, rank);
  {
    const int dst[1] = {rank};
    x_stream<1, 2>(p, blockIdx.x, NGEMM, 8 * KSTEPS, call, a, dst);
  }
  if (wave == 0) STAMP(1);

  const char* const buf = ws + OFF_BUF + (size_t)(call & 1) * NPANEL * PANEL_B;
  const unsigned lds0 = (unsigned)(uintptr_t)(LAS char*)smem;

  for (int job = 0; job < 3; ++job) {
    int pa, pb;
    if (!get_job(xcd, slot, job, a, pa, pb)) break;
    if (wave >= 4) {
      const int l = wave - 4;
      const int panel = (l >> 1) ? pb : pa;
      const char* src = buf + (size_t)panel * PANEL_B + (l & 1) * 8192 + lane * 16;
      const char* fsrc = ws + OFF_FLAG + panel * (KSTEPS * 4) + lane * 16;
      const unsigned mb = lds0 + LDS_MB + (job * 4 + l) * 512;
      const unsigned dst0 = lds0 + l * 8192;
      int ready = 0;
      auto poll = [&]() {
        if (lane < 32) {
          unsigned keep;
          asm volatile("s_mov_b32 %0, m0\n\ts_mov_b32 m0, %2\n\ts_nop 0\n\t"
                       "global_load_lds_dwordx4 %1, off sc0 sc1\n\ts_mov_b32 m0, %0"
                       : "=&s"(keep) : "v"(fsrc), "s"(mb) : "memory");
        }
      };
      auto refresh = [&]() {
        u32x2 f;
        asm volatile("ds_read_b64 %0, %1\n\ts_waitcnt lgkmcnt(0)" : "=v"(f) : "v"(mb + lane * 8) : "memory");
        const unsigned long long b0 = __builtin_amdgcn_ballot_w64(f[0] >= call);
        const unsigned long long b1 = __builtin_amdgcn_ballot_w64(f[1] >= call);
        const unsigned long long miss = ~(b0 & b1);
        if (miss == 0ull) ready = KSTEPS;
        else { const int n = __builtin_ctzll(miss); ready = 2 * n + (int)((b0 >> n) & 1ull); }
      };
      auto issue = [&](int s) {
        if (s >= ready) {
          refresh();
          if (s >= ready) {
            const unsigned long long t0 = __builtin_amdgcn_s_memrealtime();
            for (unsigned it = 1;; ++it) {
              poll();
              wait_vm0();
              refresh();
              if (s < ready) break;
              if ((it & 63u) == 0u) {
                if (__builtin_amdgcn_readfirstlane(__hip_atomic_load(tmo, __ATOMIC_RELAXED, __HIP_MEMORY_SCOPE_AGENT)) == call
                    || __builtin_amdgcn_s_memrealtime() - t0 > DEADLINE) {
                  if (lane == 0) __hip_atomic_store(tmo, call, __ATOMIC_RELAXED, __HIP_MEMORY_SCOPE_AGENT);
                  wait_vm0();
                  ready = KSTEPS;
                  break;
                }
              }
              __builtin_amdgcn_s_sleep(4);
            }
          }
        }
        if (ready < KSTEPS && ready - s < 12) poll();
        const char* g = src + (size_t)s * STEP_B;
        const unsigned d = dst0 + (s & (NSTAGE - 1)) * STAGE_B;
        unsigned keep;
        asm volatile("s_mov_b32 %0, m0\n\ts_mov_b32 m0, %3\n\ts_nop 0\n\t"
                     "global_load_lds_dwordx4 %1, off" GLDS_BITS "\n\t"
                     "global_load_lds_dwordx4 %1, off offset:1024" GLDS_BITS "\n\t"
                     "global_load_lds_dwordx4 %1, off offset:2048" GLDS_BITS "\n\t"
                     "global_load_lds_dwordx4 %1, off offset:3072" GLDS_BITS "\n\t"
                     "s_mov_b32 m0, %4\n\ts_nop 0\n\t"
                     "global_load_lds_dwordx4 %2, off" GLDS_BITS "\n\t"
                     "global_load_lds_dwordx4 %2, off offset:1024" GLDS_BITS "\n\t"
                     "global_load_lds_dwordx4 %2, off offset:2048" GLDS_BITS "\n\t"
                     "global_load_lds_dwordx4 %2, off offset:3072" GLDS_BITS "\n\t"
                     "s_mov_b32 m0, %0"
                     : "=&s"(keep) : "v"(g), "v"(g + 4096), "s"(d), "s"(d + 4096) : "memory");
      };
      issue(0); issue(1); issue(2);
      for (int t = 0; t < KSTEPS - 3; ++t) {
        asm volatile("s_waitcnt vmcnt(16)" ::: "memory");
        wg_barrier();
        issue(t + 3);
      }
      if (lane == 0) {
        unsigned keep;
        asm volatile("s_mov_b32 %0, m0\n\ts_mov_b32 m0, %2\n\ts_nop 0\n\t"
                     "global_load_lds_dword %1, off sc1\n\ts_mov_b32 m0, %0"
                     : "=&s"(keep) : "v"(tmo), "s"(lds0 + LDS_TMO + (job * 4 + l) * 4) : "memory");
      }
      asm volatile("s_waitcnt vmcnt(17)" ::: "memory");
      wg_barrier();
      asm volatile("s_waitcnt vmcnt(9)" ::: "memory");
      wg_barrier();
      wait_vm0();
      wg_barrier();
    } else {
      const int wr = wave >> 1, wc = wave & 1;
      f32x4 acc[4][4];
#pragma unroll
      for (int m = 0; m < 4; ++m)
#pragma unroll
        for (int n = 0; n < 4; ++n) acc[m][n] = (f32x4){0.f, 0.f, 0.f, 0.f};
      half8 fa[2][8], fb[2][8];
      const LAS char* la = (const LAS char*)smem + wr * 8192 + lane * 16;
      const LAS char* lb = (const LAS char*)smem + STEP_B + wc * 8192 + lane * 16;
#define READ_SET(X, t) do { const unsigned so_ = ((t) & (NSTAGE - 1)) * STAGE_B; \
        _Pragma("unroll") for (int i = 0; i < 8; ++i) { \
          fa[X][i] = *(const LAS half8*)(la + so_ + i * 1024); \
          fb[X][i] = *(const LAS half8*)(lb + so_ + i * 1024); } } while (0)
#define MMA_SET(X) do { \
        _Pragma("unroll") for (int kh = 0; kh < 2; ++kh) \
        _Pragma("unroll") for (int m = 0; m < 4; ++m) \
        _Pragma("unroll") for (int n = 0; n < 4; ++n) \
          acc[m][n] = __builtin_amdgcn_mfma_f32_16x16x32_f16(fa[X][m * 2 + kh], fb[X][kh * 4 + n], acc[m][n], 0, 0, 0); } while (0)
#define INTERLEAVE() do { \
        _Pragma("unroll") for (int i = 0; i < 8; ++i) { \
          __builtin_amdgcn_sched_group_barrier(0x100, 2, 0); \
          __builtin_amdgcn_sched_group_barrier(0x008, 3, 0); } \
        __builtin_amdgcn_sched_group_barrier(0x008, 8, 0); } while (0)
#define STEP_END() do { __builtin_amdgcn_sched_barrier(0); asm volatile("" ::: "memory"); \
        __builtin_amdgcn_s_waitcnt(0xC07F);   __builtin_amdgcn_sched_barrier(0); } while (0)
      wg_barrier();
      if (wave == 0) STAMP(4 + job * 4);
      READ_SET(0, 0);
      STEP_END();
      for (int t = 1; t < KSTEPS - 1; t += 2) {
        wg_barrier();
        __builtin_amdgcn_sched_barrier(0);
        READ_SET(1, t); MMA_SET(0); INTERLEAVE();
        STEP_END();
        wg_barrier();
        __builtin_amdgcn_sched_barrier(0);
        READ_SET(0, t + 1); MMA_SET(1); INTERLEAVE();
        STEP_END();
      }
      wg_barrier();
      __builtin_amdgcn_sched_barrier(0);
      READ_SET(1, KSTEPS - 1); MMA_SET(0); INTERLEAVE();
      const u32x4 tm = *(const LAS u32x4*)((const LAS char*)smem + LDS_TMO + job * 16);
      STEP_END();
      MMA_SET(1);
      if (wave == 0) STAMP(5 + job * 4);
      const bool dead = tm[0] == call || tm[1] == call || tm[2] == call || tm[3] == call;

      const int og = (rank & 4) ^ 4;
      const int row0 = pa < PA_AWAY ? ((rank & 4) + pa / 6) * NSH + (pa % 6) * 128
                     : pa < PA_OWNAWAY ? (og + ((pa - PA_AWAY) >> 1)) * NSH + 768 + (pa & 1) * 128
                                       : rank * NSH + 768 + (pa & 1) * 128;
      const bool foreign = pb >= PB_FOREIGN;
      const int owner = (foreign ? og + ((pb - PB_FOREIGN) >> 1) : rank) & 7;
      const int col0 = foreign ? a * 256 + (pb & 1) * 128 : (pb - PB) * 128;
      float* o = p.out[owner] + (size_t)(row0 + wr * 64 + (lane >> 4) * 4) * NSH + col0 + wc * 64 + (lane & 15) * 4;
      const float qnan = __builtin_nanf("");
#pragma unroll
      for (int m = 0; m < 4; ++m)
#pragma unroll
        for (int r = 0; r < 4; ++r) {
          f32x4 v = {silu(acc[m][0][r]), silu(acc[m][1][r]), silu(acc[m][2][r]), silu(acc[m][3][r])};
          if (dead) v = (f32x4){qnan, qnan, qnan, qnan};
          store_sys(o + (size_t)(m * 16 + r) * NSH, __builtin_bit_cast(u32x4, v));
        }
      wait_vm0();
      if (foreign && lane == 0)
        __hip_atomic_fetch_add((unsigned*)(p.ws[owner] + OFF_DONE), 1u, __ATOMIC_RELAXED, __HIP_MEMORY_SCOPE_SYSTEM);
      if (wave == 0) STAMP(6 + job * 4);

      if (blockIdx.x == 2 && job == 2 && wave == 0) {
        const unsigned* done = (const unsigned*)(ws + OFF_DONE);
        const unsigned want = call * DONE_PER_CALL;
        const unsigned long long t0 = __builtin_amdgcn_s_memrealtime();
        while ((int)(__builtin_amdgcn_readfirstlane(__hip_atomic_load(done, __ATOMIC_RELAXED, __HIP_MEMORY_SCOPE_SYSTEM)) - want) < 0) {
          if (__builtin_amdgcn_readfirstlane(__hip_atomic_load(tmo, __ATOMIC_RELAXED, __HIP_MEMORY_SCOPE_AGENT)) == call
              || __builtin_amdgcn_s_memrealtime() - t0 > DEADLINE) {
            if (lane == 0) {
              __hip_atomic_store(tmo, call, __ATOMIC_RELAXED, __HIP_MEMORY_SCOPE_AGENT);
              *o = qnan;
            }
            break;
          }
          __builtin_amdgcn_s_sleep(2);
        }
        STAMP(3);
      }
    }
  }
}

}

extern "C" void kernel_launch(int rank, int nranks, ncclComm_t comm, hipStream_t stream,
                              void* const* d_in, const int* in_sizes, int n_in,
                              void* d_out, int out_size, const int* in_dtypes, int out_dtype,
                              void* const* d_out_all, void* const* d_ws_all, size_t ws_bytes) {
  (void)comm; (void)in_dtypes; (void)out_dtype; (void)d_out;
  if (nranks != NR || (unsigned)rank >= (unsigned)NR || n_in != 2 || in_sizes[0] != NSH * KDIM ||
      in_sizes[1] != KDIM * NSH || out_size != KDIM * NSH || ws_bytes < WS_NEED) {
    fprintf(stderr, "kernel_launch: unexpected problem shape\n");
    return;
  }
  Params p;
  p.x = (const float*)d_in[0];
  p.w = (const float*)d_in[1];
  for (int i = 0; i < NR; ++i) { p.out[i] = (float*)d_out_all[i]; p.ws[i] = (char*)d_ws_all[i]; }
  p.rank = rank;
  p.pad = 0;
  ag_gemm_silu<<<dim3(NWG), dim3(NTHREADS), 0, stream>>>(p);
  HIP_CHECK(hipGetLastError());
}
```

```cpp
#include <hip/hip_runtime.h>
#include <rccl/rccl.h>
#include <cstdio>
#include <cstdint>

#define HIP_CHECK(expr) do { hipError_t _e = (expr); if (_e != hipSuccess) { \
    fprintf(stderr, "HIP error %s at %s:%d\n", hipGetErrorString(_e), __FILE__, __LINE__); \
    return; } } while (0)

namespace {

typedef _Float16 half8 __attribute__((ext_vector_type(8)));
typedef float f32x4 __attribute__((ext_vector_type(4)));
typedef unsigned u32x4 __attribute__((ext_vector_type(4)));
typedef unsigned u32x2 __attribute__((ext_vector_type(2)));
#define LAS __attribute__((address_space(3)))
#define GAS __attribute__((address_space(1)))

constexpr int NR = 8;
#define RANK_AT_V 0x75416320u
constexpr unsigned RANK_AT = RANK_AT_V;
constexpr int KDIM = 8192;
constexpr int NSH = 1024;
constexpr int KSTEPS = 128;
constexpr unsigned STEP_B = 16384;
#define SKEW 0
constexpr size_t PANEL_B = (size_t)KSTEPS * STEP_B + SKEW;
constexpr int PA_AWAY = 24, PA_OWNAWAY = 32, PB = 34, PB_FOREIGN = 42, NPANEL = 50;

constexpr unsigned BLK_B = 1792;
constexpr unsigned PSTEP_B = 8 * BLK_B;
constexpr size_t PPANEL_B = (size_t)KSTEPS * PSTEP_B;

constexpr size_t OFF_CALL = 0;
constexpr size_t OFF_TMO = 4096;
constexpr size_t OFF_FLAG = 8192;
constexpr size_t OFF_RDY = 65536;
constexpr int RDY_N = 1024, RDY_AWAY = 128, RDY_OUT = 384;
constexpr size_t OFF_DBG = 131072;
constexpr size_t OFF_OWN = 2u << 20;
constexpr size_t OFF_PACK = OFF_OWN + 32 * PANEL_B;
constexpr size_t OFF_PULL = OFF_PACK + 32 * PPANEL_B;
constexpr int OUT_CHUNKS = 480;
constexpr unsigned CHUNK_B = 8192;
constexpr size_t OFF_STG = OFF_PULL + NPANEL * PANEL_B;
constexpr size_t WS_NEED = OFF_STG + (size_t)4 * OUT_CHUNKS * CHUNK_B;
static_assert(OFF_FLAG + NPANEL * KSTEPS * 4 <= OFF_RDY && OFF_RDY + NR * RDY_N * 4 <= OFF_DBG, "");

constexpr int NWG = 256, NTHREADS = 512;
constexpr int GSLOTS = 24;
constexpr int NGEMM = GSLOTS * 8 + 8;
constexpr int PW = 8;
#define NWV 5
#define PULL_LAG 1
constexpr int NCVT_W = 104, NCVT_X = 96;
constexpr int NJOBS = 4;
constexpr unsigned long long DEADLINE = 100000000ull;

constexpr int NSTAGE = 6;
constexpr unsigned STAGE_B = 3 * 8192;
constexpr unsigned LDS_MB = NSTAGE * STAGE_B;
constexpr unsigned LDS_TMO = LDS_MB + NJOBS * 4 * 512;
constexpr unsigned LDS_LUT = LDS_TMO + NJOBS * 16;
constexpr unsigned LDS_BYTES = LDS_LUT + 64;
constexpr unsigned WAVE_LDS = 8192;

struct Params {
  const float* x;
  const float* w;
  float* out[NR];
  char* ws[NR];
  int rank;
  int pad;
  unsigned short jobs[NGEMM][NJOBS];
};

#define STAMP(idx) do {} while (0)

#define EXP 0
#define EXPON(bit) ((EXP & (bit)) && call > 2)
__device__ __forceinline__ void store_sys(void* ptr, u32x4 v) {
  asm volatile("global_store_dwordx4 %0, %1, off sc0 sc1\n\ts_nop 1" :: "v"(ptr), "v"(v) : "memory");
}
__device__ __forceinline__ void wait_vm0() { asm volatile("s_waitcnt vmcnt(0)" ::: "memory"); }
__device__ __forceinline__ void wg_barrier() {
  asm volatile("" ::: "memory");
  __builtin_amdgcn_s_barrier();
  asm volatile("" ::: "memory");
}
__device__ __forceinline__ u32x4 cvt8(f32x4 a, f32x4 b) {
  half8 h;
  h[0] = (_Float16)a[0]; h[1] = (_Float16)a[1]; h[2] = (_Float16)a[2]; h[3] = (_Float16)a[3];
  h[4] = (_Float16)b[0]; h[5] = (_Float16)b[1]; h[6] = (_Float16)b[2]; h[7] = (_Float16)b[3];
  return __builtin_bit_cast(u32x4, h);
}
__device__ __forceinline__ f32x4 ld4(const float* ptr) { return *(const GAS f32x4*)ptr; }
#define OUT_NT ""
__device__ __forceinline__ void store_out(void* ptr, u32x4 v) {
  asm volatile("global_store_dwordx4 %0, %1, off sc0 sc1" OUT_NT "\n\ts_nop 1" :: "v"(ptr), "v"(v) : "memory");
}

__device__ __forceinline__ void set_flag(char* base, int idx, unsigned v) {
  __hip_atomic_store((unsigned*)base + idx, v, __ATOMIC_RELAXED, __HIP_MEMORY_SCOPE_SYSTEM);
}
__device__ __forceinline__ unsigned own_idx(unsigned call, int idx) { return (call & 1) * 16 + idx; }
__device__ __forceinline__ int fresh_lane() {
  int l;
  asm volatile("v_mbcnt_lo_u32_b32 %0, -1, 0\n\tv_mbcnt_hi_u32_b32 %0, -1, %0" : "=v"(l));
  return l;
}
__device__ __forceinline__ void cbar() { asm volatile("" ::: "memory"); }
__device__ __forceinline__ unsigned wave_scan(unsigned v) {
  v += __builtin_amdgcn_update_dpp(0u, v, 0x111, 0xf, 0xf, false);
  v += __builtin_amdgcn_update_dpp(0u, v, 0x112, 0xf, 0xf, false);
  v += __builtin_amdgcn_update_dpp(0u, v, 0x114, 0xf, 0xf, false);
  v += __builtin_amdgcn_update_dpp(0u, v, 0x118, 0xf, 0xf, false);
  v += __builtin_amdgcn_update_dpp(0u, v, 0x142, 0xa, 0xf, false);
  v += __builtin_amdgcn_update_dpp(0u, v, 0x143, 0xc, 0xf, false);
  return v;
}

__device__ __forceinline__ unsigned codec_lut(int m, bool expand) {
  unsigned sel = 0x0c0c0c0cu;
  int n = 0;
  for (int i = 0; i < 4; ++i)
    if ((m >> i) & 1) {
      const int at = expand ? i : n, v = expand ? n : i;
      sel = (sel & ~(0xffu << (8 * at))) | ((unsigned)v << (8 * at));
      ++n;
    }
  return sel;
}

__device__ __forceinline__ void pack_block(u32x4 f0, u32x4 f1, LAS char* wl, const LAS char* lut, char* dst) {
  const int lane = threadIdx.x & 63;
  const unsigned h[8] = {f0[0], f0[1], f0[2], f0[3], f1[0], f1[1], f1[2], f1[3]};
  if (lane < 36) *(LAS unsigned*)(wl + 640 + lane * 4) = 0u;
  u32x4 D;
  unsigned H[4], sum = 0;
#pragma unroll
  for (int k = 0; k < 4; ++k) {
    D[k] = __builtin_amdgcn_perm(h[2 * k + 1], h[2 * k], 0x06040200u);
    H[k] = __builtin_amdgcn_perm(h[2 * k + 1], h[2 * k], 0x07050301u);
    sum = __builtin_amdgcn_udot4(H[k] & 0x7c7c7c7cu, 0x01010101u, sum, false);
  }
  store_sys(dst + lane * 16, D);
  const unsigned tot = __builtin_amdgcn_readlane(wave_scan(sum), 63);
  const unsigned base = min((tot + 256u) >> 12, 28u);
  const unsigned bB = base * 0x04040404u;
  unsigned t[4], pk[4], n[4], EM = 0, XM = 0, sg = 0;
#pragma unroll
  for (int k = 0; k < 4; ++k) {
    const unsigned g = ((H[k] & 0x7c7c7c7cu) | 0x80808080u) - bB;
    const unsigned a7 = g & 0x7f7f7f7fu, hi = a7 + 0x70707070u, lo = a7 + 0x1c1c1c1cu;
    const unsigned o7 = (~g | hi) & 0x80808080u;
    t[k] = ((g | (o7 >> 4) | (o7 >> 5)) & 0x0c0c0c0cu) | (H[k] & 0x03030303u);
    const unsigned E = t[k] & (t[k] >> 1) & 0x04040404u;
    const unsigned nb = ~g & 0x80808080u;
    const unsigned c4 = g & ((nb >> 2) - (nb >> 5));
    const unsigned X = ((g & hi) | (~g & ~lo)) & 0x80808080u;
    const unsigned sel = *(const LAS unsigned*)(lut + __builtin_amdgcn_udot4(E, 0x08040201u, 0u, false));
    const unsigned z = __builtin_amdgcn_perm(0u, c4, sel);
    pk[k] = __builtin_amdgcn_udot4(z, 0x00400801u, (z >> 24) << 9, false);
    n[k] = __builtin_popcount(E);
    EM |= (E >> 2) << k;
    XM |= X >> (7 - k);
    sg |= (H[k] & 0x80808080u) >> (7 - k);
  }
  const unsigned mine = (n[0] + n[1] + n[2] + n[3]) | (__builtin_popcount(XM) << 16);
  const unsigned incl = wave_scan(mine);
  const unsigned all = __builtin_amdgcn_readlane(incl, 63), n2 = all & 0xffffu, n3 = all >> 16;
  const bool raw = 3u * n2 + 16u * n3 > 1008u;
  *(LAS u32x2*)(wl + lane * 8) = (u32x2){t[0] | (t[1] << 4), t[2] | (t[3] << 4)};
  const unsigned sgn = __builtin_amdgcn_update_dpp(0u, sg, 0xb1, 0xf, 0xf, false);
  if (!(lane & 1)) *(LAS unsigned*)(wl + 512 + lane * 2) = sg | (sgn << 4);
  cbar();
  if (!raw) {
    const unsigned pre = (incl - mine) & 0xffffu;
    unsigned pre3 = (incl - mine) >> 16;
    const unsigned long long W = ((unsigned long long)(pk[0] | (pk[1] << (3 * n[0])))
        | ((unsigned long long)(pk[2] | (pk[3] << (3 * n[2]))) << (3 * (n[0] + n[1])))) >> 2;
    const unsigned o = 3 * pre, sh = o & 31u;
    LAS unsigned* q = (LAS unsigned*)(wl + 640 + (o >> 5) * 4);
    const unsigned long long Ws = W << sh;
    __hip_atomic_fetch_or(q, (unsigned)Ws, __ATOMIC_RELAXED, __HIP_MEMORY_SCOPE_WORKGROUP);
    __hip_atomic_fetch_or(q + 1, (unsigned)(Ws >> 32), __ATOMIC_RELAXED, __HIP_MEMORY_SCOPE_WORKGROUP);
    __hip_atomic_fetch_or(q + 2, ((unsigned)(W >> 32) >> 1) >> (31u - sh), __ATOMIC_RELAXED, __HIP_MEMORY_SCOPE_WORKGROUP);
    cbar();
    while (XM) {
      const unsigned pos = __builtin_ctz(XM), k = pos & 3u, i8 = pos & 24u;
      const unsigned Hk = k & 2u ? (k & 1u ? H[3] : H[2]) : (k & 1u ? H[1] : H[0]);
      const unsigned kb = 0x01010101u << k;
      const unsigned rk = pre + __builtin_popcount(EM & ((kb - 0x01010101u) | (kb & ((1u << i8) - 1u))));
      *(LAS unsigned short*)(wl + 640 + 124 - 2 * pre3) = (unsigned short)((rk << 5) | (((Hk >> (i8 + 2)) & 31u) ^ (base + 3u)));
      ++pre3;
      XM &= XM - 1u;
    }
  }
  cbar();
  if (lane == 0) *(LAS unsigned short*)(wl + 640 + 126) = (unsigned short)(raw ? 0x8000u : base | (n3 << 5));
  cbar();
  if (lane < 48) store_sys(dst + 1024 + lane * 16, *(const LAS u32x4*)(wl + lane * 16));
  cbar();
}

__device__ __forceinline__ bool unpack_block(const LAS char* blk, LAS char* ex, const LAS char* lut, u32x4& f0, u32x4& f1) {
  const int lane = threadIdx.x & 63;
  const unsigned ent = *(const LAS unsigned short*)(blk + 1790 - 2 * lane);
  const LAS unsigned* sp = (const LAS unsigned*)(blk + 1664 + ((3 * lane) & ~3));
  const unsigned s0 = sp[0], s1 = sp[1];
  const u32x2 N = *(const LAS u32x2*)(blk + 1024 + lane * 8);
  const unsigned sg = *(const LAS unsigned*)(blk + 1536 + (lane >> 1) * 4) >> ((lane & 1) * 4);
  const u32x4 D = *(const LAS u32x4*)(blk + lane * 16);
  const unsigned hdr = __builtin_amdgcn_readfirstlane(ent);
  if (hdr & 0x8000u) return false;
  const unsigned base = hdr & 31u, n3 = hdr >> 5;
  const unsigned b3 = (base + 3u) * 0x04040404u, b8 = (base + 24u) * 0x04040404u;
  const unsigned T0 = (((b8 + 0x0c080400u) ^ b3) & 0x7c7c7c00u), T1 = (((b8 + 0x1c181410u) ^ b3) & 0x7c7c7c7cu);
  const unsigned x = __builtin_amdgcn_alignbyte(s1, s0, 3 * lane);
  u32x2 X;
#pragma unroll
  for (int i = 0; i < 2; ++i) {
    const unsigned y = ((x >> (12 * i)) & 0x3fu) | ((i ? x >> 2 : x << 10) & 0x3f0000u);
    X[i] = __builtin_amdgcn_perm(T1, T0, (y | (y << 5)) & 0x07070707u);
  }
  *(LAS u32x2*)(ex + lane * 8) = X;
  cbar();
  if ((unsigned)(lane - 1) < n3) *(LAS unsigned char*)(ex + ((ent >> 5) & 511u)) = (unsigned char)((ent & 31u) << 2);
  cbar();
  const unsigned E0 = N[0] & (N[0] >> 1) & 0x44444444u, E1 = N[1] & (N[1] >> 1) & 0x44444444u;
  const unsigned mine = __builtin_popcount(E0) + __builtin_popcount(E1);
  unsigned pos = wave_scan(mine) - mine;
  const unsigned bB = base * 0x04040404u;
  unsigned h[8];
#pragma unroll
  for (int k = 0; k < 4; ++k) {
    const unsigned E = ((k & 2 ? E1 : E0) >> (4 * (k & 1))) & 0x04040404u;
    const unsigned sel = *(const LAS unsigned*)(lut + __builtin_amdgcn_udot4(E, 0x08040201u, 0u, false));
    const LAS unsigned* wp = (const LAS unsigned*)(ex + (pos & ~3u));
    const unsigned win = __builtin_amdgcn_alignbyte(wp[1], wp[0], pos);
    pos += __builtin_popcount(E);
    const unsigned t = (N[k >> 1] >> (4 * (k & 1))) & 0x0f0f0f0fu;
    const unsigned H = ((t + bB) ^ __builtin_amdgcn_perm(0u, win, sel)) | ((sg << (7 - k)) & 0x80808080u);
    h[2 * k] = __builtin_amdgcn_perm(H, D[k], 0x05010400u);
    h[2 * k + 1] = __builtin_amdgcn_perm(H, D[k], 0x07030602u);
  }
  cbar();
  f0 = (u32x4){h[0], h[1], h[2], h[3]};
  f1 = (u32x4){h[4], h[5], h[6], h[7]};
  return true;
}

__device__ __forceinline__ void convert_x(const Params& p, int first, unsigned call, int rank, LAS char* smem) {
  const int lane = threadIdx.x & 63, wave = threadIdx.x >> 6;
  const int count = 8 * KSTEPS, stride = NCVT_X;
  char* const ws = p.ws[rank];
  const float* src0 = p.x + (size_t)((wave >> 2) * 64 + (wave & 1) * 32 + (lane & 15)) * KDIM + (wave & 2) * 16 + (lane >> 4) * 4;
  f32x4 v[4];
  int u = first;
  {
    const float* s = src0 + (size_t)(u >> 7) * 128 * KDIM + (u & 127) * 64;
#pragma unroll
    for (int i = 0; i < 4; ++i) v[i] = ld4(s + (i >> 1) * 16 * KDIM + (i & 1) * 16);
  }
  for (; u < count; u += stride) {
    const int tile = u >> 7, ks = u & 127;
    const u32x4 h0 = cvt8(v[0], v[1]), h1 = cvt8(v[2], v[3]);
    {
      const int un = u + stride < count ? u + stride : u;
      const float* s = src0 + (size_t)(un >> 7) * 128 * KDIM + (un & 127) * 64;
#pragma unroll
      for (int i = 0; i < 4; ++i) v[i] = ld4(s + (i >> 1) * 16 * KDIM + (i & 1) * 16);
    }
    char* b = ws + OFF_OWN + own_idx(call, tile) * PANEL_B + ks * STEP_B + wave * 2048 + lane * 16;
    store_sys(b, h0);
    store_sys(b + 1024, h1);
    pack_block(h0, h1, smem + wave * WAVE_LDS, smem + LDS_LUT,
               ws + OFF_PACK + own_idx(call, tile) * PPANEL_B + ks * PSTEP_B + wave * BLK_B);
    wait_vm0();
    wg_barrier();
    if (threadIdx.x < 5) {
      const int t = threadIdx.x;
      const bool home = tile < 6;
      const int panel = home ? (rank & 3) * 6 + tile : PA_OWNAWAY + tile - 6;
      const int dst = (home ? (rank & 4) : (rank & 4) ^ 4) | (t & 3);
      if (t == 4) set_flag(ws + OFF_FLAG, panel * KSTEPS + ks, call);
      else if (dst != rank) set_flag(p.ws[dst & 7] + OFF_RDY, rank * RDY_N + (home ? u : RDY_AWAY + u - 6 * KSTEPS), call);
    }
  }
}

__device__ __forceinline__ void convert_w(const Params& p, int first, unsigned call, int rank, LAS char* smem) {
  const int lane = threadIdx.x & 63, wave = threadIdx.x >> 6;
  const int count = KSTEPS * 4, stride = NCVT_W;
  char* const ws = p.ws[rank];
  const float* src0 = p.w + (size_t)((wave & 1) * 32 + (lane >> 4) * 4) * NSH + (wave >> 1) * 64 + (lane & 15) * 4;
  f32x4 v[8];
  int u = first;
  {
    const float* s = src0 + (size_t)(u >> 2) * 64 * NSH + (u & 3) * 256;
#pragma unroll
    for (int j = 0; j < 8; ++j) v[j] = ld4(s + ((j & 3) + (j >> 2) * 16) * NSH);
  }
  for (; u < count; u += stride) {
    const int ks = u >> 2, q = u & 3;
    u32x4 h[4];
#pragma unroll
    for (int f = 0; f < 4; ++f)
      h[f] = cvt8((f32x4){v[0][f], v[1][f], v[2][f], v[3][f]}, (f32x4){v[4][f], v[5][f], v[6][f], v[7][f]});
    {
      const int un = u + stride < count ? u + stride : u;
      const float* s = src0 + (size_t)(un >> 2) * 64 * NSH + (un & 3) * 256;
#pragma unroll
      for (int j = 0; j < 8; ++j) v[j] = ld4(s + ((j & 3) + (j >> 2) * 16) * NSH);
    }
    const unsigned oi = own_idx(call, 8 + q * 2 + (wave >> 2));
    char* b = ws + OFF_OWN + oi * PANEL_B + ks * STEP_B + (wave & 3) * 4096 + lane * 16;
#pragma unroll
    for (int f = 0; f < 4; ++f) store_sys(b + f * 1024, h[f]);
    char* pb = ws + OFF_PACK + oi * PPANEL_B + ks * PSTEP_B + (wave & 3) * 2 * BLK_B;
    pack_block(h[0], h[1], smem + wave * WAVE_LDS, smem + LDS_LUT, pb);
    pack_block(h[2], h[3], smem + wave * WAVE_LDS + 1024, smem + LDS_LUT, pb + BLK_B);
    wait_vm0();
    wg_barrier();
    if (threadIdx.x < 3) {
      const int t = threadIdx.x;
      if (t < 2) set_flag(ws + OFF_FLAG, (PB + q * 2 + t) * KSTEPS + ks, call);
      else set_flag(p.ws[(((rank & 4) ^ 4) | q) & 7] + OFF_RDY, rank * RDY_N + ks, call);
    }
  }
}

__device__ __forceinline__ int row_of(int r) { return ((RANK_AT >> (4 * r)) & 7) * NSH; }

__device__ __forceinline__ int batch_row(int s, int o, int bt, int i) {
  if (bt < 2) return row_of(s) + (bt * 3 + i) * 128;
  if (bt == 5 || bt == 8) return row_of((o & 4) | ((o + 1 + i) & 3)) + 768 + (bt == 8 ? 128 : 0);
  const int t = bt < 5 ? bt - 2 : bt == 9 ? 5 : bt - 3;
  return row_of((s & 4) | ((s + 1 + i) & 3)) + t * 128;
}

__device__ __forceinline__ int unit_of(int it, int w) { return it * PW + ((w + it) & (PW - 1)); }

template <class F>
__device__ __forceinline__ int wait_ready(const unsigned* rdy, F flag_of, int it0, int w, int iters, unsigned call,
                                          unsigned* tmo, bool& dead) {
  const int lane = threadIdx.x & 63;
  const unsigned* f = rdy + flag_of(unit_of(it0 + lane < iters ? it0 + lane : it0, w));
  const unsigned long long t0 = __builtin_amdgcn_s_memrealtime();
  for (unsigned it = 1;; ++it) {
    unsigned v;
    asm volatile("global_load_dword %0, %1, off sc0 sc1\n\ts_waitcnt vmcnt(0)" : "=v"(v) : "v"(f) : "memory");
    const unsigned long long miss = __builtin_amdgcn_ballot_w64(v < call);
    if (miss == 0ull) return 64;
    if (!(miss & 1ull)) return __builtin_ctzll(miss);
    if ((it & 63u) == 0u) {
      if (__builtin_amdgcn_readfirstlane(__hip_atomic_load(tmo, __ATOMIC_RELAXED, __HIP_MEMORY_SCOPE_AGENT)) == call
          || __builtin_amdgcn_s_memrealtime() - t0 > DEADLINE) {
        if (lane == 0) __hip_atomic_store(tmo, call, __ATOMIC_RELAXED, __HIP_MEMORY_SCOPE_AGENT);
        wait_vm0();
        dead = true;
        return 1 << 20;
      }
    }
    __builtin_amdgcn_s_sleep(2);
  }
}

__device__ __forceinline__ void copy_step(u32x4& next, const void* src, void* dst, u32x4 cur) {
  asm volatile("global_load_dwordx4 %0, %1, off sc0 sc1\n\t"
               "global_store_dwordx4 %2, %3, off sc0 sc1" OUT_NT "\n\t"
               "s_nop 0\n\ts_waitcnt vmcnt(1)"
               : "=&v"(next) : "v"(src), "v"(dst), "v"(cur) : "memory");
}
__device__ __forceinline__ u32x4 load_sys(const void* src) {
  u32x4 v;
  asm volatile("global_load_dwordx4 %0, %1, off sc0 sc1\n\ts_waitcnt vmcnt(0)" : "=v"(v) : "v"(src) : "memory");
  return v;
}
__device__ __forceinline__ unsigned lds_ticket(LAS unsigned* c) {
  unsigned t = 0;
  if ((threadIdx.x & 63) == 0) t = __hip_atomic_fetch_add(c, 1u, __ATOMIC_RELAXED, __HIP_MEMORY_SCOPE_WORKGROUP);
  return __builtin_amdgcn_readfirstlane(t);
}

template <bool CROSS>
__device__ __forceinline__ void pull_inputs(const Params& p, int w, unsigned call, int rank, int peer, LAS char* smem,
                                            bool& dead) {
  const int lane = threadIdx.x & 63, wave = __builtin_amdgcn_readfirstlane(threadIdx.x >> 6);
  const int iters = (CROSS ? 512 : 768) / PW;
  char* const ws = p.ws[rank];
  const char* const psrc = p.ws[peer] + OFF_PACK + own_idx(call, 0) * PPANEL_B + lane * 16;
  const char* const rsrc = p.ws[peer] + OFF_OWN + own_idx(call, 0) * PANEL_B + lane * 16;
  char* const dst = ws + OFF_PULL + lane * 16;
  const unsigned* rdy = (const unsigned*)(ws + OFF_RDY) + peer * RDY_N;
  unsigned* const tmo = (unsigned*)(ws + OFF_TMO);
  LAS unsigned* const cnt = (LAS unsigned*)(smem + LDS_MB);
  LAS char* const wl = smem + wave * WAVE_LDS;
  const unsigned wl0 = (unsigned)(uintptr_t)wl;
  auto flag_of = [](int u) { return CROSS ? (u < 256 ? u >> 1 : RDY_AWAY + u - 256) : u; };
  auto decode = [&](int it, unsigned& ss, unsigned& ds) {
    const int u = unit_of(it, w);
    if (!CROSS) { ss = u; ds = (peer & 3) * 6 * KSTEPS + u; }
    else if (u < 256) { const int o = (u & 1) * KSTEPS + (u >> 1); ss = (8 + (rank & 3) * 2) * KSTEPS + o; ds = (PB_FOREIGN + (peer & 3) * 2) * KSTEPS + o; }
    else { ss = 6 * KSTEPS + u - 256; ds = (PA_AWAY + (peer & 3) * 2) * KSTEPS + u - 256; }
  };
  int ready = EXPON(1) ? 1 << 20 : 0;
  auto fetch = [&](unsigned t, unsigned buf) {
    const int it = t >> 3;
    if (it >= ready) ready = it + wait_ready(rdy, flag_of, it, w, iters, call, tmo, dead);
    unsigned ss, ds, keep;
    decode(it, ss, ds);
    asm volatile("s_mov_b32 %0, m0\n\ts_mov_b32 m0, %2\n\ts_nop 0\n\t"
                 "global_load_lds_dwordx4 %1, off sc0 sc1\n\t"
                 "s_mov_b32 exec_hi, 0xffff\n\t"
                 "global_load_lds_dwordx4 %1, off offset:1024 sc0 sc1\n\t"
                 "s_mov_b32 exec_hi, -1\n\ts_mov_b32 m0, %0"
                 : "=&s"(keep) : "v"(psrc + (size_t)(ss * 8 + (t & 7)) * BLK_B), "s"(wl0 + buf) : "memory");
  };
  auto done = [&](unsigned t) {
    unsigned ss, ds;
    decode(t >> 3, ss, ds);
    if (lane == 0 && __hip_atomic_fetch_add(cnt + 2 + (t >> 3), 1u, __ATOMIC_RELAXED, __HIP_MEMORY_SCOPE_WORKGROUP) == 7u)
      set_flag(ws + OFF_FLAG, ds, call);
  };
  const unsigned total = iters * 8, none = ~0u;
  if (wave >= NWV) {
    if (CROSS) {
      const unsigned long long t0 = __builtin_amdgcn_s_memrealtime();
      while (__builtin_amdgcn_readfirstlane(__hip_atomic_load(cnt, __ATOMIC_RELAXED, __HIP_MEMORY_SCOPE_WORKGROUP)) < total
             && __builtin_amdgcn_s_memrealtime() - t0 < 2 * DEADLINE)
        __builtin_amdgcn_s_sleep(16);
    }
    return;
  }
  unsigned t = lds_ticket(cnt), tn = lds_ticket(cnt), p1 = none, p2 = none, buf = 0;
  fetch(t, 0);
  wait_vm0();
  if (wave == 0) STAMP(3);
  while (t < total) {
    if ((t & 63) == 56) STAMP(4 + (t >> 6));
    unsigned tnn = total;
    if (tn < total) { fetch(tn, buf ^ 2048); tnn = lds_ticket(cnt); }
    if (PULL_LAG) { if (p2 != none) done(p2); } else if (p1 != none) done(p1);
    u32x4 f0, f1;
    unsigned ss, ds;
    decode(t >> 3, ss, ds);
    if (!unpack_block(wl + buf, wl + 4096, smem + LDS_LUT, f0, f1)) {
      const char* r = rsrc + (size_t)(ss * 8 + (t & 7)) * 2048 + (ss >> 7) * SKEW;
      asm volatile("global_load_dwordx4 %0, %2, off sc0 sc1\n\t"
                   "global_load_dwordx4 %1, %2, off offset:1024 sc0 sc1\n\ts_waitcnt vmcnt(0)"
                   : "=&v"(f0), "=&v"(f1) : "v"(r) : "memory");
    }
    char* d = dst + (size_t)(ds * 8 + (t & 7)) * 2048 + (ds >> 7) * SKEW;
    store_sys(d, f0);
    store_sys(d + 1024, f1);
    if (PULL_LAG) asm volatile("s_waitcnt vmcnt(2)" ::: "memory"); else wait_vm0();
    p2 = p1; p1 = t; t = tn; tn = tnn; buf ^= 2048;
  }
  if (PULL_LAG) { wait_vm0(); if (p2 != none) done(p2); }
  if (p1 != none) done(p1);
}

__device__ __forceinline__ void pull_outputs(const Params& p, int w, unsigned call, int rank, int peer, LAS char* smem,
                                             bool& dead) {
  const int lane = threadIdx.x & 63;
  const int iters = OUT_CHUNKS / PW;
  char* const ws = p.ws[rank];
  const char* const src = p.ws[peer] + OFF_STG + (size_t)(rank & 3) * OUT_CHUNKS * CHUNK_B + lane * 16;
  float* const dst = p.out[rank] + (size_t)((lane >> 4) * 4) * NSH + (peer & 3) * 256 + (lane & 15) * 4;
  const unsigned* rdy = (const unsigned*)(ws + OFF_RDY) + peer * RDY_N + RDY_OUT;
  unsigned* const tmo = (unsigned*)(ws + OFF_TMO);
  LAS unsigned* const cnt = (LAS unsigned*)(smem + LDS_MB) + 1;
  auto flag_of = [](int u) { return u; };
  auto where = [&](unsigned t) {
    const int c = unit_of(t >> 3, w), tile = c >> 3, bt = tile / 6, r = tile - bt * 6;
    return dst + (size_t)(batch_row(peer, rank, bt, r >> 1) + ((c >> 2) & 1) * 64 + (c & 1) * 32 + (t & 4) * 4 + (t & 3)) * NSH
         + (r & 1) * 128 + ((c >> 1) & 1) * 64;
  };
  int ready = EXPON(17) ? 1 << 20 : 0;
  auto from = [&](unsigned t) {
    const int it = t >> 3;
    if (it >= ready) ready = it + wait_ready(rdy, flag_of, it, w, iters, call, tmo, dead);
    return src + (size_t)(unit_of(it, w) * 8 + (t & 7)) * 1024;
  };
  const u32x4 qnan = {0x7fc00000u, 0x7fc00000u, 0x7fc00000u, 0x7fc00000u};
  const unsigned total = iters * 8;
  unsigned t = lds_ticket(cnt);
  if (t >= total) return;
  unsigned tn = lds_ticket(cnt);
  u32x4 cur = load_sys(from(t));
  while (tn < total) {
    if (t % 48 == 40) STAMP(20 + t / 48);
    u32x4 next;
    const char* s = from(tn);
    float* d = where(t);
    t = tn;
    tn = lds_ticket(cnt);
    copy_step(next, s, d, dead ? qnan : cur);
    cur = next;
  }
  if (__builtin_amdgcn_readfirstlane(__hip_atomic_load(tmo, __ATOMIC_RELAXED, __HIP_MEMORY_SCOPE_AGENT)) == call) dead = true;
  store_sys(where(t), dead ? qnan : cur);
}

struct Job { int pa, pb, half, bt, i; };
__device__ __forceinline__ void decode_job(unsigned code, int a, Job& o) {
  const int ra = code & 63;
  o.pb = PB + ((code >> 6) & 15);
  o.half = (int)((code >> 10) & 3) - 1;
  if (ra < 18) {
    const int t = ra % 6;
    o.i = ra / 6; o.pa = ((a + 1 + o.i) & 3) * 6 + t; o.bt = t < 3 ? 2 + t : t == 5 ? 9 : 3 + t;
  } else if (ra < 24) {
    o.pa = a * 6 + ra - 18; o.bt = (ra - 18) / 3; o.i = (ra - 18) % 3;
  } else {
    o.pa = ra; o.bt = 5 + 3 * (ra & 1); o.i = (((ra - PA_AWAY) >> 1) - ((o.pb - PB_FOREIGN) >> 1) - 1) & 3;
  }
}

__device__ __forceinline__ float silu(float y) {
  const float c = fminf(fmaxf(y, -60.0f), 60.0f);
  return y * __builtin_amdgcn_rcpf(1.0f + __builtin_amdgcn_exp2f(c * -1.44269504088896341f));
}

#define GLDS_BITS " sc0 sc1"

__device__ __forceinline__ unsigned next_stage(unsigned so) { return so == LDS_MB - STAGE_B ? 0u : so + STAGE_B; }

template <bool WIDE>
__device__ __forceinline__ void mfma_wave(const Params& p, const LAS char* smem, const Job& jb, int job, int wave,
                                          int rank, unsigned call, unsigned& so) {
  constexpr int MT = WIDE ? 4 : 2, NT = WIDE ? 8 : 4, STEPS = WIDE ? 2 * KSTEPS : KSTEPS;
  int lane = fresh_lane();
  const int wr = wave >> 1, wc = wave & 1;
  f32x4 acc[MT][NT];
#pragma unroll
  for (int m = 0; m < MT; ++m)
#pragma unroll
    for (int n = 0; n < NT; ++n) acc[m][n] = (f32x4){0.f, 0.f, 0.f, 0.f};
  half8 fa[2][4], fb[2][8];
  const LAS char* la = smem + wr * MT * 1024 + lane * 16;
  const LAS char* lb = smem + 8192 + wc * 8192 + lane * 16;
#define READ_SET(X) do { \
    _Pragma("unroll") for (int i = 0; i < 8; ++i) { \
      if (i < 4) fa[X][i] = *(const LAS half8*)(la + so + (WIDE ? i * 1024 : (i >> 1) * 4096 + (i & 1) * 1024)); \
      fb[X][i] = *(const LAS half8*)(lb + so + i * 1024); } \
    so = next_stage(so); } while (0)
#define MMA_SET(X) do { \
    _Pragma("unroll") for (int kh = 0; kh < (WIDE ? 1 : 2); ++kh) \
    _Pragma("unroll") for (int m = 0; m < MT; ++m) \
    _Pragma("unroll") for (int n = 0; n < NT; ++n) \
      acc[m][n] = __builtin_amdgcn_mfma_f32_16x16x32_f16(fa[X][kh * 2 + m], fb[X][kh * 4 + n], acc[m][n], 0, 0, 0); } while (0)
#define INTERLEAVE() do { \
    _Pragma("unroll") for (int i = 0; i < (WIDE ? 12 : 6); ++i) { \
      __builtin_amdgcn_sched_group_barrier(0x100, WIDE ? 1 : 2, 0); \
      __builtin_amdgcn_sched_group_barrier(0x008, 2, 0); } \
    __builtin_amdgcn_sched_group_barrier(0x008, WIDE ? 8 : 4, 0); } while (0)
#define STEP_END() do { __builtin_amdgcn_sched_barrier(0); asm volatile("" ::: "memory"); \
    __builtin_amdgcn_s_waitcnt(0xC07F);   __builtin_amdgcn_sched_barrier(0); } while (0)
  wg_barrier();
  if (wave == 0) STAMP(4 + job * 4);
  READ_SET(0);
  STEP_END();
  for (int t = 1; t < STEPS - 1; t += 2) {
    wg_barrier();
    __builtin_amdgcn_sched_barrier(0);
    READ_SET(1); MMA_SET(0); INTERLEAVE();
    STEP_END();
    wg_barrier();
    __builtin_amdgcn_sched_barrier(0);
    READ_SET(0); MMA_SET(1); INTERLEAVE();
    STEP_END();
  }
  wg_barrier();
  __builtin_amdgcn_sched_barrier(0);
  READ_SET(1); MMA_SET(0); INTERLEAVE();
  const u32x4 tm = *(const LAS u32x4*)(smem + LDS_TMO + job * 16);
  STEP_END();
  MMA_SET(1);
  __builtin_amdgcn_sched_barrier(0);
  if (wave == 0) STAMP(5 + job * 4);
  lane = fresh_lane();
  const bool dead = tm[0] == call || tm[1] == call || tm[2] == call;

  const bool foreign = jb.pb >= PB_FOREIGN;
  const int ob = (jb.pb - PB_FOREIGN) >> 1;
  const int r32 = WIDE ? wr * 2 : jb.half * 2 + wr;
  const int c64 = WIDE ? wc * 2 : (jb.pb & 1) * 2 + wc;
  const int chunk = ((jb.bt * 3 + jb.i) * 2 + (c64 >> 1)) * 8 + (r32 >> 1) * 4 + (c64 & 1) * 2 + (r32 & 1);
  char* o;
  unsigned sm, sr, sh;
  if (foreign) {
    o = p.ws[rank] + OFF_STG + (size_t)(ob * OUT_CHUNKS + chunk) * CHUNK_B + lane * 16;
    sm = 4096; sr = 1024; sh = 2 * CHUNK_B;
  } else {
    const int pa = jb.pa;
    const int row0 = pa < PA_AWAY ? row_of((rank & 4) + pa / 6) + (pa % 6) * 128
                   : row_of(pa < PA_OWNAWAY ? ((rank & 4) ^ 4) + ((pa - PA_AWAY) >> 1) : rank) + 768 + (pa & 1) * 128;
    o = (char*)(p.out[rank] + (size_t)(row0 + r32 * 32 + (lane >> 4) * 4) * NSH + ((jb.pb - PB) >> 1) * 256 + c64 * 64 + (lane & 15) * 4);
    sm = 16 * NSH * 4; sr = NSH * 4; sh = 256;
  }
  const float qnan = __builtin_nanf("");
#pragma unroll
  for (int h = 0; h < NT / 4; ++h)
#pragma unroll
    for (int m = 0; m < MT; ++m)
#pragma unroll
      for (int r = 0; r < 4; ++r) {
        f32x4 v = {silu(acc[m][h * 4][r]), silu(acc[m][h * 4 + 1][r]), silu(acc[m][h * 4 + 2][r]), silu(acc[m][h * 4 + 3][r])};
        if (dead) v = (f32x4){qnan, qnan, qnan, qnan};
        if (foreign) store_sys(o + h * sh + m * sm + r * sr, __builtin_bit_cast(u32x4, v));
        else store_out(o + h * sh + m * sm + r * sr, __builtin_bit_cast(u32x4, v));
      }
  if (foreign) {
    wait_vm0();
    if (lane < (WIDE ? 4 : 1)) set_flag(p.ws[(((rank & 4) ^ 4) | ob) & 7] + OFF_RDY, rank * RDY_N + RDY_OUT + chunk + lane, call);
  }
  if (wave == 0) STAMP(6 + job * 4);
}

__global__ void __launch_bounds__(NTHREADS, 2) ag_gemm_silu(const Params p) {
  __shared__ __attribute__((aligned(16))) char smem[LDS_BYTES];
  const int rank = p.rank & 7, a = rank & 3;
  const int wave = __builtin_amdgcn_readfirstlane(threadIdx.x >> 6);
#define XROT 4
  const int xcd = (blockIdx.x - XROT) & 7, slot = blockIdx.x >> 3;
  char* const ws = p.ws[rank];
  unsigned* const tmo = (unsigned*)(ws + OFF_TMO);

  unsigned* const cnt = (unsigned*)(ws + OFF_CALL) + blockIdx.x;
  const unsigned call = __builtin_amdgcn_readfirstlane(
      __hip_atomic_load(cnt, __ATOMIC_RELAXED, __HIP_MEMORY_SCOPE_AGENT)) + 1u;
  if (wave == 0) STAMP(0);
  const bool puller = slot >= GSLOTS && xcd < 7;
  for (unsigned i = threadIdx.x; i < (LDS_LUT - LDS_MB) / 4; i += NTHREADS) ((LAS unsigned*)(smem + LDS_MB))[i] = 0u;
  if (threadIdx.x < 16) ((LAS unsigned*)(smem + LDS_LUT))[threadIdx.x] = codec_lut(threadIdx.x, puller);
  asm volatile("s_waitcnt vmcnt(0) lgkmcnt(0)" ::: "memory");
  wg_barrier();
  if (threadIdx.x == 0) __hip_atomic_store(cnt, call, __ATOMIC_RELAXED, __HIP_MEMORY_SCOPE_AGENT);

  if (puller) {
    const int w = slot - GSLOTS;
    bool dead = false;
    if (EXPON(2)) return;
    if (xcd < 4) {
      const int peer = ((rank & 4) ^ 4) | ((a + xcd) & 3);
      pull_inputs<true>(p, w, call, rank, peer, (LAS char*)smem, dead);
      if (wave == 0) STAMP(1);
      pull_outputs(p, w, call, rank, peer, (LAS char*)smem, dead);
    } else {
      pull_inputs<false>(p, w, call, rank, (rank & 4) | ((a + xcd - 3) & 3), (LAS char*)smem, dead);
    }
    if (wave == 0) STAMP(2);
    return;
  }

  const int gi = slot < GSLOTS ? slot * 8 + xcd : GSLOTS * 8 + slot - GSLOTS;
  if (EXPON(1)) {} else
  if (gi < NCVT_X) convert_x(p, gi, call, rank, (LAS char*)smem);
  else convert_w(p, gi - NCVT_X, call, rank, (LAS char*)smem);
  if (wave == 0) STAMP(1);

  const unsigned lds0 = (unsigned)(uintptr_t)(LAS char*)smem;
  unsigned so = 0;
  for (int job = 0; job < NJOBS; ++job) {
    const unsigned code = p.jobs[gi][job];
    if (!(code & 0x8000u)) break;
    if (EXPON(4) && (job > 0 || slot >= 12)) break;
    if (EXPON(8) && job > 0) break;
    Job jb;
    decode_job(code, a, jb);
    const bool wide = jb.half < 0;
    if (wave >= 4) {
      const int l = wave - 4, lane = fresh_lane();
      const int steps = wide ? 2 * KSTEPS : KSTEPS, ksh = wide ? 1 : 0;
      if (l == 3) {
        for (int t = 0; t < steps; ++t) wg_barrier();
        continue;
      }
      const int panel = l == 0 ? jb.pa : wide ? jb.pb + l - 1 : jb.pb;
      const int own = panel >= PB_FOREIGN ? -1 : panel >= PA_OWNAWAY ? panel - PA_OWNAWAY + 6
                    : (unsigned)(panel - a * 6) < 6u ? panel - a * 6 : -1;
      const char* src = ws + (own >= 0 ? OFF_OWN + own_idx(call, own) * PANEL_B : OFF_PULL + (size_t)panel * PANEL_B)
                      + (wide ? 0 : (l == 0 ? jb.half : l - 1) * 8192) + lane * 16;
      const unsigned gap = wide ? 8192 : 4096;
      const char* fsrc = ws + OFF_FLAG + panel * (KSTEPS * 4) + lane * 16;
      const unsigned mb = lds0 + LDS_MB + (job * 4 + l) * 512;
      const unsigned dst0 = lds0 + l * 8192;
      int ready = EXPON(1) ? KSTEPS : 0;
      auto poll = [&]() {
        if (lane < 32) {
          unsigned keep;
          asm volatile("s_mov_b32 %0, m0\n\ts_mov_b32 m0, %2\n\ts_nop 0\n\t"
                       "global_load_lds_dwordx4 %1, off sc0 sc1\n\ts_mov_b32 m0, %0"
                       : "=&s"(keep) : "v"(fsrc), "s"(mb) : "memory");
        }
      };
      auto refresh = [&]() {
        u32x2 f;
        asm volatile("ds_read_b64 %0, %1\n\ts_waitcnt lgkmcnt(0)" : "=v"(f) : "v"(mb + lane * 8) : "memory");
        const unsigned long long b0 = __builtin_amdgcn_ballot_w64(f[0] >= call);
        const unsigned long long b1 = __builtin_amdgcn_ballot_w64(f[1] >= call);
        const unsigned long long miss = ~(b0 & b1);
        if (miss == 0ull) ready = KSTEPS;
        else { const int n = __builtin_ctzll(miss); ready = 2 * n + (int)((b0 >> n) & 1ull); }
      };
      auto issue = [&](int s) {
        const int ks = s >> ksh;
        if (ks >= ready) {
          refresh();
          if (ks >= ready) {
            const unsigned long long t0 = __builtin_amdgcn_s_memrealtime();
            for (unsigned it = 1;; ++it) {
              poll();
              wait_vm0();
              refresh();
              if (ks < ready) break;
              if ((it & 63u) == 0u) {
                if (__builtin_amdgcn_readfirstlane(__hip_atomic_load(tmo, __ATOMIC_RELAXED, __HIP_MEMORY_SCOPE_AGENT)) == call
                    || __builtin_amdgcn_s_memrealtime() - t0 > DEADLINE) {
                  if (lane == 0) __hip_atomic_store(tmo, call, __ATOMIC_RELAXED, __HIP_MEMORY_SCOPE_AGENT);
                  wait_vm0();
                  ready = KSTEPS;
                  break;
                }
              }
              __builtin_amdgcn_s_sleep(4);
            }
          }
        }
        if (ready < KSTEPS && ready - ks < 8 && (s & 1)) poll();
        const char* g = src + (size_t)ks * STEP_B + (s & ksh) * 4096;
        const unsigned d = dst0 + so;
        so = next_stage(so);
        unsigned keep;
        asm volatile("s_mov_b32 %0, m0\n\ts_mov_b32 m0, %3\n\ts_nop 0\n\t"
                     "global_load_lds_dwordx4 %1, off" GLDS_BITS "\n\t"
                     "global_load_lds_dwordx4 %1, off offset:1024" GLDS_BITS "\n\t"
                     "global_load_lds_dwordx4 %1, off offset:2048" GLDS_BITS "\n\t"
                     "global_load_lds_dwordx4 %1, off offset:3072" GLDS_BITS "\n\t"
                     "s_mov_b32 m0, %4\n\ts_nop 0\n\t"
                     "global_load_lds_dwordx4 %2, off" GLDS_BITS "\n\t"
                     "global_load_lds_dwordx4 %2, off offset:1024" GLDS_BITS "\n\t"
                     "global_load_lds_dwordx4 %2, off offset:2048" GLDS_BITS "\n\t"
                     "global_load_lds_dwordx4 %2, off offset:3072" GLDS_BITS "\n\t"
                     "s_mov_b32 m0, %0"
                     : "=&s"(keep) : "v"(g), "v"(g + gap), "s"(d), "s"(d + 4096) : "memory");
      };
      for (int t = 0; t < steps; ++t) {
        if (t >= NSTAGE - 1) {
          asm volatile("s_waitcnt vmcnt(32)" ::: "memory");
          wg_barrier();
        }
        issue(t);
      }
      if (lane == 0) {
        unsigned keep;
        asm volatile("s_mov_b32 %0, m0\n\ts_mov_b32 m0, %2\n\ts_nop 0\n\t"
                     "global_load_lds_dword %1, off sc1\n\ts_mov_b32 m0, %0"
                     : "=&s"(keep) : "v"(tmo), "s"(lds0 + LDS_TMO + (job * 4 + l) * 4) : "memory");
      }
      asm volatile("s_waitcnt vmcnt(33)\n\ts_barrier\n\ts_waitcnt vmcnt(25)\n\ts_barrier\n\t"
                   "s_waitcnt vmcnt(17)\n\ts_barrier\n\ts_waitcnt vmcnt(9)\n\ts_barrier\n\t"
                   "s_waitcnt vmcnt(0)\n\ts_barrier" ::: "memory");
    } else if (wide) {
      mfma_wave<true>(p, (const LAS char*)smem, jb, job, wave, rank, call, so);
    } else {
      mfma_wave<false>(p, (const LAS char*)smem, jb, job, wave, rank, call, so);
    }
  }
}

void make_schedule(Params& p) {
  int n[NGEMM] = {};
  for (int g = 0; g < NGEMM; ++g)
    for (int j = 0; j < NJOBS; ++j) p.jobs[g][j] = 0;
  auto put = [&](int xcd, int s, int ra, int b, int half) {
    const int g = s < GSLOTS ? s * 8 + xcd : GSLOTS * 8 + s - GSLOTS;
    p.jobs[g][n[g]++] = (unsigned short)(0x8000 | ((half + 1) << 10) | (b << 6) | ra);
  };
  auto P = [&](int xcd, int s0, int t, int b0) { for (int s = 0; s < 12; ++s) put(xcd, s0 + s, s % 3 * 6 + t, b0 + s / 3 * 2, -1); };
  auto L = [&](int xcd, int h, int b0) { for (int s = 0; s < 12; ++s) put(xcd, s, 18 + h * 3 + s % 3, b0 + s / 3 * 2, -1); };
  auto Af = [&](int xcd, int j) {
    for (int s = 0; s < 12; ++s) put(xcd, 12 + s, PA_AWAY + (s & 3) * 2 + j, 8 + (((s & 3) + 1 + (s >> 2)) & 3) * 2, -1);
  };
  auto H = [&](int xcd, int s0, int t, int b0) {
    for (int s = 0; s < 12; ++s) put(xcd, s0 + s, s % 3 * 6 + t, b0 + s / 6, (s / 3) & 1);
  };
  P(0, 0, 0, 0); P(1, 0, 0, 8); P(2, 0, 1, 0); P(3, 0, 1, 8);
  L(4, 0, 8); L(5, 1, 8); L(6, 0, 0); L(7, 1, 0);
  for (int s = 0; s < 8; ++s) put(3, 12 + s, PA_OWNAWAY + (s & 1), s / 2 * 2, -1);
  P(0, 12, 2, 0); P(1, 12, 2, 8); Af(4, 0);
  for (int s = 0; s < 16; ++s) put(7, 12 + s, PA_AWAY + (s & 3) * 2, s / 4 * 2, -1);
  P(2, 12, 3, 0); Af(5, 1);
  for (int s = 0; s < 16; ++s) put(s < 12 ? 6 : 7, s < 12 ? 12 + s : 16 + s, PA_AWAY + (s & 3) * 2 + 1, s / 4 * 2, -1);
  const int early[4] = {0, 1, 4, 7}, late[4] = {2, 3, 5, 6};
  for (int q = 0; q < 4; ++q) {
    H(early[q], 0, 3, 8 + q * 2);
    H(early[q], 0, 4, q * 2);
    H(early[q], 12, 5, 8 + q * 2);
    H(late[q], 0, 4, 8 + q * 2);
    H(late[q], q == 1 ? 12 : 0, 5, q * 2);
  }
}

}

extern "C" void kernel_launch(int rank, int nranks, ncclComm_t comm, hipStream_t stream,
                              void* const* d_in, const int* in_sizes, int n_in,
                              void* d_out, int out_size, const int* in_dtypes, int out_dtype,
                              void* const* d_out_all, void* const* d_ws_all, size_t ws_bytes) {
  (void)comm; (void)in_dtypes; (void)out_dtype; (void)d_out;
  if (nranks != NR || (unsigned)rank >= (unsigned)NR || n_in != 2 || in_sizes[0] != NSH * KDIM ||
      in_sizes[1] != KDIM * NSH || out_size != KDIM * NSH || ws_bytes < WS_NEED) {
    fprintf(stderr, "kernel_launch: unexpected problem shape\n");
    return;
  }
  Params p;
  p.x = (const float*)d_in[0];
  p.w = (const float*)d_in[1];
  p.rank = 0;
  for (int i = 0; i < NR; ++i) {
    const int r = (RANK_AT >> (4 * i)) & 7;
    p.out[i] = (float*)d_out_all[r];
    p.ws[i] = (char*)d_ws_all[r];
    if (r == rank) p.rank = i;
  }
  p.pad = 0;
  make_schedule(p);
  ag_gemm_silu<<<dim3(NWG), dim3(NTHREADS), 0, stream>>>(p);
  HIP_CHECK(hipGetLastError());
}
```

```cpp
#include <hip/hip_runtime.h>
#include <rccl/rccl.h>
#include <cstdio>
#include <cstdint>

#define HIP_CHECK(expr) do { hipError_t _e = (expr); if (_e != hipSuccess) { \
    fprintf(stderr, "HIP error %s at %s:%d\n", hipGetErrorString(_e), __FILE__, __LINE__); \
    return; } } while (0)

namespace {

typedef _Float16 half8 __attribute__((ext_vector_type(8)));
typedef float f32x4 __attribute__((ext_vector_type(4)));
typedef unsigned u32x4 __attribute__((ext_vector_type(4)));
typedef unsigned u32x2 __attribute__((ext_vector_type(2)));
#define LAS __attribute__((address_space(3)))
#define GAS __attribute__((address_space(1)))

constexpr int NR = 8;
#define RANK_AT_V 0x75416320u
constexpr unsigned RANK_AT = RANK_AT_V;
constexpr int KDIM = 8192;
constexpr int NSH = 1024;
constexpr int KSTEPS = 128;
constexpr unsigned STEP_B = 16384;
#define SKEW 0
constexpr size_t PANEL_B = (size_t)KSTEPS * STEP_B + SKEW;
constexpr int PA_AWAY = 24, PA_OWNAWAY = 32, PB = 34, PB_FOREIGN = 42, NPANEL = 50;

constexpr unsigned BLK_B = 1792;
constexpr unsigned PSTEP_B = 8 * BLK_B;
constexpr size_t PPANEL_B = (size_t)KSTEPS * PSTEP_B;

constexpr size_t OFF_CALL = 0;
constexpr size_t OFF_TMO = 4096;
constexpr size_t OFF_FLAG = 8192;
constexpr size_t OFF_RDY = 1u << 20;
constexpr int RDY_N = 1024, RDY_AWAY = 128, RDY_OUT = 384;
constexpr size_t OFF_DBG = 65536;
constexpr size_t OFF_OWN = 2u << 20;
constexpr size_t OFF_PACK = OFF_OWN + 32 * PANEL_B;
constexpr size_t OFF_PULL = OFF_PACK + 32 * PPANEL_B;
constexpr int OUT_CHUNKS = 480;
constexpr unsigned CHUNK_B = 8192;
constexpr size_t OFF_STG = OFF_PULL + NPANEL * PANEL_B;
constexpr size_t WS_NEED = OFF_STG + (size_t)4 * OUT_CHUNKS * CHUNK_B;
static_assert(OFF_FLAG + NPANEL * 1024 <= OFF_DBG && OFF_RDY + NR * RDY_N * 128 <= OFF_OWN, "");

constexpr int NWG = 256, NTHREADS = 512;
constexpr int GSLOTS = 24;
constexpr int NGEMM = GSLOTS * 8 + 8;
constexpr int PW = 8;
#define NWV 5
#define PULL_LAG 1
constexpr int NCVT_W = 104, NCVT_X = 96;
constexpr int NJOBS = 4;
constexpr unsigned long long DEADLINE = 100000000ull;

constexpr unsigned LDS_MB = 18 * 8192;
constexpr unsigned LDS_TMO = LDS_MB + NJOBS * 4 * 512;
constexpr unsigned LDS_CNT = LDS_TMO + NJOBS * 16;
constexpr unsigned LDS_LUT = LDS_CNT + NJOBS * 8;
constexpr unsigned LDS_BYTES = LDS_LUT + 64;
constexpr unsigned WAVE_LDS = 8192;

struct Params {
  const float* x;
  const float* w;
  float* out[NR];
  char* ws[NR];
  int rank;
  int pad;
  unsigned short jobs[NGEMM][NJOBS];
};

#define STAMP(idx) do {} while (0)

#define EXP 0
#define EXPON(bit) ((EXP & (bit)) && call > 2)
__device__ __forceinline__ void store_sys(void* ptr, u32x4 v) {
  asm volatile("global_store_dwordx4 %0, %1, off sc0 sc1\n\ts_nop 1" :: "v"(ptr), "v"(v) : "memory");
}
__device__ __forceinline__ void wait_vm0() { asm volatile("s_waitcnt vmcnt(0)" ::: "memory"); }
__device__ __forceinline__ void wg_barrier() {
  asm volatile("" ::: "memory");
  __builtin_amdgcn_s_barrier();
  asm volatile("" ::: "memory");
}
__device__ __forceinline__ u32x4 cvt8(f32x4 a, f32x4 b) {
  half8 h;
  h[0] = (_Float16)a[0]; h[1] = (_Float16)a[1]; h[2] = (_Float16)a[2]; h[3] = (_Float16)a[3];
  h[4] = (_Float16)b[0]; h[5] = (_Float16)b[1]; h[6] = (_Float16)b[2]; h[7] = (_Float16)b[3];
  return __builtin_bit_cast(u32x4, h);
}
__device__ __forceinline__ f32x4 ld4(const float* ptr) { return __builtin_nontemporal_load((const GAS f32x4*)ptr); }
#define OUT_NT " nt"
__device__ __forceinline__ void store_out(void* ptr, u32x4 v) {
  asm volatile("global_store_dwordx4 %0, %1, off sc0 sc1" OUT_NT "\n\ts_nop 1" :: "v"(ptr), "v"(v) : "memory");
}

__device__ __forceinline__ void set_local(char* ws, int step, unsigned v) {
  __hip_atomic_store((unsigned*)(ws + OFF_FLAG) + (step >> 7) * 256 + (step & 7) * 32 + ((step & 127) >> 3), v,
                     __ATOMIC_RELAXED, __HIP_MEMORY_SCOPE_SYSTEM);
}
__device__ __forceinline__ void set_rdy(char* peer_ws, int rank, int idx, unsigned v) {
  __hip_atomic_store((unsigned*)(peer_ws + OFF_RDY) + (rank * RDY_N + idx) * 32, v, __ATOMIC_RELAXED, __HIP_MEMORY_SCOPE_SYSTEM);
}
__device__ __forceinline__ unsigned own_idx(unsigned call, int idx) { return (call & 1) * 16 + idx; }
__device__ __forceinline__ int fresh_lane() {
  int l;
  asm volatile("v_mbcnt_lo_u32_b32 %0, -1, 0\n\tv_mbcnt_hi_u32_b32 %0, -1, %0" : "=v"(l));
  return l;
}
__device__ __forceinline__ void cbar() { asm volatile("" ::: "memory"); }
__device__ __forceinline__ unsigned wave_scan(unsigned v) {
  v += __builtin_amdgcn_update_dpp(0u, v, 0x111, 0xf, 0xf, false);
  v += __builtin_amdgcn_update_dpp(0u, v, 0x112, 0xf, 0xf, false);
  v += __builtin_amdgcn_update_dpp(0u, v, 0x114, 0xf, 0xf, false);
  v += __builtin_amdgcn_update_dpp(0u, v, 0x118, 0xf, 0xf, false);
  v += __builtin_amdgcn_update_dpp(0u, v, 0x142, 0xa, 0xf, false);
  v += __builtin_amdgcn_update_dpp(0u, v, 0x143, 0xc, 0xf, false);
  return v;
}

__device__ __forceinline__ unsigned codec_lut(int m, bool expand) {
  unsigned sel = 0x0c0c0c0cu;
  int n = 0;
  for (int i = 0; i < 4; ++i)
    if ((m >> i) & 1) {
      const int at = expand ? i : n, v = expand ? n : i;
      sel = (sel & ~(0xffu << (8 * at))) | ((unsigned)v << (8 * at));
      ++n;
    }
  return sel;
}

__device__ __forceinline__ void pack_block(u32x4 f0, u32x4 f1, LAS char* wl, const LAS char* lut, char* dst) {
  const int lane = threadIdx.x & 63;
  const unsigned h[8] = {f0[0], f0[1], f0[2], f0[3], f1[0], f1[1], f1[2], f1[3]};
  if (lane < 36) *(LAS unsigned*)(wl + 640 + lane * 4) = 0u;
  u32x4 D;
  unsigned H[4], sum = 0;
#pragma unroll
  for (int k = 0; k < 4; ++k) {
    D[k] = __builtin_amdgcn_perm(h[2 * k + 1], h[2 * k], 0x06040200u);
    H[k] = __builtin_amdgcn_perm(h[2 * k + 1], h[2 * k], 0x07050301u);
    sum = __builtin_amdgcn_udot4(H[k] & 0x7c7c7c7cu, 0x01010101u, sum, false);
  }
  store_sys(dst + lane * 16, D);
  const unsigned tot = __builtin_amdgcn_readlane(wave_scan(sum), 63);
  const unsigned base = min((tot + 256u) >> 12, 28u);
  const unsigned bB = base * 0x04040404u;
  unsigned t[4], pk[4], n[4], EM = 0, XM = 0, sg = 0;
#pragma unroll
  for (int k = 0; k < 4; ++k) {
    const unsigned g = ((H[k] & 0x7c7c7c7cu) | 0x80808080u) - bB;
    const unsigned a7 = g & 0x7f7f7f7fu, hi = a7 + 0x70707070u, lo = a7 + 0x1c1c1c1cu;
    const unsigned o7 = (~g | hi) & 0x80808080u;
    t[k] = ((g | (o7 >> 4) | (o7 >> 5)) & 0x0c0c0c0cu) | (H[k] & 0x03030303u);
    const unsigned E = t[k] & (t[k] >> 1) & 0x04040404u;
    const unsigned nb = ~g & 0x80808080u;
    const unsigned c4 = g & ((nb >> 2) - (nb >> 5));
    const unsigned X = ((g & hi) | (~g & ~lo)) & 0x80808080u;
    const unsigned sel = *(const LAS unsigned*)(lut + __builtin_amdgcn_udot4(E, 0x08040201u, 0u, false));
    const unsigned z = __builtin_amdgcn_perm(0u, c4, sel);
    pk[k] = __builtin_amdgcn_udot4(z, 0x00400801u, (z >> 24) << 9, false);
    n[k] = __builtin_popcount(E);
    EM |= (E >> 2) << k;
    XM |= X >> (7 - k);
    sg |= (H[k] & 0x80808080u) >> (7 - k);
  }
  const unsigned mine = (n[0] + n[1] + n[2] + n[3]) | (__builtin_popcount(XM) << 16);
  const unsigned incl = wave_scan(mine);
  const unsigned all = __builtin_amdgcn_readlane(incl, 63), n2 = all & 0xffffu, n3 = all >> 16;
  const bool raw = 3u * n2 + 16u * n3 > 1008u;
  *(LAS u32x2*)(wl + lane * 8) = (u32x2){t[0] | (t[1] << 4), t[2] | (t[3] << 4)};
  const unsigned sgn = __builtin_amdgcn_update_dpp(0u, sg, 0xb1, 0xf, 0xf, false);
  if (!(lane & 1)) *(LAS unsigned*)(wl + 512 + lane * 2) = sg | (sgn << 4);
  cbar();
  if (!raw) {
    const unsigned pre = (incl - mine) & 0xffffu;
    unsigned pre3 = (incl - mine) >> 16;
    const unsigned long long W = ((unsigned long long)(pk[0] | (pk[1] << (3 * n[0])))
        | ((unsigned long long)(pk[2] | (pk[3] << (3 * n[2]))) << (3 * (n[0] + n[1])))) >> 2;
    const unsigned o = 3 * pre, sh = o & 31u;
    LAS unsigned* q = (LAS unsigned*)(wl + 640 + (o >> 5) * 4);
    const unsigned long long Ws = W << sh;
    __hip_atomic_fetch_or(q, (unsigned)Ws, __ATOMIC_RELAXED, __HIP_MEMORY_SCOPE_WORKGROUP);
    __hip_atomic_fetch_or(q + 1, (unsigned)(Ws >> 32), __ATOMIC_RELAXED, __HIP_MEMORY_SCOPE_WORKGROUP);
    __hip_atomic_fetch_or(q + 2, ((unsigned)(W >> 32) >> 1) >> (31u - sh), __ATOMIC_RELAXED, __HIP_MEMORY_SCOPE_WORKGROUP);
    cbar();
    while (XM) {
      const unsigned pos = __builtin_ctz(XM), k = pos & 3u, i8 = pos & 24u;
      const unsigned Hk = k & 2u ? (k & 1u ? H[3] : H[2]) : (k & 1u ? H[1] : H[0]);
      const unsigned kb = 0x01010101u << k;
      const unsigned rk = pre + __builtin_popcount(EM & ((kb - 0x01010101u) | (kb & ((1u << i8) - 1u))));
      *(LAS unsigned short*)(wl + 640 + 124 - 2 * pre3) = (unsigned short)((rk << 5) | (((Hk >> (i8 + 2)) & 31u) ^ (base + 3u)));
      ++pre3;
      XM &= XM - 1u;
    }
  }
  cbar();
  if (lane == 0) *(LAS unsigned short*)(wl + 640 + 126) = (unsigned short)(raw ? 0x8000u : base | (n3 << 5));
  cbar();
  if (lane < 48) store_sys(dst + 1024 + lane * 16, *(const LAS u32x4*)(wl + lane * 16));
  cbar();
}

__device__ __forceinline__ bool unpack_block(const LAS char* blk, LAS char* ex, const LAS char* lut, u32x4& f0, u32x4& f1) {
  const int lane = threadIdx.x & 63;
  const unsigned ent = *(const LAS unsigned short*)(blk + 1790 - 2 * lane);
  const LAS unsigned* sp = (const LAS unsigned*)(blk + 1664 + ((3 * lane) & ~3));
  const unsigned s0 = sp[0], s1 = sp[1];
  const u32x2 N = *(const LAS u32x2*)(blk + 1024 + lane * 8);
  const unsigned sg = *(const LAS unsigned*)(blk + 1536 + (lane >> 1) * 4) >> ((lane & 1) * 4);
  const u32x4 D = *(const LAS u32x4*)(blk + lane * 16);
  const unsigned hdr = __builtin_amdgcn_readfirstlane(ent);
  if (hdr & 0x8000u) return false;
  const unsigned base = hdr & 31u, n3 = hdr >> 5;
  const unsigned b3 = (base + 3u) * 0x04040404u, b8 = (base + 24u) * 0x04040404u;
  const unsigned T0 = (((b8 + 0x0c080400u) ^ b3) & 0x7c7c7c00u), T1 = (((b8 + 0x1c181410u) ^ b3) & 0x7c7c7c7cu);
  const unsigned x = __builtin_amdgcn_alignbyte(s1, s0, 3 * lane);
  u32x2 X;
#pragma unroll
  for (int i = 0; i < 2; ++i) {
    const unsigned y = ((x >> (12 * i)) & 0x3fu) | ((i ? x >> 2 : x << 10) & 0x3f0000u);
    X[i] = __builtin_amdgcn_perm(T1, T0, (y | (y << 5)) & 0x07070707u);
  }
  *(LAS u32x2*)(ex + lane * 8) = X;
  cbar();
  if ((unsigned)(lane - 1) < n3) *(LAS unsigned char*)(ex + ((ent >> 5) & 511u)) = (unsigned char)((ent & 31u) << 2);
  cbar();
  const unsigned E0 = N[0] & (N[0] >> 1) & 0x44444444u, E1 = N[1] & (N[1] >> 1) & 0x44444444u;
  const unsigned mine = __builtin_popcount(E0) + __builtin_popcount(E1);
  unsigned pos = wave_scan(mine) - mine;
  const unsigned bB = base * 0x04040404u;
  unsigned h[8];
#pragma unroll
  for (int k = 0; k < 4; ++k) {
    const unsigned E = ((k & 2 ? E1 : E0) >> (4 * (k & 1))) & 0x04040404u;
    const unsigned sel = *(const LAS unsigned*)(lut + __builtin_amdgcn_udot4(E, 0x08040201u, 0u, false));
    const LAS unsigned* wp = (const LAS unsigned*)(ex + (pos & ~3u));
    const unsigned win = __builtin_amdgcn_alignbyte(wp[1], wp[0], pos);
    pos += __builtin_popcount(E);
    const unsigned t = (N[k >> 1] >> (4 * (k & 1))) & 0x0f0f0f0fu;
    const unsigned H = ((t + bB) ^ __builtin_amdgcn_perm(0u, win, sel)) | ((sg << (7 - k)) & 0x80808080u);
    h[2 * k] = __builtin_amdgcn_perm(H, D[k], 0x05010400u);
    h[2 * k + 1] = __builtin_amdgcn_perm(H, D[k], 0x07030602u);
  }
  cbar();
  f0 = (u32x4){h[0], h[1], h[2], h[3]};
  f1 = (u32x4){h[4], h[5], h[6], h[7]};
  return true;
}

__device__ __forceinline__ void convert_x(const Params& p, int first, unsigned call, int rank, LAS char* smem) {
  const int lane = threadIdx.x & 63, wave = threadIdx.x >> 6;
  const int count = 8 * KSTEPS, stride = NCVT_X;
  char* const ws = p.ws[rank];
  const float* src0 = p.x + (size_t)((wave >> 2) * 64 + (wave & 1) * 32 + (lane & 15)) * KDIM + (wave & 2) * 16 + (lane >> 4) * 4;
  f32x4 v[4];
  int u = first;
  {
    const float* s = src0 + (size_t)(u >> 7) * 128 * KDIM + (u & 127) * 64;
#pragma unroll
    for (int i = 0; i < 4; ++i) v[i] = ld4(s + (i >> 1) * 16 * KDIM + (i & 1) * 16);
  }
  for (; u < count; u += stride) {
    const int tile = u >> 7, ks = u & 127;
    const u32x4 h0 = cvt8(v[0], v[1]), h1 = cvt8(v[2], v[3]);
    {
      const int un = u + stride < count ? u + stride : u;
      const float* s = src0 + (size_t)(un >> 7) * 128 * KDIM + (un & 127) * 64;
#pragma unroll
      for (int i = 0; i < 4; ++i) v[i] = ld4(s + (i >> 1) * 16 * KDIM + (i & 1) * 16);
    }
    char* b = ws + OFF_OWN + own_idx(call, tile) * PANEL_B + ks * STEP_B + wave * 2048 + lane * 16;
    store_sys(b, h0);
    store_sys(b + 1024, h1);
    pack_block(h0, h1, smem + wave * WAVE_LDS, smem + LDS_LUT,
               ws + OFF_PACK + own_idx(call, tile) * PPANEL_B + ks * PSTEP_B + wave * BLK_B);
    wait_vm0();
    wg_barrier();
    if (threadIdx.x < 5) {
      const int t = threadIdx.x;
      const bool home = tile < 6;
      const int panel = home ? (rank & 3) * 6 + tile : PA_OWNAWAY + tile - 6;
      const int dst = (home ? (rank & 4) : (rank & 4) ^ 4) | (t & 3);
      if (t == 4) set_local(ws, panel * KSTEPS + ks, call);
      else if (dst != rank) set_rdy(p.ws[dst & 7], rank, home ? u : RDY_AWAY + u - 6 * KSTEPS, call);
    }
  }
}

__device__ __forceinline__ void convert_w(const Params& p, int first, unsigned call, int rank, LAS char* smem) {
  const int lane = threadIdx.x & 63, wave = threadIdx.x >> 6;
  const int count = KSTEPS * 4, stride = NCVT_W;
  char* const ws = p.ws[rank];
  const float* src0 = p.w + (size_t)((wave & 1) * 32 + (lane >> 4) * 4) * NSH + (wave >> 1) * 64 + (lane & 15) * 4;
  f32x4 v[8];
  int u = first;
  {
    const float* s = src0 + (size_t)(u >> 2) * 64 * NSH + (u & 3) * 256;
#pragma unroll
    for (int j = 0; j < 8; ++j) v[j] = ld4(s + ((j & 3) + (j >> 2) * 16) * NSH);
  }
  for (; u < count; u += stride) {
    const int ks = u >> 2, q = u & 3;
    u32x4 h[4];
#pragma unroll
    for (int f = 0; f < 4; ++f)
      h[f] = cvt8((f32x4){v[0][f], v[1][f], v[2][f], v[3][f]}, (f32x4){v[4][f], v[5][f], v[6][f], v[7][f]});
    {
      const int un = u + stride < count ? u + stride : u;
      const float* s = src0 + (size_t)(un >> 2) * 64 * NSH + (un & 3) * 256;
#pragma unroll
      for (int j = 0; j < 8; ++j) v[j] = ld4(s + ((j & 3) + (j >> 2) * 16) * NSH);
    }
    const unsigned oi = own_idx(call, 8 + q * 2 + (wave >> 2));
    char* b = ws + OFF_OWN + oi * PANEL_B + ks * STEP_B + (wave & 3) * 4096 + lane * 16;
#pragma unroll
    for (int f = 0; f < 4; ++f) store_sys(b + f * 1024, h[f]);
    char* pb = ws + OFF_PACK + oi * PPANEL_B + ks * PSTEP_B + (wave & 3) * 2 * BLK_B;
    pack_block(h[0], h[1], smem + wave * WAVE_LDS, smem + LDS_LUT, pb);
    pack_block(h[2], h[3], smem + wave * WAVE_LDS + 1024, smem + LDS_LUT, pb + BLK_B);
    wait_vm0();
    wg_barrier();
    if (threadIdx.x < 3) {
      const int t = threadIdx.x;
      if (t < 2) set_local(ws, (PB + q * 2 + t) * KSTEPS + ks, call);
      else set_rdy(p.ws[(((rank & 4) ^ 4) | q) & 7], rank, ks, call);
    }
  }
}

__device__ __forceinline__ int row_of(int r) { return ((RANK_AT >> (4 * r)) & 7) * NSH; }

#define ORDER_V 0x7648953210ull
constexpr unsigned long long BATCH_AT = ORDER_V;
constexpr unsigned long long batch_pos() {
  unsigned long long r = 0;
  for (int i = 0; i < 10; ++i) r |= (unsigned long long)i << (4 * ((BATCH_AT >> (4 * i)) & 15));
  return r;
}
constexpr unsigned long long BATCH_POS = batch_pos();
__device__ __forceinline__ int batch_row(int s, int o, int pos, int i) {
  const int bt = (BATCH_AT >> (4 * pos)) & 15;
  if (bt < 2) return row_of(s) + (bt * 3 + i) * 128;
  const int g = bt < 8 ? s : o;
  return row_of((g & 4) | ((g + 1 + i) & 3)) + (bt - 2) * 128;
}

__device__ __forceinline__ int unit_of(int it, int w) { return it * PW + ((w + it) & (PW - 1)); }

template <class F>
__device__ __forceinline__ int wait_ready(const unsigned* rdy, F flag_of, int it0, int w, int iters, unsigned call,
                                          unsigned* tmo, bool& dead) {
  const int lane = threadIdx.x & 63;
  const unsigned* f = rdy + flag_of(unit_of(it0 + lane < iters ? it0 + lane : it0, w)) * 32;
  const unsigned long long t0 = __builtin_amdgcn_s_memrealtime();
  for (unsigned it = 1;; ++it) {
    unsigned v;
    asm volatile("global_load_dword %0, %1, off sc0 sc1\n\ts_waitcnt vmcnt(0)" : "=v"(v) : "v"(f) : "memory");
    const unsigned long long miss = __builtin_amdgcn_ballot_w64(v < call);
    if (miss == 0ull) return 64;
    if (!(miss & 1ull)) return __builtin_ctzll(miss);
    if ((it & 63u) == 0u) {
      if (__builtin_amdgcn_readfirstlane(__hip_atomic_load(tmo, __ATOMIC_RELAXED, __HIP_MEMORY_SCOPE_AGENT)) == call
          || __builtin_amdgcn_s_memrealtime() - t0 > DEADLINE) {
        if (lane == 0) __hip_atomic_store(tmo, call, __ATOMIC_RELAXED, __HIP_MEMORY_SCOPE_AGENT);
        wait_vm0();
        dead = true;
        return 1 << 20;
      }
    }
    __builtin_amdgcn_s_sleep(2);
  }
}

__device__ __forceinline__ void copy_step(u32x4& next, const void* src, void* dst, u32x4 cur) {
  asm volatile("global_load_dwordx4 %0, %1, off sc0 sc1\n\t"
               "global_store_dwordx4 %2, %3, off sc0 sc1" OUT_NT "\n\t"
               "s_nop 0\n\ts_waitcnt vmcnt(1)"
               : "=&v"(next) : "v"(src), "v"(dst), "v"(cur) : "memory");
}
__device__ __forceinline__ u32x4 load_sys(const void* src) {
  u32x4 v;
  asm volatile("global_load_dwordx4 %0, %1, off sc0 sc1\n\ts_waitcnt vmcnt(0)" : "=v"(v) : "v"(src) : "memory");
  return v;
}
__device__ __forceinline__ unsigned lds_ticket(LAS unsigned* c) {
  unsigned t = 0;
  if ((threadIdx.x & 63) == 0) t = __hip_atomic_fetch_add(c, 1u, __ATOMIC_RELAXED, __HIP_MEMORY_SCOPE_WORKGROUP);
  return __builtin_amdgcn_readfirstlane(t);
}

template <bool CROSS>
__device__ __forceinline__ void pull_inputs(const Params& p, int w, unsigned call, int rank, int peer, LAS char* smem,
                                            bool& dead) {
  const int lane = threadIdx.x & 63, wave = __builtin_amdgcn_readfirstlane(threadIdx.x >> 6);
  const int iters = (CROSS ? 512 : 768) / PW;
  char* const ws = p.ws[rank];
  const char* const psrc = p.ws[peer] + OFF_PACK + own_idx(call, 0) * PPANEL_B + lane * 16;
  const char* const rsrc = p.ws[peer] + OFF_OWN + own_idx(call, 0) * PANEL_B + lane * 16;
  char* const dst = ws + OFF_PULL + lane * 16;
  const unsigned* rdy = (const unsigned*)(ws + OFF_RDY) + peer * RDY_N * 32;
  unsigned* const tmo = (unsigned*)(ws + OFF_TMO);
  LAS unsigned* const cnt = (LAS unsigned*)(smem + LDS_MB);
  LAS char* const wl = smem + wave * WAVE_LDS;
  const unsigned wl0 = (unsigned)(uintptr_t)wl;
  auto flag_of = [](int u) { return CROSS ? (u < 256 ? u >> 1 : RDY_AWAY + u - 256) : u; };
  auto decode = [&](int it, unsigned& ss, unsigned& ds) {
    const int u = unit_of(it, w);
    if (!CROSS) { ss = u; ds = (peer & 3) * 6 * KSTEPS + u; }
    else if (u < 256) { const int o = (u & 1) * KSTEPS + (u >> 1); ss = (8 + (rank & 3) * 2) * KSTEPS + o; ds = (PB_FOREIGN + (peer & 3) * 2) * KSTEPS + o; }
    else { ss = 6 * KSTEPS + u - 256; ds = (PA_AWAY + (peer & 3) * 2) * KSTEPS + u - 256; }
  };
  int ready = EXPON(1) ? 1 << 20 : 0;
  auto fetch = [&](unsigned t, unsigned buf) {
    const int it = t >> 3;
    if (it >= ready) ready = it + wait_ready(rdy, flag_of, it, w, iters, call, tmo, dead);
    unsigned ss, ds, keep;
    decode(it, ss, ds);
    asm volatile("s_mov_b32 %0, m0\n\ts_mov_b32 m0, %2\n\ts_nop 0\n\t"
                 "global_load_lds_dwordx4 %1, off sc0 sc1\n\t"
                 "s_mov_b32 exec_hi, 0xffff\n\t"
                 "global_load_lds_dwordx4 %1, off offset:1024 sc0 sc1\n\t"
                 "s_mov_b32 exec_hi, -1\n\ts_mov_b32 m0, %0"
                 : "=&s"(keep) : "v"(psrc + (size_t)(ss * 8 + (t & 7)) * BLK_B), "s"(wl0 + buf) : "memory");
  };
  auto done = [&](unsigned t) {
    unsigned ss, ds;
    decode(t >> 3, ss, ds);
    if (lane == 0 && __hip_atomic_fetch_add(cnt + 2 + (t >> 3), 1u, __ATOMIC_RELAXED, __HIP_MEMORY_SCOPE_WORKGROUP) == 7u)
      set_local(ws, ds, call);
  };
  const unsigned total = iters * 8, none = ~0u;
  if (wave >= NWV) {
    if (CROSS) {
      const unsigned long long t0 = __builtin_amdgcn_s_memrealtime();
      while (__builtin_amdgcn_readfirstlane(__hip_atomic_load(cnt, __ATOMIC_RELAXED, __HIP_MEMORY_SCOPE_WORKGROUP)) < total
             && __builtin_amdgcn_s_memrealtime() - t0 < 2 * DEADLINE)
        __builtin_amdgcn_s_sleep(16);
    }
    return;
  }
  unsigned t = lds_ticket(cnt), tn = lds_ticket(cnt), p1 = none, p2 = none, buf = 0;
  fetch(t, 0);
  wait_vm0();
  if (wave == 0) STAMP(3);
  while (t < total) {
    if ((t & 63) == 56) STAMP(4 + (t >> 6));
    unsigned tnn = total;
    if (tn < total) { fetch(tn, buf ^ 2048); tnn = lds_ticket(cnt); }
    if (PULL_LAG) { if (p2 != none) done(p2); } else if (p1 != none) done(p1);
    u32x4 f0, f1;
    unsigned ss, ds;
    decode(t >> 3, ss, ds);
    if (!unpack_block(wl + buf, wl + 4096, smem + LDS_LUT, f0, f1)) {
      const char* r = rsrc + (size_t)(ss * 8 + (t & 7)) * 2048 + (ss >> 7) * SKEW;
      asm volatile("global_load_dwordx4 %0, %2, off sc0 sc1\n\t"
                   "global_load_dwordx4 %1, %2, off offset:1024 sc0 sc1\n\ts_waitcnt vmcnt(0)"
                   : "=&v"(f0), "=&v"(f1) : "v"(r) : "memory");
    }
    char* d = dst + (size_t)(ds * 8 + (t & 7)) * 2048 + (ds >> 7) * SKEW;
    store_sys(d, f0);
    store_sys(d + 1024, f1);
    if (PULL_LAG) asm volatile("s_waitcnt vmcnt(2)" ::: "memory"); else wait_vm0();
    p2 = p1; p1 = t; t = tn; tn = tnn; buf ^= 2048;
  }
  if (PULL_LAG) { wait_vm0(); if (p2 != none) done(p2); }
  if (p1 != none) done(p1);
}

__device__ __forceinline__ void pull_outputs(const Params& p, int w, unsigned call, int rank, int peer, LAS char* smem,
                                             bool& dead) {
  const int lane = threadIdx.x & 63;
  const int iters = OUT_CHUNKS / PW;
  char* const ws = p.ws[rank];
  const char* const src = p.ws[peer] + OFF_STG + (size_t)(rank & 3) * OUT_CHUNKS * CHUNK_B + lane * 16;
  float* const dst = p.out[rank] + (size_t)((lane >> 4) * 4) * NSH + (peer & 3) * 256 + (lane & 15) * 4;
  const unsigned* rdy = (const unsigned*)(ws + OFF_RDY) + (peer * RDY_N + RDY_OUT) * 32;
  unsigned* const tmo = (unsigned*)(ws + OFF_TMO);
  LAS unsigned* const cnt = (LAS unsigned*)(smem + LDS_MB) + 1;
  auto flag_of = [](int u) { return u; };
  auto where = [&](unsigned t) {
    const int c = unit_of(t >> 3, w), tile = c >> 3, bt = tile / 6, r = tile - bt * 6;
    return dst + (size_t)(batch_row(peer, rank, bt, r >> 1) + ((c >> 2) & 1) * 64 + (c & 1) * 32 + (t & 4) * 4 + (t & 3)) * NSH
         + (r & 1) * 128 + ((c >> 1) & 1) * 64;
  };
  int ready = EXPON(17) ? 1 << 20 : 0;
  auto from = [&](unsigned t) {
    const int it = t >> 3;
    if (it >= ready) ready = it + wait_ready(rdy, flag_of, it, w, iters, call, tmo, dead);
    return src + (size_t)(unit_of(it, w) * 8 + (t & 7)) * 1024;
  };
  const u32x4 qnan = {0x7fc00000u, 0x7fc00000u, 0x7fc00000u, 0x7fc00000u};
  const unsigned total = iters * 8;
  unsigned t = lds_ticket(cnt);
  if (t >= total) return;
  unsigned tn = lds_ticket(cnt);
  u32x4 cur = load_sys(from(t));
  while (tn < total) {
    if (t % 48 == 40) STAMP(20 + t / 48);
    u32x4 next;
    const char* s = from(tn);
    float* d = where(t);
    t = tn;
    tn = lds_ticket(cnt);
    copy_step(next, s, d, dead ? qnan : cur);
    cur = next;
  }
  if (__builtin_amdgcn_readfirstlane(__hip_atomic_load(tmo, __ATOMIC_RELAXED, __HIP_MEMORY_SCOPE_AGENT)) == call) dead = true;
  store_sys(where(t), dead ? qnan : cur);
}

struct Job { int pa, pb, ah, bh, tile; };
__device__ __forceinline__ void decode_job(unsigned code, int a, Job& o) {
  const int ra = code & 63;
  int bt, i;
  o.pb = PB + ((code >> 6) & 15);
  o.ah = (code >> 10) & 1;
  o.bh = (code >> 11) & 1;
  if (ra < 18) {
    i = ra / 6; bt = 2 + ra % 6; o.pa = ((a + 1 + i) & 3) * 6 + bt - 2;
  } else if (ra < 24) {
    o.pa = a * 6 + ra - 18; bt = (ra - 18) / 3; i = (ra - 18) % 3;
  } else {
    o.pa = ra; bt = 8 + (ra & 1); i = (((ra - PA_AWAY) >> 1) - ((o.pb - PB_FOREIGN) >> 1) - 1) & 3;
  }
  o.tile = (int)((BATCH_POS >> (4 * bt)) & 15) * 3 + i;
}

__device__ __forceinline__ float silu(float y) {
  const float c = fminf(fmaxf(y, -60.0f), 60.0f);
  return y * __builtin_amdgcn_rcpf(1.0f + __builtin_amdgcn_exp2f(c * -1.44269504088896341f));
}

#define GLDS_BITS " sc0 sc1"

template <int NB>
__device__ __forceinline__ unsigned next_stage(unsigned so) { return so == LDS_MB - (NB + 1) * 8192 ? 0u : so + (NB + 1) * 8192; }

template <bool TALL, int NB>
__device__ __forceinline__ void mfma_wave(const Params& p, LAS char* smem, const Job& jb, int job, int wave,
                                          int rank, unsigned call) {
  constexpr int MT = TALL ? 4 : NB, NT = TALL && NB == 2 ? 8 : 4, KH = TALL ? 1 : 2, STEPS = 2 * KSTEPS / KH;
  constexpr int NRA = MT * KH, NRB = NT * KH, NRD = NRA + NRB, NMM = MT * NT * KH;
  int lane = fresh_lane();
  const int wr = MT == 1 ? wave : wave >> 1, wc = MT == 1 ? 0 : wave & 1;
  f32x4 acc[MT][NT];
#pragma unroll
  for (int m = 0; m < MT; ++m)
#pragma unroll
    for (int n = 0; n < NT; ++n) acc[m][n] = (f32x4){0.f, 0.f, 0.f, 0.f};
  half8 fa[2][NRA], fb[2][NRB];
  const LAS char* la = smem + wr * MT * 1024 + lane * 16;
  const LAS char* lb = smem + 8192 + wc * NRB * 1024 + lane * 16;
  unsigned so = 0;
#define READ_SET(X) do { \
    _Pragma("unroll") for (int i = 0; i < NRB; ++i) { \
      if (i < NRA) fa[X][i] = *(const LAS half8*)(la + so + (i / MT) * 4096 + (i % MT) * 1024); \
      fb[X][i] = *(const LAS half8*)(lb + so + i * 1024); } \
    so = next_stage<NB>(so); } while (0)
#define MMA_SET(X) do { \
    _Pragma("unroll") for (int kh = 0; kh < KH; ++kh) \
    _Pragma("unroll") for (int m = 0; m < MT; ++m) \
    _Pragma("unroll") for (int n = 0; n < NT; ++n) \
      acc[m][n] = __builtin_amdgcn_mfma_f32_16x16x32_f16(fa[X][kh * MT + m], fb[X][kh * NT + n], acc[m][n], 0, 0, 0); } while (0)
  constexpr int RG = TALL ? 1 : 2, MG = NB, G = NRD / RG;
#define INTERLEAVE() do { \
    _Pragma("unroll") for (int i = 0; i < G; ++i) { \
      __builtin_amdgcn_sched_group_barrier(0x100, RG, 0); \
      __builtin_amdgcn_sched_group_barrier(0x008, MG, 0); } \
    __builtin_amdgcn_sched_group_barrier(0x008, NMM - G * MG, 0); } while (0)
#define STEP_END() do { __builtin_amdgcn_sched_barrier(0); asm volatile("" ::: "memory"); \
    __builtin_amdgcn_s_waitcnt(0xC07F);   __builtin_amdgcn_sched_barrier(0); \
    wg_barrier(); __builtin_amdgcn_sched_barrier(0); } while (0)
  wg_barrier();
  if (wave == 0) STAMP(4 + job * 4);
  READ_SET(0);
  STEP_END();
  for (int t = 1; t < STEPS - 1; t += 2) {
    READ_SET(1); MMA_SET(0); INTERLEAVE();
    STEP_END();
    READ_SET(0); MMA_SET(1); INTERLEAVE();
    STEP_END();
  }
  READ_SET(1); MMA_SET(0); INTERLEAVE();
  const u32x4 tm = *(const LAS u32x4*)(smem + LDS_TMO + job * 16);
  STEP_END();
  MMA_SET(1);
  __builtin_amdgcn_sched_barrier(0);
  if (wave == 0) STAMP(5 + job * 4);
  lane = fresh_lane();
  asm volatile("" : "+s"(call));
  const bool dead = tm[0] == call || tm[1] == call || tm[2] == call || tm[3] == call;

  const bool foreign = jb.pb >= PB_FOREIGN;
  const int ob = (jb.pb - PB_FOREIGN) >> 1;
  const int r16 = jb.ah * 4 + wr * MT;
  const int c64 = (jb.pb & 1) * 2 + jb.bh + wc * (NT / 4);
  const int chunk = (jb.tile * 2 + (c64 >> 1)) * 8 + (r16 & 4) + (c64 & 1) * 2 + ((r16 >> 1) & 1);
  char* o;
  unsigned sm, sr, sh;
  if (foreign) {
    o = p.ws[rank] + OFF_STG + (size_t)(ob * OUT_CHUNKS + chunk) * CHUNK_B + (r16 & 1) * 4096 + lane * 16;
    sm = 4096; sr = 1024; sh = 2 * CHUNK_B;
  } else {
    const int pa = jb.pa;
    const int row0 = pa < PA_AWAY ? row_of((rank & 4) + pa / 6) + (pa % 6) * 128
                   : row_of(pa < PA_OWNAWAY ? ((rank & 4) ^ 4) + ((pa - PA_AWAY) >> 1) : rank) + 768 + (pa & 1) * 128;
    o = (char*)(p.out[rank] + (size_t)(row0 + r16 * 16 + (lane >> 4) * 4) * NSH + ((jb.pb - PB) >> 1) * 256 + c64 * 64 + (lane & 15) * 4);
    sm = 16 * NSH * 4; sr = NSH * 4; sh = 256;
  }
  const float qnan = __builtin_nanf("");
#pragma unroll
  for (int h = 0; h < NT / 4; ++h)
#pragma unroll
    for (int m = 0; m < MT; ++m)
#pragma unroll
      for (int r = 0; r < 4; ++r) {
        f32x4 v = {silu(acc[m][h * 4][r]), silu(acc[m][h * 4 + 1][r]), silu(acc[m][h * 4 + 2][r]), silu(acc[m][h * 4 + 3][r])};
        if (dead) v = (f32x4){qnan, qnan, qnan, qnan};
        if (foreign) store_sys(o + h * sh + m * sm + r * sr, __builtin_bit_cast(u32x4, v));
        else store_out(o + h * sh + m * sm + r * sr, __builtin_bit_cast(u32x4, v));
      }
  if (foreign) {
    wait_vm0();
    bool last = lane < (MT * NT + 7) / 8;
    if (MT == 1)
      last = last && __hip_atomic_fetch_add((LAS unsigned*)(smem + LDS_CNT) + job * 2 + (wave >> 1), 1u,
                                            __ATOMIC_RELAXED, __HIP_MEMORY_SCOPE_WORKGROUP) == 1u;
    if (last) set_rdy(p.ws[(((rank & 4) ^ 4) | ob) & 7], rank, RDY_OUT + chunk + lane, call);
  }
  if (wave == 0) STAMP(6 + job * 4);
}

template <int NB>
__device__ __forceinline__ void load_wave(const Params& p, LAS char* smem, const Job& jb, int job, int l, bool tall,
                                          int rank, unsigned call) {
  constexpr int DEPTH = 18 / (NB + 1) - 1;
  const int steps = tall ? 2 * KSTEPS : KSTEPS, ksh = tall ? 1 : 0;
  if (NB == 2 && l == 3) {
    for (int t = 0; t <= steps; ++t) wg_barrier();
    return;
  }
  const int lane = fresh_lane(), a = rank & 3;
  char* const ws = p.ws[rank];
  unsigned* const tmo = (unsigned*)(ws + OFF_TMO);
  const unsigned lds0 = (unsigned)(uintptr_t)smem;
  const int pc = NB == 2 ? l : l >> 1;
  const int panel = pc == 0 ? jb.pa : tall ? jb.pb + pc - 1 : jb.pb;
  const int own = panel >= PB_FOREIGN ? -1 : panel >= PA_OWNAWAY ? panel - PA_OWNAWAY + 6
                : (unsigned)(panel - a * 6) < 6u ? panel - a * 6 : -1;
  const unsigned gap = tall ? 8192 : 4096;
  const char* src = ws + (own >= 0 ? OFF_OWN + own_idx(call, own) * PANEL_B : OFF_PULL + (size_t)panel * PANEL_B)
                  + (tall ? 0 : (pc == 0 ? jb.ah : jb.bh + pc - 1) * 8192) + (NB == 2 ? 0 : (l & 1) * gap) + lane * 16;
  const char* fsrc = ws + OFF_FLAG + panel * 1024 + (lane >> 2) * 128 + (lane & 3) * 16;
  const unsigned mb = lds0 + LDS_MB + (job * 4 + l) * 512;
  const unsigned dst0 = lds0 + l * (NB * 4096);
  unsigned so = 0;
  int ready = EXPON(1) ? KSTEPS : 0;
  auto poll = [&]() {
    if (lane < 32) {
      unsigned keep;
      asm volatile("s_mov_b32 %0, m0\n\ts_mov_b32 m0, %2\n\ts_nop 0\n\t"
                   "global_load_lds_dwordx4 %1, off sc0 sc1\n\ts_mov_b32 m0, %0"
                   : "=&s"(keep) : "v"(fsrc), "s"(mb) : "memory");
    }
  };
  auto refresh = [&]() {
    u32x2 f;
    asm volatile("ds_read2_b32 %0, %1 offset1:8\n\ts_waitcnt lgkmcnt(0)"
                 : "=v"(f) : "v"(mb + (lane & 7) * 64 + (lane >> 3) * 4) : "memory");
    const unsigned long long m0 = __builtin_amdgcn_ballot_w64(f[0] < call);
    const unsigned long long m1 = __builtin_amdgcn_ballot_w64(f[1] < call);
    ready = m0 ? __builtin_ctzll(m0) : m1 ? 64 + __builtin_ctzll(m1) : KSTEPS;
  };
  for (int s = 0; s < steps; ++s) {
    if (s >= DEPTH) {
      if (NB == 2) asm volatile("s_waitcnt vmcnt(32)" ::: "memory");
      else asm volatile("s_waitcnt vmcnt(28)" ::: "memory");
      wg_barrier();
    }
    const int ks = s >> ksh;
    if (ks >= ready) {
      refresh();
      if (ks >= ready) {
        const unsigned long long t0 = __builtin_amdgcn_s_memrealtime();
        for (unsigned it = 1;; ++it) {
          poll();
          wait_vm0();
          refresh();
          if (ks < ready) break;
          if ((it & 63u) == 0u) {
            if (__builtin_amdgcn_readfirstlane(__hip_atomic_load(tmo, __ATOMIC_RELAXED, __HIP_MEMORY_SCOPE_AGENT)) == call
                || __builtin_amdgcn_s_memrealtime() - t0 > DEADLINE) {
              if (lane == 0) __hip_atomic_store(tmo, call, __ATOMIC_RELAXED, __HIP_MEMORY_SCOPE_AGENT);
              wait_vm0();
              ready = KSTEPS;
              break;
            }
          }
          __builtin_amdgcn_s_sleep(4);
        }
      }
    }
    if (ready < KSTEPS && ready - ks < 8 && (s & 1)) poll();
    const char* g = src + (size_t)ks * STEP_B + (s & ksh) * 4096;
    const unsigned d = dst0 + so;
    so = next_stage<NB>(so);
    unsigned keep;
#define GLDS4(R) "global_load_lds_dwordx4 " R ", off" GLDS_BITS "\n\t" \
                 "global_load_lds_dwordx4 " R ", off offset:1024" GLDS_BITS "\n\t" \
                 "global_load_lds_dwordx4 " R ", off offset:2048" GLDS_BITS "\n\t" \
                 "global_load_lds_dwordx4 " R ", off offset:3072" GLDS_BITS "\n\t"
    if (NB == 2)
      asm volatile("s_mov_b32 %0, m0\n\ts_mov_b32 m0, %3\n\ts_nop 0\n\t" GLDS4("%1")
                   "s_mov_b32 m0, %4\n\ts_nop 0\n\t" GLDS4("%2") "s_mov_b32 m0, %0"
                   : "=&s"(keep) : "v"(g), "v"(g + gap), "s"(d), "s"(d + 4096) : "memory");
    else
      asm volatile("s_mov_b32 %0, m0\n\ts_mov_b32 m0, %2\n\ts_nop 0\n\t" GLDS4("%1") "s_mov_b32 m0, %0"
                   : "=&s"(keep) : "v"(g), "s"(d) : "memory");
  }
  if (lane == 0) {
    unsigned keep;
    asm volatile("s_mov_b32 %0, m0\n\ts_mov_b32 m0, %2\n\ts_nop 0\n\t"
                 "global_load_lds_dword %1, off sc1\n\ts_mov_b32 m0, %0"
                 : "=&s"(keep) : "v"(tmo), "s"(lds0 + LDS_TMO + (job * 4 + l) * 4) : "memory");
  }
#define WB(N) "s_waitcnt vmcnt(" #N ")\n\ts_barrier\n\t"
  if (NB == 2) asm volatile(WB(33) WB(25) WB(17) WB(9) WB(0) "s_barrier" ::: "memory");
  else asm volatile(WB(29) WB(25) WB(21) WB(17) WB(13) WB(9) WB(5) WB(0) "s_barrier" ::: "memory");
}

__global__ void __launch_bounds__(NTHREADS, 2) ag_gemm_silu(const Params p) {
  __shared__ __attribute__((aligned(16))) char smem[LDS_BYTES];
  const int rank = p.rank & 7, a = rank & 3;
  const int wave = __builtin_amdgcn_readfirstlane(threadIdx.x >> 6);
#define XROT 4
  const int xcd = (blockIdx.x - XROT) & 7, slot = blockIdx.x >> 3;
  char* const ws = p.ws[rank];

  unsigned* const cnt = (unsigned*)(ws + OFF_CALL) + blockIdx.x;
  const unsigned call = __builtin_amdgcn_readfirstlane(
      __hip_atomic_load(cnt, __ATOMIC_RELAXED, __HIP_MEMORY_SCOPE_AGENT)) + 1u;
  if (wave == 0) STAMP(0);
  const bool puller = slot >= GSLOTS && xcd < 7;
  for (unsigned i = threadIdx.x; i < (LDS_LUT - LDS_MB) / 4; i += NTHREADS) ((LAS unsigned*)(smem + LDS_MB))[i] = 0u;
  if (threadIdx.x < 16) ((LAS unsigned*)(smem + LDS_LUT))[threadIdx.x] = codec_lut(threadIdx.x, puller);
  asm volatile("s_waitcnt vmcnt(0) lgkmcnt(0)" ::: "memory");
  wg_barrier();
  if (threadIdx.x == 0) __hip_atomic_store(cnt, call, __ATOMIC_RELAXED, __HIP_MEMORY_SCOPE_AGENT);

  if (puller) {
    const int w = slot - GSLOTS;
    bool dead = false;
    if (EXPON(2)) return;
    if (xcd < 4) {
      const int peer = ((rank & 4) ^ 4) | ((a + xcd) & 3);
      pull_inputs<true>(p, w, call, rank, peer, (LAS char*)smem, dead);
      if (wave == 0) STAMP(1);
      pull_outputs(p, w, call, rank, peer, (LAS char*)smem, dead);
    } else {
      pull_inputs<false>(p, w, call, rank, (rank & 4) | ((a + xcd - 3) & 3), (LAS char*)smem, dead);
    }
    if (wave == 0) STAMP(2);
    return;
  }

  const int gi = slot < GSLOTS ? slot * 8 + xcd : GSLOTS * 8 + slot - GSLOTS;
  if (EXPON(1)) {} else
  if (gi < NCVT_X) convert_x(p, gi, call, rank, (LAS char*)smem);
  else convert_w(p, gi - NCVT_X, call, rank, (LAS char*)smem);
  if (wave == 0) STAMP(1);

  for (int job = 0; job < NJOBS; ++job) {
    const unsigned code = p.jobs[gi][job];
    if (!(code & 0x8000u)) break;
    if (EXPON(4) && (job > 0 || slot >= 12)) break;
    if (EXPON(8) && job > 0) break;
    Job jb;
    decode_job(code, a, jb);
    const int kind = (code >> 12) & 3;
    if (wave >= 4) {
      if (kind & 1) load_wave<1>(p, (LAS char*)smem, jb, job, wave - 4, kind < 2, rank, call);
      else load_wave<2>(p, (LAS char*)smem, jb, job, wave - 4, kind < 2, rank, call);
    } else if (kind == 0) {
      mfma_wave<true, 2>(p, (LAS char*)smem, jb, job, wave, rank, call);
    } else if (kind == 1) {
      mfma_wave<true, 1>(p, (LAS char*)smem, jb, job, wave, rank, call);
    } else if (kind == 2) {
      mfma_wave<false, 2>(p, (LAS char*)smem, jb, job, wave, rank, call);
    } else {
      mfma_wave<false, 1>(p, (LAS char*)smem, jb, job, wave, rank, call);
    }
  }
}

void make_schedule(Params& p) {
  int n[NGEMM] = {};
  for (int g = 0; g < NGEMM; ++g)
    for (int j = 0; j < NJOBS; ++j) p.jobs[g][j] = 0;
  auto put = [&](int xcd, int s, int kind, int ra, int b, int ah = 0, int bh = 0) {
    const int g = s < GSLOTS ? s * 8 + xcd : GSLOTS * 8 + s - GSLOTS;
    p.jobs[g][n[g]++] = (unsigned short)(0x8000 | (kind << 12) | (bh << 11) | (ah << 10) | (b << 6) | ra);
  };
  auto P = [&](int xcd, int s0, int kind, int t, int b0) {
    for (int s = 0; s < 12; ++s) {
      if (kind < 2) put(xcd, s0 + s, kind, s % 3 * 6 + t, b0 + s / 3 * (2 - kind));
      else if (kind == 2) put(xcd, s0 + s, kind, s % 3 * 6 + t, b0 + s / 6, (s / 3) & 1);
      else put(xcd, s0 + s, kind, s % 3 * 6 + t, b0, (s / 3) & 1, s / 6);
    }
  };
  auto L = [&](int xcd, int h, int b0) { for (int s = 0; s < 12; ++s) put(xcd, s, 0, 18 + h * 3 + s % 3, b0 + s / 3 * 2); };
  P(0, 0, 0, 0, 0); P(1, 0, 0, 0, 8); P(2, 0, 0, 1, 0); P(3, 0, 0, 1, 8);
  L(4, 0, 8); L(5, 1, 8); L(6, 0, 0); L(7, 1, 0);
  for (int s = 0; s < 16; ++s) put(2 + s / 8, 12 + s % 8, 1, PA_OWNAWAY + (s & 1), s / 2);
  P(0, 12, 0, 2, 0); P(1, 12, 0, 2, 8);
  for (int s = 0; s < 12; ++s) put(4, 12 + s, 0, PA_AWAY + (s & 3) * 2, 8 + (((s & 3) + 1 + (s >> 2)) & 3) * 2);
  for (int s = 0; s < 16; ++s) put(7, 12 + s, 0, PA_AWAY + (s & 3) * 2, s / 4 * 2);
  P(2, 12, 1, 3, 8); P(3, 12, 1, 3, 12);
  for (int s = 0; s < 24; ++s) put(5 + s / 12, 12 + s % 12, 1, PA_AWAY + ((s / 6 + 1 + s % 3) & 3) * 2 + 1, 8 + s / 3);
  for (int s = 0; s < 4; ++s) put(7, 28 + s, 0, PA_AWAY + s * 2 + 1, 6);
  for (int s = 0; s < 24; ++s) put(s < 12 ? 4 : 7, s % 12, 1, PA_AWAY + (s & 3) * 2 + 1, s / 4);
  P(0, 0, 1, 3, 0); P(1, 0, 1, 3, 4);
  P(2, 0, 1, 4, 8); P(3, 0, 1, 4, 12); P(5, 0, 1, 4, 0); P(6, 0, 1, 4, 4);
  for (int x = 0; x < 8; ++x) P(x, 12, 3, 5, 8 + x);
  P(0, 0, 2, 5, 0); P(1, 0, 2, 5, 2); P(4, 0, 2, 5, 4); P(7, 0, 2, 5, 6);
}

}

extern "C" void kernel_launch(int rank, int nranks, ncclComm_t comm, hipStream_t stream,
                              void* const* d_in, const int* in_sizes, int n_in,
                              void* d_out, int out_size, const int* in_dtypes, int out_dtype,
                              void* const* d_out_all, void* const* d_ws_all, size_t ws_bytes) {
  (void)comm; (void)in_dtypes; (void)out_dtype; (void)d_out;
  if (nranks != NR || (unsigned)rank >= (unsigned)NR || n_in != 2 || in_sizes[0] != NSH * KDIM ||
      in_sizes[1] != KDIM * NSH || out_size != KDIM * NSH || ws_bytes < WS_NEED) {
    fprintf(stderr, "kernel_launch: unexpected problem shape\n");
    return;
  }
  Params p;
  p.x = (const float*)d_in[0];
  p.w = (const float*)d_in[1];
  p.rank = 0;
  for (int i = 0; i < NR; ++i) {
    const int r = (RANK_AT >> (4 * i)) & 7;
    p.out[i] = (float*)d_out_all[r];
    p.ws[i] = (char*)d_ws_all[r];
    if (r == rank) p.rank = i;
  }
  p.pad = 0;
  make_schedule(p);
  ag_gemm_silu<<<dim3(NWG), dim3(NTHREADS), 0, stream>>>(p);
  HIP_CHECK(hipGetLastError());
}
```

```cpp
#include <hip/hip_runtime.h>
#include <rccl/rccl.h>
#include <cstdio>
#include <cstdint>

#define HIP_CHECK(expr) do { hipError_t _e = (expr); if (_e != hipSuccess) { \
    fprintf(stderr, "HIP error %s at %s:%d\n", hipGetErrorString(_e), __FILE__, __LINE__); \
    return; } } while (0)

namespace {

typedef _Float16 half8 __attribute__((ext_vector_type(8)));
typedef float f32x4 __attribute__((ext_vector_type(4)));
typedef unsigned u32x4 __attribute__((ext_vector_type(4)));
typedef unsigned u32x2 __attribute__((ext_vector_type(2)));
#define LAS __attribute__((address_space(3)))
#define GAS __attribute__((address_space(1)))

constexpr int NR = 8;
#define RANK_AT_V 0x75416320u
constexpr unsigned RANK_AT = RANK_AT_V;
constexpr int KDIM = 8192;
constexpr int NSH = 1024;
constexpr int KSTEPS = 128;
constexpr unsigned STEP_B = 16384;
constexpr size_t PANEL_B = (size_t)KSTEPS * STEP_B;
constexpr int PA_AWAY = 24, PA_OWNAWAY = 32, PB = 34, PB_FOREIGN = 42, NPANEL = 50;

constexpr unsigned BLK_B = 1792;
constexpr unsigned PSTEP_B = 8 * BLK_B;
constexpr size_t PPANEL_B = (size_t)KSTEPS * PSTEP_B;

constexpr size_t OFF_CALL = 0;
constexpr size_t OFF_TMO = 4096;
constexpr size_t OFF_TICKET = 5120;
#define FCOPY 8
constexpr size_t OFF_FLAG = 1u << 18;
constexpr size_t OFF_RDY = 1u << 20;
constexpr int RDY_N = 1024, RDY_AWAY = 128, RDY_OUT = 384, RDY_HOP1 = 864, RDY_HOP2 = 896;
#define RELAY 12
constexpr size_t OFF_DBG = 65536;
constexpr size_t OFF_OWN = 2u << 20;
constexpr size_t OFF_PACK = OFF_OWN + 32 * PANEL_B;
constexpr size_t OFF_RELAY = OFF_PACK + 32 * PPANEL_B;
constexpr size_t OFF_PULL = OFF_RELAY + PANEL_B;
constexpr int OUT_CHUNKS = 480;
constexpr unsigned CHUNK_B = 8192;
constexpr size_t OFF_STG = OFF_PULL + NPANEL * PANEL_B;
constexpr size_t WS_NEED = OFF_STG + (size_t)4 * OUT_CHUNKS * CHUNK_B;
static_assert(RELAY % 4 == 0 && RELAY <= 64, "");
static_assert(NPANEL <= 64 && OFF_FLAG + FCOPY * 65536 <= OFF_RDY && OFF_RDY + NR * RDY_N * 128 <= OFF_OWN, "");

constexpr int NWG = 256, NTHREADS = 512;
constexpr int GSLOTS = 24;
constexpr int NGEMM = GSLOTS * 8 + 8;
constexpr int PW = 8;
#define WIN 6
constexpr int NJOBS = 3;
constexpr unsigned long long DEADLINE = 100000000ull;

constexpr unsigned LDS_MB = 18 * 8192;
constexpr unsigned LDS_TMO = LDS_MB + NJOBS * 4 * 512;
constexpr unsigned LDS_LUT = LDS_TMO + NJOBS * 16;
constexpr unsigned LDS_BYTES = LDS_LUT + 64;
constexpr unsigned WAVE_LDS = 8192;

struct Params {
  const float* x;
  const float* w;
  float* out[NR];
  char* ws[NR];
  int rank;
  unsigned short ncvt_x, ncvt_w;
  u32x4 plan[NGEMM];
};

#define STAMP(idx) do {} while (0)

#define EXP 0
#define EXPON(bit) ((EXP & (bit)) && call > 2)
__device__ __forceinline__ void store_sys(void* ptr, u32x4 v) {
  asm volatile("global_store_dwordx4 %0, %1, off sc0 sc1\n\ts_nop 1" :: "v"(ptr), "v"(v) : "memory");
}
__device__ __forceinline__ void wait_vm0() { asm volatile("s_waitcnt vmcnt(0)" ::: "memory"); }
__device__ __forceinline__ void wg_barrier() {
  asm volatile("" ::: "memory");
  __builtin_amdgcn_s_barrier();
  asm volatile("" ::: "memory");
}
__device__ __forceinline__ u32x4 cvt8(f32x4 a, f32x4 b) {
  half8 h;
  h[0] = (_Float16)a[0]; h[1] = (_Float16)a[1]; h[2] = (_Float16)a[2]; h[3] = (_Float16)a[3];
  h[4] = (_Float16)b[0]; h[5] = (_Float16)b[1]; h[6] = (_Float16)b[2]; h[7] = (_Float16)b[3];
  return __builtin_bit_cast(u32x4, h);
}
__device__ __forceinline__ f32x4 ld4(const float* ptr) { return __builtin_nontemporal_load((const GAS f32x4*)ptr); }
#define OUT_NT " nt"
__device__ __forceinline__ void store_out(void* ptr, u32x4 v) {
  asm volatile("global_store_dwordx4 %0, %1, off sc0 sc1" OUT_NT "\n\ts_nop 1" :: "v"(ptr), "v"(v) : "memory");
}

__device__ __forceinline__ void set_local(char* ws, int copy, int step, unsigned v) {
  __hip_atomic_store((unsigned*)(ws + OFF_FLAG) + copy * 16384 + (step >> 7) * 256 + (step & 7) * 32 + ((step & 127) >> 3), v,
                     __ATOMIC_RELAXED, __HIP_MEMORY_SCOPE_SYSTEM);
}
__device__ __forceinline__ void set_rdy(char* peer_ws, int rank, int idx, unsigned v) {
  __hip_atomic_store((unsigned*)(peer_ws + OFF_RDY) + (rank * RDY_N + idx) * 32, v, __ATOMIC_RELAXED, __HIP_MEMORY_SCOPE_SYSTEM);
}
__device__ __forceinline__ unsigned own_idx(unsigned call, int idx) { return (call & 1) * 16 + idx; }
__device__ __forceinline__ int fresh_lane() {
  int l;
  asm volatile("v_mbcnt_lo_u32_b32 %0, -1, 0\n\tv_mbcnt_hi_u32_b32 %0, -1, %0" : "=v"(l));
  return l;
}
__device__ __forceinline__ void cbar() { asm volatile("" ::: "memory"); }
__device__ __forceinline__ unsigned wave_scan(unsigned v) {
  v += __builtin_amdgcn_update_dpp(0u, v, 0x111, 0xf, 0xf, false);
  v += __builtin_amdgcn_update_dpp(0u, v, 0x112, 0xf, 0xf, false);
  v += __builtin_amdgcn_update_dpp(0u, v, 0x114, 0xf, 0xf, false);
  v += __builtin_amdgcn_update_dpp(0u, v, 0x118, 0xf, 0xf, false);
  v += __builtin_amdgcn_update_dpp(0u, v, 0x142, 0xa, 0xf, false);
  v += __builtin_amdgcn_update_dpp(0u, v, 0x143, 0xc, 0xf, false);
  return v;
}

__device__ __forceinline__ unsigned codec_lut(int m, bool expand) {
  unsigned sel = 0x0c0c0c0cu;
  int n = 0;
  for (int i = 0; i < 4; ++i)
    if ((m >> i) & 1) {
      const int at = expand ? i : n, v = expand ? n : i;
      sel = (sel & ~(0xffu << (8 * at))) | ((unsigned)v << (8 * at));
      ++n;
    }
  return sel;
}

__device__ __forceinline__ void pack_block(u32x4 f0, u32x4 f1, LAS char* wl, const LAS char* lut, char* dst) {
  const int lane = threadIdx.x & 63;
  const unsigned h[8] = {f0[0], f0[1], f0[2], f0[3], f1[0], f1[1], f1[2], f1[3]};
  if (lane < 36) *(LAS unsigned*)(wl + 640 + lane * 4) = 0u;
  u32x4 D;
  unsigned H[4], sum = 0;
#pragma unroll
  for (int k = 0; k < 4; ++k) {
    D[k] = __builtin_amdgcn_perm(h[2 * k + 1], h[2 * k], 0x06040200u);
    H[k] = __builtin_amdgcn_perm(h[2 * k + 1], h[2 * k], 0x07050301u);
    sum = __builtin_amdgcn_udot4(H[k] & 0x7c7c7c7cu, 0x01010101u, sum, false);
  }
  store_sys(dst + lane * 16, D);
  const unsigned tot = __builtin_amdgcn_readlane(wave_scan(sum), 63);
  const unsigned base = min((tot + 256u) >> 12, 28u);
  const unsigned bB = base * 0x04040404u;
  unsigned t[4], pk[4], n[4], EM = 0, XM = 0, sg = 0;
#pragma unroll
  for (int k = 0; k < 4; ++k) {
    const unsigned g = ((H[k] & 0x7c7c7c7cu) | 0x80808080u) - bB;
    const unsigned a7 = g & 0x7f7f7f7fu, hi = a7 + 0x70707070u, lo = a7 + 0x1c1c1c1cu;
    const unsigned o7 = (~g | hi) & 0x80808080u;
    t[k] = ((g | (o7 >> 4) | (o7 >> 5)) & 0x0c0c0c0cu) | (H[k] & 0x03030303u);
    const unsigned E = t[k] & (t[k] >> 1) & 0x04040404u;
    const unsigned nb = ~g & 0x80808080u;
    const unsigned c4 = g & ((nb >> 2) - (nb >> 5));
    const unsigned X = ((g & hi) | (~g & ~lo)) & 0x80808080u;
    const unsigned sel = *(const LAS unsigned*)(lut + __builtin_amdgcn_udot4(E, 0x08040201u, 0u, false));
    const unsigned z = __builtin_amdgcn_perm(0u, c4, sel);
    pk[k] = __builtin_amdgcn_udot4(z, 0x00400801u, (z >> 24) << 9, false);
    n[k] = __builtin_popcount(E);
    EM |= (E >> 2) << k;
    XM |= X >> (7 - k);
    sg |= (H[k] & 0x80808080u) >> (7 - k);
  }
  const unsigned mine = (n[0] + n[1] + n[2] + n[3]) | (__builtin_popcount(XM) << 16);
  const unsigned incl = wave_scan(mine);
  const unsigned all = __builtin_amdgcn_readlane(incl, 63), n2 = all & 0xffffu, n3 = all >> 16;
  const bool raw = 3u * n2 + 16u * n3 > 1008u;
  *(LAS u32x2*)(wl + lane * 8) = (u32x2){t[0] | (t[1] << 4), t[2] | (t[3] << 4)};
  const unsigned sgn = __builtin_amdgcn_update_dpp(0u, sg, 0xb1, 0xf, 0xf, false);
  if (!(lane & 1)) *(LAS unsigned*)(wl + 512 + lane * 2) = sg | (sgn << 4);
  cbar();
  if (!raw) {
    const unsigned pre = (incl - mine) & 0xffffu;
    unsigned pre3 = (incl - mine) >> 16;
    const unsigned long long W = ((unsigned long long)(pk[0] | (pk[1] << (3 * n[0])))
        | ((unsigned long long)(pk[2] | (pk[3] << (3 * n[2]))) << (3 * (n[0] + n[1])))) >> 2;
    const unsigned o = 3 * pre, sh = o & 31u;
    LAS unsigned* q = (LAS unsigned*)(wl + 640 + (o >> 5) * 4);
    const unsigned long long Ws = W << sh;
    __hip_atomic_fetch_or(q, (unsigned)Ws, __ATOMIC_RELAXED, __HIP_MEMORY_SCOPE_WORKGROUP);
    __hip_atomic_fetch_or(q + 1, (unsigned)(Ws >> 32), __ATOMIC_RELAXED, __HIP_MEMORY_SCOPE_WORKGROUP);
    __hip_atomic_fetch_or(q + 2, ((unsigned)(W >> 32) >> 1) >> (31u - sh), __ATOMIC_RELAXED, __HIP_MEMORY_SCOPE_WORKGROUP);
    cbar();
    while (XM) {
      const unsigned pos = __builtin_ctz(XM), k = pos & 3u, i8 = pos & 24u;
      const unsigned Hk = k & 2u ? (k & 1u ? H[3] : H[2]) : (k & 1u ? H[1] : H[0]);
      const unsigned kb = 0x01010101u << k;
      const unsigned rk = pre + __builtin_popcount(EM & ((kb - 0x01010101u) | (kb & ((1u << i8) - 1u))));
      *(LAS unsigned short*)(wl + 640 + 124 - 2 * pre3) = (unsigned short)((rk << 5) | (((Hk >> (i8 + 2)) & 31u) ^ (base + 3u)));
      ++pre3;
      XM &= XM - 1u;
    }
  }
  cbar();
  if (lane == 0) *(LAS unsigned short*)(wl + 640 + 126) = (unsigned short)(raw ? 0x8000u : base | (n3 << 5));
  cbar();
  if (lane < 48) store_sys(dst + 1024 + lane * 16, *(const LAS u32x4*)(wl + lane * 16));
  cbar();
}

__device__ __forceinline__ bool unpack_block(const LAS char* blk, LAS char* ex, const LAS char* lut, u32x4& f0, u32x4& f1) {
  const int lane = threadIdx.x & 63;
  const unsigned ent = *(const LAS unsigned short*)(blk + 1790 - 2 * lane);
  const LAS unsigned* sp = (const LAS unsigned*)(blk + 1664 + ((3 * lane) & ~3));
  const unsigned s0 = sp[0], s1 = sp[1];
  const u32x2 N = *(const LAS u32x2*)(blk + 1024 + lane * 8);
  const unsigned sg = *(const LAS unsigned*)(blk + 1536 + (lane >> 1) * 4) >> ((lane & 1) * 4);
  const u32x4 D = *(const LAS u32x4*)(blk + lane * 16);
  const unsigned hdr = __builtin_amdgcn_readfirstlane(ent);
  if (hdr & 0x8000u) return false;
  const unsigned base = hdr & 31u, n3 = hdr >> 5;
  const unsigned b3 = (base + 3u) * 0x04040404u, b8 = (base + 24u) * 0x04040404u;
  const unsigned T0 = (((b8 + 0x0c080400u) ^ b3) & 0x7c7c7c00u), T1 = (((b8 + 0x1c181410u) ^ b3) & 0x7c7c7c7cu);
  const unsigned x = __builtin_amdgcn_alignbyte(s1, s0, 3 * lane);
  u32x2 X;
#pragma unroll
  for (int i = 0; i < 2; ++i) {
    const unsigned y = ((x >> (12 * i)) & 0x3fu) | ((i ? x >> 2 : x << 10) & 0x3f0000u);
    X[i] = __builtin_amdgcn_perm(T1, T0, (y | (y << 5)) & 0x07070707u);
  }
  *(LAS u32x2*)(ex + lane * 8) = X;
  cbar();
  if ((unsigned)(lane - 1) < n3) *(LAS unsigned char*)(ex + ((ent >> 5) & 511u)) = (unsigned char)((ent & 31u) << 2);
  cbar();
  const unsigned E0 = N[0] & (N[0] >> 1) & 0x44444444u, E1 = N[1] & (N[1] >> 1) & 0x44444444u;
  const unsigned mine = __builtin_popcount(E0) + __builtin_popcount(E1);
  unsigned pos = wave_scan(mine) - mine;
  const unsigned bB = base * 0x04040404u;
  unsigned h[8];
#pragma unroll
  for (int k = 0; k < 4; ++k) {
    const unsigned E = ((k & 2 ? E1 : E0) >> (4 * (k & 1))) & 0x04040404u;
    const unsigned sel = *(const LAS unsigned*)(lut + __builtin_amdgcn_udot4(E, 0x08040201u, 0u, false));
    const LAS unsigned* wp = (const LAS unsigned*)(ex + (pos & ~3u));
    const unsigned win = __builtin_amdgcn_alignbyte(wp[1], wp[0], pos);
    pos += __builtin_popcount(E);
    const unsigned t = (N[k >> 1] >> (4 * (k & 1))) & 0x0f0f0f0fu;
    const unsigned H = ((t + bB) ^ __builtin_amdgcn_perm(0u, win, sel)) | ((sg << (7 - k)) & 0x80808080u);
    h[2 * k] = __builtin_amdgcn_perm(H, D[k], 0x05010400u);
    h[2 * k + 1] = __builtin_amdgcn_perm(H, D[k], 0x07030602u);
  }
  cbar();
  f0 = (u32x4){h[0], h[1], h[2], h[3]};
  f1 = (u32x4){h[4], h[5], h[6], h[7]};
  return true;
}

__device__ __forceinline__ int x_unit(int u) {
  if (u < 448) { const int j = u / 7, r = u - j * 7; return r < 2 ? 2 * j + r : (r - 1) * KSTEPS + j; }
  if (u < 768) { const int j = (u - 448) / 5; return (u - 447 - j * 5) * KSTEPS + 64 + j; }
  return u;
}
__device__ __forceinline__ void convert_x(const Params& p, int first, unsigned call, int rank, LAS char* smem) {
  const int lane = threadIdx.x & 63, wave = threadIdx.x >> 6;
  const int count = 8 * KSTEPS, stride = p.ncvt_x;
  char* const ws = p.ws[rank];
  const float* src0 = p.x + (size_t)((wave >> 2) * 64 + (wave & 1) * 32 + (lane & 15)) * KDIM + (wave & 2) * 16 + (lane >> 4) * 4;
  f32x4 v[4];
  int un = x_unit(first);
  {
    const float* s = src0 + (size_t)(un >> 7) * 128 * KDIM + (un & 127) * 64;
#pragma unroll
    for (int i = 0; i < 4; ++i) v[i] = ld4(s + (i >> 1) * 16 * KDIM + (i & 1) * 16);
  }
  for (int it = first; it < count; it += stride) {
    const int u = un, tile = u >> 7, ks = u & 127;
    const u32x4 h0 = cvt8(v[0], v[1]), h1 = cvt8(v[2], v[3]);
    {
      if (it + stride < count) un = x_unit(it + stride);
      const float* s = src0 + (size_t)(un >> 7) * 128 * KDIM + (un & 127) * 64;
#pragma unroll
      for (int i = 0; i < 4; ++i) v[i] = ld4(s + (i >> 1) * 16 * KDIM + (i & 1) * 16);
    }
    char* b = ws + OFF_OWN + own_idx(call, tile) * PANEL_B + ks * STEP_B + wave * 2048 + lane * 16;
    store_sys(b, h0);
    store_sys(b + 1024, h1);
    pack_block(h0, h1, smem + wave * WAVE_LDS, smem + LDS_LUT,
               ws + OFF_PACK + own_idx(call, tile) * PPANEL_B + ks * PSTEP_B + wave * BLK_B);
    wait_vm0();
    wg_barrier();
    if (threadIdx.x < 4 + FCOPY) {
      const int t = threadIdx.x;
      const bool home = tile < 6;
      const int panel = home ? (rank & 3) * 6 + tile : PA_OWNAWAY + tile - 6;
      const int dst = (home ? (rank & 4) : (rank & 4) ^ 4) | (t & 3);
      if (t >= 4) set_local(ws, t - 4, panel * KSTEPS + ks, call);
      else if (tile == 5 && ks < RELAY) { if (t == (ks & 3)) set_rdy(p.ws[(dst ^ 4) & 7], rank, RDY_HOP1 + (ks >> 2), call); }
      else if (dst != rank) set_rdy(p.ws[dst & 7], rank, home ? u : RDY_AWAY + u - 6 * KSTEPS, call);
    }
  }
}

__device__ __forceinline__ void convert_w(const Params& p, int first, unsigned call, int rank, LAS char* smem) {
  const int lane = threadIdx.x & 63, wave = threadIdx.x >> 6;
  const int count = KSTEPS * 4, stride = p.ncvt_w;
  char* const ws = p.ws[rank];
  const float* src0 = p.w + (size_t)((wave & 1) * 32 + (lane >> 4) * 4) * NSH + (wave >> 1) * 64 + (lane & 15) * 4;
  f32x4 v[8];
  int u = first;
  {
    const float* s = src0 + (size_t)(u >> 2) * 64 * NSH + (u & 3) * 256;
#pragma unroll
    for (int j = 0; j < 8; ++j) v[j] = ld4(s + ((j & 3) + (j >> 2) * 16) * NSH);
  }
  for (; u < count; u += stride) {
    const int ks = u >> 2, q = u & 3;
    u32x4 h[4];
#pragma unroll
    for (int f = 0; f < 4; ++f)
      h[f] = cvt8((f32x4){v[0][f], v[1][f], v[2][f], v[3][f]}, (f32x4){v[4][f], v[5][f], v[6][f], v[7][f]});
    {
      const int un = u + stride < count ? u + stride : u;
      const float* s = src0 + (size_t)(un >> 2) * 64 * NSH + (un & 3) * 256;
#pragma unroll
      for (int j = 0; j < 8; ++j) v[j] = ld4(s + ((j & 3) + (j >> 2) * 16) * NSH);
    }
    const unsigned oi = own_idx(call, 8 + q * 2 + (wave >> 2));
    char* b = ws + OFF_OWN + oi * PANEL_B + ks * STEP_B + (wave & 3) * 4096 + lane * 16;
#pragma unroll
    for (int f = 0; f < 4; ++f) store_sys(b + f * 1024, h[f]);
    char* pb = ws + OFF_PACK + oi * PPANEL_B + ks * PSTEP_B + (wave & 3) * 2 * BLK_B;
    pack_block(h[0], h[1], smem + wave * WAVE_LDS, smem + LDS_LUT, pb);
    pack_block(h[2], h[3], smem + wave * WAVE_LDS + 1024, smem + LDS_LUT, pb + BLK_B);
    wait_vm0();
    wg_barrier();
    if (threadIdx.x <= 2 * FCOPY) {
      const int t = threadIdx.x;
      if (t < 2 * FCOPY) set_local(ws, t >> 1, (PB + q * 2 + (t & 1)) * KSTEPS + ks, call);
      else set_rdy(p.ws[(((rank & 4) ^ 4) | q) & 7], rank, ks, call);
    }
  }
}

__device__ __forceinline__ int row_of(int r) { return ((RANK_AT >> (4 * r)) & 7) * NSH; }

#define ORDER_V 0x7648953210ull
constexpr unsigned long long BATCH_AT = ORDER_V;
constexpr unsigned long long batch_pos() {
  unsigned long long r = 0;
  for (int i = 0; i < 10; ++i) r |= (unsigned long long)i << (4 * ((BATCH_AT >> (4 * i)) & 15));
  return r;
}
constexpr unsigned long long BATCH_POS = batch_pos();
__device__ __forceinline__ int batch_row(int s, int o, int pos, int i) {
  const int bt = (BATCH_AT >> (4 * pos)) & 15;
  if (bt < 2) return row_of(s) + (bt * 3 + i) * 128;
  const int g = bt < 8 ? s : o;
  return row_of((g & 4) | ((g + 1 + i) & 3)) + (bt - 2) * 128;
}

__device__ __forceinline__ int unit_of(int it, int w) { return it * PW + ((w + it) & (PW - 1)); }

__device__ __forceinline__ unsigned lds_get(const LAS unsigned* c) {
  return __builtin_amdgcn_readfirstlane(__hip_atomic_load(c, __ATOMIC_RELAXED, __HIP_MEMORY_SCOPE_WORKGROUP));
}
__device__ __forceinline__ unsigned lds_ticket(LAS unsigned* c) {
  unsigned t = 0;
  if ((threadIdx.x & 63) == 0) t = __hip_atomic_fetch_add(c, 1u, __ATOMIC_RELAXED, __HIP_MEMORY_SCOPE_WORKGROUP);
  return __builtin_amdgcn_readfirstlane(t);
}

constexpr int RING = 8;
constexpr unsigned PL_EX = RING * PSTEP_B;
constexpr int W_TICKET = 0, W_READY = 1, W_DEAD = 2, W_DONE = 8, W_LANDED = 64, W_QUEUE = 128;
constexpr unsigned Q_N = 128, Q_OUT = 1024, Q_LAST = 1u << 31;
static_assert(PL_EX + 8 * 512 <= LDS_MB && (W_QUEUE + Q_N) * 4 <= LDS_TMO - LDS_MB, "");

template <int N>
__device__ __forceinline__ void stream_kib(const char* src, unsigned dst, unsigned& cnt, unsigned word) {
  unsigned keep;
#define SK(OFF) "global_load_lds_dwordx4 %2, off offset:" #OFF " sc0 sc1\n\tv_add_u32 %1, 1, %1\n\t" \
                "s_waitcnt vmcnt(%5)\n\tds_max_u32 %4, %1\n\t"
  if (N == 4)
    asm volatile("s_mov_b32 %0, m0\n\ts_mov_b32 m0, %3\n\ts_nop 0\n\t" SK(0) SK(1024) SK(2048) SK(3072) "s_mov_b32 m0, %0"
                 : "=&s"(keep), "+v"(cnt) : "v"(src), "s"(dst), "v"(word), "n"(WIN - 1) : "memory");
  else
    asm volatile("s_mov_b32 %0, m0\n\ts_mov_b32 m0, %3\n\ts_nop 0\n\t" SK(0) SK(1024) "s_mov_b32 m0, %0"
                 : "=&s"(keep), "+v"(cnt) : "v"(src), "s"(dst), "v"(word), "n"(WIN - 1) : "memory");
}
template <int K>
__device__ __forceinline__ void stream_drain(unsigned cnt, unsigned word) {
  if constexpr (K >= 0) {
    asm volatile("s_waitcnt vmcnt(%2)\n\tds_max_u32 %1, %0" :: "v"(cnt + 1), "v"(word), "n"(K) : "memory");
    stream_drain<K - 1>(cnt + 1, word);
  }
}

struct Unit { unsigned src, raw, dst; int from, flag; };
template <bool CROSS>
__device__ __forceinline__ Unit input_unit(int u, unsigned call, int rank, int peer) {
  constexpr int H1 = RELAY / 4, RSTEP = 32 * KSTEPS;
  const unsigned par = own_idx(call, 0) * KSTEPS;
  Unit o;
  o.from = peer;
  if (!CROSS) {
    const int v = u < 5 * KSTEPS ? u : u + RELAY;
    o.raw = par + v; o.dst = (peer & 3) * 6 * KSTEPS + v; o.flag = v;
  } else if (u < 256) {
    const int v = (u & 1) * KSTEPS + (u >> 1);
    o.raw = par + (8 + (rank & 3) * 2) * KSTEPS + v; o.dst = (PB_FOREIGN + (peer & 3) * 2) * KSTEPS + v; o.flag = u >> 1;
  } else if (u < 256 + H1) {
    o.raw = par + 5 * KSTEPS + (u - 256) * 4 + (rank & 3); o.dst = ~((peer & 3) * 16 + u - 256); o.flag = RDY_HOP1 + u - 256;
  } else if (u < 512 + H1) {
    const int v = u - 256 - H1;
    o.raw = par + 6 * KSTEPS + v; o.dst = (PA_AWAY + (peer & 3) * 2) * KSTEPS + v; o.flag = RDY_AWAY + v;
  } else {
    const int v = u - 512 - H1, j = v / 3, s = (rank + 1 + v - j * 3) & 3, ks = 5 * KSTEPS + j * 4 + (peer & 3);
    o.from = (rank & 4) | s;
    o.raw = par + ks; o.dst = s * 6 * KSTEPS + ks; o.flag = RDY_HOP2 + s * 16 + j;
    o.src = RSTEP + s * 16 + j;
    return o;
  }
  o.src = o.raw;
  return o;
}

template <bool CROSS>
__device__ __forceinline__ void pull(const Params& p, int w, unsigned call, int rank, int peer, LAS char* smem) {
  constexpr int NIN = CROSS ? 512 + RELAY : 768 - RELAY, NOUT = CROSS ? OUT_CHUNKS / PW : 0;
  const int lane = threadIdx.x & 63, wave = __builtin_amdgcn_readfirstlane(threadIdx.x >> 6);
  char* const ws = p.ws[rank];
  const char* const pws = p.ws[peer];
  LAS unsigned* const sh = (LAS unsigned*)(smem + LDS_MB);
  float* const out = p.out[rank] + (size_t)((lane >> 4) * 4) * NSH + (peer & 3) * 256 + (lane & 15) * 4;
  auto where = [&](int c, int pc) {
    const int tile = c >> 3, bt = tile / 6, r = tile - bt * 6;
    return out + (size_t)(batch_row(peer, rank, bt, r >> 1) + ((c >> 2) & 1) * 64 + (c & 1) * 32 + (pc & 4) * 4 + (pc & 3)) * NSH
         + (r & 1) * 128 + ((c >> 1) & 1) * 64;
  };
  if (wave == 6) {
    const unsigned* const rdy = (const unsigned*)(ws + OFF_RDY) + peer * RDY_N * 32;
    unsigned* const tmo = (unsigned*)(ws + OFF_TMO);
    unsigned* const ticket = (unsigned*)(ws + OFF_TICKET) + peer * 32;
    unsigned long long t0 = __builtin_amdgcn_s_memrealtime();
    bool dead = false;
    unsigned k = 0, kib = 0, spin = 0;
    auto unset = [&](bool look, int fl) {
      unsigned v = call;
      if (look && !dead)
        asm volatile("global_load_dword %0, %1, off sc0 sc1\n\ts_waitcnt vmcnt(0)" : "=v"(v) : "v"(rdy + fl * 32) : "memory");
      return __builtin_amdgcn_ballot_w64(v < call);
    };
    auto idle = [&]() {
      if ((++spin & 63u) == 0u
          && (__builtin_amdgcn_readfirstlane(__hip_atomic_load(tmo, __ATOMIC_RELAXED, __HIP_MEMORY_SCOPE_AGENT)) == call
              || __builtin_amdgcn_s_memrealtime() - t0 > DEADLINE)) {
        if (lane == 0) {
          __hip_atomic_store(tmo, call, __ATOMIC_RELAXED, __HIP_MEMORY_SCOPE_AGENT);
          __hip_atomic_store(sh + W_DEAD, 1u, __ATOMIC_RELAXED, __HIP_MEMORY_SCOPE_WORKGROUP);
        }
        wait_vm0();
        dead = true;
      }
      __builtin_amdgcn_s_sleep(1);
    };
    auto publish = [&](unsigned v, unsigned near) {
      cbar();
      if (lane == 0) __hip_atomic_store(sh + W_READY, v, __ATOMIC_RELAXED, __HIP_MEMORY_SCOPE_WORKGROUP);
      while (lds_get(sh + W_LANDED) < near) __builtin_amdgcn_s_sleep(2);
      t0 = __builtin_amdgcn_s_memrealtime();
    };
    int known = EXPON(1) ? NIN : 0;
    auto look = [&](int from, int width) {
      const unsigned long long miss = unset(lane < width, input_unit<CROSS>(min(from + lane, NIN - 1), call, rank, peer).flag);
      known = from + (miss ? (int)__builtin_ctzll(miss) : width);
    };
    for (int u = w; u < NIN;) {
      while (u >= known) { look(u, 16); if (u >= known) idle(); }
      if (lane == 0) sh[W_QUEUE + (k & (Q_N - 1))] = kib << 11 | u;
      publish(++k, kib);
      kib += 14;
      unsigned tk = 0;
      if (lane == 0) tk = __hip_atomic_fetch_add(ticket, 1u, __ATOMIC_RELAXED, __HIP_MEMORY_SCOPE_AGENT);
      if (known < NIN && known - u < 32) look(known, 64);
      u = min(__builtin_amdgcn_readfirstlane(tk) - (call - 1u) * NIN + PW, (unsigned)NIN);
    }
    const unsigned long long all = (1ull << NOUT) - 1ull;
    for (unsigned long long left = all; left;) {
      const unsigned long long fresh = left & ~unset(!EXPON(17) && ((left >> lane) & 1), RDY_OUT + unit_of(lane, w));
      if (!fresh) { idle(); continue; }
      const unsigned pos = __builtin_amdgcn_mbcnt_hi(fresh >> 32, __builtin_amdgcn_mbcnt_lo(fresh, 0));
      if ((fresh >> lane) & 1) sh[W_QUEUE + ((k + pos) & (Q_N - 1))] = (kib + pos * 8) << 11 | (Q_OUT + unit_of(lane, w));
      k += __builtin_popcountll(fresh);
      kib += __builtin_popcountll(fresh) * 8;
      left &= ~fresh;
      if (left) publish(k, kib - 16);
    }
    publish(k | Q_LAST, 0);
    if (CROSS) {
      dead = __builtin_amdgcn_readfirstlane(__hip_atomic_load(tmo, __ATOMIC_RELAXED, __HIP_MEMORY_SCOPE_AGENT)) == call;
      wg_barrier();
      if (dead) store_sys(where(unit_of(0, w), 0), (u32x4){0x7fc00000u, 0x7fc00000u, 0x7fc00000u, 0x7fc00000u});
    }
    return;
  }
  if (wave == 7) {
    const unsigned lds0 = (unsigned)(uintptr_t)smem, word = (unsigned)(uintptr_t)(sh + W_LANDED + lane);
    unsigned cnt = 0;
    for (unsigned k = 0;; ++k) {
      unsigned ready = lds_get(sh + W_READY);
      if (k >= (ready & ~Q_LAST) || lds_get(sh + W_DONE + (k & (RING - 1))) < (k & ~(RING - 1u))) {
        stream_drain<WIN - 2>(cnt, word);
        while (k >= ready) { __builtin_amdgcn_s_sleep(1); ready = lds_get(sh + W_READY); }
        if (k >= (ready & ~Q_LAST)) break;
        while (lds_get(sh + W_DONE + (k & (RING - 1))) < (k & ~(RING - 1u))) __builtin_amdgcn_s_sleep(1);
      }
      cbar();
      const unsigned u = lds_get(sh + W_QUEUE + (k & (Q_N - 1))) & 2047u, d = lds0 + (k & (RING - 1)) * PSTEP_B;
      if (u < Q_OUT) {
        if ((u & 63) >= 56) STAMP(4 + (u >> 6));
        const char* s = pws + OFF_PACK + (size_t)input_unit<CROSS>(u, call, rank, peer).src * PSTEP_B + lane * 16;
        stream_kib<4>(s, d, cnt, word);
        stream_kib<4>(s + 4096, d + 4096, cnt, word);
        stream_kib<4>(s + 8192, d + 8192, cnt, word);
        stream_kib<2>(s + 12288, d + 12288, cnt, word);
        if (k == 0) STAMP(3);
      } else {
        if ((u - Q_OUT) % 48 >= 40) STAMP(20 + (u - Q_OUT) / 48);
        const char* s = pws + OFF_STG + (size_t)((rank & 3) * OUT_CHUNKS + u - Q_OUT) * CHUNK_B + lane * 16;
        stream_kib<4>(s, d, cnt, word);
        stream_kib<4>(s + 4096, d + 4096, cnt, word);
      }
    }
    STAMP(1);
    if (CROSS) wg_barrier();
    return;
  }
  static_assert(RING == 8, "");
  auto landed = [&](unsigned t, unsigned& u) {
    const unsigned ready = lds_get(sh + W_READY);
    cbar();
    const unsigned q = lds_get(sh + W_QUEUE + ((t >> 3) & (Q_N - 1))), n = lds_get(sh + W_LANDED);
    u = q & 2047u;
    if ((t >> 3) >= (ready & ~Q_LAST)) { u = ~0u; return ready >= Q_LAST; }
    return n >= (q >> 11) + (u < Q_OUT ? ((t & 7) * 7 + 10) >> 2 : (t & 7) + 1) + WIN - 1;
  };
  const unsigned none = ~0u;
  unsigned prev = none, pdst = 0;
  auto done = [&]() {
    const bool last = (lds_ticket(sh + W_DONE + ((prev >> 3) & (RING - 1))) & 7u) == 7u;
    prev = none;
    if (!last) return;
    if (!CROSS && (pdst & 127) >= 112) STAMP(17 + ((~pdst >> 3) & 1));
    if (!CROSS || (int)pdst >= 0) { if (lane < FCOPY) set_local(ws, lane, pdst, call); }
    else if (lane < 3) set_rdy(p.ws[((peer & 4) | ((peer + 1 + lane) & 3)) & 7], rank, RDY_HOP2 + ~pdst, call);
  };
  for (;;) {
    unsigned u;
    if (prev != none && !landed(lds_get(sh + W_TICKET), u)) { wait_vm0(); done(); }
    const unsigned t = lds_ticket(sh + W_TICKET);
    if (!landed(t, u)) {
      if (prev != none) { wait_vm0(); done(); }
      while (!landed(t, u)) __builtin_amdgcn_s_sleep(1);
    }
    cbar();
    const LAS char* slot = smem + ((t >> 3) & (RING - 1)) * PSTEP_B;
    if (u >= Q_OUT) {
      if (prev != none) { wait_vm0(); done(); }
      if (!CROSS || u == none) break;
      u32x4 v = *(const LAS u32x4*)(slot + (t & 7) * 1024 + lane * 16);
      if (lds_get(sh + W_DEAD)) v = (u32x4){0x7fc00000u, 0x7fc00000u, 0x7fc00000u, 0x7fc00000u};
      store_out(where(u - Q_OUT, t & 7), v);
      if (lane == 0) __hip_atomic_fetch_add(sh + W_DONE + ((t >> 3) & (RING - 1)), 1u, __ATOMIC_RELAXED, __HIP_MEMORY_SCOPE_WORKGROUP);
      continue;
    }
    const Unit un = input_unit<CROSS>(u, call, rank, peer);
    const LAS char* blk = slot + (t & 7) * BLK_B;
    u32x4 f0, f1;
    char* d;
    if (CROSS && RELAY && (int)un.dst < 0) {
      const int l1 = min(lane, 47);
      f0 = *(const LAS u32x4*)(blk + lane * 16);
      f1 = *(const LAS u32x4*)(blk + 1024 + l1 * 16);
      d = ws + OFF_RELAY + (size_t)(~un.dst * 8 + (t & 7)) * BLK_B;
      store_sys(d + lane * 16, f0);
      store_sys(d + 1024 + l1 * 16, f1);
    } else {
      if (!unpack_block(blk, smem + PL_EX + wave * 512, smem + LDS_LUT, f0, f1)) {
        const char* r = p.ws[un.from & 7] + OFF_OWN + (size_t)(un.raw * 8 + (t & 7)) * 2048 + lane * 16;
        asm volatile("global_load_dwordx4 %0, %2, off sc0 sc1\n\t"
                     "global_load_dwordx4 %1, %2, off offset:1024 sc0 sc1\n\ts_waitcnt vmcnt(0)"
                     : "=&v"(f0), "=&v"(f1) : "v"(r) : "memory");
      }
      d = ws + OFF_PULL + (size_t)(un.dst * 8 + (t & 7)) * 2048 + lane * 16;
      store_sys(d, f0);
      store_sys(d + 1024, f1);
    }
    asm volatile("s_waitcnt vmcnt(2)" ::: "memory");
    if (prev != none) done();
    prev = t;
    pdst = un.dst;
  }
  if (CROSS) { wait_vm0(); wg_barrier(); }
}

struct Job { int pa, pb, ah, bh, tile; };
__device__ __forceinline__ void decode_job(unsigned code, int a, Job& o) {
  const int ra = code & 63;
  int bt, i;
  o.pb = PB + ((code >> 6) & 15);
  o.ah = (code >> 10) & 1;
  o.bh = (code >> 11) & 1;
  if (ra < 18) {
    i = ra / 6; bt = 2 + ra % 6; o.pa = ((a + 1 + i) & 3) * 6 + bt - 2;
  } else if (ra < 24) {
    o.pa = a * 6 + ra - 18; bt = (ra - 18) / 3; i = (ra - 18) % 3;
  } else {
    o.pa = ra; bt = 8 + (ra & 1); i = (((ra - PA_AWAY) >> 1) - ((o.pb - PB_FOREIGN) >> 1) - 1) & 3;
  }
  o.tile = (int)((BATCH_POS >> (4 * bt)) & 15) * 3 + i;
}

__device__ __forceinline__ float silu(float y) {
  const float c = fminf(fmaxf(y, -60.0f), 60.0f);
  return y * __builtin_amdgcn_rcpf(1.0f + __builtin_amdgcn_exp2f(c * -1.44269504088896341f));
}

#define POLL_EVERY 1
#define POLL_CAD 2
#define POLL_SLEEP 2
#define GLDS_BITS " sc0 sc1"

template <int NB>
__device__ __forceinline__ unsigned next_stage(unsigned so) { return so == LDS_MB - (NB + 1) * 8192 ? 0u : so + (NB + 1) * 8192; }

template <bool TALL, int NB>
__device__ __forceinline__ void mfma_wave(const Params& p, LAS char* smem, const Job& jb, int job, int wave,
                                          int rank, unsigned call) {
  constexpr int MT = TALL ? 4 : NB, NT = TALL && NB == 2 ? 8 : 4, KH = TALL ? 1 : 2, STEPS = 2 * KSTEPS / KH;
  constexpr int NRA = MT * KH, NRB = NT * KH, NRD = NRA + NRB, NMM = MT * NT * KH;
  int lane = fresh_lane();
  const int wr = MT == 1 ? wave : wave >> 1, wc = MT == 1 ? 0 : wave & 1;
  f32x4 acc[MT][NT];
#pragma unroll
  for (int m = 0; m < MT; ++m)
#pragma unroll
    for (int n = 0; n < NT; ++n) acc[m][n] = (f32x4){0.f, 0.f, 0.f, 0.f};
  half8 fa[2][NRA], fb[2][NRB];
  const LAS char* la = smem + wr * MT * 1024 + lane * 16;
  const LAS char* lb = smem + 8192 + wc * NRB * 1024 + lane * 16;
  unsigned so = 0;
#define READ_SET(X) do { \
    _Pragma("unroll") for (int i = 0; i < NRB; ++i) { \
      if (i < NRA) fa[X][i] = *(const LAS half8*)(la + so + (i / MT) * 4096 + (i % MT) * 1024); \
      fb[X][i] = *(const LAS half8*)(lb + so + i * 1024); } \
    so = next_stage<NB>(so); } while (0)
#define MMA_SET(X) do { \
    _Pragma("unroll") for (int kh = 0; kh < KH; ++kh) \
    _Pragma("unroll") for (int m = 0; m < MT; ++m) \
    _Pragma("unroll") for (int n = 0; n < NT; ++n) \
      acc[m][n] = __builtin_amdgcn_mfma_f32_16x16x32_f16(fa[X][kh * MT + m], fb[X][kh * NT + n], acc[m][n], 0, 0, 0); } while (0)
  constexpr int RG = TALL ? 1 : 2, MG = NB, G = NRD / RG;
#define INTERLEAVE() do { \
    _Pragma("unroll") for (int i = 0; i < G; ++i) { \
      __builtin_amdgcn_sched_group_barrier(0x100, RG, 0); \
      __builtin_amdgcn_sched_group_barrier(0x008, MG, 0); } \
    __builtin_amdgcn_sched_group_barrier(0x008, NMM - G * MG, 0); } while (0)
#define STEP_END() do { __builtin_amdgcn_sched_barrier(0); asm volatile("" ::: "memory"); \
    __builtin_amdgcn_s_waitcnt(0xC07F);   __builtin_amdgcn_sched_barrier(0); \
    wg_barrier(); __builtin_amdgcn_sched_barrier(0); } while (0)
  wg_barrier();
  if (wave == 0) STAMP(4 + job * 4);
  READ_SET(0);
  STEP_END();
  for (int t = 1; t < STEPS - 1; t += 2) {
    READ_SET(1); MMA_SET(0); INTERLEAVE();
    STEP_END();
    READ_SET(0); MMA_SET(1); INTERLEAVE();
    STEP_END();
  }
  READ_SET(1); MMA_SET(0); INTERLEAVE();
  const unsigned tm = *(const LAS unsigned*)(smem + LDS_TMO + job * 16);
  STEP_END();
  MMA_SET(1);
  __builtin_amdgcn_sched_barrier(0);
  if (wave == 0) STAMP(5 + job * 4);
  lane = fresh_lane();
  asm volatile("" : "+s"(call));
  const bool dead = tm == call;

  const bool foreign = jb.pb >= PB_FOREIGN;
  const int ob = (jb.pb - PB_FOREIGN) >> 1;
  const int r16 = jb.ah * 4 + wr * MT;
  const int c64 = (jb.pb & 1) * 2 + jb.bh + wc * (NT / 4);
  const int chunk = (jb.tile * 2 + (c64 >> 1)) * 8 + (r16 & 4) + (c64 & 1) * 2 + ((r16 >> 1) & 1);
  char* o;
  unsigned sm, sr, sh;
  if (foreign) {
    o = p.ws[rank] + OFF_STG + (size_t)(ob * OUT_CHUNKS + chunk) * CHUNK_B + (r16 & 1) * 4096 + lane * 16;
    sm = 4096; sr = 1024; sh = 2 * CHUNK_B;
  } else {
    const int pa = jb.pa;
    const int row0 = pa < PA_AWAY ? row_of((rank & 4) + pa / 6) + (pa % 6) * 128
                   : row_of(pa < PA_OWNAWAY ? ((rank & 4) ^ 4) + ((pa - PA_AWAY) >> 1) : rank) + 768 + (pa & 1) * 128;
    o = (char*)(p.out[rank] + (size_t)(row0 + r16 * 16 + (lane >> 4) * 4) * NSH + ((jb.pb - PB) >> 1) * 256 + c64 * 64 + (lane & 15) * 4);
    sm = 16 * NSH * 4; sr = NSH * 4; sh = 256;
  }
  const float qnan = __builtin_nanf("");
#pragma unroll
  for (int h = 0; h < NT / 4; ++h)
#pragma unroll
    for (int m = 0; m < MT; ++m)
#pragma unroll
      for (int r = 0; r < 4; ++r) {
        f32x4 v = {silu(acc[m][h * 4][r]), silu(acc[m][h * 4 + 1][r]), silu(acc[m][h * 4 + 2][r]), silu(acc[m][h * 4 + 3][r])};
        if (dead) v = (f32x4){qnan, qnan, qnan, qnan};
        if (foreign) store_sys(o + h * sh + m * sm + r * sr, __builtin_bit_cast(u32x4, v));
        else store_out(o + h * sh + m * sm + r * sr, __builtin_bit_cast(u32x4, v));
      }
  if (foreign) {
    wait_vm0();
    bool last = lane < (MT * NT + 7) / 8;
    if (MT == 1)
      last = last && __hip_atomic_fetch_add((LAS unsigned*)(smem + LDS_TMO) + job * 4 + 2 + (wave >> 1), 1u,
                                            __ATOMIC_RELAXED, __HIP_MEMORY_SCOPE_WORKGROUP) == 1u;
    if (last) set_rdy(p.ws[(((rank & 4) ^ 4) | ob) & 7], rank, RDY_OUT + chunk + lane, call);
  }
  if (wave == 0) STAMP(6 + job * 4);
}

#define NEAR 4
template <int NB>
__device__ __forceinline__ void load_wave(const Params& p, LAS char* smem, const Job& jb, int job, unsigned code, int l,
                                          bool tall, int rank, unsigned call) {
  constexpr int DEPTH = NB == 2 ? 5 : 7;
  const int steps = tall ? 2 * KSTEPS : KSTEPS, ksh = tall ? 1 : 0;
  if (l > NB && l < 3) {
    for (int t = 0; t <= steps; ++t) wg_barrier();
    return;
  }
  const int lane = fresh_lane(), a = rank & 3;
  char* const ws = p.ws[rank];
  const unsigned lds0 = (unsigned)(uintptr_t)smem;
  LAS unsigned* const rdyw = (LAS unsigned*)(smem + LDS_TMO) + job * 4 + 1;
  const unsigned gap = tall ? 8192 : 4096;
  auto panel_of = [&](int pc) { return pc == 0 ? jb.pa : tall ? jb.pb + pc - 1 : jb.pb; };
  auto source = [&](int pc) {
    const int panel = panel_of(pc);
    const int own = panel >= PB_FOREIGN ? -1 : panel >= PA_OWNAWAY ? panel - PA_OWNAWAY + 6
                  : (unsigned)(panel - a * 6) < 6u ? panel - a * 6 : -1;
    return ws + (own >= 0 ? OFF_OWN + own_idx(call, own) * PANEL_B : OFF_PULL + (size_t)panel * PANEL_B)
         + (tall ? 0 : (pc == 0 ? jb.ah : jb.bh + pc - 1) * 8192);
  };
  int ready = EXPON(1) ? KSTEPS : 0;
  if (l == 3) {
    unsigned* const tmo = (unsigned*)(ws + OFF_TMO);
    const char* fsrc = ws + OFF_FLAG + (blockIdx.x & (FCOPY - 1)) * 65536 + panel_of(lane >> 5) * 1024 + ((lane >> 2) & 7) * 128 + (lane & 3) * 16;
    const unsigned mb = lds0 + LDS_MB + job * 2048;
    int ra = ready, rb = ready;
    auto poll = [&]() {
      if ((lane < 32 ? ra : rb) < KSTEPS) {
        unsigned keep;
        asm volatile("s_mov_b32 %0, m0\n\ts_mov_b32 m0, %2\n\ts_nop 0\n\t"
                     "global_load_lds_dwordx4 %1, off sc0 sc1\n\ts_mov_b32 m0, %0"
                     : "=&s"(keep) : "v"(fsrc), "s"(mb) : "memory");
        if (NB == 2 && tall && lane >= 32)
          asm volatile("s_mov_b32 %0, m0\n\ts_mov_b32 m0, %2\n\ts_nop 0\n\t"
                       "global_load_lds_dwordx4 %1, off offset:1024 sc0 sc1\n\ts_mov_b32 m0, %0"
                       : "=&s"(keep) : "v"(fsrc), "s"(mb - 512) : "memory");
      }
    };
    auto refresh = [&]() {
      u32x2 f, g, h;
      asm volatile("ds_read2_b32 %0, %3 offset1:8\n\tds_read2_b32 %1, %3 offset0:128 offset1:136\n\t"
                   "ds_read2_b32 %2, %4 offset1:8\n\ts_waitcnt lgkmcnt(0)"
                   : "=&v"(f), "=&v"(g), "=&v"(h)
                   : "v"(mb + (lane & 7) * 64 + (lane >> 3) * 4), "v"(mb + (NB == 2 && tall ? 1024 : 512) + (lane & 7) * 64 + (lane >> 3) * 4)
                   : "memory");
      const unsigned long long a0 = __builtin_amdgcn_ballot_w64(f[0] < call), a1 = __builtin_amdgcn_ballot_w64(f[1] < call);
      const unsigned long long b0 = __builtin_amdgcn_ballot_w64(min(g[0], h[0]) < call), b1 = __builtin_amdgcn_ballot_w64(min(g[1], h[1]) < call);
      ra = a0 ? __builtin_ctzll(a0) : a1 ? 64 + __builtin_ctzll(a1) : KSTEPS;
      rb = b0 ? __builtin_ctzll(b0) : b1 ? 64 + __builtin_ctzll(b1) : KSTEPS;
      ready = min(ra, rb);
      if (lane == 0) __hip_atomic_store(rdyw, (unsigned)ready, __ATOMIC_RELAXED, __HIP_MEMORY_SCOPE_WORKGROUP);
    };
    auto advance = [&]() {
      const int k = min(ready + lane, KSTEPS - 1);
      u32x2 f;
      unsigned h;
      asm volatile("ds_read2_b32 %0, %2 offset1:128\n\tds_read_b32 %1, %3\n\ts_waitcnt lgkmcnt(0)"
                   : "=&v"(f), "=&v"(h)
                   : "v"(mb + (k & 7) * 64 + (k >> 3) * 4), "v"(mb + (NB == 2 && tall ? 1024 : 512) + (k & 7) * 64 + (k >> 3) * 4)
                   : "memory");
      const unsigned long long m = __builtin_amdgcn_ballot_w64(min(min(f[0], f[1]), h) < call);
      ready = min(ready + (m ? (int)__builtin_ctzll(m) : 64), KSTEPS);
      __hip_atomic_store(rdyw, (unsigned)ready, __ATOMIC_RELAXED, __HIP_MEMORY_SCOPE_WORKGROUP);
    };
    for (int s = 0; s < steps; ++s) {
      if (s >= DEPTH) wg_barrier();
      const int ks = s >> ksh;
      if (ready < KSTEPS && ready - ks < NEAR) {
        advance();
        if (ks >= ready) {
          const unsigned long long t0 = __builtin_amdgcn_s_memrealtime();
          for (unsigned it = 1;; ++it) {
            if (it >= 32u || s == 0) { poll(); wait_vm0(); refresh(); }
            else { if ((it & (POLL_CAD - 1)) == 1u) poll(); advance(); }
            if (ks < ready) break;
            if ((it & 63u) == 0u) {
              if (__builtin_amdgcn_readfirstlane(__hip_atomic_load(tmo, __ATOMIC_RELAXED, __HIP_MEMORY_SCOPE_AGENT)) == call
                  || __builtin_amdgcn_s_memrealtime() - t0 > DEADLINE) {
                if (lane == 0) __hip_atomic_store(tmo, call, __ATOMIC_RELAXED, __HIP_MEMORY_SCOPE_AGENT);
                wait_vm0();
                ready = KSTEPS;
                if (lane == 0) __hip_atomic_store(rdyw, (unsigned)ready, __ATOMIC_RELAXED, __HIP_MEMORY_SCOPE_WORKGROUP);
                break;
              }
            }
            if (it >= 32u) __builtin_amdgcn_s_sleep(POLL_SLEEP); else __builtin_amdgcn_s_sleep(1);
          }
        } else if (!(s & (((POLL_EVERY) << ksh) - 1))) {
          poll();
        }
      }
    }
    if (lane == 0) {
      unsigned keep;
      asm volatile("s_mov_b32 %0, m0\n\ts_mov_b32 m0, %2\n\ts_nop 0\n\t"
                   "global_load_lds_dword %1, off sc1\n\ts_mov_b32 m0, %0"
                   : "=&s"(keep) : "v"(tmo), "s"(lds0 + LDS_TMO + job * 16) : "memory");
    }
    for (int t = 1; t < DEPTH; ++t) wg_barrier();
    wait_vm0();
    wg_barrier();
    wg_barrier();
    return;
  }
  const char* src = source(l) + lane * 16;
  const unsigned dst0 = lds0 + l * 8192;
  unsigned so = 0;
  for (int s = 0; s < steps; ++s) {
    if (s >= DEPTH) {
      asm volatile("s_waitcnt vmcnt(%0)" :: "n"(8 * (DEPTH - 1)) : "memory");
      wg_barrier();
    }
    const int ks = s >> ksh;
    if (ks >= ready)
      while ((ready = lds_get(rdyw)) <= ks) __builtin_amdgcn_s_sleep(1);
    const char* g = src + (size_t)ks * STEP_B + (s & ksh) * 4096;
    const unsigned d = dst0 + so;
    so = next_stage<NB>(so);
    unsigned keep;
#define GLDS4(R) "global_load_lds_dwordx4 " R ", off" GLDS_BITS "\n\t" \
                 "global_load_lds_dwordx4 " R ", off offset:1024" GLDS_BITS "\n\t" \
                 "global_load_lds_dwordx4 " R ", off offset:2048" GLDS_BITS "\n\t" \
                 "global_load_lds_dwordx4 " R ", off offset:3072" GLDS_BITS "\n\t"
    asm volatile("s_mov_b32 %0, m0\n\ts_mov_b32 m0, %3\n\ts_nop 0\n\t" GLDS4("%1")
                 "s_mov_b32 m0, %4\n\ts_nop 0\n\t" GLDS4("%2") "s_mov_b32 m0, %0"
                 : "=&s"(keep) : "v"(g), "v"(g + gap), "s"(d), "s"(d + 4096) : "memory");
  }
#define WB(N) "s_waitcnt vmcnt(" #N ")\n\ts_barrier\n\t"
  if (NB == 1) asm volatile(WB(48) WB(40) ::: "memory");
  asm volatile(WB(32) WB(24) WB(16) WB(8) WB(0) "s_barrier" ::: "memory");
}

__global__ void __launch_bounds__(NTHREADS, 2) ag_gemm_silu(const Params p) {
  __shared__ __attribute__((aligned(16))) char smem[LDS_BYTES];
  const int rank = p.rank & 7, a = rank & 3;
  const int wave = __builtin_amdgcn_readfirstlane(threadIdx.x >> 6);
#define XROT 4
  const int xcd = (blockIdx.x - XROT) & 7, slot = blockIdx.x >> 3;
  char* const ws = p.ws[rank];
  const u32x4 plan = p.plan[slot < GSLOTS ? slot * 8 + xcd : GSLOTS * 8 + slot - GSLOTS];

  unsigned* const cnt = (unsigned*)(ws + OFF_CALL) + blockIdx.x;
  const unsigned call = __builtin_amdgcn_readfirstlane(
      __hip_atomic_load(cnt, __ATOMIC_RELAXED, __HIP_MEMORY_SCOPE_AGENT)) + 1u;
  if (wave == 0) STAMP(0);
  const bool puller = slot >= GSLOTS && xcd < 7;
  for (unsigned i = threadIdx.x; i < (LDS_LUT - LDS_MB) / 4; i += NTHREADS) ((LAS unsigned*)(smem + LDS_MB))[i] = 0u;
  if (threadIdx.x < 16) ((LAS unsigned*)(smem + LDS_LUT))[threadIdx.x] = codec_lut(threadIdx.x, puller);
  asm volatile("s_waitcnt vmcnt(0) lgkmcnt(0)" ::: "memory");
  wg_barrier();
  if (threadIdx.x == 0) __hip_atomic_store(cnt, call, __ATOMIC_RELAXED, __HIP_MEMORY_SCOPE_AGENT);

  if (puller) {
    const int w = slot - GSLOTS;
    if (EXPON(2)) return;
    if (xcd < 4) pull<true>(p, w, call, rank, ((rank & 4) ^ 4) | ((a + xcd) & 3), (LAS char*)smem);
    else pull<false>(p, w, call, rank, (rank & 4) | ((a + xcd - 3) & 3), (LAS char*)smem);
    if (wave == 0) STAMP(2);
    return;
  }

  if (EXPON(1)) {} else
  if (plan[0] & 0x4000u) convert_x(p, plan[0] & 255u, call, rank, (LAS char*)smem);
  else if (plan[0] & 0x8000u) convert_w(p, plan[0] & 255u, call, rank, (LAS char*)smem);
  if (wave == 0) STAMP(1);
  __builtin_amdgcn_s_waitcnt(0x0F70);

  for (int job = 0; job < NJOBS; ++job) {
    const unsigned code = job == 0 ? plan[1] : job == 1 ? plan[2] : plan[3];
    if (!(code & 0x8000u)) break;
    if (EXPON(4) && (job > 0 || slot >= 12)) break;
    if (EXPON(8) && job > 0) break;
    Job jb;
    decode_job(code, a, jb);
    const int kind = (code >> 12) & 3;
    if (kind == 3 && (code & 0x200) && wave == 0) STAMP(21 + (code & 63) / 6);
    if (wave >= 4) {
      if (kind & 1) load_wave<1>(p, (LAS char*)smem, jb, job, code, wave - 4, kind < 2, rank, call);
      else load_wave<2>(p, (LAS char*)smem, jb, job, code, wave - 4, kind < 2, rank, call);
    } else if (kind == 0) {
      mfma_wave<true, 2>(p, (LAS char*)smem, jb, job, wave, rank, call);
    } else if (kind == 1) {
      mfma_wave<true, 1>(p, (LAS char*)smem, jb, job, wave, rank, call);
    } else if (kind == 2) {
      mfma_wave<false, 2>(p, (LAS char*)smem, jb, job, wave, rank, call);
    } else {
      mfma_wave<false, 1>(p, (LAS char*)smem, jb, job, wave, rank, call);
    }
  }
}

void make_schedule(Params& p) {
  int n[NGEMM] = {};
  bool early[NGEMM] = {};
  for (int g = 0; g < NGEMM; ++g) p.plan[g] = (u32x4){0u, 0u, 0u, 0u};
  int team[32], nteam = 0;
  auto id = [](int xcd, int s) { return s < GSLOTS ? s * 8 + xcd : GSLOTS * 8 + s - GSLOTS; };
  auto put = [&](int xcd, int s, int kind, int ra, int b, int ah = 0, int bh = 0) {
    const int g = id(xcd, s);
    if (n[g] >= NJOBS || nteam >= 32) return;
    p.plan[g][++n[g]] = 0x8000 | (kind << 12) | (bh << 11) | (ah << 10) | (b << 6) | ra;
    team[nteam++] = g;
  };
  auto close = [&]() {
    for (int side = 0; side < 2; ++side)
      for (int i = 0; i < nteam; ++i) {
        const unsigned mask = side ? 0xbc0u : 0x43fu;
        int cnt = 0, idx = 0;
        for (int k = 0; k < nteam; ++k)
          if (((p.plan[team[i]][n[team[i]]] ^ p.plan[team[k]][n[team[k]]]) & mask) == 0) { idx += k < i; ++cnt; }
        p.plan[team[i]][n[team[i]]] |= (unsigned)((cnt - 1) | (idx << 3)) << (16 + 6 * side);
      }
    nteam = 0;
  };
  auto P = [&](int xcd, int s0, int kind, int t, int b0) {
    for (int s = 0; s < 12; ++s) {
      if (kind < 2) put(xcd, s0 + s, kind, s % 3 * 6 + t, b0 + s / 3 * (2 - kind));
      else if (kind == 2) put(xcd, s0 + s, kind, s % 3 * 6 + t, b0 + s / 6, (s / 3) & 1);
      else put(xcd, s0 + s, kind, s % 3 * 6 + t, b0, (s / 3) & 1, s / 6);
    }
    close();
  };
  auto L = [&](int xcd, int s0, int h, int b0) { for (int s = 0; s < 12; ++s) put(xcd, s0 + s, 0, 18 + h * 3 + s % 3, b0 + s / 3 * 2); close(); };
  auto Afw = [&](int xcd, int s0, int t) {
    for (int s = 0; s < 12; ++s) put(xcd, s0 + s, 0, PA_AWAY + (s & 3) * 2 + t, 8 + (((s & 3) + 1 + (s >> 2)) & 3) * 2);
    close();
  };
  auto Afs = [&](int xcd, int s0, int t, int h) {
    for (int s = h * 12; s < h * 12 + 12; ++s) put(xcd, s0 + s % 12, 1, PA_AWAY + ((s / 6 + 1 + s % 3) & 3) * 2 + t, 8 + s / 3);
    close();
  };
  auto Aow = [&](int xcd, int s0, int t, int c0, int cnt) {
    for (int s = 0; s < cnt; ++s) put(xcd, s0 + s, 0, PA_AWAY + ((c0 + s) & 3) * 2 + t, (c0 + s) / 4 * 2);
    if (c0 + cnt == 16) close();
  };
  auto E = [&](int xcd, int s0) { for (int s = 0; s < 12; ++s) early[id(xcd, s0 + s)] = true; };
#define SCHED 5
  auto Aos = [&](int xcd, int s0, int t, int h) {
    for (int s = h * 12; s < h * 12 + 12; ++s) put(xcd, s0 + s % 12, 1, PA_AWAY + (s & 3) * 2 + t, s / 4);
    close();
  };
  auto OA = [&](int xcd, int s0, int h) { for (int s = h * 8; s < h * 8 + 8; ++s) put(xcd, s0 + s % 8, 1, PA_OWNAWAY + (s & 1), s / 2); close(); };
  if (SCHED == 1) {
    P(0, 0, 0, 0, 0); P(1, 0, 0, 0, 8); L(2, 0, 0, 8); L(3, 0, 1, 8);
    E(0, 0); E(1, 0); E(2, 0); E(3, 0);
    L(0, 12, 0, 0); L(1, 12, 1, 0);
    P(5, 0, 0, 1, 0); P(6, 0, 0, 1, 8);
    OA(2, 12, 0); OA(3, 12, 1);
    P(4, 12, 0, 2, 0); P(5, 12, 0, 2, 8); Afw(6, 12, 0); Aow(7, 12, 0, 0, 16);
    P(0, 0, 1, 3, 8); P(1, 0, 1, 3, 12); Afs(2, 0, 1, 0); Afs(3, 0, 1, 1);
    P(4, 0, 0, 3, 0); Aow(7, 0, 1, 0, 12); Aow(7, 28, 1, 12, 4);
    for (int x = 0; x < 4; ++x) P(x, 12, 2, 4, 8 + 2 * x);
    P(5, 0, 1, 4, 0); P(6, 0, 1, 4, 4);
    for (int x = 0; x < 8; ++x) P(x, x < 4 ? 0 : 12, 3, 5, 8 + x);
    for (int x = 0; x < 4; ++x) P(x, 12, 2, 5, 2 * x);
  } else if (SCHED >= 4) {
    P(0, 0, 0, 0, 0); P(1, 0, 0, 0, 8); L(2, 0, 0, 8); L(3, 0, 1, 8); L(4, 0, 0, 0); L(5, 0, 1, 0);
    for (int x = 0; x < 6; ++x) E(x, 0);
    OA(6, 0, 0); OA(7, 0, 1);
    P(5, 12, 0, 1, 0); P(6, 12, 0, 1, 8);
    P(0, 12, 1, 2, 8); P(1, 12, 1, 2, 12); Afs(2, 12, 0, 0); Afs(3, 12, 0, 1); P(4, 12, 0, 2, 0); Aow(7, 12, 0, 0, 16);
    P(0, 0, 1, 3, 8); P(1, 0, 1, 3, 12); Afs(2, 0, 1, 0); Afs(3, 0, 1, 1);
    P(4, 0, 1, 3, 0); P(5, 0, 1, 3, 4); Aos(6, 0, 1, 0); Aos(7, 0, 1, 1); Aow(7, 28, 1, 12, 4);
    for (int x = 0; x < 4; ++x) P(x, 12, 2, 4, 8 + 2 * x);
    P(5, 12, 1, 4, 0); P(6, 12, 1, 4, 4);
    for (int x = 0; x < 4; ++x) P(x, 12, 3, 5, 4 + x);
    const int h0 = SCHED == 4 ? 5 : 0;
    P(h0, 0, 2, 5, 0); P(h0 + 1, 0, 2, 5, 2);
    int b = 8;
    for (int x = 0; x < 8; ++x) if (x != h0 && x != h0 + 1) P(x, 0, 3, 5, b++);
    P(4, 12, 3, 5, b++); P(7, 12, 3, 5, b++);
  } else {
    P(0, 0, 0, 0, 0); P(1, 0, 0, 0, 8); L(4, 0, 0, 8); L(5, 0, 1, 8);
    E(0, 0); E(1, 0); E(4, 0); E(5, 0);
    if (SCHED == 2) { L(0, 12, 0, 0); L(1, 12, 1, 0); OA(2, 0, 0); OA(3, 0, 1); }
    else { L(2, 0, 0, 0); L(3, 0, 1, 0); E(2, 0); E(3, 0); OA(0, 12, 0); OA(1, 12, 1); }
    P(2, 12, 0, 1, 0); P(3, 12, 0, 1, 8);
    P(4, 12, 0, 2, 0); P(5, 12, 0, 2, 8); Afw(6, 12, 0); Aow(7, 12, 0, 0, 16);
    P(0, 0, 1, 3, 8); P(1, 0, 1, 3, 12); Afs(2, 0, 1, 0); Afs(3, 0, 1, 1);
    P(4, 0, 1, 3, 0); P(5, 0, 1, 3, 4); Aos(6, 0, 1, 0); Aos(7, 0, 1, 1); Aow(7, 28, 1, 12, 4);
    P(0, 12, 1, 4, 8); P(1, 12, 1, 4, 12); P(2, 12, 1, 4, 0); P(3, 12, 1, 4, 4);
    for (int x = 0; x < 4; ++x) P(x, 0, 2, 5, 2 * x);
    for (int x = 0; x < 8; ++x) P(4 + x / 2, (x & 1) * 12, 3, 5, 8 + x);
  }
#define CVT_W 7
  int nx = 0, nw = 0, c = 0;
  for (int g = 0; g < NGEMM; ++g) {
    if (early[g]) continue;
    if ((c++ * CVT_W & 15) < CVT_W) p.plan[g][0] = 0x8000u | nw++;
    else p.plan[g][0] = 0x4000u | nx++;
  }
  p.ncvt_x = (unsigned short)nx;
  p.ncvt_w = (unsigned short)nw;
}

}

extern "C" void kernel_launch(int rank, int nranks, ncclComm_t comm, hipStream_t stream,
                              void* const* d_in, const int* in_sizes, int n_in,
                              void* d_out, int out_size, const int* in_dtypes, int out_dtype,
                              void* const* d_out_all, void* const* d_ws_all, size_t ws_bytes) {
  (void)comm; (void)in_dtypes; (void)out_dtype; (void)d_out;
  if (nranks != NR || (unsigned)rank >= (unsigned)NR || n_in != 2 || in_sizes[0] != NSH * KDIM ||
      in_sizes[1] != KDIM * NSH || out_size != KDIM * NSH || ws_bytes < WS_NEED) {
    fprintf(stderr, "kernel_launch: unexpected problem shape\n");
    return;
  }
  Params p;
  p.x = (const float*)d_in[0];
  p.w = (const float*)d_in[1];
  p.rank = 0;
  for (int i = 0; i < NR; ++i) {
    const int r = (RANK_AT >> (4 * i)) & 7;
    p.out[i] = (float*)d_out_all[r];
    p.ws[i] = (char*)d_ws_all[r];
    if (r == rank) p.rank = i;
  }
  make_schedule(p);
  ag_gemm_silu<<<dim3(NWG), dim3(NTHREADS), 0, stream>>>(p);
  HIP_CHECK(hipGetLastError());
}
```

```cpp
#include <hip/hip_runtime.h>
#include <rccl/rccl.h>
#include <cstdio>
#include <cstdint>

#define HIP_CHECK(expr) do { hipError_t _e = (expr); if (_e != hipSuccess) { \
    fprintf(stderr, "HIP error %s at %s:%d\n", hipGetErrorString(_e), __FILE__, __LINE__); \
    return; } } while (0)

namespace {

typedef _Float16 half8 __attribute__((ext_vector_type(8)));
typedef float f32x4 __attribute__((ext_vector_type(4)));
typedef unsigned u32x4 __attribute__((ext_vector_type(4)));
typedef unsigned u32x2 __attribute__((ext_vector_type(2)));
#define LAS __attribute__((address_space(3)))
#define GAS __attribute__((address_space(1)))

constexpr int NR = 8;
#define RANK_AT_V 0x75416320u
constexpr unsigned RANK_AT = RANK_AT_V;
constexpr int KDIM = 8192;
constexpr int NSH = 1024;
constexpr int KSTEPS = 128;
constexpr unsigned STEP_B = 16384;
constexpr size_t PANEL_B = (size_t)KSTEPS * STEP_B;
constexpr int PA_AWAY = 24, PA_OWNAWAY = 32, PB = 34, PB_FOREIGN = 42, NPANEL = 50;

constexpr unsigned BLK_B = 1792;
constexpr unsigned PSTEP_B = 8 * BLK_B;
constexpr size_t PPANEL_B = (size_t)KSTEPS * PSTEP_B;

constexpr size_t OFF_CALL = 0;
constexpr size_t OFF_TMO = 4096;
constexpr size_t OFF_TICKET = 5120;
#define FCOPY 8
constexpr size_t OFF_FLAG = 1u << 18;
constexpr size_t OFF_RDY = 1u << 20;
constexpr int RDY_N = 1024, RDY_AWAY = 128, RDY_OUT = 384, RDY_HOP1 = 864, RDY_HOP2 = 896;
#define RELAY 12
constexpr size_t OFF_DBG = 65536;
constexpr size_t OFF_OWN = 2u << 20;
constexpr size_t OFF_PACK = OFF_OWN + 32 * PANEL_B;
constexpr size_t OFF_RELAY = OFF_PACK + 32 * PPANEL_B;
constexpr size_t OFF_PULL = OFF_RELAY + PANEL_B;
constexpr int OUT_CHUNKS = 480;
constexpr unsigned CHUNK_B = 8192;
constexpr size_t OFF_STG = OFF_PULL + NPANEL * PANEL_B;
constexpr size_t WS_NEED = OFF_STG + (size_t)4 * OUT_CHUNKS * CHUNK_B;
static_assert(RELAY % 4 == 0 && RELAY <= 64, "");
static_assert(NPANEL <= 64 && OFF_FLAG + FCOPY * 65536 <= OFF_RDY && OFF_RDY + NR * RDY_N * 128 <= OFF_OWN, "");

constexpr int NWG = 256, NTHREADS = 512;
constexpr int GSLOTS = 24;
constexpr int NGEMM = GSLOTS * 8 + 8;
constexpr int PW = 8;
#define WIN 6
constexpr int NJOBS = 3;
constexpr unsigned long long DEADLINE = 100000000ull;

constexpr unsigned LDS_MB = 18 * 8192;
constexpr unsigned LDS_TMO = LDS_MB + NJOBS * 4 * 512;
constexpr unsigned LDS_LUT = LDS_TMO + NJOBS * 16;
constexpr unsigned LDS_BYTES = LDS_LUT + 64;
constexpr unsigned WAVE_LDS = 8192;

struct Params {
  const float* x;
  const float* w;
  float* out[NR];
  char* ws[NR];
  int rank;
  unsigned short ncvt_x, ncvt_w;
  u32x4 plan[NGEMM];
};

#define STAMP(idx) do {} while (0)

#define EXP 0
#define EXPON(bit) ((EXP & (bit)) && call > 2)
__device__ __forceinline__ void store_sys(void* ptr, u32x4 v) {
  asm volatile("global_store_dwordx4 %0, %1, off sc0 sc1\n\ts_nop 1" :: "v"(ptr), "v"(v) : "memory");
}
__device__ __forceinline__ void wait_vm0() { asm volatile("s_waitcnt vmcnt(0)" ::: "memory"); }
__device__ __forceinline__ void wg_barrier() {
  asm volatile("" ::: "memory");
  __builtin_amdgcn_s_barrier();
  asm volatile("" ::: "memory");
}
__device__ __forceinline__ u32x4 cvt8(f32x4 a, f32x4 b) {
  half8 h;
  h[0] = (_Float16)a[0]; h[1] = (_Float16)a[1]; h[2] = (_Float16)a[2]; h[3] = (_Float16)a[3];
  h[4] = (_Float16)b[0]; h[5] = (_Float16)b[1]; h[6] = (_Float16)b[2]; h[7] = (_Float16)b[3];
  return __builtin_bit_cast(u32x4, h);
}
__device__ __forceinline__ f32x4 ld4(const float* ptr) { return __builtin_nontemporal_load((const GAS f32x4*)ptr); }
#define OUT_NT " nt"
__device__ __forceinline__ void store_out(void* ptr, u32x4 v) {
  asm volatile("global_store_dwordx4 %0, %1, off sc0 sc1" OUT_NT "\n\ts_nop 1" :: "v"(ptr), "v"(v) : "memory");
}

__device__ __forceinline__ void set_local(char* ws, int copy, int step, unsigned v) {
  __hip_atomic_store((unsigned*)(ws + OFF_FLAG) + copy * 16384 + (step >> 7) * 256 + (step & 7) * 32 + ((step & 127) >> 3), v,
                     __ATOMIC_RELAXED, __HIP_MEMORY_SCOPE_SYSTEM);
}
__device__ __forceinline__ void set_rdy(char* peer_ws, int rank, int idx, unsigned v) {
  __hip_atomic_store((unsigned*)(peer_ws + OFF_RDY) + (rank * RDY_N + idx) * 32, v, __ATOMIC_RELAXED, __HIP_MEMORY_SCOPE_SYSTEM);
}
__device__ __forceinline__ unsigned own_idx(unsigned call, int idx) { return (call & 1) * 16 + idx; }
__device__ __forceinline__ int fresh_lane() {
  int l;
  asm volatile("v_mbcnt_lo_u32_b32 %0, -1, 0\n\tv_mbcnt_hi_u32_b32 %0, -1, %0" : "=v"(l));
  return l;
}
__device__ __forceinline__ void cbar() { asm volatile("" ::: "memory"); }
__device__ __forceinline__ unsigned wave_scan(unsigned v) {
  v += __builtin_amdgcn_update_dpp(0u, v, 0x111, 0xf, 0xf, false);
  v += __builtin_amdgcn_update_dpp(0u, v, 0x112, 0xf, 0xf, false);
  v += __builtin_amdgcn_update_dpp(0u, v, 0x114, 0xf, 0xf, false);
  v += __builtin_amdgcn_update_dpp(0u, v, 0x118, 0xf, 0xf, false);
  v += __builtin_amdgcn_update_dpp(0u, v, 0x142, 0xa, 0xf, false);
  v += __builtin_amdgcn_update_dpp(0u, v, 0x143, 0xc, 0xf, false);
  return v;
}

__device__ __forceinline__ unsigned codec_lut(int m, bool expand) {
  unsigned sel = 0x0c0c0c0cu;
  int n = 0;
  for (int i = 0; i < 4; ++i)
    if ((m >> i) & 1) {
      const int at = expand ? i : n, v = expand ? n : i;
      sel = (sel & ~(0xffu << (8 * at))) | ((unsigned)v << (8 * at));
      ++n;
    }
  return sel;
}

__device__ __forceinline__ void pack_block(u32x4 f0, u32x4 f1, LAS char* wl, const LAS char* lut, char* dst) {
  const int lane = threadIdx.x & 63;
  const unsigned h[8] = {f0[0], f0[1], f0[2], f0[3], f1[0], f1[1], f1[2], f1[3]};
  if (lane < 36) *(LAS unsigned*)(wl + 640 + lane * 4) = 0u;
  u32x4 D;
  unsigned H[4], sum = 0;
#pragma unroll
  for (int k = 0; k < 4; ++k) {
    D[k] = __builtin_amdgcn_perm(h[2 * k + 1], h[2 * k], 0x06040200u);
    H[k] = __builtin_amdgcn_perm(h[2 * k + 1], h[2 * k], 0x07050301u);
    sum = __builtin_amdgcn_udot4(H[k] & 0x7c7c7c7cu, 0x01010101u, sum, false);
  }
  store_sys(dst + lane * 16, D);
  const unsigned tot = __builtin_amdgcn_readlane(wave_scan(sum), 63);
  const unsigned base = min((tot + 256u) >> 12, 28u);
  const unsigned bB = base * 0x04040404u;
  unsigned t[4], pk[4], n[4], EM = 0, XM = 0, sg = 0;
#pragma unroll
  for (int k = 0; k < 4; ++k) {
    const unsigned g = ((H[k] & 0x7c7c7c7cu) | 0x80808080u) - bB;
    const unsigned a7 = g & 0x7f7f7f7fu, hi = a7 + 0x70707070u, lo = a7 + 0x1c1c1c1cu;
    const unsigned o7 = (~g | hi) & 0x80808080u;
    t[k] = ((g | (o7 >> 4) | (o7 >> 5)) & 0x0c0c0c0cu) | (H[k] & 0x03030303u);
    const unsigned E = t[k] & (t[k] >> 1) & 0x04040404u;
    const unsigned nb = ~g & 0x80808080u;
    const unsigned c4 = g & ((nb >> 2) - (nb >> 5));
    const unsigned X = ((g & hi) | (~g & ~lo)) & 0x80808080u;
    const unsigned sel = *(const LAS unsigned*)(lut + __builtin_amdgcn_udot4(E, 0x08040201u, 0u, false));
    const unsigned z = __builtin_amdgcn_perm(0u, c4, sel);
    pk[k] = __builtin_amdgcn_udot4(z, 0x00400801u, (z >> 24) << 9, false);
    n[k] = __builtin_popcount(E);
    EM |= (E >> 2) << k;
    XM |= X >> (7 - k);
    sg |= (H[k] & 0x80808080u) >> (7 - k);
  }
  const unsigned mine = (n[0] + n[1] + n[2] + n[3]) | (__builtin_popcount(XM) << 16);
  const unsigned incl = wave_scan(mine);
  const unsigned all = __builtin_amdgcn_readlane(incl, 63), n2 = all & 0xffffu, n3 = all >> 16;
  const bool raw = 3u * n2 + 16u * n3 > 1008u;
  *(LAS u32x2*)(wl + lane * 8) = (u32x2){t[0] | (t[1] << 4), t[2] | (t[3] << 4)};
  const unsigned sgn = __builtin_amdgcn_update_dpp(0u, sg, 0xb1, 0xf, 0xf, false);
  if (!(lane & 1)) *(LAS unsigned*)(wl + 512 + lane * 2) = sg | (sgn << 4);
  cbar();
  if (!raw) {
    const unsigned pre = (incl - mine) & 0xffffu;
    unsigned pre3 = (incl - mine) >> 16;
    const unsigned long long W = ((unsigned long long)(pk[0] | (pk[1] << (3 * n[0])))
        | ((unsigned long long)(pk[2] | (pk[3] << (3 * n[2]))) << (3 * (n[0] + n[1])))) >> 2;
    const unsigned o = 3 * pre, sh = o & 31u;
    LAS unsigned* q = (LAS unsigned*)(wl + 640 + (o >> 5) * 4);
    const unsigned long long Ws = W << sh;
    __hip_atomic_fetch_or(q, (unsigned)Ws, __ATOMIC_RELAXED, __HIP_MEMORY_SCOPE_WORKGROUP);
    __hip_atomic_fetch_or(q + 1, (unsigned)(Ws >> 32), __ATOMIC_RELAXED, __HIP_MEMORY_SCOPE_WORKGROUP);
    __hip_atomic_fetch_or(q + 2, ((unsigned)(W >> 32) >> 1) >> (31u - sh), __ATOMIC_RELAXED, __HIP_MEMORY_SCOPE_WORKGROUP);
    cbar();
    while (XM) {
      const unsigned pos = __builtin_ctz(XM), k = pos & 3u, i8 = pos & 24u;
      const unsigned Hk = k & 2u ? (k & 1u ? H[3] : H[2]) : (k & 1u ? H[1] : H[0]);
      const unsigned kb = 0x01010101u << k;
      const unsigned rk = pre + __builtin_popcount(EM & ((kb - 0x01010101u) | (kb & ((1u << i8) - 1u))));
      *(LAS unsigned short*)(wl + 640 + 124 - 2 * pre3) = (unsigned short)((rk << 5) | (((Hk >> (i8 + 2)) & 31u) ^ (base + 3u)));
      ++pre3;
      XM &= XM - 1u;
    }
  }
  cbar();
  if (lane == 0) *(LAS unsigned short*)(wl + 640 + 126) = (unsigned short)(raw ? 0x8000u : base | (n3 << 5));
  cbar();
  if (lane < 48) store_sys(dst + 1024 + lane * 16, *(const LAS u32x4*)(wl + lane * 16));
  cbar();
}

__device__ __forceinline__ bool unpack_block(int lane, const LAS char* blk, LAS char* ex, const LAS char* lut, u32x4& f0, u32x4& f1) {
  const unsigned ent = *(const LAS unsigned short*)(blk + 1790 - 2 * lane);
  const LAS unsigned* sp = (const LAS unsigned*)(blk + 1664 + ((3 * lane) & ~3));
  const unsigned s0 = sp[0], s1 = sp[1];
  const u32x2 N = *(const LAS u32x2*)(blk + 1024 + lane * 8);
  const unsigned sg = *(const LAS unsigned*)(blk + 1536 + (lane >> 1) * 4) >> ((lane & 1) * 4);
  const u32x4 D = *(const LAS u32x4*)(blk + lane * 16);
  const unsigned hdr = __builtin_amdgcn_readfirstlane(ent);
  if (hdr & 0x8000u) return false;
  const unsigned base = hdr & 31u, n3 = hdr >> 5;
  const unsigned b3 = (base + 3u) * 0x04040404u, b8 = (base + 24u) * 0x04040404u;
  const unsigned T0 = (((b8 + 0x0c080400u) ^ b3) & 0x7c7c7c00u), T1 = (((b8 + 0x1c181410u) ^ b3) & 0x7c7c7c7cu);
  const unsigned x = __builtin_amdgcn_alignbyte(s1, s0, 3 * lane);
  u32x2 X;
#pragma unroll
  for (int i = 0; i < 2; ++i) {
    const unsigned y = ((x >> (12 * i)) & 0x3fu) | ((i ? x >> 2 : x << 10) & 0x3f0000u);
    X[i] = __builtin_amdgcn_perm(T1, T0, (y | (y << 5)) & 0x07070707u);
  }
  *(LAS u32x2*)(ex + lane * 8) = X;
  cbar();
  if ((unsigned)(lane - 1) < n3) *(LAS unsigned char*)(ex + ((ent >> 5) & 511u)) = (unsigned char)((ent & 31u) << 2);
  cbar();
  const unsigned E0 = N[0] & (N[0] >> 1) & 0x44444444u, E1 = N[1] & (N[1] >> 1) & 0x44444444u;
  const unsigned mine = __builtin_popcount(E0) + __builtin_popcount(E1);
  unsigned pos = wave_scan(mine) - mine;
  const unsigned bB = base * 0x04040404u;
  unsigned h[8];
#pragma unroll
  for (int k = 0; k < 4; ++k) {
    const unsigned E = ((k & 2 ? E1 : E0) >> (4 * (k & 1))) & 0x04040404u;
    const unsigned sel = *(const LAS unsigned*)(lut + __builtin_amdgcn_udot4(E, 0x08040201u, 0u, false));
    const LAS unsigned* wp = (const LAS unsigned*)(ex + (pos & ~3u));
    const unsigned win = __builtin_amdgcn_alignbyte(wp[1], wp[0], pos);
    pos += __builtin_popcount(E);
    const unsigned t = (N[k >> 1] >> (4 * (k & 1))) & 0x0f0f0f0fu;
    const unsigned H = ((t + bB) ^ __builtin_amdgcn_perm(0u, win, sel)) | ((sg << (7 - k)) & 0x80808080u);
    h[2 * k] = __builtin_amdgcn_perm(H, D[k], 0x05010400u);
    h[2 * k + 1] = __builtin_amdgcn_perm(H, D[k], 0x07030602u);
  }
  cbar();
  f0 = (u32x4){h[0], h[1], h[2], h[3]};
  f1 = (u32x4){h[4], h[5], h[6], h[7]};
  return true;
}

__device__ __forceinline__ int x_unit(int u) {
  if (u < 448) { const int j = u / 7, r = u - j * 7; return r < 2 ? 2 * j + r : (r - 1) * KSTEPS + j; }
  if (u < 768) { const int j = (u - 448) / 5; return (u - 447 - j * 5) * KSTEPS + 64 + j; }
  return u;
}
__device__ __forceinline__ void convert_x(const Params& p, int first, unsigned call, int rank, LAS char* smem) {
  const int lane = threadIdx.x & 63, wave = threadIdx.x >> 6;
  const int count = 8 * KSTEPS, stride = p.ncvt_x;
  char* const ws = p.ws[rank];
  const float* src0 = p.x + (size_t)((wave >> 2) * 64 + (wave & 1) * 32 + (lane & 15)) * KDIM + (wave & 2) * 16 + (lane >> 4) * 4;
  f32x4 v[4];
  int un = x_unit(first);
  {
    const float* s = src0 + (size_t)(un >> 7) * 128 * KDIM + (un & 127) * 64;
#pragma unroll
    for (int i = 0; i < 4; ++i) v[i] = ld4(s + (i >> 1) * 16 * KDIM + (i & 1) * 16);
  }
  for (int it = first; it < count; it += stride) {
    const int u = un, tile = u >> 7, ks = u & 127;
    const u32x4 h0 = cvt8(v[0], v[1]), h1 = cvt8(v[2], v[3]);
    {
      if (it + stride < count) un = x_unit(it + stride);
      const float* s = src0 + (size_t)(un >> 7) * 128 * KDIM + (un & 127) * 64;
#pragma unroll
      for (int i = 0; i < 4; ++i) v[i] = ld4(s + (i >> 1) * 16 * KDIM + (i & 1) * 16);
    }
    char* b = ws + OFF_OWN + own_idx(call, tile) * PANEL_B + ks * STEP_B + wave * 2048 + lane * 16;
    store_sys(b, h0);
    store_sys(b + 1024, h1);
    pack_block(h0, h1, smem + wave * WAVE_LDS, smem + LDS_LUT,
               ws + OFF_PACK + own_idx(call, tile) * PPANEL_B + ks * PSTEP_B + wave * BLK_B);
    wait_vm0();
    wg_barrier();
    if (threadIdx.x < 4 + FCOPY) {
      const int t = threadIdx.x;
      const bool home = tile < 6;
      const int panel = home ? (rank & 3) * 6 + tile : PA_OWNAWAY + tile - 6;
      const int dst = (home ? (rank & 4) : (rank & 4) ^ 4) | (t & 3);
      if (t >= 4) set_local(ws, t - 4, panel * KSTEPS + ks, call);
      else if (tile == 5 && ks < RELAY) { if (t == (ks & 3)) set_rdy(p.ws[(dst ^ 4) & 7], rank, RDY_HOP1 + (ks >> 2), call); }
      else if (dst != rank) set_rdy(p.ws[dst & 7], rank, home ? u : RDY_AWAY + u - 6 * KSTEPS, call);
    }
  }
}

__device__ __forceinline__ void convert_w(const Params& p, int first, unsigned call, int rank, LAS char* smem) {
  const int lane = threadIdx.x & 63, wave = threadIdx.x >> 6;
  const int count = KSTEPS * 4, stride = p.ncvt_w;
  char* const ws = p.ws[rank];
  const float* src0 = p.w + (size_t)((wave & 1) * 32 + (lane >> 4) * 4) * NSH + (wave >> 1) * 64 + (lane & 15) * 4;
  f32x4 v[8];
  int u = first;
  {
    const float* s = src0 + (size_t)(u >> 2) * 64 * NSH + (u & 3) * 256;
#pragma unroll
    for (int j = 0; j < 8; ++j) v[j] = ld4(s + ((j & 3) + (j >> 2) * 16) * NSH);
  }
  for (; u < count; u += stride) {
    const int ks = u >> 2, q = u & 3;
    u32x4 h[4];
#pragma unroll
    for (int f = 0; f < 4; ++f)
      h[f] = cvt8((f32x4){v[0][f], v[1][f], v[2][f], v[3][f]}, (f32x4){v[4][f], v[5][f], v[6][f], v[7][f]});
    {
      const int un = u + stride < count ? u + stride : u;
      const float* s = src0 + (size_t)(un >> 2) * 64 * NSH + (un & 3) * 256;
#pragma unroll
      for (int j = 0; j < 8; ++j) v[j] = ld4(s + ((j & 3) + (j >> 2) * 16) * NSH);
    }
    const unsigned oi = own_idx(call, 8 + q * 2 + (wave >> 2));
    char* b = ws + OFF_OWN + oi * PANEL_B + ks * STEP_B + (wave & 3) * 4096 + lane * 16;
#pragma unroll
    for (int f = 0; f < 4; ++f) store_sys(b + f * 1024, h[f]);
    char* pb = ws + OFF_PACK + oi * PPANEL_B + ks * PSTEP_B + (wave & 3) * 2 * BLK_B;
    pack_block(h[0], h[1], smem + wave * WAVE_LDS, smem + LDS_LUT, pb);
    pack_block(h[2], h[3], smem + wave * WAVE_LDS + 1024, smem + LDS_LUT, pb + BLK_B);
    wait_vm0();
    wg_barrier();
    if (threadIdx.x <= 2 * FCOPY) {
      const int t = threadIdx.x;
      if (t < 2 * FCOPY) set_local(ws, t >> 1, (PB + q * 2 + (t & 1)) * KSTEPS + ks, call);
      else set_rdy(p.ws[(((rank & 4) ^ 4) | q) & 7], rank, ks, call);
    }
  }
}

__device__ __forceinline__ int row_of(int r) { return ((RANK_AT >> (4 * r)) & 7) * NSH; }

#define ORDER_V 0x7648953210ull
constexpr unsigned long long BATCH_AT = ORDER_V;
constexpr unsigned long long batch_pos() {
  unsigned long long r = 0;
  for (int i = 0; i < 10; ++i) r |= (unsigned long long)i << (4 * ((BATCH_AT >> (4 * i)) & 15));
  return r;
}
constexpr unsigned long long BATCH_POS = batch_pos();
__device__ __forceinline__ int batch_row(int s, int o, int pos, int i) {
  const int bt = (BATCH_AT >> (4 * pos)) & 15;
  if (bt < 2) return row_of(s) + (bt * 3 + i) * 128;
  const int g = bt < 8 ? s : o;
  return row_of((g & 4) | ((g + 1 + i) & 3)) + (bt - 2) * 128;
}

__device__ __forceinline__ int unit_of(int it, int w) { return it * PW + ((w + it) & (PW - 1)); }

__device__ __forceinline__ unsigned lds_get(const LAS unsigned* c) {
  return __builtin_amdgcn_readfirstlane(__hip_atomic_load(c, __ATOMIC_RELAXED, __HIP_MEMORY_SCOPE_WORKGROUP));
}
__device__ __forceinline__ unsigned lds_ticket(int lane, LAS unsigned* c) {
  unsigned t = 0;
  if (lane == 0) t = __hip_atomic_fetch_add(c, 1u, __ATOMIC_RELAXED, __HIP_MEMORY_SCOPE_WORKGROUP);
  return __builtin_amdgcn_readfirstlane(t);
}

constexpr int RING = 8;
constexpr unsigned PL_EX = RING * PSTEP_B;
constexpr int W_TICKET = 0, W_READY = 1, W_DEAD = 2, W_DONE = 8, W_LANDED = 64, W_QUEUE = 128;
constexpr unsigned Q_N = 128, Q_OUT = 1024, Q_LAST = 1u << 31;
static_assert(PL_EX + 8 * 512 <= LDS_MB && (W_QUEUE + Q_N) * 4 <= LDS_TMO - LDS_MB, "");

template <int N>
__device__ __forceinline__ void stream_kib(const char* src, unsigned dst, unsigned& cnt, unsigned word) {
  unsigned keep;
#define SK(OFF) "global_load_lds_dwordx4 %2, off offset:" #OFF " sc0 sc1\n\tv_add_u32 %1, 1, %1\n\t" \
                "s_waitcnt vmcnt(%5)\n\tds_max_u32 %4, %1\n\t"
  if (N == 4)
    asm volatile("s_mov_b32 %0, m0\n\ts_mov_b32 m0, %3\n\ts_nop 0\n\t" SK(0) SK(1024) SK(2048) SK(3072) "s_mov_b32 m0, %0"
                 : "=&s"(keep), "+v"(cnt) : "v"(src), "s"(dst), "v"(word), "n"(WIN - 1) : "memory");
  else
    asm volatile("s_mov_b32 %0, m0\n\ts_mov_b32 m0, %3\n\ts_nop 0\n\t" SK(0) SK(1024) "s_mov_b32 m0, %0"
                 : "=&s"(keep), "+v"(cnt) : "v"(src), "s"(dst), "v"(word), "n"(WIN - 1) : "memory");
}
template <int K>
__device__ __forceinline__ void stream_drain(unsigned cnt, unsigned word) {
  if constexpr (K >= 0) {
    asm volatile("s_waitcnt vmcnt(%2)\n\tds_max_u32 %1, %0" :: "v"(cnt + 1), "v"(word), "n"(K) : "memory");
    stream_drain<K - 1>(cnt + 1, word);
  }
}

struct Unit { unsigned src, raw, dst; int from, flag; };
template <bool CROSS>
__device__ __forceinline__ Unit input_unit(int u, unsigned call, int rank, int peer) {
  constexpr int H1 = RELAY / 4, RSTEP = 32 * KSTEPS;
  const unsigned par = own_idx(call, 0) * KSTEPS;
  Unit o;
  o.from = peer;
  if (!CROSS) {
    const int v = u < 5 * KSTEPS ? u : u + RELAY;
    o.raw = par + v; o.dst = (peer & 3) * 6 * KSTEPS + v; o.flag = v;
  } else if (u < 256) {
    const int v = (u & 1) * KSTEPS + (u >> 1);
    o.raw = par + (8 + (rank & 3) * 2) * KSTEPS + v; o.dst = (PB_FOREIGN + (peer & 3) * 2) * KSTEPS + v; o.flag = u >> 1;
  } else if (u < 256 + H1) {
    o.raw = par + 5 * KSTEPS + (u - 256) * 4 + (rank & 3); o.dst = ~((peer & 3) * 16 + u - 256); o.flag = RDY_HOP1 + u - 256;
  } else if (u < 512 + H1) {
    const int v = u - 256 - H1;
    o.raw = par + 6 * KSTEPS + v; o.dst = (PA_AWAY + (peer & 3) * 2) * KSTEPS + v; o.flag = RDY_AWAY + v;
  } else {
    const int v = u - 512 - H1, j = v / 3, s = (rank + 1 + v - j * 3) & 3, ks = 5 * KSTEPS + j * 4 + (peer & 3);
    o.from = (rank & 4) | s;
    o.raw = par + ks; o.dst = s * 6 * KSTEPS + ks; o.flag = RDY_HOP2 + s * 16 + j;
    o.src = RSTEP + s * 16 + j;
    return o;
  }
  o.src = o.raw;
  return o;
}

template <bool CROSS>
__device__ __forceinline__ void pull(const Params& p, int w, int wave, unsigned call, int rank, int peer, LAS char* smem) {
  constexpr int NIN = CROSS ? 512 + RELAY : 768 - RELAY, NOUT = CROSS ? OUT_CHUNKS / PW : 0;
  const int lane = fresh_lane();
  char* const ws = p.ws[rank];
  const char* const pws = p.ws[peer];
  LAS unsigned* const sh = (LAS unsigned*)(smem + LDS_MB);
  float* const out = p.out[rank] + (size_t)((lane >> 4) * 4) * NSH + (peer & 3) * 256 + (lane & 15) * 4;
  auto where = [&](int c, int pc) {
    const int tile = c >> 3, bt = tile / 6, r = tile - bt * 6;
    return out + (size_t)(batch_row(peer, rank, bt, r >> 1) + ((c >> 2) & 1) * 64 + (c & 1) * 32 + (pc & 4) * 4 + (pc & 3)) * NSH
         + (r & 1) * 128 + ((c >> 1) & 1) * 64;
  };
  if (wave == 6) {
    const unsigned* const rdy = (const unsigned*)(ws + OFF_RDY) + peer * RDY_N * 32;
    unsigned* const tmo = (unsigned*)(ws + OFF_TMO);
    unsigned* const ticket = (unsigned*)(ws + OFF_TICKET) + peer * 32;
    unsigned long long t0 = __builtin_amdgcn_s_memrealtime();
    bool dead = false;
    unsigned k = 0, kib = 0, spin = 0;
    auto unset = [&](bool look, int fl) {
      unsigned v = call;
      if (look && !dead)
        asm volatile("global_load_dword %0, %1, off sc0 sc1\n\ts_waitcnt vmcnt(0)" : "=v"(v) : "v"(rdy + fl * 32) : "memory");
      return __builtin_amdgcn_ballot_w64(v < call);
    };
    auto idle = [&]() {
      if ((++spin & 63u) == 0u
          && (__builtin_amdgcn_readfirstlane(__hip_atomic_load(tmo, __ATOMIC_RELAXED, __HIP_MEMORY_SCOPE_AGENT)) == call
              || __builtin_amdgcn_s_memrealtime() - t0 > DEADLINE)) {
        if (lane == 0) {
          __hip_atomic_store(tmo, call, __ATOMIC_RELAXED, __HIP_MEMORY_SCOPE_AGENT);
          __hip_atomic_store(sh + W_DEAD, 1u, __ATOMIC_RELAXED, __HIP_MEMORY_SCOPE_WORKGROUP);
        }
        wait_vm0();
        dead = true;
      }
      __builtin_amdgcn_s_sleep(1);
    };
    auto publish = [&](unsigned v, unsigned near) {
      cbar();
      if (lane == 0) __hip_atomic_store(sh + W_READY, v, __ATOMIC_RELAXED, __HIP_MEMORY_SCOPE_WORKGROUP);
      while (lds_get(sh + W_LANDED) < near) __builtin_amdgcn_s_sleep(2);
      t0 = __builtin_amdgcn_s_memrealtime();
    };
    int known = EXPON(1) ? NIN : 0;
    auto look = [&](int from, int width) {
      const unsigned long long miss = unset(lane < width, input_unit<CROSS>(min(from + lane, NIN - 1), call, rank, peer).flag);
      known = from + (miss ? (int)__builtin_ctzll(miss) : width);
    };
    for (int u = w; u < NIN;) {
      while (u >= known) { look(u, 16); if (u >= known) idle(); }
      if (lane == 0) sh[W_QUEUE + (k & (Q_N - 1))] = kib << 11 | u;
      publish(++k, kib);
      kib += 14;
      unsigned tk = 0;
      if (lane == 0) tk = __hip_atomic_fetch_add(ticket, 1u, __ATOMIC_RELAXED, __HIP_MEMORY_SCOPE_AGENT);
      if (known < NIN && known - u < 32) look(known, 64);
      u = min(__builtin_amdgcn_readfirstlane(tk) - (call - 1u) * NIN + PW, (unsigned)NIN);
    }
    const unsigned long long all = (1ull << NOUT) - 1ull;
    for (unsigned long long left = all; left;) {
      const unsigned long long fresh = left & ~unset(!EXPON(17) && ((left >> lane) & 1), RDY_OUT + unit_of(lane, w));
      if (!fresh) { idle(); continue; }
      const unsigned pos = __builtin_amdgcn_mbcnt_hi(fresh >> 32, __builtin_amdgcn_mbcnt_lo(fresh, 0));
      if ((fresh >> lane) & 1) sh[W_QUEUE + ((k + pos) & (Q_N - 1))] = (kib + pos * 8) << 11 | (Q_OUT + unit_of(lane, w));
      k += __builtin_popcountll(fresh);
      kib += __builtin_popcountll(fresh) * 8;
      left &= ~fresh;
      if (left) publish(k, kib - 16);
    }
    publish(k | Q_LAST, 0);
    if (CROSS) {
      dead = __builtin_amdgcn_readfirstlane(__hip_atomic_load(tmo, __ATOMIC_RELAXED, __HIP_MEMORY_SCOPE_AGENT)) == call;
      wg_barrier();
      if (dead) store_sys(where(unit_of(0, w), 0), (u32x4){0x7fc00000u, 0x7fc00000u, 0x7fc00000u, 0x7fc00000u});
    }
    return;
  }
  if (wave == 7) {
    const unsigned lds0 = (unsigned)(uintptr_t)smem, word = (unsigned)(uintptr_t)(sh + W_LANDED + lane);
    unsigned cnt = 0;
    for (unsigned k = 0;; ++k) {
      unsigned ready = lds_get(sh + W_READY);
      if (k >= (ready & ~Q_LAST) || lds_get(sh + W_DONE + (k & (RING - 1))) < (k & ~(RING - 1u))) {
        stream_drain<WIN - 2>(cnt, word);
        while (k >= ready) { __builtin_amdgcn_s_sleep(1); ready = lds_get(sh + W_READY); }
        if (k >= (ready & ~Q_LAST)) break;
        while (lds_get(sh + W_DONE + (k & (RING - 1))) < (k & ~(RING - 1u))) __builtin_amdgcn_s_sleep(1);
      }
      cbar();
      const unsigned u = lds_get(sh + W_QUEUE + (k & (Q_N - 1))) & 2047u, d = lds0 + (k & (RING - 1)) * PSTEP_B;
      if (u < Q_OUT) {
        if ((u & 63) >= 56) STAMP(4 + (u >> 6));
        const char* s = pws + OFF_PACK + (size_t)input_unit<CROSS>(u, call, rank, peer).src * PSTEP_B + lane * 16;
        stream_kib<4>(s, d, cnt, word);
        stream_kib<4>(s + 4096, d + 4096, cnt, word);
        stream_kib<4>(s + 8192, d + 8192, cnt, word);
        stream_kib<2>(s + 12288, d + 12288, cnt, word);
        if (k == 0) STAMP(3);
      } else {
        if ((u - Q_OUT) % 48 >= 40) STAMP(20 + (u - Q_OUT) / 48);
        const char* s = pws + OFF_STG + (size_t)((rank & 3) * OUT_CHUNKS + u - Q_OUT) * CHUNK_B + lane * 16;
        stream_kib<4>(s, d, cnt, word);
        stream_kib<4>(s + 4096, d + 4096, cnt, word);
      }
    }
    STAMP(1);
    if (CROSS) wg_barrier();
    return;
  }
  static_assert(RING == 8, "");
  auto landed = [&](unsigned t, unsigned& u) {
    const unsigned ready = lds_get(sh + W_READY);
    cbar();
    const unsigned q = lds_get(sh + W_QUEUE + ((t >> 3) & (Q_N - 1))), n = lds_get(sh + W_LANDED);
    u = q & 2047u;
    if ((t >> 3) >= (ready & ~Q_LAST)) { u = ~0u; return ready >= Q_LAST; }
    return n >= (q >> 11) + (u < Q_OUT ? ((t & 7) * 7 + 10) >> 2 : (t & 7) + 1) + WIN - 1;
  };
  const unsigned none = ~0u;
  unsigned prev = none, pdst = 0;
  auto done = [&]() {
    const bool last = (lds_ticket(lane, sh + W_DONE + ((prev >> 3) & (RING - 1))) & 7u) == 7u;
    prev = none;
    if (!last) return;
    if (!CROSS && (pdst & 127) >= 112) STAMP(17 + ((~pdst >> 3) & 1));
    if (!CROSS || (int)pdst >= 0) { if (lane < FCOPY) set_local(ws, lane, pdst, call); }
    else if (lane < 3) set_rdy(p.ws[((peer & 4) | ((peer + 1 + lane) & 3)) & 7], rank, RDY_HOP2 + ~pdst, call);
  };
  for (;;) {
    unsigned u;
    if (prev != none && !landed(lds_get(sh + W_TICKET), u)) { wait_vm0(); done(); }
    const unsigned t = lds_ticket(lane, sh + W_TICKET);
    if (!landed(t, u)) {
      if (prev != none) { wait_vm0(); done(); }
      while (!landed(t, u)) __builtin_amdgcn_s_sleep(1);
    }
    cbar();
    const LAS char* slot = smem + ((t >> 3) & (RING - 1)) * PSTEP_B;
    if (u >= Q_OUT) {
      if (prev != none) { wait_vm0(); done(); }
      if (!CROSS || u == none) break;
      u32x4 v = *(const LAS u32x4*)(slot + (t & 7) * 1024 + lane * 16);
      if (lds_get(sh + W_DEAD)) v = (u32x4){0x7fc00000u, 0x7fc00000u, 0x7fc00000u, 0x7fc00000u};
      store_out(where(u - Q_OUT, t & 7), v);
      if (lane == 0) __hip_atomic_fetch_add(sh + W_DONE + ((t >> 3) & (RING - 1)), 1u, __ATOMIC_RELAXED, __HIP_MEMORY_SCOPE_WORKGROUP);
      continue;
    }
    const Unit un = input_unit<CROSS>(u, call, rank, peer);
    const LAS char* blk = slot + (t & 7) * BLK_B;
    u32x4 f0, f1;
    char* d;
    if (CROSS && RELAY && (int)un.dst < 0) {
      const int l1 = min(lane, 47);
      f0 = *(const LAS u32x4*)(blk + lane * 16);
      f1 = *(const LAS u32x4*)(blk + 1024 + l1 * 16);
      d = ws + OFF_RELAY + (size_t)(~un.dst * 8 + (t & 7)) * BLK_B;
      store_sys(d + lane * 16, f0);
      store_sys(d + 1024 + l1 * 16, f1);
    } else {
      if (!unpack_block(lane, blk, smem + PL_EX + wave * 512, smem + LDS_LUT, f0, f1)) {
        const char* r = p.ws[un.from & 7] + OFF_OWN + (size_t)(un.raw * 8 + (t & 7)) * 2048 + lane * 16;
        asm volatile("global_load_dwordx4 %0, %2, off sc0 sc1\n\t"
                     "global_load_dwordx4 %1, %2, off offset:1024 sc0 sc1\n\ts_waitcnt vmcnt(0)"
                     : "=&v"(f0), "=&v"(f1) : "v"(r) : "memory");
      }
      d = ws + OFF_PULL + (size_t)(un.dst * 8 + (t & 7)) * 2048 + lane * 16;
      store_sys(d, f0);
      store_sys(d + 1024, f1);
    }
    asm volatile("s_waitcnt vmcnt(2)" ::: "memory");
    if (prev != none) done();
    prev = t;
    pdst = un.dst;
  }
  if (CROSS) { wait_vm0(); wg_barrier(); }
}

struct Job { int pa, pb, ah, bh, tile; };
__device__ __forceinline__ void decode_job(unsigned code, int a, Job& o) {
  const int ra = code & 63;
  int bt, i;
  o.pb = PB + ((code >> 6) & 15);
  o.ah = (code >> 10) & 1;
  o.bh = (code >> 11) & 1;
  if (ra < 18) {
    i = ra / 6; bt = 2 + ra % 6; o.pa = ((a + 1 + i) & 3) * 6 + bt - 2;
  } else if (ra < 24) {
    o.pa = a * 6 + ra - 18; bt = (ra - 18) / 3; i = (ra - 18) % 3;
  } else {
    o.pa = ra; bt = 8 + (ra & 1); i = (((ra - PA_AWAY) >> 1) - ((o.pb - PB_FOREIGN) >> 1) - 1) & 3;
  }
  o.tile = (int)((BATCH_POS >> (4 * bt)) & 15) * 3 + i;
}

__device__ __forceinline__ float silu(float y) {
  const float c = fminf(fmaxf(y, -60.0f), 60.0f);
  return y * __builtin_amdgcn_rcpf(1.0f + __builtin_amdgcn_exp2f(c * -1.44269504088896341f));
}

#define POLL_EVERY 1
#define POLL_CAD 2
#define POLL_SLEEP 2
#define GLDS_BITS " sc0 sc1"

template <int NB>
__device__ __forceinline__ unsigned next_stage(unsigned so) { return so == LDS_MB - (NB + 1) * 8192 ? 0u : so + (NB + 1) * 8192; }

template <bool TALL, int NB>
__device__ __forceinline__ void mfma_wave(const Params& p, LAS char* smem, const Job& jb, int job, int wave,
                                          int rank, unsigned call) {
  constexpr int MT = TALL ? 4 : NB, NT = TALL && NB == 2 ? 8 : 4, KH = TALL ? 1 : 2, STEPS = 2 * KSTEPS / KH;
  constexpr int NRA = MT * KH, NRB = NT * KH, NRD = NRA + NRB, NMM = MT * NT * KH;
  int lane = fresh_lane();
  const int wr = MT == 1 ? wave : wave >> 1, wc = MT == 1 ? 0 : wave & 1;
  f32x4 acc[MT][NT];
#pragma unroll
  for (int m = 0; m < MT; ++m)
#pragma unroll
    for (int n = 0; n < NT; ++n) acc[m][n] = (f32x4){0.f, 0.f, 0.f, 0.f};
  half8 fa[2][NRA], fb[2][NRB];
  const LAS char* la = smem + wr * MT * 1024 + lane * 16;
  const LAS char* lb = smem + 8192 + wc * NRB * 1024 + lane * 16;
  unsigned so = 0;
#define READ_SET(X) do { \
    _Pragma("unroll") for (int i = 0; i < NRB; ++i) { \
      if (i < NRA) fa[X][i] = *(const LAS half8*)(la + so + (i / MT) * 4096 + (i % MT) * 1024); \
      fb[X][i] = *(const LAS half8*)(lb + so + i * 1024); } \
    so = next_stage<NB>(so); } while (0)
#define MMA_SET(X) do { \
    _Pragma("unroll") for (int kh = 0; kh < KH; ++kh) \
    _Pragma("unroll") for (int m = 0; m < MT; ++m) \
    _Pragma("unroll") for (int n = 0; n < NT; ++n) \
      acc[m][n] = __builtin_amdgcn_mfma_f32_16x16x32_f16(fa[X][kh * MT + m], fb[X][kh * NT + n], acc[m][n], 0, 0, 0); } while (0)
  constexpr int RG = TALL ? 1 : 2, MG = NB, G = NRD / RG;
#define INTERLEAVE() do { \
    _Pragma("unroll") for (int i = 0; i < G; ++i) { \
      __builtin_amdgcn_sched_group_barrier(0x100, RG, 0); \
      __builtin_amdgcn_sched_group_barrier(0x008, MG, 0); } \
    __builtin_amdgcn_sched_group_barrier(0x008, NMM - G * MG, 0); } while (0)
#define STEP_END() do { __builtin_amdgcn_sched_barrier(0); asm volatile("" ::: "memory"); \
    __builtin_amdgcn_s_waitcnt(0xC07F);   __builtin_amdgcn_sched_barrier(0); \
    wg_barrier(); __builtin_amdgcn_sched_barrier(0); } while (0)
  wg_barrier();
  if (wave == 0) STAMP(4 + job * 4);
  READ_SET(0);
  STEP_END();
  for (int t = 1; t < STEPS - 1; t += 2) {
    READ_SET(1); MMA_SET(0); INTERLEAVE();
    STEP_END();
    READ_SET(0); MMA_SET(1); INTERLEAVE();
    STEP_END();
  }
  READ_SET(1); MMA_SET(0); INTERLEAVE();
  const unsigned tm = *(const LAS unsigned*)(smem + LDS_TMO + job * 16);
  STEP_END();
  MMA_SET(1);
  __builtin_amdgcn_sched_barrier(0);
  if (wave == 0) STAMP(5 + job * 4);
  lane = fresh_lane();
  asm volatile("" : "+s"(call));
  const bool dead = tm == call;

  const bool foreign = jb.pb >= PB_FOREIGN;
  const int ob = (jb.pb - PB_FOREIGN) >> 1;
  const int r16 = jb.ah * 4 + wr * MT;
  const int c64 = (jb.pb & 1) * 2 + jb.bh + wc * (NT / 4);
  const int chunk = (jb.tile * 2 + (c64 >> 1)) * 8 + (r16 & 4) + (c64 & 1) * 2 + ((r16 >> 1) & 1);
  char* o;
  unsigned sm, sr, sh;
  if (foreign) {
    o = p.ws[rank] + OFF_STG + (size_t)(ob * OUT_CHUNKS + chunk) * CHUNK_B + (r16 & 1) * 4096 + lane * 16;
    sm = 4096; sr = 1024; sh = 2 * CHUNK_B;
  } else {
    const int pa = jb.pa;
    const int row0 = pa < PA_AWAY ? row_of((rank & 4) + pa / 6) + (pa % 6) * 128
                   : row_of(pa < PA_OWNAWAY ? ((rank & 4) ^ 4) + ((pa - PA_AWAY) >> 1) : rank) + 768 + (pa & 1) * 128;
    o = (char*)(p.out[rank] + (size_t)(row0 + r16 * 16 + (lane >> 4) * 4) * NSH + ((jb.pb - PB) >> 1) * 256 + c64 * 64 + (lane & 15) * 4);
    sm = 16 * NSH * 4; sr = NSH * 4; sh = 256;
  }
  const float qnan = __builtin_nanf("");
#pragma unroll
  for (int h = 0; h < NT / 4; ++h)
#pragma unroll
    for (int m = 0; m < MT; ++m)
#pragma unroll
      for (int r = 0; r < 4; ++r) {
        f32x4 v = {silu(acc[m][h * 4][r]), silu(acc[m][h * 4 + 1][r]), silu(acc[m][h * 4 + 2][r]), silu(acc[m][h * 4 + 3][r])};
        if (dead) v = (f32x4){qnan, qnan, qnan, qnan};
        if (foreign) store_sys(o + h * sh + m * sm + r * sr, __builtin_bit_cast(u32x4, v));
        else store_out(o + h * sh + m * sm + r * sr, __builtin_bit_cast(u32x4, v));
      }
  if (foreign) {
    wait_vm0();
    bool last = lane < (MT * NT + 7) / 8;
    if (MT == 1)
      last = last && __hip_atomic_fetch_add((LAS unsigned*)(smem + LDS_TMO) + job * 4 + 2 + (wave >> 1), 1u,
                                            __ATOMIC_RELAXED, __HIP_MEMORY_SCOPE_WORKGROUP) == 1u;
    if (last) set_rdy(p.ws[(((rank & 4) ^ 4) | ob) & 7], rank, RDY_OUT + chunk + lane, call);
  }
  if (wave == 0) STAMP(6 + job * 4);
}

#define NEAR 4
template <int NB>
__device__ __forceinline__ void load_wave(const Params& p, LAS char* smem, const Job& jb, int job, unsigned code, int l,
                                          bool tall, int rank, unsigned call) {
  constexpr int DEPTH = NB == 2 ? 5 : 7;
  const int steps = tall ? 2 * KSTEPS : KSTEPS, ksh = tall ? 1 : 0;
  if (l > NB && l < 3) {
    for (int t = 0; t <= steps; ++t) wg_barrier();
    return;
  }
  const int lane = fresh_lane(), a = rank & 3;
  char* const ws = p.ws[rank];
  const unsigned lds0 = (unsigned)(uintptr_t)smem;
  LAS unsigned* const rdyw = (LAS unsigned*)(smem + LDS_TMO) + job * 4 + 1;
  const unsigned gap = tall ? 8192 : 4096;
  auto panel_of = [&](int pc) { return pc == 0 ? jb.pa : tall ? jb.pb + pc - 1 : jb.pb; };
  auto source = [&](int pc) {
    const int panel = panel_of(pc);
    const int own = panel >= PB_FOREIGN ? -1 : panel >= PA_OWNAWAY ? panel - PA_OWNAWAY + 6
                  : (unsigned)(panel - a * 6) < 6u ? panel - a * 6 : -1;
    return ws + (own >= 0 ? OFF_OWN + own_idx(call, own) * PANEL_B : OFF_PULL + (size_t)panel * PANEL_B)
         + (tall ? 0 : (pc == 0 ? jb.ah : jb.bh + pc - 1) * 8192);
  };
  int ready = EXPON(1) ? KSTEPS : 0;
  if (l == 3) {
    unsigned* const tmo = (unsigned*)(ws + OFF_TMO);
    const char* fsrc = ws + OFF_FLAG + (blockIdx.x & (FCOPY - 1)) * 65536 + panel_of(lane >> 5) * 1024 + ((lane >> 2) & 7) * 128 + (lane & 3) * 16;
    const unsigned mb = lds0 + LDS_MB + job * 2048;
    int ra = ready, rb = ready;
    auto poll = [&]() {
      if ((lane < 32 ? ra : rb) < KSTEPS) {
        unsigned keep;
        asm volatile("s_mov_b32 %0, m0\n\ts_mov_b32 m0, %2\n\ts_nop 0\n\t"
                     "global_load_lds_dwordx4 %1, off sc0 sc1\n\ts_mov_b32 m0, %0"
                     : "=&s"(keep) : "v"(fsrc), "s"(mb) : "memory");
        if (NB == 2 && tall && lane >= 32)
          asm volatile("s_mov_b32 %0, m0\n\ts_mov_b32 m0, %2\n\ts_nop 0\n\t"
                       "global_load_lds_dwordx4 %1, off offset:1024 sc0 sc1\n\ts_mov_b32 m0, %0"
                       : "=&s"(keep) : "v"(fsrc), "s"(mb - 512) : "memory");
      }
    };
    auto refresh = [&]() {
      u32x2 f, g, h;
      asm volatile("ds_read2_b32 %0, %3 offset1:8\n\tds_read2_b32 %1, %3 offset0:128 offset1:136\n\t"
                   "ds_read2_b32 %2, %4 offset1:8\n\ts_waitcnt lgkmcnt(0)"
                   : "=&v"(f), "=&v"(g), "=&v"(h)
                   : "v"(mb + (lane & 7) * 64 + (lane >> 3) * 4), "v"(mb + (NB == 2 && tall ? 1024 : 512) + (lane & 7) * 64 + (lane >> 3) * 4)
                   : "memory");
      const unsigned long long a0 = __builtin_amdgcn_ballot_w64(f[0] < call), a1 = __builtin_amdgcn_ballot_w64(f[1] < call);
      const unsigned long long b0 = __builtin_amdgcn_ballot_w64(min(g[0], h[0]) < call), b1 = __builtin_amdgcn_ballot_w64(min(g[1], h[1]) < call);
      ra = a0 ? __builtin_ctzll(a0) : a1 ? 64 + __builtin_ctzll(a1) : KSTEPS;
      rb = b0 ? __builtin_ctzll(b0) : b1 ? 64 + __builtin_ctzll(b1) : KSTEPS;
      ready = min(ra, rb);
      if (lane == 0) __hip_atomic_store(rdyw, (unsigned)ready, __ATOMIC_RELAXED, __HIP_MEMORY_SCOPE_WORKGROUP);
    };
    auto advance = [&]() {
      const int k = min(ready + lane, KSTEPS - 1);
      u32x2 f;
      unsigned h;
      asm volatile("ds_read2_b32 %0, %2 offset1:128\n\tds_read_b32 %1, %3\n\ts_waitcnt lgkmcnt(0)"
                   : "=&v"(f), "=&v"(h)
                   : "v"(mb + (k & 7) * 64 + (k >> 3) * 4), "v"(mb + (NB == 2 && tall ? 1024 : 512) + (k & 7) * 64 + (k >> 3) * 4)
                   : "memory");
      const unsigned long long m = __builtin_amdgcn_ballot_w64(min(min(f[0], f[1]), h) < call);
      ready = min(ready + (m ? (int)__builtin_ctzll(m) : 64), KSTEPS);
      __hip_atomic_store(rdyw, (unsigned)ready, __ATOMIC_RELAXED, __HIP_MEMORY_SCOPE_WORKGROUP);
    };
    for (int s = 0; s < steps; ++s) {
      if (s >= DEPTH) wg_barrier();
      const int ks = s >> ksh;
      if (ready < KSTEPS && ready - ks < NEAR) {
        advance();
        if (ks >= ready) {
          const unsigned long long t0 = __builtin_amdgcn_s_memrealtime();
          for (unsigned it = 1;; ++it) {
            if (it >= 32u || s == 0) { poll(); wait_vm0(); refresh(); }
            else { if ((it & (POLL_CAD - 1)) == 1u) poll(); advance(); }
            if (ks < ready) break;
            if ((it & 63u) == 0u) {
              if (__builtin_amdgcn_readfirstlane(__hip_atomic_load(tmo, __ATOMIC_RELAXED, __HIP_MEMORY_SCOPE_AGENT)) == call
                  || __builtin_amdgcn_s_memrealtime() - t0 > DEADLINE) {
                if (lane == 0) __hip_atomic_store(tmo, call, __ATOMIC_RELAXED, __HIP_MEMORY_SCOPE_AGENT);
                wait_vm0();
                ready = KSTEPS;
                if (lane == 0) __hip_atomic_store(rdyw, (unsigned)ready, __ATOMIC_RELAXED, __HIP_MEMORY_SCOPE_WORKGROUP);
                break;
              }
            }
            if (it >= 32u) __builtin_amdgcn_s_sleep(POLL_SLEEP); else __builtin_amdgcn_s_sleep(1);
          }
        } else if (!(s & (((POLL_EVERY) << ksh) - 1))) {
          poll();
        }
      }
    }
    if (lane == 0) {
      unsigned keep;
      asm volatile("s_mov_b32 %0, m0\n\ts_mov_b32 m0, %2\n\ts_nop 0\n\t"
                   "global_load_lds_dword %1, off sc1\n\ts_mov_b32 m0, %0"
                   : "=&s"(keep) : "v"(tmo), "s"(lds0 + LDS_TMO + job * 16) : "memory");
    }
    for (int t = 1; t < DEPTH; ++t) wg_barrier();
    wait_vm0();
    wg_barrier();
    wg_barrier();
    return;
  }
  const char* src = source(l) + lane * 16;
  const unsigned dst0 = lds0 + l * 8192;
  unsigned so = 0;
  for (int s = 0; s < steps; ++s) {
    if (s >= DEPTH) {
      asm volatile("s_waitcnt vmcnt(%0)" :: "n"(8 * (DEPTH - 1)) : "memory");
      wg_barrier();
    }
    const int ks = s >> ksh;
    if (ks >= ready)
      while ((ready = lds_get(rdyw)) <= ks) __builtin_amdgcn_s_sleep(1);
    const char* g = src + (size_t)ks * STEP_B + (s & ksh) * 4096;
    const unsigned d = dst0 + so;
    so = next_stage<NB>(so);
    unsigned keep;
#define GLDS4(R) "global_load_lds_dwordx4 " R ", off" GLDS_BITS "\n\t" \
                 "global_load_lds_dwordx4 " R ", off offset:1024" GLDS_BITS "\n\t" \
                 "global_load_lds_dwordx4 " R ", off offset:2048" GLDS_BITS "\n\t" \
                 "global_load_lds_dwordx4 " R ", off offset:3072" GLDS_BITS "\n\t"
    asm volatile("s_mov_b32 %0, m0\n\ts_mov_b32 m0, %3\n\ts_nop 0\n\t" GLDS4("%1")
                 "s_mov_b32 m0, %4\n\ts_nop 0\n\t" GLDS4("%2") "s_mov_b32 m0, %0"
                 : "=&s"(keep) : "v"(g), "v"(g + gap), "s"(d), "s"(d + 4096) : "memory");
  }
#define WB(N) "s_waitcnt vmcnt(" #N ")\n\ts_barrier\n\t"
  if (NB == 1) asm volatile(WB(48) WB(40) ::: "memory");
  asm volatile(WB(32) WB(24) WB(16) WB(8) WB(0) "s_barrier" ::: "memory");
}

__global__ void __launch_bounds__(NTHREADS, 2) ag_gemm_silu(const Params p) {
  __shared__ __attribute__((aligned(16))) char smem[LDS_BYTES];
  const int rank = p.rank & 7, a = rank & 3;
  const int wave = __builtin_amdgcn_readfirstlane(threadIdx.x >> 6);
#define XROT 4
  const int xcd = (blockIdx.x - XROT) & 7, slot = blockIdx.x >> 3;
  char* const ws = p.ws[rank];
  const u32x4 plan = p.plan[slot < GSLOTS ? slot * 8 + xcd : GSLOTS * 8 + slot - GSLOTS];

  unsigned* const cnt = (unsigned*)(ws + OFF_CALL) + blockIdx.x;
  const unsigned call = __builtin_amdgcn_readfirstlane(
      __hip_atomic_load(cnt, __ATOMIC_RELAXED, __HIP_MEMORY_SCOPE_AGENT)) + 1u;
  if (wave == 0) STAMP(0);
  const bool puller = slot >= GSLOTS && xcd < 7;
  for (unsigned i = threadIdx.x; i < (LDS_LUT - LDS_MB) / 4; i += NTHREADS) ((LAS unsigned*)(smem + LDS_MB))[i] = 0u;
  if (threadIdx.x < 16) ((LAS unsigned*)(smem + LDS_LUT))[threadIdx.x] = codec_lut(threadIdx.x, puller);
  asm volatile("s_waitcnt vmcnt(0) lgkmcnt(0)" ::: "memory");
  wg_barrier();
  if (threadIdx.x == 0) __hip_atomic_store(cnt, call, __ATOMIC_RELAXED, __HIP_MEMORY_SCOPE_AGENT);

  if (puller) {
    const int w = slot - GSLOTS;
    if (EXPON(2)) return;
    if (xcd < 4) pull<true>(p, w, wave, call, rank, ((rank & 4) ^ 4) | ((a + xcd) & 3), (LAS char*)smem);
    else pull<false>(p, w, wave, call, rank, (rank & 4) | ((a + xcd - 3) & 3), (LAS char*)smem);
    if (wave == 0) STAMP(2);
    return;
  }

  if (EXPON(1)) {} else
  if (plan[0] & 0x4000u) convert_x(p, plan[0] & 255u, call, rank, (LAS char*)smem);
  else if (plan[0] & 0x8000u) convert_w(p, plan[0] & 255u, call, rank, (LAS char*)smem);
  if (wave == 0) STAMP(1);
  __builtin_amdgcn_s_waitcnt(0x0F70);

  for (int job = 0; job < NJOBS; ++job) {
    const unsigned code = job == 0 ? plan[1] : job == 1 ? plan[2] : plan[3];
    if (!(code & 0x8000u)) break;
    if (EXPON(4) && (job > 0 || slot >= 12)) break;
    if (EXPON(8) && job > 0) break;
    Job jb;
    decode_job(code, a, jb);
    const int kind = (code >> 12) & 3;
    if (kind == 3 && (code & 0x200) && wave == 0) STAMP(21 + (code & 63) / 6);
    if (wave >= 4) {
      if (kind & 1) load_wave<1>(p, (LAS char*)smem, jb, job, code, wave - 4, kind < 2, rank, call);
      else load_wave<2>(p, (LAS char*)smem, jb, job, code, wave - 4, kind < 2, rank, call);
    } else if (kind == 0) {
      mfma_wave<true, 2>(p, (LAS char*)smem, jb, job, wave, rank, call);
    } else if (kind == 1) {
      mfma_wave<true, 1>(p, (LAS char*)smem, jb, job, wave, rank, call);
    } else if (kind == 2) {
      mfma_wave<false, 2>(p, (LAS char*)smem, jb, job, wave, rank, call);
    } else {
      mfma_wave<false, 1>(p, (LAS char*)smem, jb, job, wave, rank, call);
    }
  }
}

void make_schedule(Params& p) {
  int n[NGEMM] = {};
  bool early[NGEMM] = {};
  for (int g = 0; g < NGEMM; ++g) p.plan[g] = (u32x4){0u, 0u, 0u, 0u};
  int team[32], nteam = 0;
  auto id = [](int xcd, int s) { return s < GSLOTS ? s * 8 + xcd : GSLOTS * 8 + s - GSLOTS; };
  auto put = [&](int xcd, int s, int kind, int ra, int b, int ah = 0, int bh = 0) {
    const int g = id(xcd, s);
    if (n[g] >= NJOBS || nteam >= 32) return;
    p.plan[g][++n[g]] = 0x8000 | (kind << 12) | (bh << 11) | (ah << 10) | (b << 6) | ra;
    team[nteam++] = g;
  };
  auto close = [&]() {
    for (int side = 0; side < 2; ++side)
      for (int i = 0; i < nteam; ++i) {
        const unsigned mask = side ? 0xbc0u : 0x43fu;
        int cnt = 0, idx = 0;
        for (int k = 0; k < nteam; ++k)
          if (((p.plan[team[i]][n[team[i]]] ^ p.plan[team[k]][n[team[k]]]) & mask) == 0) { idx += k < i; ++cnt; }
        p.plan[team[i]][n[team[i]]] |= (unsigned)((cnt - 1) | (idx << 3)) << (16 + 6 * side);
      }
    nteam = 0;
  };
  auto P = [&](int xcd, int s0, int kind, int t, int b0) {
    for (int s = 0; s < 12; ++s) {
      if (kind < 2) put(xcd, s0 + s, kind, s % 3 * 6 + t, b0 + s / 3 * (2 - kind));
      else if (kind == 2) put(xcd, s0 + s, kind, s % 3 * 6 + t, b0 + s / 6, (s / 3) & 1);
      else put(xcd, s0 + s, kind, s % 3 * 6 + t, b0, (s / 3) & 1, s / 6);
    }
    close();
  };
  auto L = [&](int xcd, int s0, int h, int b0) { for (int s = 0; s < 12; ++s) put(xcd, s0 + s, 0, 18 + h * 3 + s % 3, b0 + s / 3 * 2); close(); };
  auto Afw = [&](int xcd, int s0, int t) {
    for (int s = 0; s < 12; ++s) put(xcd, s0 + s, 0, PA_AWAY + (s & 3) * 2 + t, 8 + (((s & 3) + 1 + (s >> 2)) & 3) * 2);
    close();
  };
  auto Afs = [&](int xcd, int s0, int t, int h) {
    for (int s = h * 12; s < h * 12 + 12; ++s) put(xcd, s0 + s % 12, 1, PA_AWAY + ((s / 6 + 1 + s % 3) & 3) * 2 + t, 8 + s / 3);
    close();
  };
  auto Aow = [&](int xcd, int s0, int t, int c0, int cnt) {
    for (int s = 0; s < cnt; ++s) put(xcd, s0 + s, 0, PA_AWAY + ((c0 + s) & 3) * 2 + t, (c0 + s) / 4 * 2);
    if (c0 + cnt == 16) close();
  };
  auto E = [&](int xcd, int s0) { for (int s = 0; s < 12; ++s) early[id(xcd, s0 + s)] = true; };
#define SCHED 5
  auto Aos = [&](int xcd, int s0, int t, int h) {
    for (int s = h * 12; s < h * 12 + 12; ++s) put(xcd, s0 + s % 12, 1, PA_AWAY + (s & 3) * 2 + t, s / 4);
    close();
  };
  auto OA = [&](int xcd, int s0, int h) { for (int s = h * 8; s < h * 8 + 8; ++s) put(xcd, s0 + s % 8, 1, PA_OWNAWAY + (s & 1), s / 2); close(); };
  if (SCHED == 1) {
    P(0, 0, 0, 0, 0); P(1, 0, 0, 0, 8); L(2, 0, 0, 8); L(3, 0, 1, 8);
    E(0, 0); E(1, 0); E(2, 0); E(3, 0);
    L(0, 12, 0, 0); L(1, 12, 1, 0);
    P(5, 0, 0, 1, 0); P(6, 0, 0, 1, 8);
    OA(2, 12, 0); OA(3, 12, 1);
    P(4, 12, 0, 2, 0); P(5, 12, 0, 2, 8); Afw(6, 12, 0); Aow(7, 12, 0, 0, 16);
    P(0, 0, 1, 3, 8); P(1, 0, 1, 3, 12); Afs(2, 0, 1, 0); Afs(3, 0, 1, 1);
    P(4, 0, 0, 3, 0); Aow(7, 0, 1, 0, 12); Aow(7, 28, 1, 12, 4);
    for (int x = 0; x < 4; ++x) P(x, 12, 2, 4, 8 + 2 * x);
    P(5, 0, 1, 4, 0); P(6, 0, 1, 4, 4);
    for (int x = 0; x < 8; ++x) P(x, x < 4 ? 0 : 12, 3, 5, 8 + x);
    for (int x = 0; x < 4; ++x) P(x, 12, 2, 5, 2 * x);
  } else if (SCHED >= 4) {
    P(0, 0, 0, 0, 0); P(1, 0, 0, 0, 8); L(2, 0, 0, 8); L(3, 0, 1, 8); L(4, 0, 0, 0); L(5, 0, 1, 0);
    for (int x = 0; x < 6; ++x) E(x, 0);
    OA(6, 0, 0); OA(7, 0, 1);
    P(5, 12, 0, 1, 0); P(6, 12, 0, 1, 8);
    P(0, 12, 1, 2, 8); P(1, 12, 1, 2, 12); Afs(2, 12, 0, 0); Afs(3, 12, 0, 1); P(4, 12, 0, 2, 0); Aow(7, 12, 0, 0, 16);
    P(0, 0, 1, 3, 8); P(1, 0, 1, 3, 12); Afs(2, 0, 1, 0); Afs(3, 0, 1, 1);
    P(4, 0, 1, 3, 0); P(5, 0, 1, 3, 4); Aos(6, 0, 1, 0); Aos(7, 0, 1, 1); Aow(7, 28, 1, 12, 4);
    for (int x = 0; x < 4; ++x) P(x, 12, 2, 4, 8 + 2 * x);
    P(5, 12, 1, 4, 0); P(6, 12, 1, 4, 4);
    for (int x = 0; x < 4; ++x) P(x, 12, 3, 5, 4 + x);
    const int h0 = SCHED == 4 ? 5 : 0;
    P(h0, 0, 2, 5, 0); P(h0 + 1, 0, 2, 5, 2);
    int b = 8;
    for (int x = 0; x < 8; ++x) if (x != h0 && x != h0 + 1) P(x, 0, 3, 5, b++);
    P(4, 12, 3, 5, b++); P(7, 12, 3, 5, b++);
  } else {
    P(0, 0, 0, 0, 0); P(1, 0, 0, 0, 8); L(4, 0, 0, 8); L(5, 0, 1, 8);
    E(0, 0); E(1, 0); E(4, 0); E(5, 0);
    if (SCHED == 2) { L(0, 12, 0, 0); L(1, 12, 1, 0); OA(2, 0, 0); OA(3, 0, 1); }
    else { L(2, 0, 0, 0); L(3, 0, 1, 0); E(2, 0); E(3, 0); OA(0, 12, 0); OA(1, 12, 1); }
    P(2, 12, 0, 1, 0); P(3, 12, 0, 1, 8);
    P(4, 12, 0, 2, 0); P(5, 12, 0, 2, 8); Afw(6, 12, 0); Aow(7, 12, 0, 0, 16);
    P(0, 0, 1, 3, 8); P(1, 0, 1, 3, 12); Afs(2, 0, 1, 0); Afs(3, 0, 1, 1);
    P(4, 0, 1, 3, 0); P(5, 0, 1, 3, 4); Aos(6, 0, 1, 0); Aos(7, 0, 1, 1); Aow(7, 28, 1, 12, 4);
    P(0, 12, 1, 4, 8); P(1, 12, 1, 4, 12); P(2, 12, 1, 4, 0); P(3, 12, 1, 4, 4);
    for (int x = 0; x < 4; ++x) P(x, 0, 2, 5, 2 * x);
    for (int x = 0; x < 8; ++x) P(4 + x / 2, (x & 1) * 12, 3, 5, 8 + x);
  }
#define CVT_W 7
  int nx = 0, nw = 0, c = 0;
  for (int g = 0; g < NGEMM; ++g) {
    if (early[g]) continue;
    if ((c++ * CVT_W & 15) < CVT_W) p.plan[g][0] = 0x8000u | nw++;
    else p.plan[g][0] = 0x4000u | nx++;
  }
  p.ncvt_x = (unsigned short)nx;
  p.ncvt_w = (unsigned short)nw;
}

}

extern "C" void kernel_launch(int rank, int nranks, ncclComm_t comm, hipStream_t stream,
                              void* const* d_in, const int* in_sizes, int n_in,
                              void* d_out, int out_size, const int* in_dtypes, int out_dtype,
                              void* const* d_out_all, void* const* d_ws_all, size_t ws_bytes) {
  (void)comm; (void)in_dtypes; (void)out_dtype; (void)d_out;
  if (nranks != NR || (unsigned)rank >= (unsigned)NR || n_in != 2 || in_sizes[0] != NSH * KDIM ||
      in_sizes[1] != KDIM * NSH || out_size != KDIM * NSH || ws_bytes < WS_NEED) {
    fprintf(stderr, "kernel_launch: unexpected problem shape\n");
    return;
  }
  Params p;
  p.x = (const float*)d_in[0];
  p.w = (const float*)d_in[1];
  p.rank = 0;
  for (int i = 0; i < NR; ++i) {
    const int r = (RANK_AT >> (4 * i)) & 7;
    p.out[i] = (float*)d_out_all[r];
    p.ws[i] = (char*)d_ws_all[r];
    if (r == rank) p.rank = i;
  }
  make_schedule(p);
  ag_gemm_silu<<<dim3(NWG), dim3(NTHREADS), 0, stream>>>(p);
  HIP_CHECK(hipGetLastError());
}
```

```cpp
#include <hip/hip_runtime.h>
#include <rccl/rccl.h>
#include <cstdio>
#include <cstdint>
#include <cstddef>

#define HIP_CHECK(expr) do { hipError_t _e = (expr); if (_e != hipSuccess) { \
    fprintf(stderr, "HIP error %s at %s:%d\n", hipGetErrorString(_e), __FILE__, __LINE__); \
    return; } } while (0)

namespace {

typedef _Float16 half8 __attribute__((ext_vector_type(8)));
typedef float f32x4 __attribute__((ext_vector_type(4)));
typedef unsigned u32x4 __attribute__((ext_vector_type(4)));
typedef unsigned u32x2 __attribute__((ext_vector_type(2)));
#define LAS __attribute__((address_space(3)))
#define GAS __attribute__((address_space(1)))

constexpr int NR = 8;
#define RANK_AT_V 0x75416320u
constexpr unsigned RANK_AT = RANK_AT_V;
constexpr int KDIM = 8192;
constexpr int NSH = 1024;
constexpr int KSTEPS = 128;
constexpr unsigned STEP_B = 16384;
constexpr size_t PANEL_B = (size_t)KSTEPS * STEP_B;
constexpr int PA_AWAY = 24, PA_OWNAWAY = 32, PB = 34, PB_FOREIGN = 42, NPANEL = 50;

constexpr unsigned BLK_B = 1792;
constexpr unsigned PSTEP_B = 8 * BLK_B;
constexpr size_t PPANEL_B = (size_t)KSTEPS * PSTEP_B;

constexpr size_t OFF_CALL = 0;
constexpr size_t OFF_TMO = 4096;
constexpr size_t OFF_TICKET = 5120;
#define FCOPY 8
constexpr size_t OFF_FLAG = 1u << 18;
constexpr size_t OFF_RDY = 1u << 20;
constexpr int RDY_N = 1024, RDY_AWAY = 128, RDY_OUT = 384, RDY_HOP1 = 864, RDY_HOP2 = 896;
#define RELAY 32
constexpr size_t OFF_DBG = 65536;
constexpr size_t OFF_OWN = 2u << 20;
constexpr size_t OFF_PACK = OFF_OWN + 32 * PANEL_B;
constexpr size_t OFF_RELAY = OFF_PACK + 32 * PPANEL_B;
constexpr size_t OFF_PULL = OFF_RELAY + PANEL_B;
#define HX 20
#define HW 10
constexpr int OUT_CHUNKS = 480;
#define CSTRIDE 8192
constexpr unsigned SUM_B = 3456, CHUNK_B = CSTRIDE;
constexpr size_t OFF_STG = OFF_PULL + 2 * NPANEL * PANEL_B;
constexpr size_t OFF_SPARE = OFF_STG + (size_t)4 * OUT_CHUNKS * CHUNK_B;
constexpr size_t WS_NEED = OFF_SPARE + (size_t)4 * OUT_CHUNKS * 2048;
static_assert(RELAY % 4 == 0 && RELAY <= 64, "");
static_assert(NPANEL <= 64 && OFF_FLAG + FCOPY * 65536 <= OFF_RDY && OFF_RDY + NR * RDY_N * 128 <= OFF_OWN, "");

constexpr int NWG = 256, NTHREADS = 512;
constexpr int GSLOTS = 24;
constexpr int NGEMM = GSLOTS * 8 + 8;
constexpr int PW = 8;
#define WIN 5
#define WIN1 6
#define WMASK 0
constexpr int NJOBS = 3;
constexpr unsigned long long DEADLINE = 100000000ull;

constexpr unsigned LDS_MB = 18 * 8192;
constexpr unsigned LDS_TMO = LDS_MB + NJOBS * 4 * 512;
constexpr unsigned LDS_LUT = LDS_TMO + NJOBS * 16;
constexpr unsigned LDS_EPI = LDS_LUT + 64;
constexpr unsigned LDS_BYTES = LDS_EPI + 4 * 512;
constexpr unsigned WAVE_LDS = 8192;

struct Params {
  const float* x;
  const float* w;
  char* mine;
  int rank;
  unsigned char ncvt_x, nlong_x, ncvt_w;
  float* out;
  char* ws[NR];
  u32x4 plan[NGEMM];
};

__device__ __forceinline__ int xcc_id() {
  return __builtin_amdgcn_s_getreg(20 | (3 << 11)) & 7;
}
#define STAMP(idx) do {} while (0)

#define EXP 0
#define EXPON(bit) ((EXP & (bit)) && call > 2)
__device__ __forceinline__ void store_sys(void* ptr, u32x4 v) {
  asm volatile("global_store_dwordx4 %0, %1, off sc0 sc1\n\ts_nop 1" :: "v"(ptr), "v"(v) : "memory");
}
__device__ __forceinline__ void wait_vm0() { asm volatile("s_waitcnt vmcnt(0)" ::: "memory"); }
__device__ __forceinline__ void wg_barrier() {
  asm volatile("" ::: "memory");
  __builtin_amdgcn_s_barrier();
  asm volatile("" ::: "memory");
}
__device__ __forceinline__ u32x4 cvt8(f32x4 a, f32x4 b) {
  half8 h;
  h[0] = (_Float16)a[0]; h[1] = (_Float16)a[1]; h[2] = (_Float16)a[2]; h[3] = (_Float16)a[3];
  h[4] = (_Float16)b[0]; h[5] = (_Float16)b[1]; h[6] = (_Float16)b[2]; h[7] = (_Float16)b[3];
  return __builtin_bit_cast(u32x4, h);
}
__device__ __forceinline__ f32x4 ld4(const float* ptr) { return __builtin_nontemporal_load((const GAS f32x4*)ptr); }
#define OUT_NT " nt"
__device__ __forceinline__ void store_out(void* ptr, u32x4 v) {
  asm volatile("global_store_dwordx4 %0, %1, off sc0 sc1" OUT_NT "\n\ts_nop 1" :: "v"(ptr), "v"(v) : "memory");
}

__device__ __forceinline__ void set_local(char* ws, int copy, int step, unsigned v) {
  __hip_atomic_store((unsigned*)(ws + OFF_FLAG) + copy * 16384 + (step >> 7) * 256 + (step & 7) * 32 + ((step & 127) >> 3), v,
                     __ATOMIC_RELAXED, __HIP_MEMORY_SCOPE_SYSTEM);
}
__device__ __forceinline__ void set_rdy(char* peer_ws, int rank, int idx, unsigned v) {
  __hip_atomic_store((unsigned*)(peer_ws + OFF_RDY) + (rank * RDY_N + idx) * 32, v, __ATOMIC_RELAXED, __HIP_MEMORY_SCOPE_SYSTEM);
}
__device__ __forceinline__ unsigned own_idx(unsigned call, int idx) { return (call & 1) * 16 + idx; }
__device__ __forceinline__ size_t pulled(unsigned call) { return OFF_PULL + (call & 1) * (NPANEL * PANEL_B); }
__device__ __forceinline__ int fresh_lane() {
  int l;
  asm volatile("v_mbcnt_lo_u32_b32 %0, -1, 0\n\tv_mbcnt_hi_u32_b32 %0, -1, %0" : "=v"(l));
  return l;
}
__device__ __forceinline__ void cbar() { asm volatile("" ::: "memory"); }
__device__ __forceinline__ unsigned wave_scan(unsigned v) {
  v += __builtin_amdgcn_update_dpp(0u, v, 0x111, 0xf, 0xf, false);
  v += __builtin_amdgcn_update_dpp(0u, v, 0x112, 0xf, 0xf, false);
  v += __builtin_amdgcn_update_dpp(0u, v, 0x114, 0xf, 0xf, false);
  v += __builtin_amdgcn_update_dpp(0u, v, 0x118, 0xf, 0xf, false);
  v += __builtin_amdgcn_update_dpp(0u, v, 0x142, 0xa, 0xf, false);
  v += __builtin_amdgcn_update_dpp(0u, v, 0x143, 0xc, 0xf, false);
  return v;
}

__device__ __forceinline__ unsigned codec_lut(int m, bool expand) {
  unsigned sel = 0x0c0c0c0cu;
  int n = 0;
  for (int i = 0; i < 4; ++i)
    if ((m >> i) & 1) {
      const int at = expand ? i : n, v = expand ? n : i;
      sel = (sel & ~(0xffu << (8 * at))) | ((unsigned)v << (8 * at));
      ++n;
    }
  return sel;
}

__device__ __forceinline__ bool code_exponents(int lane, const unsigned (&H)[4], unsigned base, unsigned head, bool bad,
                                               LAS char* esc, const LAS char* lut, unsigned (&t)[4]) {
  const unsigned bB = base * 0x04040404u;
  unsigned pk[4], n[4], EM = 0, XM = 0;
#pragma unroll
  for (int k = 0; k < 4; ++k) {
    const unsigned g = ((H[k] & 0x7c7c7c7cu) | 0x80808080u) - bB;
    const unsigned a7 = g & 0x7f7f7f7fu, hi = a7 + 0x70707070u, lo = a7 + 0x1c1c1c1cu;
    const unsigned o7 = (~g | hi) & 0x80808080u;
    t[k] = (g | (o7 >> 4) | (o7 >> 5)) & 0x0c0c0c0cu;
    const unsigned E = t[k] & (t[k] >> 1);
    const unsigned nb = ~g & 0x80808080u;
    const unsigned c4 = g & ((nb >> 2) - (nb >> 5));
    const unsigned X = ((g & hi) | (~g & ~lo)) & 0x80808080u;
    const unsigned sel = *(const LAS unsigned*)(lut + __builtin_amdgcn_udot4(E, 0x08040201u, 0u, false));
    const unsigned z = __builtin_amdgcn_perm(0u, c4, sel);
    pk[k] = __builtin_amdgcn_udot4(z, 0x00400801u, (z >> 24) << 9, false);
    n[k] = __builtin_popcount(E);
    EM |= (E >> 2) << k;
    XM |= X >> (7 - k);
  }
  const unsigned mine = (n[0] + n[1] + n[2] + n[3]) | (__builtin_popcount(XM) << 16);
  const unsigned incl = wave_scan(mine);
  const unsigned all = __builtin_amdgcn_readlane(incl, 63), n2 = all & 0xffffu, n3 = all >> 16;
  const bool raw = bad || 3u * n2 + 16u * n3 > 1008u;
  cbar();
  if (!raw) {
    const unsigned pre = (incl - mine) & 0xffffu;
    unsigned pre3 = (incl - mine) >> 16;
    const unsigned long long W = ((unsigned long long)(pk[0] | (pk[1] << (3 * n[0])))
        | ((unsigned long long)(pk[2] | (pk[3] << (3 * n[2]))) << (3 * (n[0] + n[1])))) >> 2;
    const unsigned o = 3 * pre, sh = o & 31u;
    LAS unsigned* q = (LAS unsigned*)(esc + (o >> 5) * 4);
    const unsigned long long Ws = W << sh;
    __hip_atomic_fetch_or(q, (unsigned)Ws, __ATOMIC_RELAXED, __HIP_MEMORY_SCOPE_WORKGROUP);
    __hip_atomic_fetch_or(q + 1, (unsigned)(Ws >> 32), __ATOMIC_RELAXED, __HIP_MEMORY_SCOPE_WORKGROUP);
    __hip_atomic_fetch_or(q + 2, ((unsigned)(W >> 32) >> 1) >> (31u - sh), __ATOMIC_RELAXED, __HIP_MEMORY_SCOPE_WORKGROUP);
    cbar();
    const unsigned H0 = H[0], H1 = H[1], H2 = H[2], H3 = H[3];
    while (XM) {
      const unsigned pos = __builtin_ctz(XM), k = pos & 3u, i8 = pos & 24u;
      const unsigned Hk = k & 2u ? (k & 1u ? H3 : H2) : (k & 1u ? H1 : H0);
      const unsigned kb = 0x01010101u << k;
      const unsigned rk = pre + __builtin_popcount(EM & ((kb - 0x01010101u) | (kb & ((1u << i8) - 1u))));
      *(LAS unsigned short*)(esc + 124 - 2 * pre3) = (unsigned short)((rk << 5) | (((Hk >> (i8 + 2)) & 31u) ^ (base + 3u)));
      ++pre3;
      XM &= XM - 1u;
    }
  }
  cbar();
  if (lane == 0) *(LAS unsigned short*)(esc + 126) = (unsigned short)(raw ? 0x8000u : head | (n3 << 5));
  cbar();
  return raw;
}

__device__ __forceinline__ void pack_block(u32x4 f0, u32x4 f1, LAS char* wl, const LAS char* lut, char* dst) {
  const int lane = threadIdx.x & 63;
  const unsigned h[8] = {f0[0], f0[1], f0[2], f0[3], f1[0], f1[1], f1[2], f1[3]};
  if (lane < 36) *(LAS unsigned*)(wl + 640 + lane * 4) = 0u;
  u32x4 D;
  unsigned H[4], t[4], sum = 0, sg = 0;
#pragma unroll
  for (int k = 0; k < 4; ++k) {
    D[k] = __builtin_amdgcn_perm(h[2 * k + 1], h[2 * k], 0x06040200u);
    H[k] = __builtin_amdgcn_perm(h[2 * k + 1], h[2 * k], 0x07050301u);
    sum = __builtin_amdgcn_udot4(H[k] & 0x7c7c7c7cu, 0x01010101u, sum, false);
    sg |= (H[k] & 0x80808080u) >> (7 - k);
  }
  store_sys(dst + lane * 16, D);
  const unsigned sgn = __builtin_amdgcn_update_dpp(0u, sg, 0xb1, 0xf, 0xf, false);
  if (!(lane & 1)) *(LAS unsigned*)(wl + 512 + lane * 2) = sg | (sgn << 4);
  const unsigned tot = __builtin_amdgcn_readlane(wave_scan(sum), 63);
  const unsigned base = min((tot + 256u) >> 12, 28u);
  code_exponents(lane, H, base, base, false, wl + 640, lut, t);
#pragma unroll
  for (int k = 0; k < 4; ++k) t[k] |= H[k] & 0x03030303u;
  *(LAS u32x2*)(wl + lane * 8) = (u32x2){t[0] | (t[1] << 4), t[2] | (t[3] << 4)};
  cbar();
  if (lane < 48) store_sys(dst + 1024 + lane * 16, *(const LAS u32x4*)(wl + lane * 16));
  cbar();
}

constexpr unsigned SUM_BASE = 24;
__device__ __forceinline__ void pack_sums(int lane, const f32x4 (&a)[4], LAS char* wl, const LAS char* lut, char* dst, char* spare) {
  if (lane < 36) *(LAS unsigned*)(wl + 256 + lane * 4) = 0u;
  u32x4 P[3], E;
  unsigned H[4], t[4], sum = 0, any = 0;
#pragma unroll
  for (int k = 0; k < 4; ++k) {
    unsigned v[4];
#pragma unroll
    for (int i = 0; i < 4; ++i) v[i] = __builtin_rotateleft32(__builtin_bit_cast(u32x4, a[i])[k], 1);
#pragma unroll
    for (int i = 0; i < 3; ++i) P[i][k] = __builtin_amdgcn_perm(v[3], v[i], 0x04020100u + ((unsigned)i << 24));
    E[k] = __builtin_amdgcn_perm(v[1], v[0], 0x0c0c0703u) | __builtin_amdgcn_perm(v[3], v[2], 0x07030c0cu);
    sum = __builtin_amdgcn_udot4(E[k], 0x01010101u, sum, false);
  }
#pragma unroll
  for (int i = 0; i < 3; ++i) store_sys(dst + i * 1024 + lane * 16, P[i]);
  const unsigned tot = __builtin_amdgcn_readlane(wave_scan(sum), 63);
  const unsigned off = min(max((tot + 64u) >> 10, SUM_BASE) - SUM_BASE, 223u);
#pragma unroll
  for (int k = 0; k < 4; ++k) {
    const unsigned d = E[k] - off * 0x01010101u;
    any |= d;
    H[k] = d << 2;
  }
  const bool bad = __builtin_amdgcn_ballot_w64((any & 0xe0e0e0e0u) != 0u) != 0ull;
  const bool raw = code_exponents(lane, H, SUM_BASE, (off & 31u) | (off >> 5 << 11), bad, wl + 256, lut, t);
  *(LAS unsigned*)(wl + lane * 4) = (t[0] >> 2) | t[1] | (t[2] << 2) | (t[3] << 4);
  cbar();
  if (lane < 24) store_sys(dst + 3072 + lane * 16, *(const LAS u32x4*)(wl + lane * 16));
  if (raw) store_sys(spare + lane * 16, E);
  cbar();
}

struct Lists { unsigned ent, s0, s1; };
__device__ __forceinline__ Lists read_lists(int lane, const LAS char* esc) {
  const LAS unsigned* sp = (const LAS unsigned*)(esc + ((3 * lane) & ~3));
  return {*(const LAS unsigned short*)(esc + 126 - 2 * lane), sp[0], sp[1]};
}
__device__ __forceinline__ void decode_exponents(int lane, const Lists& l, unsigned base, unsigned n3, unsigned mine,
                                                 const unsigned (&E)[4], LAS char* ex, const LAS char* lut, unsigned (&t)[4]) {
  const unsigned b3 = (base + 3u) * 0x04040404u, b8 = (base + 24u) * 0x04040404u;
  const unsigned T0 = (((b8 + 0x0c080400u) ^ b3) & 0x7c7c7c00u), T1 = (((b8 + 0x1c181410u) ^ b3) & 0x7c7c7c7cu);
  const unsigned x = __builtin_amdgcn_alignbyte(l.s1, l.s0, 3 * lane);
  u32x2 X;
#pragma unroll
  for (int i = 0; i < 2; ++i) {
    const unsigned y = ((x >> (12 * i)) & 0x3fu) | ((i ? x >> 2 : x << 10) & 0x3f0000u);
    X[i] = __builtin_amdgcn_perm(T1, T0, (y | (y << 5)) & 0x07070707u);
  }
  *(LAS u32x2*)(ex + lane * 8) = X;
  cbar();
  if ((unsigned)(lane - 1) < n3) *(LAS unsigned char*)(ex + ((l.ent >> 5) & 511u)) = (unsigned char)((l.ent & 31u) << 2);
  cbar();
  unsigned pos = wave_scan(mine) - mine;
  const unsigned bB = base * 0x04040404u;
#pragma unroll
  for (int k = 0; k < 4; ++k) {
    const unsigned sel = *(const LAS unsigned*)(lut + __builtin_amdgcn_udot4(E[k], 0x08040201u, 0u, false));
    const LAS unsigned* wp = (const LAS unsigned*)(ex + (pos & ~3u));
    const unsigned win = __builtin_amdgcn_alignbyte(wp[1], wp[0], pos);
    pos += __builtin_popcount(E[k]);
    t[k] = (t[k] + bB) ^ __builtin_amdgcn_perm(0u, win, sel);
  }
  cbar();
}

__device__ __forceinline__ bool unpack_block(int lane, const LAS char* blk, LAS char* ex, const LAS char* lut, u32x4& f0, u32x4& f1) {
  const Lists l = read_lists(lane, blk + 1664);
  const u32x2 N = *(const LAS u32x2*)(blk + 1024 + lane * 8);
  const unsigned sg = *(const LAS unsigned*)(blk + 1536 + (lane >> 1) * 4) >> ((lane & 1) * 4);
  const u32x4 D = *(const LAS u32x4*)(blk + lane * 16);
  const unsigned hdr = __builtin_amdgcn_readfirstlane(l.ent);
  if (hdr & 0x8000u) return false;
  const unsigned E0 = N[0] & (N[0] >> 1) & 0x44444444u, E1 = N[1] & (N[1] >> 1) & 0x44444444u;
  unsigned E[4], t[4], h[8];
#pragma unroll
  for (int k = 0; k < 4; ++k) {
    E[k] = ((k & 2 ? E1 : E0) >> (4 * (k & 1))) & 0x04040404u;
    t[k] = (N[k >> 1] >> (4 * (k & 1))) & 0x0f0f0f0fu;
  }
  decode_exponents(lane, l, hdr & 31u, hdr >> 5, __builtin_popcount(E0) + __builtin_popcount(E1), E, ex, lut, t);
#pragma unroll
  for (int k = 0; k < 4; ++k) {
    const unsigned H = t[k] | ((sg << (7 - k)) & 0x80808080u);
    h[2 * k] = __builtin_amdgcn_perm(H, D[k], 0x05010400u);
    h[2 * k + 1] = __builtin_amdgcn_perm(H, D[k], 0x07030602u);
  }
  f0 = (u32x4){h[0], h[1], h[2], h[3]};
  f1 = (u32x4){h[4], h[5], h[6], h[7]};
  return true;
}

__device__ __forceinline__ void unpack_sums(int lane, const LAS char* blk, LAS char* ex, const LAS char* lut, const char* spare,
                                            f32x4 (&y)[4]) {
  const Lists l = read_lists(lane, blk + 3328);
  const unsigned C = *(const LAS unsigned*)(blk + 3072 + lane * 4);
  u32x4 P[3], X;
#pragma unroll
  for (int i = 0; i < 3; ++i) P[i] = *(const LAS u32x4*)(blk + i * 1024 + lane * 16);
  const unsigned hdr = __builtin_amdgcn_readfirstlane(l.ent);
  if (hdr & 0x8000u) {
    asm volatile("global_load_dwordx4 %0, %1, off sc0 sc1\n\ts_waitcnt vmcnt(0)" : "=&v"(X) : "v"(spare + lane * 16) : "memory");
  } else {
    unsigned E[4], t[4];
#pragma unroll
    for (int k = 0; k < 4; ++k) {
      t[k] = (k ? C >> (2 * k - 2) : C << 2) & 0x0c0c0c0cu;
      E[k] = t[k] & (t[k] >> 1);
    }
    decode_exponents(lane, l, SUM_BASE, (hdr >> 5) & 63u, __builtin_popcount(C & (C >> 1) & 0x55555555u), E, ex, lut, t);
    const unsigned off = ((hdr & 31u) | (hdr >> 11 << 5)) * 0x01010101u;
#pragma unroll
    for (int k = 0; k < 4; ++k) X[k] = (t[k] >> 2) + off;
  }
#pragma unroll
  for (int k = 0; k < 4; ++k) {
    unsigned v[4];
#pragma unroll
    for (int i = 0; i < 3; ++i) v[i] = __builtin_amdgcn_perm(X[k], P[i][k], 0x04020100u + ((unsigned)i << 24));
    v[3] = __builtin_amdgcn_perm(P[1][k], P[0][k], 0x0c0c0703u) | __builtin_amdgcn_perm(X[k], P[2][k], 0x07030c0cu);
#pragma unroll
    for (int i = 0; i < 4; ++i) y[k][i] = __builtin_bit_cast(float, __builtin_rotateright32(v[i], 1));
  }
}

__device__ __forceinline__ float silu(float y) {
  const float c = fminf(fmaxf(y, -60.0f), 60.0f);
  return y * __builtin_amdgcn_rcpf(1.0f + __builtin_amdgcn_exp2f(c * -1.44269504088896341f));
}

__device__ __forceinline__ unsigned* call_word(const Params& p) { return (unsigned*)(p.mine + OFF_CALL) + blockIdx.x; }
__device__ __forceinline__ unsigned begin(const Params& p, unsigned before, LAS char* smem, bool puller) {
  *(LAS u32x4*)(smem + LDS_MB + min(threadIdx.x * 16u, LDS_LUT - LDS_MB - 16u)) = (u32x4){0u, 0u, 0u, 0u};
  if (threadIdx.x < 16) ((LAS unsigned*)(smem + LDS_LUT))[threadIdx.x] = codec_lut(threadIdx.x, puller);
  unsigned call = __builtin_amdgcn_readfirstlane(before) + 1u;
  asm volatile("s_waitcnt lgkmcnt(0)" : "+s"(call) :: "memory");
  if (threadIdx.x == 0) STAMP(0);
  wg_barrier();
  if (threadIdx.x == 0) __hip_atomic_store(call_word(p), call, __ATOMIC_RELAXED, __HIP_MEMORY_SCOPE_AGENT);
  return call;
}
static_assert(LDS_LUT - LDS_MB <= NTHREADS * 16 && (LDS_LUT - LDS_MB) % 16 == 0, "");

constexpr int XJ = (KSTEPS - HX) / 2;
static_assert(HX % 2 == 0, "");
__device__ __forceinline__ int x_unit(int u) {
  const int j = u / 7, r = u - j * 7;
  if (u >= 7 * KSTEPS) return (6 + (u & 1)) * KSTEPS + KSTEPS - XJ + ((u - 7 * KSTEPS) >> 1);
  return r >= 2 ? (r - 1) * KSTEPS + j : j < XJ ? HX + 2 * j + r : (6 + r) * KSTEPS + j - XJ;
}
#define XROUNDS 11
#define LATE 0
__device__ __forceinline__ unsigned convert_x(const Params& p, unsigned role, unsigned before, int rank, LAS char* smem) {
  const int lane = threadIdx.x & 63, wave = threadIdx.x >> 6;
  const int first = role & 255u, all = XROUNDS * p.ncvt_x, count = first < p.nlong_x ? 8 * KSTEPS - HX : all;
  const bool push = role & 0x1000u;
  char* const ws = p.mine;
  const float* src0 = p.x + (size_t)((wave >> 2) * 64 + (wave & 1) * 32 + (lane & 15)) * KDIM + (wave & 2) * 16 + (lane >> 4) * 4;
  f32x4 v[4];
  int un = push ? first : x_unit(first);
  if (LATE && !push) __builtin_amdgcn_s_sleep(LATE);
  {
    const float* s = src0 + (size_t)(un >> 7) * 128 * KDIM + (un & 127) * 64;
#pragma unroll
    for (int i = 0; i < 4; ++i) v[i] = ld4(s + (i >> 1) * 16 * KDIM + (i & 1) * 16);
  }
  const unsigned call = begin(p, before, smem, false);
  if (push) {
    const u32x4 h0 = cvt8(v[0], v[1]), h1 = cvt8(v[2], v[3]);
    const unsigned at = un * STEP_B + wave * 2048 + lane * 16;
    if (wave == 0) STAMP(24);
#pragma unroll
    for (int i = 0; i < 4; ++i) {
      char* b = i == 3 ? ws + OFF_OWN + own_idx(call, 0) * PANEL_B
                       : p.ws[(rank & 4) | ((rank + 1 + (i + un) % 3) & 3)] + pulled(call) + (rank & 3) * 6 * PANEL_B;
      store_sys(b + at, h0);
      store_sys(b + at + 1024, h1);
    }
    if (wave == 0) STAMP(25);
    wait_vm0();
    wg_barrier();
    if (wave == 0) STAMP(26);
    if (threadIdx.x < 4 * FCOPY) set_local(p.ws[(rank & 4) | (threadIdx.x & 3)], threadIdx.x >> 2, (rank & 3) * 6 * KSTEPS + un, call);
    return call;
  }
  for (int it = first, stride; it < count; it += stride) {
    const int u = un, tile = u >> 7, ks = u & 127;
    stride = it < all ? p.ncvt_x : p.nlong_x;
    const u32x4 h0 = cvt8(v[0], v[1]), h1 = cvt8(v[2], v[3]);
    {
      if (it + stride < count) un = x_unit(it + stride);
      const float* s = src0 + (size_t)(un >> 7) * 128 * KDIM + (un & 127) * 64;
#pragma unroll
      for (int i = 0; i < 4; ++i) v[i] = ld4(s + (i >> 1) * 16 * KDIM + (i & 1) * 16);
    }
    char* b = ws + OFF_OWN + own_idx(call, tile) * PANEL_B + ks * STEP_B + wave * 2048 + lane * 16;
    store_sys(b, h0);
    store_sys(b + 1024, h1);
    pack_block(h0, h1, smem + wave * WAVE_LDS, smem + LDS_LUT,
               ws + OFF_PACK + own_idx(call, tile) * PPANEL_B + ks * PSTEP_B + wave * BLK_B);
    wait_vm0();
    wg_barrier();
    if (it == first && wave == 0) STAMP(27);
    if (threadIdx.x < 4 + FCOPY) {
      const int t = threadIdx.x;
      const bool home = tile < 6;
      const int panel = home ? (rank & 3) * 6 + tile : PA_OWNAWAY + tile - 6;
      const int dst = (home ? (rank & 4) : (rank & 4) ^ 4) | (t & 3);
      if (t >= 4) set_local(ws, t - 4, panel * KSTEPS + ks, call);
      else if (tile == 5 && ks < RELAY) { if (t == (ks & 3)) set_rdy(p.ws[(dst ^ 4) & 7], rank, RDY_HOP1 + (ks >> 2), call); }
      else if (dst != rank) set_rdy(p.ws[dst & 7], rank, home ? u : RDY_AWAY + u - 6 * KSTEPS, call);
    }
  }
  return call;
}

__device__ __forceinline__ unsigned convert_w(const Params& p, unsigned role, unsigned before, int rank, LAS char* smem) {
  const int lane = threadIdx.x & 63, wave = threadIdx.x >> 6;
  const int count = KSTEPS * 4, stride = p.ncvt_w;
  const bool push = role & 0x1000u;
  char* const ws = p.mine;
  const float* src0 = p.w + (size_t)((wave & 1) * 32 + (lane >> 4) * 4) * NSH + (wave >> 1) * 64 + (lane & 15) * 4;
  f32x4 v[8];
  int u = role & 255u;
  if (LATE && !push) __builtin_amdgcn_s_sleep(LATE);
  {
    const float* s = src0 + (size_t)(u >> 2) * 64 * NSH + (u & 3) * 256;
#pragma unroll
    for (int j = 0; j < 8; ++j) v[j] = ld4(s + ((j & 3) + (j >> 2) * 16) * NSH);
  }
  const unsigned call = begin(p, before, smem, false);
  for (; u < count; u += stride) {
    const int ks = u >> 2, q = u & 3;
    u32x4 h[4];
#pragma unroll
    for (int f = 0; f < 4; ++f)
      h[f] = cvt8((f32x4){v[0][f], v[1][f], v[2][f], v[3][f]}, (f32x4){v[4][f], v[5][f], v[6][f], v[7][f]});
    const unsigned oi = own_idx(call, 8 + q * 2 + (wave >> 2)), at = ks * STEP_B + (wave & 3) * 4096 + lane * 16;
    char* const peer = p.ws[(((rank & 4) ^ 4) | q) & 7];
    char* b = ws + OFF_OWN + oi * PANEL_B + at;
    if (push) {
      char* r = peer + pulled(call) + (PB_FOREIGN + (rank & 3) * 2 + (wave >> 2)) * PANEL_B + at;
      if (wave == 0) STAMP(28);
#pragma unroll
      for (int f = 0; f < 4; ++f) store_sys(r + f * 1024, h[f]);
#pragma unroll
      for (int f = 0; f < 4; ++f) store_sys(b + f * 1024, h[f]);
      if (wave == 0) STAMP(29);
      wait_vm0();
      wg_barrier();
      if (wave == 0) STAMP(30);
      const int t = threadIdx.x;
      if (t < 4 * FCOPY)
        set_local(t & 2 ? peer : ws, t >> 2, ((t & 2 ? PB_FOREIGN + (rank & 3) * 2 : PB + q * 2) + (t & 1)) * KSTEPS + ks, call);
      break;
    }
    {
      const int un = u + stride < count ? u + stride : u;
      const float* s = src0 + (size_t)(un >> 2) * 64 * NSH + (un & 3) * 256;
#pragma unroll
      for (int j = 0; j < 8; ++j) v[j] = ld4(s + ((j & 3) + (j >> 2) * 16) * NSH);
    }
#pragma unroll
    for (int f = 0; f < 4; ++f) store_sys(b + f * 1024, h[f]);
    char* pb = ws + OFF_PACK + oi * PPANEL_B + ks * PSTEP_B + (wave & 3) * 2 * BLK_B;
    pack_block(h[0], h[1], smem + wave * WAVE_LDS, smem + LDS_LUT, pb);
    pack_block(h[2], h[3], smem + wave * WAVE_LDS + 1024, smem + LDS_LUT, pb + BLK_B);
    wait_vm0();
    wg_barrier();
    if (u == (role & 255u) && wave == 0) STAMP(31);
    if (threadIdx.x <= 2 * FCOPY) {
      const int t = threadIdx.x;
      if (t < 2 * FCOPY) set_local(ws, t >> 1, (PB + q * 2 + (t & 1)) * KSTEPS + ks, call);
      else set_rdy(peer, rank, ks, call);
    }
  }
  return call;
}

__device__ __forceinline__ int row_of(int r) { return ((RANK_AT >> (4 * r)) & 7) * NSH; }

#define ORDER_V 0x7648953210ull
constexpr unsigned long long BATCH_AT = ORDER_V;
constexpr unsigned long long batch_pos() {
  unsigned long long r = 0;
  for (int i = 0; i < 10; ++i) r |= (unsigned long long)i << (4 * ((BATCH_AT >> (4 * i)) & 15));
  return r;
}
constexpr unsigned long long BATCH_POS = batch_pos();
__device__ __forceinline__ int batch_row(int s, int o, int pos, int i) {
  const int bt = (BATCH_AT >> (4 * pos)) & 15;
  if (bt < 2) return row_of(s) + (bt * 3 + i) * 128;
  const int g = bt < 8 ? s : o;
  return row_of((g & 4) | ((g + 1 + i) & 3)) + (bt - 2) * 128;
}

__device__ __forceinline__ int unit_of(int it, int w) { return it * PW + ((w + it) & (PW - 1)); }

__device__ __forceinline__ unsigned lds_get(const LAS unsigned* c) {
  return __builtin_amdgcn_readfirstlane(__hip_atomic_load(c, __ATOMIC_RELAXED, __HIP_MEMORY_SCOPE_WORKGROUP));
}
__device__ __forceinline__ unsigned lds_ticket(int lane, LAS unsigned* c) {
  unsigned t = 0;
  if (lane == 0) t = __hip_atomic_fetch_add(c, 1u, __ATOMIC_RELAXED, __HIP_MEMORY_SCOPE_WORKGROUP);
  return __builtin_amdgcn_readfirstlane(t);
}

constexpr int RING = 8;
constexpr unsigned PL_EX = RING * PSTEP_B;
constexpr int W_TICKET = 0, W_READY = 1, W_DEAD = 2, W_TURN = 3, W_DONE = 8, W_LANDED = 64, W_QUEUE = 128;
constexpr unsigned Q_N = 128, Q_OUT = 1024, Q_LAST = 1u << 31;
static_assert(PL_EX + 8 * 512 <= LDS_MB && (W_QUEUE + Q_N) * 4 <= LDS_TMO - LDS_MB, "");

template <int N, int W>
__device__ __forceinline__ void stream_kib(const char* src, unsigned dst, unsigned& cnt, unsigned word) {
  unsigned keep;
#define SK(OFF) "s_waitcnt vmcnt(%5)\n\tglobal_load_lds_dwordx4 %2, off offset:" #OFF " sc0 sc1\n\t" \
                "v_add_u32 %1, 1, %1\n\tds_max_u32 %4, %1\n\t"
  if (N == 4)
    asm volatile("s_mov_b32 %0, m0\n\ts_mov_b32 m0, %3\n\ts_nop 0\n\t" SK(0) SK(1024) SK(2048) SK(3072) "s_mov_b32 m0, %0"
                 : "=&s"(keep), "+v"(cnt) : "v"(src), "s"(dst), "v"(word), "n"(W - 1) : "memory");
  else if (N == 2)
    asm volatile("s_mov_b32 %0, m0\n\ts_mov_b32 m0, %3\n\ts_nop 0\n\t" SK(0) SK(1024) "s_mov_b32 m0, %0"
                 : "=&s"(keep), "+v"(cnt) : "v"(src), "s"(dst), "v"(word), "n"(W - 1) : "memory");
  else
    asm volatile("s_mov_b32 %0, m0\n\ts_mov_b32 m0, %3\n\ts_nop 0\n\t" SK(0) "s_mov_b32 m0, %0"
                 : "=&s"(keep), "+v"(cnt) : "v"(src), "s"(dst), "v"(word), "n"(W - 1) : "memory");
}
template <int K, class F>
__device__ __forceinline__ bool stream_drain(unsigned cnt, unsigned word, F&& more) {
  if constexpr (K >= 0) {
    asm volatile("s_waitcnt vmcnt(%2)\n\tds_max_u32 %1, %0" :: "v"(cnt + 1), "v"(word), "n"(K) : "memory");
    return more() || stream_drain<K - 1>(cnt + 1, word, more);
  }
  return false;
}

struct Unit { unsigned src, raw, dst; int from, flag; };
template <bool CROSS>
__device__ __forceinline__ Unit input_unit(int u, unsigned call, int rank, int peer) {
  constexpr int H1 = RELAY / 4, RSTEP = 32 * KSTEPS;
  const unsigned par = own_idx(call, 0) * KSTEPS;
  Unit o;
  o.from = peer;
  u += CROSS ? 2 * HW : HX;
  if (!CROSS) {
    const int v = u < 5 * KSTEPS ? u : u + RELAY;
    o.raw = par + v; o.dst = (peer & 3) * 6 * KSTEPS + v; o.flag = v;
  } else if (u < 256) {
    const int v = (u & 1) * KSTEPS + (u >> 1);
    o.raw = par + (8 + (rank & 3) * 2) * KSTEPS + v; o.dst = (PB_FOREIGN + (peer & 3) * 2) * KSTEPS + v; o.flag = u >> 1;
  } else if (u < 256 + H1) {
    o.raw = par + 5 * KSTEPS + (u - 256) * 4 + (rank & 3); o.dst = ~((peer & 3) * 16 + u - 256); o.flag = RDY_HOP1 + u - 256;
  } else if (u < 512 + H1) {
    const int v = u - 256 - H1;
    o.raw = par + 6 * KSTEPS + v; o.dst = (PA_AWAY + (peer & 3) * 2) * KSTEPS + v; o.flag = RDY_AWAY + v;
  } else {
    const int v = u - 512 - H1, j = v / 3, s = (rank + 1 + v - j * 3) & 3, ks = 5 * KSTEPS + j * 4 + (peer & 3);
    o.from = (rank & 4) | s;
    o.raw = par + ks; o.dst = s * 6 * KSTEPS + ks; o.flag = RDY_HOP2 + s * 16 + j;
    o.src = RSTEP + s * 16 + j;
    return o;
  }
  o.src = o.raw;
  return o;
}

template <bool CROSS, int W>
__device__ __forceinline__ void stream(const Params& p, int lane, unsigned call, int rank, int peer, LAS char* smem) {
  LAS unsigned* const sh = (LAS unsigned*)(smem + LDS_MB);
  const char* const pws = p.ws[peer];
  const unsigned lds0 = (unsigned)(uintptr_t)smem, word = (unsigned)(uintptr_t)(sh + W_LANDED + lane);
  unsigned cnt = WIN1 - W - 1;
  for (unsigned k = 0;; ++k) {
    unsigned ready, done, u;
    auto look = [&]() {
      const unsigned a = __hip_atomic_load(sh + W_READY, __ATOMIC_RELAXED, __HIP_MEMORY_SCOPE_WORKGROUP);
      cbar();
      const unsigned b = __hip_atomic_load(sh + W_DONE + (k & (RING - 1)), __ATOMIC_RELAXED, __HIP_MEMORY_SCOPE_WORKGROUP);
      const unsigned c = __hip_atomic_load(sh + W_QUEUE + (k & (Q_N - 1)), __ATOMIC_RELAXED, __HIP_MEMORY_SCOPE_WORKGROUP);
      cbar();
      ready = __builtin_amdgcn_readfirstlane(a);
      done = __builtin_amdgcn_readfirstlane(b);
      u = __builtin_amdgcn_readfirstlane(c) & 2047u;
      return k < (ready & ~Q_LAST) && done >= (k & ~(RING - 1u));
    };
    if (!look()) {
      if (!stream_drain<W - 1>(cnt, word, look)) {
        while (!look() && (k | Q_LAST) != ready) __builtin_amdgcn_s_sleep(1);
        if ((k | Q_LAST) == ready) break;
      }
    }
    const unsigned d = lds0 + (k & (RING - 1)) * PSTEP_B;
    if (u < Q_OUT) {
      if ((u & 63) >= 56) STAMP(4 + (u >> 6));
      const char* s = pws + OFF_PACK + (size_t)input_unit<CROSS>(u, call, rank, peer).src * PSTEP_B + lane * 16;
      stream_kib<4, W>(s, d, cnt, word);
      stream_kib<4, W>(s + 4096, d + 4096, cnt, word);
      stream_kib<4, W>(s + 8192, d + 8192, cnt, word);
      stream_kib<2, W>(s + 12288, d + 12288, cnt, word);
      if (k == 0) STAMP(3);
    } else {
      if ((u - Q_OUT) % 48 >= 40) STAMP(20 + (u - Q_OUT) / 48);
      const char* s = pws + OFF_STG + (size_t)((rank & 3) * OUT_CHUNKS + u - Q_OUT) * CHUNK_B + lane * 16;
      stream_kib<4, W>(s, d, cnt, word);
      stream_kib<2, W>(s + 4096, d + 4096, cnt, word);
      if (lane < 48) stream_kib<1, W>(s + 6144, d + 6144, cnt, word);
    }
  }
}

template <bool CROSS>
__device__ __forceinline__ void pull(const Params& p, int w, bool far, int wave, unsigned call, int rank, int peer, LAS char* smem) {
  constexpr int NIN = CROSS ? 512 + RELAY - 2 * HW : 768 - RELAY - HX, NOUT = CROSS ? OUT_CHUNKS / PW : 0;
  const int lane = fresh_lane();
  char* const ws = p.mine;
  LAS unsigned* const sh = (LAS unsigned*)(smem + LDS_MB);
  float* const out = p.out + (size_t)((lane >> 4) * 4) * NSH + (peer & 3) * 256 + (lane & 15) * 4;
  auto where = [&](int c, int pc) {
    const int tile = c >> 3, bt = tile / 6, r = tile - bt * 6;
    return out + (size_t)(batch_row(peer, rank, bt, r >> 1) + ((c >> 2) & 1) * 64 + (c & 1) * 32 + (pc & 4) * 4 + (pc & 3)) * NSH
         + (r & 1) * 128 + ((c >> 1) & 1) * 64;
  };
#define PRIO 3
  if (wave >= 6) __builtin_amdgcn_s_setprio(PRIO);
  if (wave == 6) {
    const unsigned* const rdy = (const unsigned*)(ws + OFF_RDY) + peer * RDY_N * 32;
    unsigned* const tmo = (unsigned*)(ws + OFF_TMO);
    unsigned* const ticket = (unsigned*)(ws + OFF_TICKET) + peer * 32;
    unsigned long long t0 = __builtin_amdgcn_s_memrealtime();
    bool dead = false;
    unsigned k = 0, kib = 0, spin = 0;
    auto unset = [&](bool look, int fl) {
      unsigned v = call;
      if (look && !dead)
        asm volatile("global_load_dword %0, %1, off sc0 sc1\n\ts_waitcnt vmcnt(0)" : "=v"(v) : "v"(rdy + fl * 32) : "memory");
      return __builtin_amdgcn_ballot_w64(v < call);
    };
    auto idle = [&]() {
      if ((++spin & 63u) == 0u
          && (__builtin_amdgcn_readfirstlane(__hip_atomic_load(tmo, __ATOMIC_RELAXED, __HIP_MEMORY_SCOPE_AGENT)) == call
              || __builtin_amdgcn_s_memrealtime() - t0 > DEADLINE)) {
        if (lane == 0) {
          __hip_atomic_store(tmo, call, __ATOMIC_RELAXED, __HIP_MEMORY_SCOPE_AGENT);
          __hip_atomic_store(sh + W_DEAD, 1u, __ATOMIC_RELAXED, __HIP_MEMORY_SCOPE_WORKGROUP);
        }
        wait_vm0();
        dead = true;
      }
      __builtin_amdgcn_s_sleep(1);
    };
    auto publish = [&](unsigned v, unsigned near) {
      cbar();
      if (lane == 0) __hip_atomic_store(sh + W_READY, v, __ATOMIC_RELAXED, __HIP_MEMORY_SCOPE_WORKGROUP);
      while (lds_get(sh + W_LANDED) < near) __builtin_amdgcn_s_sleep(2);
      t0 = __builtin_amdgcn_s_memrealtime();
    };
    int known = EXPON(1) ? NIN : 0;
    auto look = [&](int from, int width) {
      const unsigned long long miss = unset(lane < width, input_unit<CROSS>(min(from + lane, NIN - 1), call, rank, peer).flag);
      known = from + (miss ? (int)__builtin_ctzll(miss) : width);
    };
    for (int u = w; u < NIN;) {
      while (u >= known) { look(u, 16); if (u >= known) idle(); }
      if (k == 0) STAMP(30);
      if (k == 1) STAMP(31);
      if (k == 2) STAMP(19);
      if (lane == 0) sh[W_QUEUE + (k & (Q_N - 1))] = kib << 11 | u;
#define AHEAD 4
      publish(++k, max(kib, (unsigned)AHEAD) - AHEAD);
      kib += 14;
      unsigned tk = 0;
      if (lane == 0) tk = __hip_atomic_fetch_add(ticket, 1u, __ATOMIC_RELAXED, __HIP_MEMORY_SCOPE_AGENT);
      if (known < NIN && known - u < 32) look(known, 64);
      u = min(__builtin_amdgcn_readfirstlane(tk) - (call - 1u) * NIN + PW, (unsigned)NIN);
    }
    if (CROSS && lane == 0) __hip_atomic_store(sh + W_TURN, ~(k * 8u), __ATOMIC_RELAXED, __HIP_MEMORY_SCOPE_WORKGROUP);
    const unsigned long long all = (1ull << NOUT) - 1ull;
    for (unsigned long long left = all; left;) {
      const unsigned long long fresh = left & ~unset(!EXPON(17) && ((left >> lane) & 1), RDY_OUT + unit_of(lane, w));
      if (!fresh) { idle(); continue; }
      const unsigned pos = __builtin_amdgcn_mbcnt_hi(fresh >> 32, __builtin_amdgcn_mbcnt_lo(fresh, 0));
      if ((fresh >> lane) & 1) sh[W_QUEUE + ((k + pos) & (Q_N - 1))] = (kib + pos * 7) << 11 | (Q_OUT + unit_of(lane, w));
      k += __builtin_popcountll(fresh);
      kib += __builtin_popcountll(fresh) * 7;
      left &= ~fresh;
      if (left) publish(k, kib - 14);
    }
    publish(k | Q_LAST, 0);
    if (CROSS) {
      dead = __builtin_amdgcn_readfirstlane(__hip_atomic_load(tmo, __ATOMIC_RELAXED, __HIP_MEMORY_SCOPE_AGENT)) == call;
      wg_barrier();
      if (dead) store_sys(where(unit_of(0, w), 0), (u32x4){0x7fc00000u, 0x7fc00000u, 0x7fc00000u, 0x7fc00000u});
    }
    return;
  }
  if (wave == 7) {
    if (far) stream<CROSS, WIN1>(p, lane, call, rank, peer, smem);
    else stream<CROSS, WIN>(p, lane, call, rank, peer, smem);
    STAMP(1);
    if (CROSS) wg_barrier();
    return;
  }
  static_assert(RING == 8, "");
  auto landed = [&](unsigned tk, unsigned& t, unsigned& u) {
    const unsigned ready = lds_get(sh + W_READY);
    cbar();
    t = tk;
    if (CROSS) {
      const unsigned turn = ~lds_get(sh + W_TURN);
      if (tk > turn) t = turn + (tk - turn) * 4u;
    }
    const unsigned q = lds_get(sh + W_QUEUE + ((t >> 3) & (Q_N - 1))), n = lds_get(sh + W_LANDED);
    u = q & 2047u;
    if ((t >> 3) >= (ready & ~Q_LAST)) { u = ~0u; return ready >= Q_LAST; }
    return n >= (q >> 11) + (u < Q_OUT ? ((t & 7) * 7 + 10) >> 2 : t & 4 ? 7 : 4) + WIN1 - 1;
  };
  const unsigned none = ~0u;
  unsigned prev = none, pdst = 0;
  auto done = [&]() {
    const bool last = (lds_ticket(lane, sh + W_DONE + ((prev >> 3) & (RING - 1))) & 7u) == 7u;
    prev = none;
    if (!last) return;
    if (!CROSS && (pdst & 127) >= 112) STAMP(17 + ((~pdst >> 3) & 1));
    if (!CROSS || (int)pdst >= 0) { if (lane < FCOPY) set_local(ws, lane, pdst, call); }
    else if (lane < 3) set_rdy(p.ws[((peer & 4) | ((peer + 1 + lane) & 3)) & 7], rank, RDY_HOP2 + ~pdst, call);
  };
  for (;;) {
    unsigned t, u;
    if (prev != none && !landed(lds_get(sh + W_TICKET), t, u)) { wait_vm0(); done(); }
    const unsigned tk = lds_ticket(lane, sh + W_TICKET);
    if (!landed(tk, t, u)) {
      if (prev != none) { wait_vm0(); done(); }
      while (!landed(tk, t, u)) __builtin_amdgcn_s_sleep(1);
    }
    cbar();
    const LAS char* slot = smem + ((t >> 3) & (RING - 1)) * PSTEP_B;
    if (u >= Q_OUT) {
      if (prev != none) { wait_vm0(); done(); }
      if (!CROSS || u == none) break;
      const unsigned half = (t >> 2) & 1u;
      f32x4 y[4];
      unpack_sums(lane, slot + half * SUM_B, smem + PL_EX + wave * 512, smem + LDS_LUT,
                  p.ws[peer] + OFF_SPARE + (size_t)(((rank & 3) * OUT_CHUNKS + u - Q_OUT) * 2 + half) * 1024, y);
      if (lane == 0) __hip_atomic_fetch_add(sh + W_DONE + ((t >> 3) & (RING - 1)), 4u, __ATOMIC_RELAXED, __HIP_MEMORY_SCOPE_WORKGROUP);
      const bool dead = lds_get(sh + W_DEAD);
      float* const o = where(u - Q_OUT, t & 4);
#pragma unroll
      for (int r = 0; r < 4; ++r) {
        u32x4 v = __builtin_bit_cast(u32x4, (f32x4){silu(y[r][0]), silu(y[r][1]), silu(y[r][2]), silu(y[r][3])});
        if (dead) v = (u32x4){0x7fc00000u, 0x7fc00000u, 0x7fc00000u, 0x7fc00000u};
        store_out(o + r * NSH, v);
      }
      continue;
    }
    const Unit un = input_unit<CROSS>(u, call, rank, peer);
    const LAS char* blk = slot + (t & 7) * BLK_B;
    u32x4 f0, f1;
    char* d;
    if (CROSS && RELAY && (int)un.dst < 0) {
      const int l1 = min(lane, 47);
      f0 = *(const LAS u32x4*)(blk + lane * 16);
      f1 = *(const LAS u32x4*)(blk + 1024 + l1 * 16);
      d = ws + OFF_RELAY + (size_t)(~un.dst * 8 + (t & 7)) * BLK_B;
      store_sys(d + lane * 16, f0);
      store_sys(d + 1024 + l1 * 16, f1);
    } else {
      if (!unpack_block(lane, blk, smem + PL_EX + wave * 512, smem + LDS_LUT, f0, f1)) {
        const char* r = p.ws[un.from & 7] + OFF_OWN + (size_t)(un.raw * 8 + (t & 7)) * 2048 + lane * 16;
        asm volatile("global_load_dwordx4 %0, %2, off sc0 sc1\n\t"
                     "global_load_dwordx4 %1, %2, off offset:1024 sc0 sc1\n\ts_waitcnt vmcnt(0)"
                     : "=&v"(f0), "=&v"(f1) : "v"(r) : "memory");
      }
      d = ws + pulled(call) + (size_t)(un.dst * 8 + (t & 7)) * 2048 + lane * 16;
      store_sys(d, f0);
      store_sys(d + 1024, f1);
    }
    asm volatile("s_waitcnt vmcnt(2)" ::: "memory");
    if (prev != none) done();
    prev = t;
    pdst = un.dst;
  }
  if (CROSS) { wait_vm0(); wg_barrier(); }
}

struct Job { int pa, pb, ah, bh, tile; };
__device__ __forceinline__ void decode_job(unsigned code, int a, Job& o) {
  const int ra = code & 63;
  int bt, i;
  o.pb = PB + ((code >> 6) & 15);
  o.ah = (code >> 10) & 1;
  o.bh = (code >> 11) & 1;
  if (ra < 18) {
    i = ra / 6; bt = 2 + ra % 6; o.pa = ((a + 1 + i) & 3) * 6 + bt - 2;
  } else if (ra < 24) {
    o.pa = a * 6 + ra - 18; bt = (ra - 18) / 3; i = (ra - 18) % 3;
  } else {
    o.pa = ra; bt = 8 + (ra & 1); i = (((ra - PA_AWAY) >> 1) - ((o.pb - PB_FOREIGN) >> 1) - 1) & 3;
  }
  o.tile = (int)((BATCH_POS >> (4 * bt)) & 15) * 3 + i;
}

#define POLL_EVERY 1
#define POLL_CAD 2
#define POLL_SLEEP 2
#define GLDS_BITS " sc0 sc1"

template <int NB>
__device__ __forceinline__ unsigned next_stage(unsigned so) { return so == LDS_MB - (NB + 1) * 8192 ? 0u : so + (NB + 1) * 8192; }

template <bool TALL, int NB>
__device__ __forceinline__ void mfma_wave(const Params& p, LAS char* smem, const Job& jb, int job, int wave,
                                          int rank, unsigned call) {
  constexpr int MT = TALL ? 4 : NB, NT = TALL && NB == 2 ? 8 : 4, KH = TALL ? 1 : 2, STEPS = 2 * KSTEPS / KH;
  constexpr int NRA = MT * KH, NRB = NT * KH, NRD = NRA + NRB, NMM = MT * NT * KH;
  int lane = fresh_lane();
  const int wr = MT == 1 ? wave : wave >> 1, wc = MT == 1 ? 0 : wave & 1;
  f32x4 acc[MT][NT];
#pragma unroll
  for (int m = 0; m < MT; ++m)
#pragma unroll
    for (int n = 0; n < NT; ++n) acc[m][n] = (f32x4){0.f, 0.f, 0.f, 0.f};
  half8 fa[2][NRA], fb[2][NRB];
  const LAS char* la = smem + wr * MT * 1024 + lane * 16;
  const LAS char* lb = smem + 8192 + wc * NRB * 1024 + lane * 16;
  unsigned so = 0;
#define READ_SET(X) do { \
    _Pragma("unroll") for (int i = 0; i < NRB; ++i) { \
      if (i < NRA) fa[X][i] = *(const LAS half8*)(la + so + (i / MT) * 4096 + (i % MT) * 1024); \
      fb[X][i] = *(const LAS half8*)(lb + so + i * 1024); } \
    so = next_stage<NB>(so); } while (0)
#define MMA_SET(X) do { \
    _Pragma("unroll") for (int kh = 0; kh < KH; ++kh) \
    _Pragma("unroll") for (int m = 0; m < MT; ++m) \
    _Pragma("unroll") for (int n = 0; n < NT; ++n) \
      acc[m][n] = __builtin_amdgcn_mfma_f32_16x16x32_f16(fa[X][kh * MT + m], fb[X][kh * NT + n], acc[m][n], 0, 0, 0); } while (0)
  constexpr int RG = TALL ? 1 : 2, MG = NB, G = NRD / RG;
#define INTERLEAVE() do { \
    _Pragma("unroll") for (int i = 0; i < G; ++i) { \
      __builtin_amdgcn_sched_group_barrier(0x100, RG, 0); \
      __builtin_amdgcn_sched_group_barrier(0x008, MG, 0); } \
    __builtin_amdgcn_sched_group_barrier(0x008, NMM - G * MG, 0); } while (0)
#define STEP_END() do { __builtin_amdgcn_sched_barrier(0); asm volatile("" ::: "memory"); \
    __builtin_amdgcn_s_waitcnt(0xC07F);   __builtin_amdgcn_sched_barrier(0); \
    wg_barrier(); __builtin_amdgcn_sched_barrier(0); } while (0)
  wg_barrier();
  if (wave == 0) STAMP(4 + job * 4);
  READ_SET(0);
  STEP_END();
  for (int t = 1; t < STEPS - 1; t += 2) {
    READ_SET(1); MMA_SET(0); INTERLEAVE();
    STEP_END();
    READ_SET(0); MMA_SET(1); INTERLEAVE();
    STEP_END();
  }
  READ_SET(1); MMA_SET(0); INTERLEAVE();
  const unsigned tm = *(const LAS unsigned*)(smem + LDS_TMO + job * 16);
  STEP_END();
  MMA_SET(1);
  __builtin_amdgcn_sched_barrier(0);
  if (wave == 0) STAMP(5 + job * 4);
  lane = fresh_lane();
  asm volatile("" : "+s"(call));
  const bool dead = tm == call;

  const bool foreign = jb.pb >= PB_FOREIGN;
  const int ob = (jb.pb - PB_FOREIGN) >> 1;
  const int r16 = jb.ah * 4 + wr * MT;
  const int c64 = (jb.pb & 1) * 2 + jb.bh + wc * (NT / 4);
  const int chunk = (jb.tile * 2 + (c64 >> 1)) * 8 + (r16 & 4) + (c64 & 1) * 2 + ((r16 >> 1) & 1);
  const float qnan = __builtin_nanf("");
  if (foreign) {
    const unsigned blk = (ob * OUT_CHUNKS + chunk) * 2 + (r16 & 1);
#pragma unroll 1
    for (int i = 0; i < MT * NT / 4; ++i) {
      f32x4 v[4];
#pragma unroll
      for (int j = 0; j < MT * NT / 4; ++j)
        if (j == 0 || i == j) {
#pragma unroll
          for (int n = 0; n < 4; ++n) v[n] = acc[j % MT][j / MT * 4 + n];
          asm volatile("" : "+v"(v[0]), "+v"(v[1]), "+v"(v[2]), "+v"(v[3]));
        }
      if (dead) v[0] = v[1] = v[2] = v[3] = (f32x4){qnan, qnan, qnan, qnan};
      pack_sums(lane, v, smem + LDS_EPI + wave * 512, smem + LDS_LUT, p.mine + OFF_STG + (size_t)((blk + i) >> 1) * CHUNK_B + ((blk + i) & 1) * SUM_B,
                p.mine + OFF_SPARE + (size_t)(blk + i) * 1024);
    }
  } else {
    const int pa = jb.pa;
    const int row0 = pa < PA_AWAY ? row_of((rank & 4) + pa / 6) + (pa % 6) * 128
                   : row_of(pa < PA_OWNAWAY ? ((rank & 4) ^ 4) + ((pa - PA_AWAY) >> 1) : rank) + 768 + (pa & 1) * 128;
    float* o = p.out + (size_t)(row0 + r16 * 16 + (lane >> 4) * 4) * NSH + ((jb.pb - PB) >> 1) * 256 + c64 * 64 + (lane & 15) * 4;
#pragma unroll
    for (int h = 0; h < NT / 4; ++h)
#pragma unroll
      for (int m = 0; m < MT; ++m)
#pragma unroll
        for (int r = 0; r < 4; ++r) {
          f32x4 v = {silu(acc[m][h * 4][r]), silu(acc[m][h * 4 + 1][r]), silu(acc[m][h * 4 + 2][r]), silu(acc[m][h * 4 + 3][r])};
          if (dead) v = (f32x4){qnan, qnan, qnan, qnan};
          store_out(o + h * 64 + (m * 16 + r) * NSH, __builtin_bit_cast(u32x4, v));
        }
  }
  if (foreign) {
    wait_vm0();
    bool last = lane < (MT * NT + 7) / 8;
    if (MT == 1)
      last = last && __hip_atomic_fetch_add((LAS unsigned*)(smem + LDS_TMO) + job * 4 + 2 + (wave >> 1), 1u,
                                            __ATOMIC_RELAXED, __HIP_MEMORY_SCOPE_WORKGROUP) == 1u;
    if (last) set_rdy(p.ws[(((rank & 4) ^ 4) | ob) & 7], rank, RDY_OUT + chunk + lane, call);
  }
  if (wave == 0) STAMP(6 + job * 4);
}

#define NEAR 4
template <int NB>
__device__ __forceinline__ void load_wave(const Params& p, LAS char* smem, const Job& jb, int job, unsigned code, int l,
                                          bool tall, int rank, unsigned call) {
  constexpr int DEPTH = NB == 2 ? 5 : 7;
  const int steps = tall ? 2 * KSTEPS : KSTEPS, ksh = tall ? 1 : 0;
  if (l > NB && l < 3) {
    for (int t = 0; t <= steps; ++t) wg_barrier();
    return;
  }
  const int lane = fresh_lane(), a = rank & 3;
  char* const ws = p.mine;
  const unsigned lds0 = (unsigned)(uintptr_t)smem;
  LAS unsigned* const rdyw = (LAS unsigned*)(smem + LDS_TMO) + job * 4 + 1;
  const unsigned gap = tall ? 8192 : 4096;
  auto panel_of = [&](int pc) { return pc == 0 ? jb.pa : tall ? jb.pb + pc - 1 : jb.pb; };
  auto source = [&](int pc) {
    const int panel = panel_of(pc);
    const int own = panel >= PB_FOREIGN ? -1 : panel >= PA_OWNAWAY ? panel - PA_OWNAWAY + 6
                  : (unsigned)(panel - a * 6) < 6u ? panel - a * 6 : -1;
    return ws + (own >= 0 ? OFF_OWN + own_idx(call, own) * PANEL_B : pulled(call) + (size_t)panel * PANEL_B)
         + (tall ? 0 : (pc == 0 ? jb.ah : jb.bh + pc - 1) * 8192);
  };
  int ready = EXPON(1) ? KSTEPS : 0;
  if (l == 3) {
    unsigned* const tmo = (unsigned*)(ws + OFF_TMO);
    const char* fsrc = ws + OFF_FLAG + (blockIdx.x & (FCOPY - 1)) * 65536 + panel_of(lane >> 5) * 1024 + ((lane >> 2) & 7) * 128 + (lane & 3) * 16;
    const unsigned mb = lds0 + LDS_MB + job * 2048;
    int ra = ready, rb = ready;
    auto poll = [&]() {
      if ((lane < 32 ? ra : rb) < KSTEPS) {
        unsigned keep;
        asm volatile("s_mov_b32 %0, m0\n\ts_mov_b32 m0, %2\n\ts_nop 0\n\t"
                     "global_load_lds_dwordx4 %1, off sc0 sc1\n\ts_mov_b32 m0, %0"
                     : "=&s"(keep) : "v"(fsrc), "s"(mb) : "memory");
        if (NB == 2 && tall && lane >= 32)
          asm volatile("s_mov_b32 %0, m0\n\ts_mov_b32 m0, %2\n\ts_nop 0\n\t"
                       "global_load_lds_dwordx4 %1, off offset:1024 sc0 sc1\n\ts_mov_b32 m0, %0"
                       : "=&s"(keep) : "v"(fsrc), "s"(mb - 512) : "memory");
      }
    };
    auto refresh = [&]() {
      u32x2 f, g, h;
      asm volatile("ds_read2_b32 %0, %3 offset1:8\n\tds_read2_b32 %1, %3 offset0:128 offset1:136\n\t"
                   "ds_read2_b32 %2, %4 offset1:8\n\ts_waitcnt lgkmcnt(0)"
                   : "=&v"(f), "=&v"(g), "=&v"(h)
                   : "v"(mb + (lane & 7) * 64 + (lane >> 3) * 4), "v"(mb + (NB == 2 && tall ? 1024 : 512) + (lane & 7) * 64 + (lane >> 3) * 4)
                   : "memory");
      const unsigned long long a0 = __builtin_amdgcn_ballot_w64(f[0] < call), a1 = __builtin_amdgcn_ballot_w64(f[1] < call);
      const unsigned long long b0 = __builtin_amdgcn_ballot_w64(min(g[0], h[0]) < call), b1 = __builtin_amdgcn_ballot_w64(min(g[1], h[1]) < call);
      ra = a0 ? __builtin_ctzll(a0) : a1 ? 64 + __builtin_ctzll(a1) : KSTEPS;
      rb = b0 ? __builtin_ctzll(b0) : b1 ? 64 + __builtin_ctzll(b1) : KSTEPS;
      ready = min(ra, rb);
      if (lane == 0) __hip_atomic_store(rdyw, (unsigned)ready, __ATOMIC_RELAXED, __HIP_MEMORY_SCOPE_WORKGROUP);
    };
    auto advance = [&]() {
      const int k = min(ready + lane, KSTEPS - 1);
      u32x2 f;
      unsigned h;
      asm volatile("ds_read2_b32 %0, %2 offset1:128\n\tds_read_b32 %1, %3\n\ts_waitcnt lgkmcnt(0)"
                   : "=&v"(f), "=&v"(h)
                   : "v"(mb + (k & 7) * 64 + (k >> 3) * 4), "v"(mb + (NB == 2 && tall ? 1024 : 512) + (k & 7) * 64 + (k >> 3) * 4)
                   : "memory");
      const unsigned long long m = __builtin_amdgcn_ballot_w64(min(min(f[0], f[1]), h) < call);
      ready = min(ready + (m ? (int)__builtin_ctzll(m) : 64), KSTEPS);
      __hip_atomic_store(rdyw, (unsigned)ready, __ATOMIC_RELAXED, __HIP_MEMORY_SCOPE_WORKGROUP);
    };
    for (int s = 0; s < steps; ++s) {
      if (s >= DEPTH) wg_barrier();
      const int ks = s >> ksh;
      if (ready < KSTEPS && ready - ks < NEAR) {
        advance();
        if (ks >= ready) {
          const unsigned long long t0 = __builtin_amdgcn_s_memrealtime();
          for (unsigned it = 1;; ++it) {
            if (it >= 32u || s == 0) { poll(); wait_vm0(); refresh(); }
            else { if ((it & (POLL_CAD - 1)) == 1u) poll(); advance(); }
            if (ks < ready) break;
            if ((it & 63u) == 0u) {
              if (__builtin_amdgcn_readfirstlane(__hip_atomic_load(tmo, __ATOMIC_RELAXED, __HIP_MEMORY_SCOPE_AGENT)) == call
                  || __builtin_amdgcn_s_memrealtime() - t0 > DEADLINE) {
                if (lane == 0) __hip_atomic_store(tmo, call, __ATOMIC_RELAXED, __HIP_MEMORY_SCOPE_AGENT);
                wait_vm0();
                ready = KSTEPS;
                if (lane == 0) __hip_atomic_store(rdyw, (unsigned)ready, __ATOMIC_RELAXED, __HIP_MEMORY_SCOPE_WORKGROUP);
                break;
              }
            }
            if (it >= 32u) __builtin_amdgcn_s_sleep(POLL_SLEEP); else __builtin_amdgcn_s_sleep(1);
          }
        } else if (!(s & (((POLL_EVERY) << ksh) - 1))) {
          poll();
        }
      }
    }
    if (lane == 0) {
      unsigned keep;
      asm volatile("s_mov_b32 %0, m0\n\ts_mov_b32 m0, %2\n\ts_nop 0\n\t"
                   "global_load_lds_dword %1, off sc1\n\ts_mov_b32 m0, %0"
                   : "=&s"(keep) : "v"(tmo), "s"(lds0 + LDS_TMO + job * 16) : "memory");
    }
    for (int t = 1; t < DEPTH; ++t) wg_barrier();
    wait_vm0();
    wg_barrier();
    wg_barrier();
    return;
  }
  const char* src = source(l) + lane * 16;
  const unsigned dst0 = lds0 + l * 8192;
  unsigned so = 0;
  for (int s = 0; s < steps; ++s) {
    if (s >= DEPTH) {
      asm volatile("s_waitcnt vmcnt(%0)" :: "n"(8 * (DEPTH - 1)) : "memory");
      wg_barrier();
    }
    const int ks = s >> ksh;
    if (ks >= ready)
      while ((ready = lds_get(rdyw)) <= ks) __builtin_amdgcn_s_sleep(1);
    const char* g = src + (size_t)ks * STEP_B + (s & ksh) * 4096;
    const unsigned d = dst0 + so;
    so = next_stage<NB>(so);
    unsigned keep;
#define GLDS4(R) "global_load_lds_dwordx4 " R ", off" GLDS_BITS "\n\t" \
                 "global_load_lds_dwordx4 " R ", off offset:1024" GLDS_BITS "\n\t" \
                 "global_load_lds_dwordx4 " R ", off offset:2048" GLDS_BITS "\n\t" \
                 "global_load_lds_dwordx4 " R ", off offset:3072" GLDS_BITS "\n\t"
    asm volatile("s_mov_b32 %0, m0\n\ts_mov_b32 m0, %3\n\ts_nop 0\n\t" GLDS4("%1")
                 "s_mov_b32 m0, %4\n\ts_nop 0\n\t" GLDS4("%2") "s_mov_b32 m0, %0"
                 : "=&s"(keep) : "v"(g), "v"(g + gap), "s"(d), "s"(d + 4096) : "memory");
  }
#define WB(N) "s_waitcnt vmcnt(" #N ")\n\ts_barrier\n\t"
  if (NB == 1) asm volatile(WB(48) WB(40) ::: "memory");
  asm volatile(WB(32) WB(24) WB(16) WB(8) WB(0) "s_barrier" ::: "memory");
}

__global__ void __launch_bounds__(NTHREADS, 2) ag_gemm_silu(const Params p) {
  __shared__ __attribute__((aligned(16))) char smem[LDS_BYTES];
  const int xcd = (xcc_id() - 4) & 7, slot = blockIdx.x >> 3;
  u32x4 plan;
  {
    unsigned l0, l1;
    asm volatile("s_load_dword %1, %3, 0\n\ts_load_dword %2, %3, 64\n\ts_load_dwordx4 %0, %3, %4\n\ts_waitcnt lgkmcnt(0)"
                 : "=&s"(plan), "=&s"(l0), "=&s"(l1)
                 : "s"(__builtin_amdgcn_kernarg_segment_ptr()),
                   "s"((unsigned)(offsetof(Params, plan) + 16 * (slot < GSLOTS ? slot * 8 + xcd : GSLOTS * 8 + slot - GSLOTS))) : "memory");
  }
  const int rank = p.rank & 7, a = rank & 3;
  const int wave = __builtin_amdgcn_readfirstlane(threadIdx.x >> 6);
  const unsigned before = __hip_atomic_load(call_word(p), __ATOMIC_RELAXED, __HIP_MEMORY_SCOPE_AGENT);
  const bool puller = slot >= GSLOTS && xcd < 7;
  unsigned call;
  if (!puller && (plan[0] & 0x4000u)) call = convert_x(p, plan[0], before, rank, (LAS char*)smem);
  else if (!puller && (plan[0] & 0x8000u)) call = convert_w(p, plan[0], before, rank, (LAS char*)smem);
  else call = begin(p, before, (LAS char*)smem, puller);
  __builtin_amdgcn_s_waitcnt(0x0F70);

  if (puller) {
    const int w = slot - GSLOTS;
    if (xcd < 4) pull<true>(p, w, w < (int)((WMASK >> (4 * xcd)) & 15), wave, call, rank, ((rank & 4) ^ 4) | ((a + xcd) & 3), (LAS char*)smem);
    else pull<false>(p, w, w < (int)((WMASK >> (4 * xcd)) & 15), wave, call, rank, (rank & 4) | ((a + xcd - 3) & 3), (LAS char*)smem);
    if (wave == 0) STAMP(2);
    return;
  }
  if (wave == 0) STAMP(1);

  for (int job = 0; job < NJOBS; ++job) {
    const unsigned code = job == 0 ? plan[1] : job == 1 ? plan[2] : plan[3];
    if (!(code & 0x8000u)) break;
    if (EXPON(4) && (job > 0 || slot >= 12)) break;
    if (EXPON(8) && job > 0) break;
    Job jb;
    decode_job(code, a, jb);
    const int kind = (code >> 12) & 3;
    if (kind == 3 && (code & 0x200) && wave == 0) STAMP(21 + (code & 63) / 6);
    if (wave >= 4) {
      if (kind & 1) load_wave<1>(p, (LAS char*)smem, jb, job, code, wave - 4, kind < 2, rank, call);
      else load_wave<2>(p, (LAS char*)smem, jb, job, code, wave - 4, kind < 2, rank, call);
    } else if (kind == 0) {
      mfma_wave<true, 2>(p, (LAS char*)smem, jb, job, wave, rank, call);
    } else if (kind == 1) {
      mfma_wave<true, 1>(p, (LAS char*)smem, jb, job, wave, rank, call);
    } else if (kind == 2) {
      mfma_wave<false, 2>(p, (LAS char*)smem, jb, job, wave, rank, call);
    } else {
      mfma_wave<false, 1>(p, (LAS char*)smem, jb, job, wave, rank, call);
    }
  }
}

void make_schedule(Params& p) {
  int n[NGEMM] = {};
  bool early[NGEMM] = {};
  for (int g = 0; g < NGEMM; ++g) p.plan[g] = (u32x4){0u, 0u, 0u, 0u};
  int team[32], nteam = 0;
  auto id = [](int xcd, int s) { return s < GSLOTS ? s * 8 + xcd : GSLOTS * 8 + s - GSLOTS; };
  auto put = [&](int xcd, int s, int kind, int ra, int b, int ah = 0, int bh = 0) {
    const int g = id(xcd, s);
    if (n[g] >= NJOBS || nteam >= 32) return;
    p.plan[g][++n[g]] = 0x8000 | (kind << 12) | (bh << 11) | (ah << 10) | (b << 6) | ra;
    team[nteam++] = g;
  };
  auto close = [&]() {
    for (int side = 0; side < 2; ++side)
      for (int i = 0; i < nteam; ++i) {
        const unsigned mask = side ? 0xbc0u : 0x43fu;
        int cnt = 0, idx = 0;
        for (int k = 0; k < nteam; ++k)
          if (((p.plan[team[i]][n[team[i]]] ^ p.plan[team[k]][n[team[k]]]) & mask) == 0) { idx += k < i; ++cnt; }
        p.plan[team[i]][n[team[i]]] |= (unsigned)((cnt - 1) | (idx << 3)) << (16 + 6 * side);
      }
    nteam = 0;
  };
  auto P = [&](int xcd, int s0, int kind, int t, int b0) {
    for (int s = 0; s < 12; ++s) {
      if (kind < 2) put(xcd, s0 + s, kind, s % 3 * 6 + t, b0 + s / 3 * (2 - kind));
      else if (kind == 2) put(xcd, s0 + s, kind, s % 3 * 6 + t, b0 + s / 6, (s / 3) & 1);
      else put(xcd, s0 + s, kind, s % 3 * 6 + t, b0, (s / 3) & 1, s / 6);
    }
    close();
  };
  auto L = [&](int xcd, int s0, int h, int b0) { for (int s = 0; s < 12; ++s) put(xcd, s0 + s, 0, 18 + h * 3 + s % 3, b0 + s / 3 * 2); close(); };
  auto Afw = [&](int xcd, int s0, int t) {
    for (int s = 0; s < 12; ++s) put(xcd, s0 + s, 0, PA_AWAY + (s & 3) * 2 + t, 8 + (((s & 3) + 1 + (s >> 2)) & 3) * 2);
    close();
  };
  auto Afs = [&](int xcd, int s0, int t, int h) {
    for (int s = h * 12; s < h * 12 + 12; ++s) put(xcd, s0 + s % 12, 1, PA_AWAY + ((s / 6 + 1 + s % 3) & 3) * 2 + t, 8 + s / 3);
    close();
  };
  auto Aow = [&](int xcd, int s0, int t, int c0, int cnt) {
    for (int s = 0; s < cnt; ++s) put(xcd, s0 + s, 0, PA_AWAY + ((c0 + s) & 3) * 2 + t, (c0 + s) / 4 * 2);
    if (c0 + cnt == 16) close();
  };
  auto E = [&](int xcd, int s0) { for (int s = 0; s < 12; ++s) early[id(xcd, s0 + s)] = true; };
#define SCHED 5
  auto Aos = [&](int xcd, int s0, int t, int h) {
    for (int s = h * 12; s < h * 12 + 12; ++s) put(xcd, s0 + s % 12, 1, PA_AWAY + (s & 3) * 2 + t, s / 4);
    close();
  };
  auto OA = [&](int xcd, int s0, int h) { for (int s = h * 8; s < h * 8 + 8; ++s) put(xcd, s0 + s % 8, 1, PA_OWNAWAY + (s & 1), s / 2); close(); };
  if (SCHED == 1) {
    P(0, 0, 0, 0, 0); P(1, 0, 0, 0, 8); L(2, 0, 0, 8); L(3, 0, 1, 8);
    E(0, 0); E(1, 0); E(2, 0); E(3, 0);
    L(0, 12, 0, 0); L(1, 12, 1, 0);
    P(5, 0, 0, 1, 0); P(6, 0, 0, 1, 8);
    OA(2, 12, 0); OA(3, 12, 1);
    P(4, 12, 0, 2, 0); P(5, 12, 0, 2, 8); Afw(6, 12, 0); Aow(7, 12, 0, 0, 16);
    P(0, 0, 1, 3, 8); P(1, 0, 1, 3, 12); Afs(2, 0, 1, 0); Afs(3, 0, 1, 1);
    P(4, 0, 0, 3, 0); Aow(7, 0, 1, 0, 12); Aow(7, 28, 1, 12, 4);
    for (int x = 0; x < 4; ++x) P(x, 12, 2, 4, 8 + 2 * x);
    P(5, 0, 1, 4, 0); P(6, 0, 1, 4, 4);
    for (int x = 0; x < 8; ++x) P(x, x < 4 ? 0 : 12, 3, 5, 8 + x);
    for (int x = 0; x < 4; ++x) P(x, 12, 2, 5, 2 * x);
  } else if (SCHED >= 4) {
    P(0, 0, 0, 0, 0); P(1, 0, 0, 0, 8); L(2, 0, 0, 8); L(3, 0, 1, 8); L(4, 0, 0, 0); L(5, 0, 1, 0);
    for (int x = 0; x < 6; ++x) E(x, 0);
    OA(6, 0, 0); OA(7, 0, 1);
    P(5, 12, 0, 1, 0); P(6, 12, 0, 1, 8);
    P(0, 12, 1, 2, 8); P(1, 12, 1, 2, 12); Afs(2, 12, 0, 0); Afs(3, 12, 0, 1); P(4, 12, 0, 2, 0); Aow(7, 12, 0, 0, 16);
    P(0, 0, 1, 3, 8); P(1, 0, 1, 3, 12); Afs(2, 0, 1, 0); Afs(3, 0, 1, 1);
    P(4, 0, 1, 3, 0); P(5, 0, 1, 3, 4); Aos(6, 0, 1, 0); Aos(7, 0, 1, 1); Aow(7, 28, 1, 12, 4);
    for (int x = 0; x < 4; ++x) P(x, 12, 2, 4, 8 + 2 * x);
    P(5, 12, 1, 4, 0); P(6, 12, 1, 4, 4);
    for (int x = 0; x < 4; ++x) P(x, 12, 3, 5, 4 + x);
    const int h0 = SCHED == 4 ? 5 : 0;
    P(h0, 0, 2, 5, 0); P(h0 + 1, 0, 2, 5, 2);
    int b = 8;
    for (int x = 0; x < 8; ++x) if (x != h0 && x != h0 + 1) P(x, 0, 3, 5, b++);
    P(4, 12, 3, 5, b++); P(7, 12, 3, 5, b++);
  } else {
    P(0, 0, 0, 0, 0); P(1, 0, 0, 0, 8); L(4, 0, 0, 8); L(5, 0, 1, 8);
    E(0, 0); E(1, 0); E(4, 0); E(5, 0);
    if (SCHED == 2) { L(0, 12, 0, 0); L(1, 12, 1, 0); OA(2, 0, 0); OA(3, 0, 1); }
    else { L(2, 0, 0, 0); L(3, 0, 1, 0); E(2, 0); E(3, 0); OA(0, 12, 0); OA(1, 12, 1); }
    P(2, 12, 0, 1, 0); P(3, 12, 0, 1, 8);
    P(4, 12, 0, 2, 0); P(5, 12, 0, 2, 8); Afw(6, 12, 0); Aow(7, 12, 0, 0, 16);
    P(0, 0, 1, 3, 8); P(1, 0, 1, 3, 12); Afs(2, 0, 1, 0); Afs(3, 0, 1, 1);
    P(4, 0, 1, 3, 0); P(5, 0, 1, 3, 4); Aos(6, 0, 1, 0); Aos(7, 0, 1, 1); Aow(7, 28, 1, 12, 4);
    P(0, 12, 1, 4, 8); P(1, 12, 1, 4, 12); P(2, 12, 1, 4, 0); P(3, 12, 1, 4, 4);
    for (int x = 0; x < 4; ++x) P(x, 0, 2, 5, 2 * x);
    for (int x = 0; x < 8; ++x) P(4 + x / 2, (x & 1) * 12, 3, 5, 8 + x);
  }
#define CVT_W 56
  int nx = 0, nw = 0, c = 0;
  for (int pass = 0; pass < 2; ++pass) {
    for (int g = 0; g < NGEMM; ++g) {
      const int xcd = g & 7, s = g >> 3, ra = p.plan[g][1] & 63;
      if (early[g]) {
        if (xcd < 4 && s < HW) p.plan[g][0] = 0x9000u | (s * 4 + ((p.rank + xcd) & 3));
        if (xcd >= 4 && s * 2 + xcd - 4 < HX) p.plan[g][0] = 0x5000u | (s * 2 + xcd - 4);
        continue;
      }
      const bool soon = ra >= PA_OWNAWAY || (ra < 18 && ra % 6 == 1);
      if (soon != (pass == 1)) continue;
      if (!soon && c++ * CVT_W % 88 < CVT_W) p.plan[g][0] = 0x8000u | (4 * HW + nw++);
      else p.plan[g][0] = 0x4000u | nx++;
    }
    if (pass == 0) p.nlong_x = (unsigned char)nx;
  }
  static_assert(HW <= 12 && HX <= 24, "");
  p.ncvt_x = (unsigned char)nx;
  p.ncvt_w = (unsigned char)nw;
}

}

extern "C" void kernel_launch(int rank, int nranks, ncclComm_t comm, hipStream_t stream,
                              void* const* d_in, const int* in_sizes, int n_in,
                              void* d_out, int out_size, const int* in_dtypes, int out_dtype,
                              void* const* d_out_all, void* const* d_ws_all, size_t ws_bytes) {
  (void)comm; (void)in_dtypes; (void)out_dtype; (void)d_out_all;
  if (nranks != NR || (unsigned)rank >= (unsigned)NR || n_in != 2 || in_sizes[0] != NSH * KDIM ||
      in_sizes[1] != KDIM * NSH || out_size != KDIM * NSH || ws_bytes < WS_NEED) {
    fprintf(stderr, "kernel_launch: unexpected problem shape\n");
    return;
  }
  Params p;
  p.x = (const float*)d_in[0];
  p.w = (const float*)d_in[1];
  p.rank = 0;
  for (int i = 0; i < NR; ++i) {
    const int r = (RANK_AT >> (4 * i)) & 7;
    p.ws[i] = (char*)d_ws_all[r];
    if (r == rank) p.rank = i;
  }
  p.mine = (char*)d_ws_all[rank];
  p.out = (float*)d_out;
  make_schedule(p);
  ag_gemm_silu<<<dim3(NWG), dim3(NTHREADS), 0, stream>>>(p);
  HIP_CHECK(hipGetLastError());
}
```

```cpp
#include <hip/hip_runtime.h>
#include <rccl/rccl.h>
#include <cstdio>
#include <cstdint>
#include <cstddef>

#define HIP_CHECK(expr) do { hipError_t _e = (expr); if (_e != hipSuccess) { \
    fprintf(stderr, "HIP error %s at %s:%d\n", hipGetErrorString(_e), __FILE__, __LINE__); \
    return; } } while (0)

namespace {

typedef _Float16 half8 __attribute__((ext_vector_type(8)));
typedef float f32x4 __attribute__((ext_vector_type(4)));
typedef unsigned u32x4 __attribute__((ext_vector_type(4)));
typedef unsigned u32x2 __attribute__((ext_vector_type(2)));
#define LAS __attribute__((address_space(3)))
#define GAS __attribute__((address_space(1)))

constexpr int NR = 8;
#define RANK_AT_V 0x75416320u
constexpr unsigned RANK_AT = RANK_AT_V;
constexpr int KDIM = 8192;
constexpr int NSH = 1024;
constexpr int KSTEPS = 128;
constexpr unsigned STEP_B = 16384;
constexpr size_t PANEL_B = (size_t)KSTEPS * STEP_B;
constexpr int PA_AWAY = 24, PA_OWNAWAY = 32, PB = 34, PB_FOREIGN = 42, NPANEL = 50;

constexpr unsigned BLK_B = 1664;
constexpr unsigned PSTEP_B = 1024 + 8 * BLK_B;
constexpr size_t PPANEL_B = (size_t)KSTEPS * PSTEP_B;

constexpr size_t OFF_CALL = 0;
constexpr size_t OFF_TMO = 4096;
constexpr size_t OFF_TICKET = 5120;
#define FCOPY 8
constexpr size_t OFF_FLAG = 1u << 18;
constexpr size_t OFF_RDY = 1u << 20;
constexpr int RDY_N = 1024, RDY_AWAY = 128, RDY_OUT = 384, RDY_HOP1 = 864, RDY_HOP2 = 896;
#define RELAY 32
#define FILL 0
#define HOPF 1
constexpr size_t OFF_DBG = 65536;
constexpr size_t OFF_OWN = 2u << 20;
constexpr size_t OFF_PACK = OFF_OWN + 32 * PANEL_B;
constexpr size_t OFF_RELAY = OFF_PACK + 32 * PPANEL_B;
constexpr size_t OFF_PULL = OFF_RELAY + PANEL_B;
#define HX 20
#define HW 6
constexpr int OUT_CHUNKS = 480;
#define CSTRIDE 8192
constexpr unsigned SUM_B = 3456, CHUNK_B = CSTRIDE;
constexpr size_t OFF_STG = OFF_PULL + 2 * NPANEL * PANEL_B;
constexpr size_t OFF_SPARE = OFF_STG + (size_t)4 * OUT_CHUNKS * CHUNK_B;
constexpr size_t WS_NEED = OFF_SPARE + (size_t)4 * OUT_CHUNKS * 2048;
static_assert(RELAY % 4 == 0 && RELAY <= 64, "");
static_assert(NPANEL <= 64 && OFF_FLAG + FCOPY * 65536 <= OFF_RDY && OFF_RDY + NR * RDY_N * 128 <= OFF_OWN, "");

constexpr int NWG = 256, NTHREADS = 512;
constexpr int GSLOTS = 24;
constexpr int NGEMM = GSLOTS * 8 + 8;
constexpr int PW = 8;
#define WIN 5
#define WIN1 6
#define WMASK 0
constexpr int NJOBS = 3;
constexpr unsigned long long DEADLINE = 100000000ull;

constexpr unsigned LDS_MB = 18 * 8192;
constexpr unsigned LDS_TMO = LDS_MB + NJOBS * 4 * 512;
constexpr unsigned LDS_LUT = LDS_TMO + NJOBS * 16;
constexpr unsigned LDS_FILL = LDS_LUT + 64;
constexpr unsigned LDS_CALL = LDS_FILL + 16;
constexpr unsigned LDS_EPI = LDS_CALL + 16;
constexpr unsigned LDS_BYTES = LDS_EPI + 4 * 512;
constexpr unsigned WAVE_LDS = LDS_MB / 8;

struct Params {
  const float* x;
  const float* w;
  char* mine;
  int rank;
  unsigned counts;
  float* out;
  char* ws[NR];
  u32x4 plan[NGEMM];
};

__device__ __forceinline__ int xcc_id() {
  return __builtin_amdgcn_s_getreg(20 | (3 << 11)) & 7;
}
#define STAMP(idx) do {} while (0)

#define EXP 0
#define EXPON(bit) ((EXP & (bit)) && call > 2)
__device__ __forceinline__ void store_sys(void* ptr, u32x4 v) {
  asm volatile("global_store_dwordx4 %0, %1, off sc0 sc1\n\ts_nop 1" :: "v"(ptr), "v"(v) : "memory");
}
__device__ __forceinline__ void wait_vm0() { asm volatile("s_waitcnt vmcnt(0)" ::: "memory"); }
__device__ __forceinline__ void wg_barrier() {
  asm volatile("" ::: "memory");
  __builtin_amdgcn_s_barrier();
  asm volatile("" ::: "memory");
}
#define OUT_NT " nt"
__device__ __forceinline__ void store_out(void* ptr, u32x4 v) {
  asm volatile("global_store_dwordx4 %0, %1, off sc0 sc1" OUT_NT "\n\ts_nop 1" :: "v"(ptr), "v"(v) : "memory");
}

__device__ __forceinline__ void set_local(char* ws, int copy, int step, unsigned v) {
  __hip_atomic_store((GAS unsigned*)(ws + OFF_FLAG) + copy * 16384 + (step >> 7) * 256 + (step & 7) * 32 + ((step & 127) >> 3), v,
                     __ATOMIC_RELAXED, __HIP_MEMORY_SCOPE_SYSTEM);
}
__device__ __forceinline__ void set_rdy(char* peer_ws, int rank, int idx, unsigned call, unsigned lines) {
  __hip_atomic_store((GAS unsigned*)(peer_ws + OFF_RDY) + (rank * RDY_N + idx) * 32, call << 6 | lines, __ATOMIC_RELAXED,
                     __HIP_MEMORY_SCOPE_SYSTEM);
}
__device__ __forceinline__ unsigned own_idx(unsigned call, int idx) { return (call & 1) * 16 + idx; }
__device__ __forceinline__ size_t pulled(unsigned call) { return OFF_PULL + (call & 1) * (NPANEL * PANEL_B); }
__device__ __forceinline__ int fresh_lane() {
  int l;
  asm volatile("v_mbcnt_lo_u32_b32 %0, -1, 0\n\tv_mbcnt_hi_u32_b32 %0, -1, %0" : "=v"(l));
  return l;
}
__device__ __forceinline__ void cbar() { asm volatile("" ::: "memory"); }
__device__ __forceinline__ unsigned wave_scan(unsigned v) {
  v += __builtin_amdgcn_update_dpp(0u, v, 0x111, 0xf, 0xf, false);
  v += __builtin_amdgcn_update_dpp(0u, v, 0x112, 0xf, 0xf, false);
  v += __builtin_amdgcn_update_dpp(0u, v, 0x114, 0xf, 0xf, false);
  v += __builtin_amdgcn_update_dpp(0u, v, 0x118, 0xf, 0xf, false);
  v += __builtin_amdgcn_update_dpp(0u, v, 0x142, 0xa, 0xf, false);
  v += __builtin_amdgcn_update_dpp(0u, v, 0x143, 0xc, 0xf, false);
  return v;
}

__device__ __forceinline__ unsigned codec_lut(int m, bool expand) {
  unsigned sel = 0x0c0c0c0cu;
  int n = 0;
  for (int i = 0; i < 4; ++i)
    if ((m >> i) & 1) {
      const int at = expand ? i : n, v = expand ? n : i;
      sel = (sel & ~(0xffu << (8 * at))) | ((unsigned)v << (8 * at));
      ++n;
    }
  return sel;
}

__device__ __forceinline__ unsigned lut_at(const LAS char* lut, unsigned E) {
  return *(const LAS unsigned*)(lut + __builtin_amdgcn_udot4(E, 0x08040201u, 0u, false));
}
__device__ __forceinline__ void lds_or(LAS unsigned* q, unsigned v) {
  __hip_atomic_fetch_or(q, v, __ATOMIC_RELAXED, __HIP_MEMORY_SCOPE_WORKGROUP);
}
struct Codes {
  unsigned pk[4], n[4], EM = 0, XM = 0;
  __device__ __forceinline__ void add(int k, const LAS char* lut, unsigned c, unsigned E, unsigned X) {
    const unsigned z = __builtin_amdgcn_perm(0u, c, lut_at(lut, E));
    pk[k] = __builtin_amdgcn_udot4(z, 0x00400801u, (z >> 24) << 9, false);
    n[k] = __builtin_popcount(E);
    EM |= (E >> 2) << k;
    XM |= X >> (7 - k);
  }
  __device__ __forceinline__ void put(LAS char* list, LAS char* patch, unsigned pre, const unsigned (&H)[4], unsigned base) {
    const unsigned long long W = (unsigned long long)(pk[0] | (pk[1] << (3 * n[0])))
        | ((unsigned long long)(pk[2] | (pk[3] << (3 * n[2]))) << (3 * (n[0] + n[1])));
    const unsigned o = 3 * pre, sh = o & 31u;
    LAS unsigned* q = (LAS unsigned*)(list + (o >> 5) * 4);
    const unsigned long long Ws = W << sh;
    lds_or(q, (unsigned)Ws);
    lds_or(q + 1, (unsigned)(Ws >> 32));
    lds_or(q + 2, ((unsigned)(W >> 32) >> 1) >> (31u - sh));
    cbar();
    const unsigned H0 = H[0], H1 = H[1], H2 = H[2], H3 = H[3];
    for (; XM; XM &= XM - 1u, patch -= 2) {
      const unsigned pos = __builtin_ctz(XM), k = pos & 3u, i8 = pos & 24u;
      const unsigned Hk = k & 2u ? (k & 1u ? H3 : H2) : (k & 1u ? H1 : H0);
      const unsigned kb = 0x01010101u << k;
      const unsigned rk = pre + __builtin_popcount(EM & ((kb - 0x01010101u) | (kb & ((1u << i8) - 1u))));
      const unsigned at = (unsigned)(uintptr_t)patch;
      lds_or((LAS unsigned*)(uintptr_t)(at & ~3u), ((rk << 5) | (((Hk >> (i8 + 2)) & 31u) ^ (base + 3u))) << ((at & 2u) * 8));
    }
    cbar();
  }
};

__device__ __forceinline__ void pack_block(u32x4 f0, u32x4 f1, LAS char* wl, const LAS char* lut, LAS char* lists,
                                           LAS unsigned* fill, char* step, int blk) {
  const int lane = threadIdx.x & 63;
  const unsigned h[8] = {f0[0], f0[1], f0[2], f0[3], f1[0], f1[1], f1[2], f1[3]};
  char* const dst = step + 1024 + blk * BLK_B;
  u32x4 D;
  unsigned H[4], t[4], sum = 0, sg = 0;
#pragma unroll
  for (int k = 0; k < 4; ++k) {
    D[k] = __builtin_amdgcn_perm(h[2 * k + 1], h[2 * k], 0x06040200u);
    H[k] = __builtin_amdgcn_perm(h[2 * k + 1], h[2 * k], 0x07050301u);
    sum = __builtin_amdgcn_udot4(H[k] & 0x7c7c7c7cu, 0x01010101u, sum, false);
    sg |= (H[k] & 0x80808080u) >> (7 - k);
  }
  store_sys(dst + lane * 16, D);
  const unsigned sgn = __builtin_amdgcn_update_dpp(0u, sg, 0xb1, 0xf, 0xf, false);
  if (!(lane & 1)) *(LAS unsigned*)(wl + 512 + lane * 2) = sg | (sgn << 4);
  const unsigned tot = __builtin_amdgcn_readlane(wave_scan(sum), 63);
  const unsigned base = min((tot + 256u) >> 12, 28u);
  Codes c;
  unsigned W = 0, n2 = 0;
#pragma unroll
  for (int k = 0; k < 4; ++k) {
    const unsigned g = ((H[k] >> 2) & 0x1f1f1f1fu) + (0x43u - base) * 0x01010101u;
    const unsigned x = g ^ 0x40404040u, y = (g + 0x07070707u) ^ 0x40404040u;
    const unsigned tt = __builtin_amdgcn_perm(0xff1c0804u, 0x001e1dffu, x | (((x & 0x78787878u) + 0x78787878u) & 0x80808080u));
    t[k] = (tt & 0x0c0c0c0cu) | (H[k] & 0x03030303u);
    const unsigned E = (tt >> 2) & 0x04040404u;
    W |= __builtin_amdgcn_udot4(__builtin_amdgcn_perm(0u, tt & 0x03030303u, lut_at(lut, E)), 0x40100401u, 0u, false) << (2 * n2);
    n2 += __builtin_popcount(E);
    c.add(k, lut, y & 0x07070707u, (tt >> 3) & 0x04040404u, ((y & 0x78787878u) + 0x78787878u) & tt & 0x80808080u);
  }
  *(LAS u32x2*)(wl + lane * 8) = (u32x2){t[0] | (t[1] << 4), t[2] | (t[3] << 4)};
  const unsigned mine = n2 | ((c.n[0] + c.n[1] + c.n[2] + c.n[3]) << 11) | (__builtin_popcount(c.XM) << 22);
  const unsigned incl = wave_scan(mine), pre = incl - mine;
  const unsigned all = __builtin_amdgcn_readlane(incl, 63), n4 = all >> 22, l2 = ((all & 2047u) + 15u) >> 4;
  const unsigned len = l2 + ((3u * ((all >> 11) & 2047u) + 16u * n4 + 31u) >> 5);
  unsigned head = 1u << 31, off = 0;
  cbar();
  if (l2 <= 32u && len < 64u && n4 < 64u) {
    asm volatile("" : "+v"(off));
    if (lane == 0) off = __hip_atomic_fetch_add(fill + off, len, __ATOMIC_RELAXED, __HIP_MEMORY_SCOPE_WORKGROUP);
    off = __builtin_amdgcn_readfirstlane(off);
    if (off + len < 256u) {
      head = base | (n4 << 5) | (l2 << 11) | (len << 17) | (off << 23);
      LAS char* const esc = lists + off * 4;
      const unsigned sh = (2u * pre) & 31u;
      LAS unsigned* q = (LAS unsigned*)esc + ((pre & 2047u) >> 4);
      lds_or(q, W << sh);
      lds_or(q + 1, (W >> 1) >> (31u - sh));
      c.put(esc + l2 * 4, esc + len * 4 - 2 - 2 * (pre >> 22), (pre >> 11) & 2047u, H, base);
    }
  }
  if (lane == 0) ((LAS unsigned*)lists)[blk] = head;
  cbar();
}
__device__ __forceinline__ void pack_flush(LAS char* wl, LAS char* lists, char* step, int blk) {
  const int lane = threadIdx.x & 63;
  if (lane < 48) {
    const unsigned line = blk * 128 + (lane - 40) * 16;
    LAS u32x4* const from = (LAS u32x4*)(lane < 40 ? wl + lane * 16 : lists + line);
    const u32x4 v = *from;
    *from = (u32x4){0u, 0u, 0u, 0u};
    store_sys(step + (lane < 40 ? 2048 + blk * BLK_B + lane * 16 : line), v);
  }
}
__device__ __forceinline__ void lds_barrier() {
  asm volatile("s_waitcnt lgkmcnt(0)\n\ts_barrier" ::: "memory");
}
__device__ __forceinline__ unsigned lines_used(LAS unsigned* fill) {
  const unsigned n = *fill;
  *fill = 8u;
  return min(n - 1u, 255u) >> 5;
}

constexpr unsigned SUM_BASE = 24;
__device__ __forceinline__ void pack_sums(int lane, const f32x4 (&a)[4], LAS char* wl, const LAS char* lut, char* dst, char* spare) {
  if (lane < 36) *(LAS unsigned*)(wl + 256 + lane * 4) = 0u;
  u32x4 P[3], E;
  unsigned H[4], t[4], sum = 0, any = 0;
#pragma unroll
  for (int k = 0; k < 4; ++k) {
    unsigned v[4];
#pragma unroll
    for (int i = 0; i < 4; ++i) v[i] = __builtin_rotateleft32(__builtin_bit_cast(u32x4, a[i])[k], 1);
#pragma unroll
    for (int i = 0; i < 3; ++i) P[i][k] = __builtin_amdgcn_perm(v[3], v[i], 0x04020100u + ((unsigned)i << 24));
    E[k] = __builtin_amdgcn_perm(v[1], v[0], 0x0c0c0703u) | __builtin_amdgcn_perm(v[3], v[2], 0x07030c0cu);
    sum = __builtin_amdgcn_udot4(E[k], 0x01010101u, sum, false);
  }
#pragma unroll
  for (int i = 0; i < 3; ++i) store_sys(dst + i * 1024 + lane * 16, P[i]);
  const unsigned tot = __builtin_amdgcn_readlane(wave_scan(sum), 63);
  const unsigned off = min(max((tot + 64u) >> 10, SUM_BASE) - SUM_BASE, 223u);
#pragma unroll
  for (int k = 0; k < 4; ++k) {
    const unsigned d = E[k] - off * 0x01010101u;
    any |= d;
    H[k] = d << 2;
  }
  Codes c;
#pragma unroll
  for (int k = 0; k < 4; ++k) {
    const unsigned g = ((H[k] & 0x7c7c7c7cu) | 0x80808080u) - SUM_BASE * 0x04040404u;
    const unsigned a7 = g & 0x7f7f7f7fu, hi = a7 + 0x70707070u, lo = a7 + 0x1c1c1c1cu;
    const unsigned o7 = (~g | hi) & 0x80808080u;
    t[k] = (g | (o7 >> 4) | (o7 >> 5)) & 0x0c0c0c0cu;
    const unsigned nb = ~g & 0x80808080u;
    c.add(k, lut, (g >> 2) & ((nb >> 4) - (nb >> 7)), t[k] & (t[k] >> 1), ((g & hi) | (~g & ~lo)) & 0x80808080u);
  }
  const unsigned mine = (c.n[0] + c.n[1] + c.n[2] + c.n[3]) | (__builtin_popcount(c.XM) << 16);
  const unsigned incl = wave_scan(mine), pre = incl - mine;
  const unsigned all = __builtin_amdgcn_readlane(incl, 63), n4 = all >> 16;
  const bool raw = __builtin_amdgcn_ballot_w64((any & 0xe0e0e0e0u) != 0u) != 0ull || 3u * (all & 0xffffu) + 16u * n4 > 1008u;
  cbar();
  if (!raw) c.put(wl + 256, wl + 380 - 2 * (pre >> 16), pre & 0xffffu, H, SUM_BASE);
  if (lane == 0) *(LAS unsigned short*)(wl + 382) = (unsigned short)(raw ? 0x8000u : (off & 31u) | (n4 << 5) | (off >> 5 << 11));
  *(LAS unsigned*)(wl + lane * 4) = (t[0] >> 2) | t[1] | (t[2] << 2) | (t[3] << 4);
  cbar();
  if (lane < 24) store_sys(dst + 3072 + lane * 16, *(const LAS u32x4*)(wl + lane * 16));
  if (raw) store_sys(spare + lane * 16, E);
  cbar();
}

__device__ __forceinline__ u32x2 read_codes(int lane, const LAS char* list) {
  const LAS unsigned* sp = (const LAS unsigned*)(list + ((3 * lane) & ~3));
  return (u32x2){sp[0], sp[1]};
}
__device__ __forceinline__ void expand_codes(int lane, u32x2 codes, unsigned base, unsigned first, unsigned keep,
                                             bool patch, unsigned ent, LAS char* ex) {
  __builtin_amdgcn_sched_barrier(0);
  const unsigned x = __builtin_amdgcn_alignbyte(codes[1], codes[0], 3 * lane);
  const unsigned b3 = (base + 3u) * 0x04040404u, b8 = (base + first) * 0x04040404u;
  const unsigned T0 = ((b8 + 0x0c080400u) ^ b3) & keep, T1 = ((b8 + 0x1c181410u) ^ b3) & 0x7c7c7c7cu;
  u32x2 X;
#pragma unroll
  for (int i = 0; i < 2; ++i) {
    const unsigned y = ((x >> (12 * i)) & 0x3fu) | ((i ? x >> 2 : x << 10) & 0x3f0000u);
    X[i] = __builtin_amdgcn_perm(T1, T0, (y | (y << 5)) & 0x07070707u);
  }
  *(LAS u32x2*)(ex + lane * 8) = X;
  cbar();
  if (patch) *(LAS unsigned char*)(ex + ((ent >> 5) & 511u)) = (unsigned char)((ent & 31u) << 2);
  cbar();
}
template <int N>
__device__ __forceinline__ void pick(const unsigned (&E)[N], const unsigned (&sel)[N], const LAS char* ex, unsigned pos,
                                     unsigned (&out)[N]) {
  unsigned at[N], w0[N], w1[N];
#pragma unroll
  for (int k = 0; k < N; ++k) { at[k] = pos; pos += __builtin_popcount(E[k]); }
  __builtin_amdgcn_sched_barrier(0);
#pragma unroll
  for (int k = 0; k < N; ++k) {
    const LAS unsigned* wp = (const LAS unsigned*)(ex + (at[k] & ~3u));
    w0[k] = wp[0];
    w1[k] = wp[1];
  }
  __builtin_amdgcn_sched_barrier(0);
#pragma unroll
  for (int k = 0; k < N; ++k) out[k] = __builtin_amdgcn_perm(0u, __builtin_amdgcn_alignbyte(w1[k], w0[k], at[k]), sel[k]);
}

__device__ __forceinline__ void unpack_block(int lane, const LAS char* step, int blk, unsigned hdr, LAS char* ex,
                                             const LAS char* lut, u32x4& f0, u32x4& f1) {
  const LAS char* const src = step + 1024 + blk * BLK_B;
  const LAS char* const esc = step + (hdr >> 23) * 4;
  const unsigned base = hdr & 31u;
  const bool patch = lane < ((hdr >> 5) & 63u);
  const u32x2 x3 = read_codes(lane, esc + ((hdr >> 11) & 63u) * 4);
  const unsigned ent = *(const LAS unsigned short*)(esc + ((hdr >> 17) & 63u) * 4 - (patch ? 2 + 2 * lane : 0));
  const unsigned x2 = *(const LAS unsigned short*)(esc + 2 * lane);
  const u32x2 N = *(const LAS u32x2*)(src + 1024 + lane * 8);
  unsigned sg = *(const LAS unsigned*)(src + 1536 + (lane >> 1) * 4);
  const u32x4 D = *(const LAS u32x4*)(src + lane * 16);
  expand_codes(lane, x3, base, 22u, 0x7c7c7c7cu, patch, ent, ex + 512);
  sg >>= (lane & 1) * 4;
  const unsigned E0 = N[0] & (N[0] >> 1) & 0x44444444u, E1 = N[1] & (N[1] >> 1) & 0x44444444u;
  unsigned c[2], F[2], sf[2], X[2], E[4], se[4], Y[4], h[8];
#pragma unroll
  for (int i = 0; i < 2; ++i) {
    const unsigned y = ((x2 >> (8 * i)) & 0xfu) | ((x2 << (12 - 8 * i)) & 0xf0000u);
    c[i] = (y | (y << 6)) & 0x03030303u;
    F[i] = (c[i] & (c[i] >> 1)) << 2;
    sf[i] = lut_at(lut, F[i]);
  }
#pragma unroll
  for (int k = 0; k < 4; ++k) {
    E[k] = ((k & 2 ? E1 : E0) >> (4 * (k & 1))) & 0x04040404u;
    se[k] = lut_at(lut, E[k]);
  }
  const unsigned mine = (__builtin_popcount(E0) + __builtin_popcount(E1)) | ((__builtin_popcount(F[0]) + __builtin_popcount(F[1])) << 16);
  const unsigned pre = wave_scan(mine) - mine;
  pick<2>(F, sf, ex + 512, pre >> 16, X);
  const unsigned T = ((base * 0x00040400u + 0x007c7800u) ^ ((base + 3u) * 0x00040400u)) & 0x007c7c00u;
  *(LAS u32x2*)(ex + lane * 8) = (u32x2){X[0] | __builtin_amdgcn_perm(0u, T, c[0]), X[1] | __builtin_amdgcn_perm(0u, T, c[1])};
  cbar();
  pick<4>(E, se, ex, pre & 0xffffu, Y);
#pragma unroll
  for (int k = 0; k < 4; ++k) {
    const unsigned t = ((N[k >> 1] >> (4 * (k & 1))) & 0x0f0f0f0fu) + base * 0x04040404u;
    const unsigned H = (t ^ Y[k]) | ((sg << (7 - k)) & 0x80808080u);
    h[2 * k] = __builtin_amdgcn_perm(H, D[k], 0x05010400u);
    h[2 * k + 1] = __builtin_amdgcn_perm(H, D[k], 0x07030602u);
  }
  cbar();
  f0 = (u32x4){h[0], h[1], h[2], h[3]};
  f1 = (u32x4){h[4], h[5], h[6], h[7]};
}

__device__ __forceinline__ void unpack_sums(int lane, const LAS char* blk, LAS char* ex, const LAS char* lut, const char* spare,
                                            f32x4 (&y)[4]) {
  const unsigned ent = *(const LAS unsigned short*)(blk + 3454 - 2 * lane);
  const u32x2 x3 = read_codes(lane, blk + 3328);
  const unsigned C = *(const LAS unsigned*)(blk + 3072 + lane * 4);
  u32x4 P[3], X;
#pragma unroll
  for (int i = 0; i < 3; ++i) P[i] = *(const LAS u32x4*)(blk + i * 1024 + lane * 16);
  const unsigned hdr = __builtin_amdgcn_readfirstlane(ent);
  if (hdr & 0x8000u) {
    asm volatile("global_load_dwordx4 %0, %1, off sc0 sc1\n\ts_waitcnt vmcnt(0)" : "=&v"(X) : "v"(spare + lane * 16) : "memory");
  } else {
    expand_codes(lane, x3, SUM_BASE, 24u, 0x7c7c7c00u, (unsigned)(lane - 1) < ((hdr >> 5) & 63u), ent, ex);
    unsigned t[4], E[4], se[4], Y[4];
#pragma unroll
    for (int k = 0; k < 4; ++k) {
      t[k] = (k ? C >> (2 * k - 2) : C << 2) & 0x0c0c0c0cu;
      E[k] = t[k] & (t[k] >> 1);
      se[k] = lut_at(lut, E[k]);
    }
    const unsigned mine = __builtin_popcount(C & (C >> 1) & 0x55555555u);
    pick<4>(E, se, ex, wave_scan(mine) - mine, Y);
    const unsigned off = ((hdr & 31u) | (hdr >> 11 << 5)) * 0x01010101u;
#pragma unroll
    for (int k = 0; k < 4; ++k) X[k] = (((t[k] + SUM_BASE * 0x04040404u) ^ Y[k]) >> 2) + off;
    cbar();
  }
#pragma unroll
  for (int k = 0; k < 4; ++k) {
    unsigned v[4];
#pragma unroll
    for (int i = 0; i < 3; ++i) v[i] = __builtin_amdgcn_perm(X[k], P[i][k], 0x04020100u + ((unsigned)i << 24));
    v[3] = __builtin_amdgcn_perm(P[1][k], P[0][k], 0x0c0c0703u) | __builtin_amdgcn_perm(X[k], P[2][k], 0x07030c0cu);
#pragma unroll
    for (int i = 0; i < 4; ++i) y[k][i] = __builtin_bit_cast(float, __builtin_rotateright32(v[i], 1));
  }
}

__device__ __forceinline__ float silu(float y) {
  const float c = fminf(fmaxf(y, -60.0f), 60.0f);
  return y * __builtin_amdgcn_rcpf(1.0f + __builtin_amdgcn_exp2f(c * -1.44269504088896341f));
}

__device__ __forceinline__ unsigned* call_word(const Params& p) { return (unsigned*)(p.mine + OFF_CALL) + blockIdx.x; }
template <int LOADS>
__device__ __forceinline__ unsigned begin(const Params& p, LAS char* smem, bool puller) {
  *(LAS u32x4*)(smem + LDS_MB + min(threadIdx.x * 16u, LDS_LUT - LDS_MB - 16u)) = (u32x4){0u, 0u, 0u, 0u};
  if (threadIdx.x < 20) ((LAS unsigned*)(smem + LDS_LUT))[threadIdx.x] = threadIdx.x < 16 ? codec_lut(threadIdx.x, puller) : 8u;
  asm volatile("s_waitcnt vmcnt(%0) lgkmcnt(0)" :: "n"(LOADS) : "memory");
  if (threadIdx.x == 0) STAMP(0);
  wg_barrier();
  return __builtin_amdgcn_readfirstlane(*(const LAS unsigned*)(smem + LDS_CALL)) + 1u;
}
static_assert(LDS_LUT - LDS_MB <= NTHREADS * 16 && (LDS_LUT - LDS_MB) % 16 == 0, "");

constexpr int XJ = (KSTEPS - HX) / 2;
static_assert(HX % 2 == 0, "");
__device__ __forceinline__ int x_unit(int u) {
  if (u < FILL) return 6 * KSTEPS + u;
  u -= FILL;
  constexpr int HF = HOPF ? RELAY : 0;
  if (u < HF) return 5 * KSTEPS + u;
  const int j = u < 7 * HF ? (u - HF) / 6 : u / 7, r = u < 7 * HF ? u - HF - j * 6 : u - j * 7;
  if (u < 7 * KSTEPS && (r >= 2 || j < XJ)) return r >= 2 ? (r - 1) * KSTEPS + j : HX + 2 * j + r;
  const int i = (u < 7 * KSTEPS ? 2 * j + r : u - 5 * KSTEPS) - 2 * XJ + FILL;
  return i < 2 * FILL ? 7 * KSTEPS + i - FILL : (6 + (i & 1)) * KSTEPS + (i >> 1);
}
#define XROUNDS 11
#define XLONG 5
__device__ __forceinline__ void load_in(const float* a, const float* b, unsigned dst) {
  unsigned keep;
  asm volatile("s_mov_b32 %0, m0\n\ts_mov_b32 m0, %3\n\ts_nop 0\n\tglobal_load_lds_dwordx4 %1, off nt\n\t"
               "s_add_u32 m0, m0, 1024\n\ts_nop 0\n\tglobal_load_lds_dwordx4 %2, off nt\n\ts_mov_b32 m0, %0"
               : "=&s"(keep) : "v"(a), "v"(b), "s"(dst) : "memory", "scc");
}
template <int N>
__device__ __forceinline__ void wait_vm() { asm volatile("s_waitcnt vmcnt(%0)" :: "n"(N) : "memory"); }
__device__ __forceinline__ u32x4 cvt8(f32x4 a, f32x4 b) {
  half8 h;
  h[0] = (_Float16)a[0]; h[1] = (_Float16)a[1]; h[2] = (_Float16)a[2]; h[3] = (_Float16)a[3];
  h[4] = (_Float16)b[0]; h[5] = (_Float16)b[1]; h[6] = (_Float16)b[2]; h[7] = (_Float16)b[3];
  u32x4 r = __builtin_bit_cast(u32x4, h);
  asm volatile("" : "+v"(r));
  return r;
}
__device__ __forceinline__ char* member(const Params& p, int group, int i) {
  char *a = p.ws[group], *b = p.ws[group + 1], *c = p.ws[group + 2], *d = p.ws[group + 3];
  asm volatile("" : "+s"(a), "+s"(b), "+s"(c), "+s"(d));
  return i & 2 ? (i & 1 ? d : c) : (i & 1 ? b : a);
}
__device__ __forceinline__ unsigned convert_x(const Params& p, unsigned role, unsigned counts, int rank, LAS char* smem) {
  const int lane = threadIdx.x & 63, wave = __builtin_amdgcn_readfirstlane(threadIdx.x >> 6), t = threadIdx.x;
  const int n0 = counts & 255u, n1 = (counts >> 8) & 255u, n2 = (counts >> 16) & 255u;
  const int first = role & 255u, all = XROUNDS * n0, most = min(all + XLONG * n1, 8 * KSTEPS - HX);
  const int count = first < n2 ? 8 * KSTEPS - HX : first < n1 ? most : all;
  const bool push = role & 0x1000u;
  char* const ws = p.mine;
  LAS char* const wl = smem + wave * WAVE_LDS;
  const float* src0 = p.x + (size_t)((wave >> 2) * 64 + (wave & 1) * 32 + (lane >> 3)) * KDIM + (wave & 2) * 16 + ((lane ^ (lane >> 4)) & 7) * 4;
  const int odd8 = 8 * KDIM + (lane & 4 ? -16 : 16);
  auto fetch = [&](unsigned buf, int u) {
    const float* s = src0 + (size_t)(u >> 7) * 128 * KDIM + (u & 127) * 64;
    const unsigned d = (unsigned)(uintptr_t)wl + 2048 + buf;
    load_in(s, s + odd8, d);
    load_in(s + 16 * KDIM, s + 16 * KDIM + odd8, d + 2048);
  };
  u32x4 h0, h1;
  auto take = [&](unsigned buf) {
    const LAS f32x4* v = (const LAS f32x4*)(wl + 2048 + buf) + (lane & 15) * 8;
    const int i = (lane >> 4) ^ ((lane >> 1) & 7);
    const f32x4 a = v[i], b = v[i ^ 4], c = v[128 + i], d = v[128 + (i ^ 4)];
    h0 = cvt8(a, b);
    h1 = cvt8(c, d);
  };
  auto next = [=](int it) { return it + (int)((counts >> (it < all ? 0 : it < most ? 8 : 16)) & 255u); };
#define LATE 4
  const bool late = (LATE & 1) && !push;
  if (!late) fetch(0, push ? first : x_unit(role >> 16)); else wait_vm0();
  const unsigned call = begin<4>(p, smem, false);
  if (late) fetch(0, x_unit(role >> 16));
  char* const in = member(p, rank & 4, t);
  if (push) {
    const unsigned at = first * STEP_B + wave * 2048 + lane * 16;
    char* b[4];
#pragma unroll
    for (int i = 0; i < 4; ++i)
      b[i] = at + (i == 3 ? ws + OFF_OWN + own_idx(call, 0) * PANEL_B
                          : member(p, rank & 4, rank + 1 + (i + first) % 3) + pulled(call) + (rank & 3) * 6 * PANEL_B);
    asm volatile("s_waitcnt vmcnt(0)" : "+v"(b[0]), "+v"(b[1]), "+v"(b[2]), "+v"(b[3]) :: "memory");
    take(0);
    if (wave == 0) STAMP(24);
#pragma unroll
    for (int i = 0; i < 4; ++i) {
      store_sys(b[i], h0);
      store_sys(b[i] + 1024, h1);
    }
    if (wave == 0) STAMP(25);
    wait_vm0();
    wg_barrier();
    if (wave == 0) STAMP(26);
    if (t < 4 * FCOPY) set_local(in, t >> 2, (rank & 3) * 6 * KSTEPS + first, call);
    return call;
  }
  char* const across = member(p, (rank & 4) ^ 4, t);
  LAS unsigned* const fill = (LAS unsigned*)(smem + LDS_FILL);
  unsigned odd = 0;
  auto ahead = [&](unsigned buf, int it) {
    if (it < count) { fetch(buf, x_unit(it)); wait_vm<4>(); } else wait_vm0();
  };
  wait_vm0();
  for (int it = first, it1 = next(it), it2; it < count; it = it1, it1 = it2, odd ^= 8192u) {
    const int u = x_unit(it == first ? role >> 16 : it);
    it2 = next(it1);
    take(odd);
    char* b = ws + OFF_OWN + (own_idx(call, 0) * KSTEPS + u) * STEP_B + wave * 2048 + lane * 16;
    store_sys(b, h0);
    store_sys(b + 1024, h1);
    char* pb = ws + OFF_PACK + (own_idx(call, 0) * KSTEPS + u) * PSTEP_B;
    pack_block(h0, h1, wl, smem + LDS_LUT, smem + LDS_MB, fill, pb, wave);
    lds_barrier();
    pack_flush(wl, smem + LDS_MB, pb, wave);
    unsigned lines = 0;
    if (wave == 0) lines = lines_used(fill);
    ahead(it == first ? 8192u : odd, it == first ? it1 : it2);
    lds_barrier();
    if (t < 4 + FCOPY) {
      const int tile = u >> 7, ks = u & 127;
      if (t >= 4) set_local(ws, t - 4, (tile < 6 ? (rank & 3) * 6 : PA_OWNAWAY - 6) * KSTEPS + u, call);
      else if (tile >= 6) set_rdy(across, rank, RDY_AWAY + u - 6 * KSTEPS, call, lines);
      else if (tile == 5 && ks < RELAY) { if (t == (ks & 3)) set_rdy(across, rank, RDY_HOP1 + (ks >> 2), call, lines); }
      else if (t != (rank & 3)) set_rdy(in, rank, u, call, lines);
    }
    if (it == first) {
      if (wave == 0) STAMP(27);
      ahead(0, it2);
    }
  }
  return call;
}

__device__ __forceinline__ unsigned convert_w(const Params& p, unsigned role, unsigned counts, int rank, LAS char* smem) {
  const int lane = threadIdx.x & 63, wave = __builtin_amdgcn_readfirstlane(threadIdx.x >> 6), t = threadIdx.x;
  const int count = KSTEPS * 4, stride = counts >> 24, first = role & 255u;
  const bool push = role & 0x1000u;
  char* const ws = p.mine;
  LAS char* const wl = smem + wave * WAVE_LDS;
  const float* src0 = p.w + (size_t)((wave & 1) * 32 + (lane >> 4) * 4) * NSH + (wave >> 1) * 64 + (lane & 15) * 4;
  auto fetch = [&](unsigned buf, int u) {
    const float* s = src0 + (size_t)(u >> 2) * 64 * NSH + (u & 3) * 256;
#pragma unroll
    for (int j = 0; j < 8; j += 2) {
      const float* r = s + ((j & 3) + (j >> 2) * 16) * NSH;
      load_in(r, r + NSH, (unsigned)(uintptr_t)wl + 2048 + buf + j * 1024);
    }
  };
  u32x4 h[4];
  auto take = [&](unsigned buf) {
    f32x4 v[8];
#pragma unroll
    for (int j = 0; j < 8; ++j) v[j] = ((const LAS f32x4*)(wl + 2048 + buf))[j * 64 + lane];
#pragma unroll
    for (int f = 0; f < 4; ++f)
      h[f] = cvt8((f32x4){v[0][f], v[1][f], v[2][f], v[3][f]}, (f32x4){v[4][f], v[5][f], v[6][f], v[7][f]});
  };
  const bool late = push ? (LATE & 4) && wave >= 4 : (LATE & 2);
  if (!late) fetch(0, first); else wait_vm0();
  const unsigned call = begin<8>(p, smem, false);
  if (late) fetch(0, first);
  char* const own = ws + OFF_OWN + own_idx(call, 8 + (wave >> 2)) * PANEL_B + (wave & 3) * 4096 + lane * 16;
  char* const across = member(p, (rank & 4) ^ 4, push ? first : t);
  if (push) {
    const int q = first & 3, ks = first >> 2;
    char* b = own + q * 2 * PANEL_B + ks * STEP_B;
    char* r = across + pulled(call) + (PB_FOREIGN + (rank & 3) * 2 + (wave >> 2)) * PANEL_B + ks * STEP_B + (wave & 3) * 4096 + lane * 16;
    asm volatile("s_waitcnt vmcnt(0)" : "+v"(b), "+v"(r) :: "memory");
    take(0);
    if (wave == 0) STAMP(28);
#pragma unroll
    for (int f = 0; f < 4; ++f) store_sys(r + f * 1024, h[f]);
#pragma unroll
    for (int f = 0; f < 4; ++f) store_sys(b + f * 1024, h[f]);
    if (wave == 0) STAMP(29);
    wait_vm0();
    wg_barrier();
    if (wave == 0) STAMP(30);
    if (t < 4 * FCOPY)
      set_local(t & 2 ? across : ws, t >> 2, ((t & 2 ? PB_FOREIGN + (rank & 3) * 2 : PB + q * 2) + (t & 1)) * KSTEPS + ks, call);
    return call;
  }
  LAS unsigned* const fill = (LAS unsigned*)(smem + LDS_FILL) + (wave >> 2);
  LAS char* const lists = smem + LDS_MB + (wave >> 2) * 2048;
  unsigned odd = 0;
  auto ahead = [&](unsigned buf, int u) {
    if (u < count) { fetch(buf, u); wait_vm<8>(); } else wait_vm0();
  };
  static_assert(FCOPY % 2 == 0, "");
  wait_vm0();
  for (int u = first; u < count; u += stride, odd ^= 8192u) {
    const int ks = u >> 2, q = u & 3, blk = (wave & 3) * 2;
    char* b = own + q * 2 * PANEL_B + ks * STEP_B;
    take(odd);
#pragma unroll
    for (int f = 0; f < 4; ++f) store_sys(b + f * 1024, h[f]);
    char* pb = ws + OFF_PACK + own_idx(call, 8 + q * 2 + (wave >> 2)) * PPANEL_B + ks * PSTEP_B;
    pack_block(h[0], h[1], wl, smem + LDS_LUT, lists, fill, pb, blk);
    pack_block(h[2], h[3], wl + 1024, smem + LDS_LUT, lists, fill, pb, blk + 1);
    lds_barrier();
    pack_flush(wl, lists, pb, blk);
    pack_flush(wl + 1024, lists, pb, blk + 1);
    unsigned lines = 0;
    if (wave == 0) lines = lines_used(fill) | lines_used(fill + 1) << 3;
    ahead(u == first ? 8192u : odd, u + (u == first ? stride : 2 * stride));
    lds_barrier();
    if (t < 2 * FCOPY) set_local(ws, t >> 1, (PB + q * 2 + (t & 1)) * KSTEPS + ks, call);
    else if (t == 2 * FCOPY + q) set_rdy(across, rank, ks, call, lines);
    if (u == first) {
      if (wave == 0) STAMP(31);
      ahead(0, u + 2 * stride);
    }
  }
  return call;
}

__device__ __forceinline__ int row_of(int r) { return ((RANK_AT >> (4 * r)) & 7) * NSH; }

#define ORDER_V 0x7648953210ull
constexpr unsigned long long BATCH_AT = ORDER_V;
constexpr unsigned long long batch_pos() {
  unsigned long long r = 0;
  for (int i = 0; i < 10; ++i) r |= (unsigned long long)i << (4 * ((BATCH_AT >> (4 * i)) & 15));
  return r;
}
constexpr unsigned long long BATCH_POS = batch_pos();
__device__ __forceinline__ int batch_row(int s, int o, int pos, int i) {
  const int bt = (BATCH_AT >> (4 * pos)) & 15;
  if (bt < 2) return row_of(s) + (bt * 3 + i) * 128;
  const int g = bt < 8 ? s : o;
  return row_of((g & 4) | ((g + 1 + i) & 3)) + (bt - 2) * 128;
}

__device__ __forceinline__ int unit_of(int it, int w) { return it * PW + ((w + it) & (PW - 1)); }

__device__ __forceinline__ unsigned lds_get(const LAS unsigned* c) {
  return __builtin_amdgcn_readfirstlane(__hip_atomic_load(c, __ATOMIC_RELAXED, __HIP_MEMORY_SCOPE_WORKGROUP));
}

constexpr int RING = 8;
constexpr unsigned PL_EX = RING * PSTEP_B;
constexpr int W_TICKET = 0, W_READY = 1, W_DEAD = 2, W_TURN = 3, W_SPARE = 4, W_DONE = 8, W_LANDED = 64, W_QUEUE = 128, W_LINES = 256;
constexpr unsigned Q_N = 128, Q_OUT = 1024, Q_LAST = 1u << 31;
static_assert(PL_EX + 8 * 1024 <= LDS_MB && W_QUEUE + Q_N <= W_LINES && W_LINES * 4 + 768 <= LDS_TMO - LDS_MB, "");

template <int N, int W>
__device__ __forceinline__ void stream_kib(const char* src, unsigned dst, unsigned& cnt, unsigned word) {
  unsigned keep;
#define SK(OFF) "s_waitcnt vmcnt(%5)\n\tglobal_load_lds_dwordx4 %2, off offset:" #OFF " sc0 sc1\n\t" \
                "v_add_u32 %1, 1, %1\n\tds_max_u32 %4, %1\n\t"
  if (N == 4)
    asm volatile("s_mov_b32 %0, m0\n\ts_mov_b32 m0, %3\n\ts_nop 0\n\t" SK(0) SK(1024) SK(2048) SK(3072) "s_mov_b32 m0, %0"
                 : "=&s"(keep), "+v"(cnt) : "v"(src), "s"(dst), "v"(word), "n"(W - 1) : "memory");
  else if (N == 2)
    asm volatile("s_mov_b32 %0, m0\n\ts_mov_b32 m0, %3\n\ts_nop 0\n\t" SK(0) SK(1024) "s_mov_b32 m0, %0"
                 : "=&s"(keep), "+v"(cnt) : "v"(src), "s"(dst), "v"(word), "n"(W - 1) : "memory");
  else
    asm volatile("s_mov_b32 %0, m0\n\ts_mov_b32 m0, %3\n\ts_nop 0\n\t" SK(0) "s_mov_b32 m0, %0"
                 : "=&s"(keep), "+v"(cnt) : "v"(src), "s"(dst), "v"(word), "n"(W - 1) : "memory");
}
template <int K, class F>
__device__ __forceinline__ bool stream_drain(unsigned cnt, unsigned word, F&& more) {
  if constexpr (K >= 0) {
    asm volatile("s_waitcnt vmcnt(%2)\n\tds_max_u32 %1, %0" :: "v"(cnt + 1), "v"(word), "n"(K) : "memory");
    return more() || stream_drain<K - 1>(cnt + 1, word, more);
  }
  return false;
}

__device__ __forceinline__ int zip(int u, int s, int d, int g) {
  const int i = u - (s - d), j = i - g;
  return (unsigned)i >= (unsigned)(2 * d + g) ? u : j < 0 ? s + i : (j & 1 ? s + g : s - d) + (j >> 1);
}
#define D2 12
#define G2 20
#define D3 0
#define G3 0
#define D4 40
#define G4 28
#define D5 52
#define G5 0
#define DA 40
#define GA 0
#define DH 48
template <bool CROSS>
__device__ __forceinline__ int in_order(int u) {
  constexpr int HOP = 4 * KSTEPS + RELAY / 4 - 2 * HW - DH;
  constexpr int AWAY = 2 * KSTEPS + RELAY / 4 - 2 * HW;
  if (CROSS && u < FILL) return AWAY + u;
  if (CROSS && u < AWAY + FILL) {
    u -= FILL;
    return !HOPF ? u : u < RELAY / 4 ? u + 2 * KSTEPS - 2 * HW : u - RELAY / 4;
  }
  if (CROSS) return u < HOP ? zip(u, 3 * KSTEPS + RELAY / 4 - 2 * HW, DA, GA) : u < HOP + RELAY / 4 * 3 ? u + DH : u - RELAY / 4 * 3;
  if (D2 && u < 2 * KSTEPS - HX + D2 + G2) return zip(u, 2 * KSTEPS - HX, D2, G2);
  if (D3 && u < 3 * KSTEPS - HX + D3 + G3) return zip(u, 3 * KSTEPS - HX, D3, G3);
  return u < 4 * KSTEPS - HX + D4 + G4 ? zip(u, 4 * KSTEPS - HX, D4, G4) : zip(u, 5 * KSTEPS - HX, D5, G5);
}
static_assert(FILL + DA <= KSTEPS && DA + GA + DH <= KSTEPS && D2 + G2 + D3 <= KSTEPS && D3 + G3 + D4 <= KSTEPS && D4 + G4 + D5 <= KSTEPS && D5 + G5 + RELAY <= KSTEPS, "");

struct Unit { unsigned src, raw, dst; int from, flag; };
template <bool CROSS>
__device__ __forceinline__ Unit input_unit(int u, unsigned call, int rank, int peer) {
  constexpr int H1 = RELAY / 4, RSTEP = 32 * KSTEPS;
  const unsigned par = own_idx(call, 0) * KSTEPS;
  Unit o;
  o.from = peer;
  u += CROSS ? 2 * HW : HX;
  if (!CROSS) {
    const int v = u < 5 * KSTEPS ? u : u + RELAY;
    o.raw = par + v; o.dst = (peer & 3) * 6 * KSTEPS + v; o.flag = v;
  } else if (u < 256) {
    const int v = (u & 1) * KSTEPS + (u >> 1);
    o.raw = par + (8 + (rank & 3) * 2) * KSTEPS + v; o.dst = (PB_FOREIGN + (peer & 3) * 2) * KSTEPS + v; o.flag = u >> 1;
  } else if (u < 256 + H1) {
    o.raw = par + 5 * KSTEPS + (u - 256) * 4 + (rank & 3); o.dst = ~((peer & 3) * 16 + u - 256); o.flag = RDY_HOP1 + u - 256;
  } else if (u < 512 + H1) {
    const int v = u - 256 - H1;
    o.raw = par + 6 * KSTEPS + v; o.dst = (PA_AWAY + (peer & 3) * 2) * KSTEPS + v; o.flag = RDY_AWAY + v;
  } else {
    const int v = u - 512 - H1, j = v / 3, s = (rank + 1 + v - j * 3) & 3, ks = 5 * KSTEPS + j * 4 + (peer & 3);
    o.from = (rank & 4) | s;
    o.raw = par + ks; o.dst = s * 6 * KSTEPS + ks; o.flag = RDY_HOP2 + s * 16 + j;
    o.src = RSTEP + s * 16 + j;
    return o;
  }
  o.src = o.raw;
  return o;
}

template <bool CROSS, int W>
__device__ __forceinline__ void stream(const Params& p, int lane, unsigned call, int rank, int peer, LAS char* smem,
                                       unsigned k, unsigned stop) {
  LAS unsigned* const sh = (LAS unsigned*)(smem + LDS_MB);
  const char* const pws = p.ws[peer];
  const unsigned lds0 = (unsigned)(uintptr_t)smem, word = (unsigned)(uintptr_t)(sh + W_LANDED + lane);
  unsigned cnt = WIN1 - W - 1 + k * 14;
  for (; k != stop; ++k) {
    unsigned ready, done, q;
    auto look = [&]() {
      const unsigned a = __hip_atomic_load(sh + W_READY, __ATOMIC_RELAXED, __HIP_MEMORY_SCOPE_WORKGROUP);
      cbar();
      const unsigned b = __hip_atomic_load(sh + W_DONE + (k & (RING - 1)), __ATOMIC_RELAXED, __HIP_MEMORY_SCOPE_WORKGROUP);
      const unsigned c = __hip_atomic_load(sh + W_QUEUE + (k & (Q_N - 1)), __ATOMIC_RELAXED, __HIP_MEMORY_SCOPE_WORKGROUP);
      cbar();
      ready = __builtin_amdgcn_readfirstlane(a);
      done = __builtin_amdgcn_readfirstlane(b);
      q = __builtin_amdgcn_readfirstlane(c);
      return k < (ready & ~Q_LAST) && done >= (k & ~(RING - 1u));
    };
    if (!look()) {
      if (!stream_drain<W - 1>(cnt, word, look)) {
        while (!look() && (k | Q_LAST) != ready) __builtin_amdgcn_s_sleep(1);
        if ((k | Q_LAST) == ready) break;
      }
    }
    const unsigned d = lds0 + (k & (RING - 1)) * PSTEP_B, u = q & 2047u;
    if (u < Q_OUT) {
      if ((u & 63) >= 56) STAMP(4 + (u >> 6));
      const char* s = pws + OFF_PACK + (size_t)input_unit<CROSS>(u, call, rank, peer).src * PSTEP_B + lane * 16;
      if ((unsigned)lane >> 3 <= ((q >> 11) & 7u)) stream_kib<1, W>(s, d, cnt, word);
      stream_kib<4, W>(s + 1024, d + 1024, cnt, word);
      stream_kib<4, W>(s + 5120, d + 5120, cnt, word);
      stream_kib<4, W>(s + 9216, d + 9216, cnt, word);
      stream_kib<1, W>(s + 13312, d + 13312, cnt, word);
      if (k == 0) STAMP(3);
    } else {
      if ((u - Q_OUT) % 48 >= 40) STAMP(20 + (u - Q_OUT) / 48);
      const char* s = pws + OFF_STG + (size_t)((rank & 3) * OUT_CHUNKS + u - Q_OUT) * CHUNK_B + lane * 16;
      stream_kib<4, W>(s, d, cnt, word);
      stream_kib<2, W>(s + 4096, d + 4096, cnt, word);
      if (lane < 48) stream_kib<1, W>(s + 6144, d + 6144, cnt, word);
    }
  }
}

template <bool CROSS>
__device__ __forceinline__ void pull(const Params& p, int w, bool far, int wave, unsigned call, int rank, int peer, LAS char* smem) {
  constexpr int NIN = CROSS ? 512 + RELAY - 2 * HW : 768 - RELAY - HX, NOUT = CROSS ? OUT_CHUNKS / PW : 0;
  const int lane = fresh_lane();
  char* const ws = p.mine;
  LAS unsigned* const sh = (LAS unsigned*)(smem + LDS_MB);
  float* const out = p.out + (size_t)((lane >> 4) * 4) * NSH + (peer & 3) * 256 + (lane & 15) * 4;
  auto where = [&](int c, int pc) {
    const int tile = c >> 3, bt = tile / 6, r = tile - bt * 6;
    return out + (size_t)(batch_row(peer, rank, bt, r >> 1) + ((c >> 2) & 1) * 64 + (c & 1) * 32 + (pc & 4) * 4 + (pc & 3)) * NSH
         + (r & 1) * 128 + ((c >> 1) & 1) * 64;
  };
#define PRIO 3
  if (wave >= 6) __builtin_amdgcn_s_setprio(PRIO);
  if (wave == 6) {
    const unsigned* const rdy = (const unsigned*)(ws + OFF_RDY) + peer * RDY_N * 32;
    unsigned* const tmo = (unsigned*)(ws + OFF_TMO);
    unsigned* const ticket = (unsigned*)(ws + OFF_TICKET) + peer * 32;
    unsigned long long t0 = __builtin_amdgcn_s_memrealtime();
    bool dead = false;
    unsigned k = 0, kib = 0, spin = 0;
    unsigned v;
    auto unset = [&](bool look, int fl) {
      v = ~0u;
      if (look && !dead)
        asm volatile("global_load_dword %0, %1, off sc0 sc1\n\ts_waitcnt vmcnt(0)" : "=v"(v) : "v"(rdy + fl * 32) : "memory");
      return __builtin_amdgcn_ballot_w64(v >> 6 < call);
    };
    auto idle = [&]() {
      if ((++spin & 63u) == 0u
          && (__builtin_amdgcn_readfirstlane(__hip_atomic_load(tmo, __ATOMIC_RELAXED, __HIP_MEMORY_SCOPE_AGENT)) == call
              || __builtin_amdgcn_s_memrealtime() - t0 > DEADLINE)) {
        if (lane == 0) {
          __hip_atomic_store(tmo, call, __ATOMIC_RELAXED, __HIP_MEMORY_SCOPE_AGENT);
          __hip_atomic_store(sh + W_DEAD, 1u, __ATOMIC_RELAXED, __HIP_MEMORY_SCOPE_WORKGROUP);
        }
        wait_vm0();
        dead = true;
      }
      __builtin_amdgcn_s_sleep(1);
    };
    auto publish = [&](unsigned v, unsigned near) {
      cbar();
      if (lane == 0) __hip_atomic_store(sh + W_READY, v, __ATOMIC_RELAXED, __HIP_MEMORY_SCOPE_WORKGROUP);
      while (lds_get(sh + W_LANDED) < near) __builtin_amdgcn_s_sleep(2);
      t0 = __builtin_amdgcn_s_memrealtime();
    };
    int known = EXPON(1) ? NIN : 0;
    LAS unsigned char* const lines = (LAS unsigned char*)(sh + W_LINES);
    auto look = [&](int from, int width) {
      const int i = min(from + lane, NIN - 1), o = in_order<CROSS>(i);
      const unsigned long long miss = unset(lane < width, input_unit<CROSS>(o, call, rank, peer).flag);
      known = from + (miss ? (int)__builtin_ctzll(miss) : width);
      if (lane < width) lines[i] = v >> 6 == call ? (v >> (CROSS && o < 256 - 2 * HW ? (o & 1) * 3 : 0)) & 7u : 7u;
    };
    for (int u = w; u < NIN;) {
      while (u >= known) { look(u, 16); if (u >= known) idle(); }
      if (k == 0) STAMP(30);
      if (k == 1) STAMP(31);
      if (k == 2) STAMP(19);
      if (lane == 0) sh[W_QUEUE + (k & (Q_N - 1))] = kib << 14 | lines[u] << 11 | in_order<CROSS>(u);
#define AHEAD 4
      publish(++k, max(kib, (unsigned)AHEAD) - AHEAD);
      kib += 14;
      unsigned tk = 0;
      if (lane == 0) tk = __hip_atomic_fetch_add(ticket, 1u, __ATOMIC_RELAXED, __HIP_MEMORY_SCOPE_AGENT);
      if (known < NIN && known - u < 32) look(known, 64);
      u = min(__builtin_amdgcn_readfirstlane(tk) - (call - 1u) * NIN + PW, (unsigned)NIN);
    }
    if (CROSS && lane == 0) __hip_atomic_store(sh + W_TURN, ~(k * 8u), __ATOMIC_RELAXED, __HIP_MEMORY_SCOPE_WORKGROUP);
    const unsigned long long all = (1ull << NOUT) - 1ull;
    for (unsigned long long left = all; left;) {
      const unsigned long long fresh = left & ~unset(!EXPON(17) && ((left >> lane) & 1), RDY_OUT + unit_of(lane, w));
      if (!fresh) { idle(); continue; }
      const unsigned pos = __builtin_amdgcn_mbcnt_hi(fresh >> 32, __builtin_amdgcn_mbcnt_lo(fresh, 0));
      if ((fresh >> lane) & 1) sh[W_QUEUE + ((k + pos) & (Q_N - 1))] = (kib + pos * 7) << 14 | (Q_OUT + unit_of(lane, w));
      k += __builtin_popcountll(fresh);
      kib += __builtin_popcountll(fresh) * 7;
      left &= ~fresh;
      if (left) publish(k, kib - 14);
    }
    publish(k | Q_LAST, 0);
    if (CROSS) {
      dead = __builtin_amdgcn_readfirstlane(__hip_atomic_load(tmo, __ATOMIC_RELAXED, __HIP_MEMORY_SCOPE_AGENT)) == call;
      wg_barrier();
      if (dead) store_sys(where(unit_of(0, w), 0), (u32x4){0x7fc00000u, 0x7fc00000u, 0x7fc00000u, 0x7fc00000u});
    }
    return;
  }
  if (wave == 7) {
#define KSW 0
    if (KSW) stream<CROSS, WIN1>(p, lane, call, rank, peer, smem, 0, KSW);
    stream<CROSS, WIN>(p, lane, call, rank, peer, smem, KSW, ~0u);
    STAMP(1);
    if (CROSS) wg_barrier();
    return;
  }
  static_assert(RING == 8 && Q_N == 128, "");
  const unsigned sh0 = (unsigned)(uintptr_t)sh, none = ~0u;
  const unsigned aq = sh0 + W_QUEUE * 4 + lane * 8, ah = (unsigned)(uintptr_t)smem + (lane >> 3) * PSTEP_B + (lane & 7) * 4;
  u32x4 st;
  u32x2 queue;
  unsigned count, heads;
  auto look = [&](int word) {
    unsigned r;
    unsigned long long keep;
    asm volatile("s_mov_b64 %5, exec\n\ts_mov_b64 exec, 1\n\tds_add_rtn_u32 %0, %6, %7\n\ts_mov_b64 exec, %5\n\t"
                 "ds_read_b128 %1, %8\n\tds_read_b64 %2, %9\n\tds_read_b32 %3, %8 offset:%11\n\tds_read_b32 %4, %10\n\t"
                 "s_waitcnt lgkmcnt(0)"
                 : "=&v"(r), "=&v"(st), "=&v"(queue), "=&v"(count), "=&v"(heads), "=&s"(keep)
                 : "v"(sh0 + word * 4), "v"(1u), "v"(sh0), "v"(aq), "v"(ah), "n"(W_LANDED * 4) : "memory");
    return __builtin_amdgcn_readfirstlane(r);
  };
  unsigned t, u, q;
  auto landed = [&](unsigned tk) {
    const unsigned ready = __builtin_amdgcn_readfirstlane(st[W_READY]);
    t = tk;
    if (CROSS) {
      const unsigned turn = ~__builtin_amdgcn_readfirstlane(st[W_TURN]);
      if (tk > turn) t = turn + (tk - turn) * 4u;
    }
    const unsigned q0 = __builtin_amdgcn_readlane(queue[0], (t >> 4) & 63), q1 = __builtin_amdgcn_readlane(queue[1], (t >> 4) & 63);
    q = t & 8 ? q1 : q0;
    const bool there = (t >> 3) < (ready & ~Q_LAST);
    u = there ? q & 2047u : none;
    const unsigned need = (q >> 14) + (!CROSS || u < Q_OUT ? ((t & 7) * 13 + 28) >> 3 : t & 4 ? 7 : 4) + WIN1 - 1;
    return there ? (unsigned)__builtin_amdgcn_readfirstlane(count) >= need : ready >= Q_LAST;
  };
  unsigned prev = none, pdst = 0, plines = 0;
  auto done = [&]() {
    const bool last = (look(prev != none ? W_DONE + ((prev >> 3) & (RING - 1)) : W_SPARE) & 7u) == 7u && prev != none;
    prev = none;
    if (!last) return;
    if (!CROSS && (pdst & 127) >= 112) STAMP(17 + ((~pdst >> 3) & 1));
    if (CROSS && (int)pdst >= PB_FOREIGN * KSTEPS && (pdst & 127) < HW + 8) STAMP(17 + ((pdst & 127) >= HW + 4));
    if (CROSS && (int)pdst < 0) STAMP(15);
    if (!CROSS || (int)pdst >= 0) { if (lane < FCOPY) set_local(ws, lane, pdst, call); }
    else if (lane < 3) set_rdy(p.ws[((peer & 4) | ((peer + 1 + lane) & 3)) & 7], rank, RDY_HOP2 + ~pdst, call, plines);
  };
  for (;;) {
    const unsigned tk = look(W_TICKET);
    if (!landed(tk)) {
      if (prev != none) wait_vm0();
      for (done(); !landed(tk); look(W_SPARE)) __builtin_amdgcn_s_sleep(1);
    }
    const LAS char* slot = smem + ((t >> 3) & (RING - 1)) * PSTEP_B;
    if (u >= Q_OUT) {
      if (prev != none) { wait_vm0(); done(); }
      if (!CROSS || u == none) break;
      const unsigned half = (t >> 2) & 1u;
      const bool dead = __builtin_amdgcn_readfirstlane(st[W_DEAD]);
      f32x4 y[4];
      unpack_sums(lane, slot + half * SUM_B, smem + PL_EX + wave * 1024, smem + LDS_LUT,
                  p.ws[peer] + OFF_SPARE + (size_t)(((rank & 3) * OUT_CHUNKS + u - Q_OUT) * 2 + half) * 1024, y);
      if (lane == 0) __hip_atomic_fetch_add(sh + W_DONE + ((t >> 3) & (RING - 1)), 4u, __ATOMIC_RELAXED, __HIP_MEMORY_SCOPE_WORKGROUP);
      float* const o = where(u - Q_OUT, t & 4);
#pragma unroll
      for (int r = 0; r < 4; ++r) {
        u32x4 v = __builtin_bit_cast(u32x4, (f32x4){silu(y[r][0]), silu(y[r][1]), silu(y[r][2]), silu(y[r][3])});
        if (dead) v = (u32x4){0x7fc00000u, 0x7fc00000u, 0x7fc00000u, 0x7fc00000u};
        store_out(o + r * NSH, v);
      }
      continue;
    }
    const Unit un = input_unit<CROSS>(u, call, rank, peer);
    u32x4 f0, f1;
    char* d;
    if (CROSS && RELAY && (int)un.dst < 0) {
      const int l1 = min(lane, 47);
      const LAS char* blk = slot + (t & 7) * (PSTEP_B / 8);
      f0 = *(const LAS u32x4*)(blk + lane * 16);
      f1 = *(const LAS u32x4*)(blk + 1024 + l1 * 16);
      d = ws + OFF_RELAY + (size_t)(~un.dst * 8 + (t & 7)) * (PSTEP_B / 8);
      store_sys(d + lane * 16, f0);
      store_sys(d + 1024 + l1 * 16, f1);
    } else {
      const unsigned hdr = __builtin_amdgcn_readlane(heads, t & 63);
      if ((int)hdr >= 0) {
        unpack_block(lane, slot, t & 7, hdr, smem + PL_EX + wave * 1024, smem + LDS_LUT, f0, f1);
      } else {
        const char* r = p.ws[un.from & 7] + OFF_OWN + (size_t)(un.raw * 8 + (t & 7)) * 2048 + lane * 16;
        asm volatile("global_load_dwordx4 %0, %2, off sc0 sc1\n\t"
                     "global_load_dwordx4 %1, %2, off offset:1024 sc0 sc1\n\ts_waitcnt vmcnt(0)"
                     : "=&v"(f0), "=&v"(f1) : "v"(r) : "memory");
      }
      d = ws + pulled(call) + (size_t)(un.dst * 8 + (t & 7)) * 2048 + lane * 16;
      store_sys(d, f0);
      store_sys(d + 1024, f1);
    }
    asm volatile("s_waitcnt vmcnt(2)" ::: "memory");
    done();
    prev = t;
    pdst = un.dst;
    plines = (q >> 11) & 7u;
    if (!landed(__builtin_amdgcn_readfirstlane(st[W_TICKET]))) { wait_vm0(); done(); }
  }
  if (CROSS) { wait_vm0(); wg_barrier(); }
}

struct Job { int pa, pb, ah, bh, tile; };
__device__ __forceinline__ void decode_job(unsigned code, int a, Job& o) {
  const int ra = code & 63;
  int bt, i;
  o.pb = PB + ((code >> 6) & 15);
  o.ah = (code >> 10) & 1;
  o.bh = (code >> 11) & 1;
  if (ra < 18) {
    i = ra / 6; bt = 2 + ra % 6; o.pa = ((a + 1 + i) & 3) * 6 + bt - 2;
  } else if (ra < 24) {
    o.pa = a * 6 + ra - 18; bt = (ra - 18) / 3; i = (ra - 18) % 3;
  } else {
    o.pa = ra; bt = 8 + (ra & 1); i = (((ra - PA_AWAY) >> 1) - ((o.pb - PB_FOREIGN) >> 1) - 1) & 3;
  }
  o.tile = (int)((BATCH_POS >> (4 * bt)) & 15) * 3 + i;
}

#define POLL_EVERY 1
#define POLL_CAD 2
#define POLL_SLEEP 2
#define GLDS_BITS " sc0 sc1"

template <int NB>
__device__ __forceinline__ unsigned next_stage(unsigned so) { return so == LDS_MB - (NB + 1) * 8192 ? 0u : so + (NB + 1) * 8192; }

template <bool TALL, int NB>
__device__ __forceinline__ void mfma_wave(const Params& p, LAS char* smem, const Job& jb, int job, int wave,
                                          int rank, unsigned call) {
  constexpr int MT = TALL ? 4 : NB, NT = TALL && NB == 2 ? 8 : 4, KH = TALL ? 1 : 2, STEPS = 2 * KSTEPS / KH;
  constexpr int NRA = MT * KH, NRB = NT * KH, NRD = NRA + NRB, NMM = MT * NT * KH;
  int lane = fresh_lane();
  const int wr = MT == 1 ? wave : wave >> 1, wc = MT == 1 ? 0 : wave & 1;
  f32x4 acc[MT][NT];
#pragma unroll
  for (int m = 0; m < MT; ++m)
#pragma unroll
    for (int n = 0; n < NT; ++n) acc[m][n] = (f32x4){0.f, 0.f, 0.f, 0.f};
  half8 fa[2][NRA], fb[2][NRB];
  const LAS char* la = smem + wr * MT * 1024 + lane * 16;
  const LAS char* lb = smem + 8192 + wc * NRB * 1024 + lane * 16;
  unsigned so = 0;
#define READ_SET(X) do { \
    _Pragma("unroll") for (int i = 0; i < NRB; ++i) { \
      if (i < NRA) fa[X][i] = *(const LAS half8*)(la + so + (i / MT) * 4096 + (i % MT) * 1024); \
      fb[X][i] = *(const LAS half8*)(lb + so + i * 1024); } \
    so = next_stage<NB>(so); } while (0)
#define MMA_SET(X) do { \
    _Pragma("unroll") for (int kh = 0; kh < KH; ++kh) \
    _Pragma("unroll") for (int m = 0; m < MT; ++m) \
    _Pragma("unroll") for (int n = 0; n < NT; ++n) \
      acc[m][n] = __builtin_amdgcn_mfma_f32_16x16x32_f16(fa[X][kh * MT + m], fb[X][kh * NT + n], acc[m][n], 0, 0, 0); } while (0)
  constexpr int RG = TALL ? 1 : 2, MG = NB, G = NRD / RG;
#define INTERLEAVE() do { \
    _Pragma("unroll") for (int i = 0; i < G; ++i) { \
      __builtin_amdgcn_sched_group_barrier(0x100, RG, 0); \
      __builtin_amdgcn_sched_group_barrier(0x008, MG, 0); } \
    __builtin_amdgcn_sched_group_barrier(0x008, NMM - G * MG, 0); } while (0)
#define STEP_END() do { __builtin_amdgcn_sched_barrier(0); asm volatile("" ::: "memory"); \
    __builtin_amdgcn_s_waitcnt(0xC07F);   __builtin_amdgcn_sched_barrier(0); \
    wg_barrier(); __builtin_amdgcn_sched_barrier(0); } while (0)
  wg_barrier();
  if (wave == 0) STAMP(4 + job * 4);
  READ_SET(0);
  STEP_END();
  for (int t = 1; t < STEPS - 1; t += 2) {
    READ_SET(1); MMA_SET(0); INTERLEAVE();
    STEP_END();
    READ_SET(0); MMA_SET(1); INTERLEAVE();
    STEP_END();
  }
  READ_SET(1); MMA_SET(0); INTERLEAVE();
  const unsigned tm = *(const LAS unsigned*)(smem + LDS_TMO + job * 16);
  STEP_END();
  MMA_SET(1);
  __builtin_amdgcn_sched_barrier(0);
  if (wave == 0) STAMP(5 + job * 4);
  lane = fresh_lane();
  asm volatile("" : "+s"(call));
  const bool dead = tm == call;

  const bool foreign = jb.pb >= PB_FOREIGN;
  const int ob = (jb.pb - PB_FOREIGN) >> 1;
  const int r16 = jb.ah * 4 + wr * MT;
  const int c64 = (jb.pb & 1) * 2 + jb.bh + wc * (NT / 4);
  const int chunk = (jb.tile * 2 + (c64 >> 1)) * 8 + (r16 & 4) + (c64 & 1) * 2 + ((r16 >> 1) & 1);
  const float qnan = __builtin_nanf("");
  if (foreign) {
    const unsigned blk = (ob * OUT_CHUNKS + chunk) * 2 + (r16 & 1);
#pragma unroll 1
    for (int i = 0; i < MT * NT / 4; ++i) {
      f32x4 v[4];
#pragma unroll
      for (int j = 0; j < MT * NT / 4; ++j)
        if (j == 0 || i == j) {
#pragma unroll
          for (int n = 0; n < 4; ++n) v[n] = acc[j % MT][j / MT * 4 + n];
          asm volatile("" : "+v"(v[0]), "+v"(v[1]), "+v"(v[2]), "+v"(v[3]));
        }
      if (dead) v[0] = v[1] = v[2] = v[3] = (f32x4){qnan, qnan, qnan, qnan};
      pack_sums(lane, v, smem + LDS_EPI + wave * 512, smem + LDS_LUT, p.mine + OFF_STG + (size_t)((blk + i) >> 1) * CHUNK_B + ((blk + i) & 1) * SUM_B,
                p.mine + OFF_SPARE + (size_t)(blk + i) * 1024);
    }
  } else {
    const int pa = jb.pa;
    const int row0 = pa < PA_AWAY ? row_of((rank & 4) + pa / 6) + (pa % 6) * 128
                   : row_of(pa < PA_OWNAWAY ? ((rank & 4) ^ 4) + ((pa - PA_AWAY) >> 1) : rank) + 768 + (pa & 1) * 128;
    float* o = p.out + (size_t)(row0 + r16 * 16 + (lane >> 4) * 4) * NSH + ((jb.pb - PB) >> 1) * 256 + c64 * 64 + (lane & 15) * 4;
#pragma unroll
    for (int h = 0; h < NT / 4; ++h)
#pragma unroll
      for (int m = 0; m < MT; ++m)
#pragma unroll
        for (int r = 0; r < 4; ++r) {
          f32x4 v = {silu(acc[m][h * 4][r]), silu(acc[m][h * 4 + 1][r]), silu(acc[m][h * 4 + 2][r]), silu(acc[m][h * 4 + 3][r])};
          if (dead) v = (f32x4){qnan, qnan, qnan, qnan};
          store_out(o + h * 64 + (m * 16 + r) * NSH, __builtin_bit_cast(u32x4, v));
        }
  }
  if (foreign) {
    wait_vm0();
    bool last = lane < (MT * NT + 7) / 8;
    if (MT == 1)
      last = last && __hip_atomic_fetch_add((LAS unsigned*)(smem + LDS_TMO) + job * 4 + 2 + (wave >> 1), 1u,
                                            __ATOMIC_RELAXED, __HIP_MEMORY_SCOPE_WORKGROUP) == 1u;
    if (last) set_rdy(p.ws[(((rank & 4) ^ 4) | ob) & 7], rank, RDY_OUT + chunk + lane, call, 0);
  }
  if (wave == 0) STAMP(6 + job * 4);
}

#define NEAR 4
template <int NB>
__device__ __forceinline__ void load_wave(const Params& p, LAS char* smem, const Job& jb, int job, unsigned code, int l,
                                          bool tall, int rank, unsigned call) {
  constexpr int DEPTH = NB == 2 ? 5 : 7;
  const int steps = tall ? 2 * KSTEPS : KSTEPS, ksh = tall ? 1 : 0;
  if (l > NB && l < 3) {
    for (int t = 0; t <= steps; ++t) wg_barrier();
    return;
  }
  const int lane = fresh_lane(), a = rank & 3;
  char* const ws = p.mine;
  const unsigned lds0 = (unsigned)(uintptr_t)smem;
  LAS unsigned* const rdyw = (LAS unsigned*)(smem + LDS_TMO) + job * 4 + 1;
  const unsigned gap = tall ? 8192 : 4096;
  auto panel_of = [&](int pc) { return pc == 0 ? jb.pa : tall ? jb.pb + pc - 1 : jb.pb; };
  auto source = [&](int pc) {
    const int panel = panel_of(pc);
    const int own = panel >= PB_FOREIGN ? -1 : panel >= PA_OWNAWAY ? panel - PA_OWNAWAY + 6
                  : (unsigned)(panel - a * 6) < 6u ? panel - a * 6 : -1;
    return ws + (own >= 0 ? OFF_OWN + own_idx(call, own) * PANEL_B : pulled(call) + (size_t)panel * PANEL_B)
         + (tall ? 0 : (pc == 0 ? jb.ah : jb.bh + pc - 1) * 8192);
  };
  int ready = EXPON(1) ? KSTEPS : 0;
  if (l == 3) {
    unsigned* const tmo = (unsigned*)(ws + OFF_TMO);
    const char* fsrc = ws + OFF_FLAG + (blockIdx.x & (FCOPY - 1)) * 65536 + panel_of(lane >> 5) * 1024 + ((lane >> 2) & 7) * 128 + (lane & 3) * 16;
    const unsigned mb = lds0 + LDS_MB + job * 2048;
    int ra = ready, rb = ready;
    auto poll = [&]() {
      if ((lane < 32 ? ra : rb) < KSTEPS) {
        unsigned keep;
        asm volatile("s_mov_b32 %0, m0\n\ts_mov_b32 m0, %2\n\ts_nop 0\n\t"
                     "global_load_lds_dwordx4 %1, off sc0 sc1\n\ts_mov_b32 m0, %0"
                     : "=&s"(keep) : "v"(fsrc), "s"(mb) : "memory");
        if (NB == 2 && tall && lane >= 32)
          asm volatile("s_mov_b32 %0, m0\n\ts_mov_b32 m0, %2\n\ts_nop 0\n\t"
                       "global_load_lds_dwordx4 %1, off offset:1024 sc0 sc1\n\ts_mov_b32 m0, %0"
                       : "=&s"(keep) : "v"(fsrc), "s"(mb - 512) : "memory");
      }
    };
    auto refresh = [&]() {
      u32x2 f, g, h;
      asm volatile("ds_read2_b32 %0, %3 offset1:8\n\tds_read2_b32 %1, %3 offset0:128 offset1:136\n\t"
                   "ds_read2_b32 %2, %4 offset1:8\n\ts_waitcnt lgkmcnt(0)"
                   : "=&v"(f), "=&v"(g), "=&v"(h)
                   : "v"(mb + (lane & 7) * 64 + (lane >> 3) * 4), "v"(mb + (NB == 2 && tall ? 1024 : 512) + (lane & 7) * 64 + (lane >> 3) * 4)
                   : "memory");
      const unsigned long long a0 = __builtin_amdgcn_ballot_w64(f[0] < call), a1 = __builtin_amdgcn_ballot_w64(f[1] < call);
      const unsigned long long b0 = __builtin_amdgcn_ballot_w64(min(g[0], h[0]) < call), b1 = __builtin_amdgcn_ballot_w64(min(g[1], h[1]) < call);
      ra = a0 ? __builtin_ctzll(a0) : a1 ? 64 + __builtin_ctzll(a1) : KSTEPS;
      rb = b0 ? __builtin_ctzll(b0) : b1 ? 64 + __builtin_ctzll(b1) : KSTEPS;
      ready = min(ra, rb);
      if (lane == 0) __hip_atomic_store(rdyw, (unsigned)ready, __ATOMIC_RELAXED, __HIP_MEMORY_SCOPE_WORKGROUP);
    };
    auto advance = [&]() {
      const int k = min(ready + lane, KSTEPS - 1);
      u32x2 f;
      unsigned h;
      asm volatile("ds_read2_b32 %0, %2 offset1:128\n\tds_read_b32 %1, %3\n\ts_waitcnt lgkmcnt(0)"
                   : "=&v"(f), "=&v"(h)
                   : "v"(mb + (k & 7) * 64 + (k >> 3) * 4), "v"(mb + (NB == 2 && tall ? 1024 : 512) + (k & 7) * 64 + (k >> 3) * 4)
                   : "memory");
      const unsigned long long m = __builtin_amdgcn_ballot_w64(min(min(f[0], f[1]), h) < call);
      ready = min(ready + (m ? (int)__builtin_ctzll(m) : 64), KSTEPS);
      __hip_atomic_store(rdyw, (unsigned)ready, __ATOMIC_RELAXED, __HIP_MEMORY_SCOPE_WORKGROUP);
    };
    for (int s = 0; s < steps; ++s) {
      if (s >= DEPTH) wg_barrier();
      const int ks = s >> ksh;
      if (ready < KSTEPS && ready - ks < NEAR) {
        advance();
        if (ks >= ready) {
          const unsigned long long t0 = __builtin_amdgcn_s_memrealtime();
          for (unsigned it = 1;; ++it) {
            if (it >= 32u || s == 0) { poll(); wait_vm0(); refresh(); }
            else { if ((it & (POLL_CAD - 1)) == 1u) poll(); advance(); }
            if (ks < ready) break;
            if ((it & 63u) == 0u) {
              if (__builtin_amdgcn_readfirstlane(__hip_atomic_load(tmo, __ATOMIC_RELAXED, __HIP_MEMORY_SCOPE_AGENT)) == call
                  || __builtin_amdgcn_s_memrealtime() - t0 > DEADLINE) {
                if (lane == 0) __hip_atomic_store(tmo, call, __ATOMIC_RELAXED, __HIP_MEMORY_SCOPE_AGENT);
                wait_vm0();
                ready = KSTEPS;
                if (lane == 0) __hip_atomic_store(rdyw, (unsigned)ready, __ATOMIC_RELAXED, __HIP_MEMORY_SCOPE_WORKGROUP);
                break;
              }
            }
            if (it >= 32u) __builtin_amdgcn_s_sleep(POLL_SLEEP); else __builtin_amdgcn_s_sleep(1);
          }
        } else if (!(s & (((POLL_EVERY) << ksh) - 1))) {
          poll();
        }
      }
    }
    if (lane == 0) {
      unsigned keep;
      asm volatile("s_mov_b32 %0, m0\n\ts_mov_b32 m0, %2\n\ts_nop 0\n\t"
                   "global_load_lds_dword %1, off sc1\n\ts_mov_b32 m0, %0"
                   : "=&s"(keep) : "v"(tmo), "s"(lds0 + LDS_TMO + job * 16) : "memory");
    }
    for (int t = 1; t < DEPTH; ++t) wg_barrier();
    wait_vm0();
    wg_barrier();
    wg_barrier();
    return;
  }
  const char* src = source(l) + lane * 16;
  const unsigned dst0 = lds0 + l * 8192;
  unsigned so = 0;
  for (int s = 0; s < steps; ++s) {
    if (s >= DEPTH) {
      asm volatile("s_waitcnt vmcnt(%0)" :: "n"(8 * (DEPTH - 1)) : "memory");
      wg_barrier();
    }
    const int ks = s >> ksh;
    if (ks >= ready)
      while ((ready = lds_get(rdyw)) <= ks) __builtin_amdgcn_s_sleep(1);
    const char* g = src + (size_t)ks * STEP_B + (s & ksh) * 4096;
    const unsigned d = dst0 + so;
    so = next_stage<NB>(so);
    unsigned keep;
#define GLDS4(R) "global_load_lds_dwordx4 " R ", off" GLDS_BITS "\n\t" \
                 "global_load_lds_dwordx4 " R ", off offset:1024" GLDS_BITS "\n\t" \
                 "global_load_lds_dwordx4 " R ", off offset:2048" GLDS_BITS "\n\t" \
                 "global_load_lds_dwordx4 " R ", off offset:3072" GLDS_BITS "\n\t"
    asm volatile("s_mov_b32 %0, m0\n\ts_mov_b32 m0, %3\n\ts_nop 0\n\t" GLDS4("%1")
                 "s_mov_b32 m0, %4\n\ts_nop 0\n\t" GLDS4("%2") "s_mov_b32 m0, %0"
                 : "=&s"(keep) : "v"(g), "v"(g + gap), "s"(d), "s"(d + 4096) : "memory");
  }
#define WB(N) "s_waitcnt vmcnt(" #N ")\n\ts_barrier\n\t"
  if (NB == 1) asm volatile(WB(48) WB(40) ::: "memory");
  asm volatile(WB(32) WB(24) WB(16) WB(8) WB(0) "s_barrier" ::: "memory");
}

__global__ void __launch_bounds__(NTHREADS, 2) ag_gemm_silu(const Params p) {
  __shared__ __attribute__((aligned(16))) char smem[LDS_BYTES];
  const int xcd = (xcc_id() - 4) & 7, slot = blockIdx.x >> 3;
  u32x4 plan;
  {
    unsigned l0, l1;
    asm volatile("s_load_dword %1, %3, 0\n\ts_load_dword %2, %3, 64\n\ts_load_dwordx4 %0, %3, %4\n\ts_waitcnt lgkmcnt(0)"
                 : "=&s"(plan), "=&s"(l0), "=&s"(l1)
                 : "s"(__builtin_amdgcn_kernarg_segment_ptr()),
                   "s"((unsigned)(offsetof(Params, plan) + 16 * (slot < GSLOTS ? slot * 8 + xcd : GSLOTS * 8 + slot - GSLOTS))) : "memory");
  }
  const int rank = p.rank & 7, a = rank & 3;
  const int wave = __builtin_amdgcn_readfirstlane(threadIdx.x >> 6);
  if (threadIdx.x == 0)
    asm volatile("s_mov_b32 m0, %1\n\ts_nop 0\n\tglobal_load_lds_dword %0, off sc1"
                 :: "v"(call_word(p)), "s"((unsigned)(uintptr_t)(LAS char*)smem + LDS_CALL) : "memory");
  const bool puller = slot >= GSLOTS && xcd < 7;
  unsigned call;
  if (!puller && (plan[0] & 0x4000u)) call = convert_x(p, plan[0], p.counts, rank, (LAS char*)smem);
  else if (!puller && (plan[0] & 0x8000u)) call = convert_w(p, plan[0], p.counts, rank, (LAS char*)smem);
  else call = begin<0>(p, (LAS char*)smem, puller);
  if (threadIdx.x == 0) __hip_atomic_store(call_word(p), call, __ATOMIC_RELAXED, __HIP_MEMORY_SCOPE_AGENT);
  __builtin_amdgcn_s_waitcnt(0x0F70);

  if (puller) {
    const int w = slot - GSLOTS;
    if (xcd < 4) pull<true>(p, w, w < (int)((WMASK >> (4 * xcd)) & 15), wave, call, rank, ((rank & 4) ^ 4) | ((a + xcd) & 3), (LAS char*)smem);
    else pull<false>(p, w, w < (int)((WMASK >> (4 * xcd)) & 15), wave, call, rank, (rank & 4) | ((a + xcd - 3) & 3), (LAS char*)smem);
    if (wave == 0) STAMP(2);
    return;
  }
  if (wave == 0) STAMP(1);

  for (int job = 0; job < NJOBS; ++job) {
    const unsigned code = job == 0 ? plan[1] : job == 1 ? plan[2] : plan[3];
    if (!(code & 0x8000u)) break;
    if (EXPON(4) && (job > 0 || slot >= 12)) break;
    if (EXPON(8) && job > 0) break;
    Job jb;
    decode_job(code, a, jb);
    const int kind = (code >> 12) & 3;
    if (kind == 3 && (code & 0x200) && wave == 0) STAMP(21 + (code & 63) / 6);
    if (wave >= 4) {
      if (kind & 1) load_wave<1>(p, (LAS char*)smem, jb, job, code, wave - 4, kind < 2, rank, call);
      else load_wave<2>(p, (LAS char*)smem, jb, job, code, wave - 4, kind < 2, rank, call);
    } else if (kind == 0) {
      mfma_wave<true, 2>(p, (LAS char*)smem, jb, job, wave, rank, call);
    } else if (kind == 1) {
      mfma_wave<true, 1>(p, (LAS char*)smem, jb, job, wave, rank, call);
    } else if (kind == 2) {
      mfma_wave<false, 2>(p, (LAS char*)smem, jb, job, wave, rank, call);
    } else {
      mfma_wave<false, 1>(p, (LAS char*)smem, jb, job, wave, rank, call);
    }
  }
}

void make_schedule(Params& p) {
  int n[NGEMM] = {};
  bool early[NGEMM] = {};
  for (int g = 0; g < NGEMM; ++g) p.plan[g] = (u32x4){0u, 0u, 0u, 0u};
  int team[32], nteam = 0;
  auto id = [](int xcd, int s) { return s < GSLOTS ? s * 8 + xcd : GSLOTS * 8 + s - GSLOTS; };
  auto put = [&](int xcd, int s, int kind, int ra, int b, int ah = 0, int bh = 0) {
    const int g = id(xcd, s);
    if (n[g] >= NJOBS || nteam >= 32) return;
    p.plan[g][++n[g]] = 0x8000 | (kind << 12) | (bh << 11) | (ah << 10) | (b << 6) | ra;
    team[nteam++] = g;
  };
  auto close = [&]() {
    for (int side = 0; side < 2; ++side)
      for (int i = 0; i < nteam; ++i) {
        const unsigned mask = side ? 0xbc0u : 0x43fu;
        int cnt = 0, idx = 0;
        for (int k = 0; k < nteam; ++k)
          if (((p.plan[team[i]][n[team[i]]] ^ p.plan[team[k]][n[team[k]]]) & mask) == 0) { idx += k < i; ++cnt; }
        p.plan[team[i]][n[team[i]]] |= (unsigned)((cnt - 1) | (idx << 3)) << (16 + 6 * side);
      }
    nteam = 0;
  };
  auto P = [&](int xcd, int s0, int kind, int t, int b0) {
    for (int s = 0; s < 12; ++s) {
      if (kind < 2) put(xcd, s0 + s, kind, s % 3 * 6 + t, b0 + s / 3 * (2 - kind));
      else if (kind == 2) put(xcd, s0 + s, kind, s % 3 * 6 + t, b0 + s / 6, (s / 3) & 1);
      else put(xcd, s0 + s, kind, s % 3 * 6 + t, b0, (s / 3) & 1, s / 6);
    }
    close();
  };
  auto L = [&](int xcd, int s0, int h, int b0) { for (int s = 0; s < 12; ++s) put(xcd, s0 + s, 0, 18 + h * 3 + s % 3, b0 + s / 3 * 2); close(); };
  auto Afw = [&](int xcd, int s0, int t) {
    for (int s = 0; s < 12; ++s) put(xcd, s0 + s, 0, PA_AWAY + (s & 3) * 2 + t, 8 + (((s & 3) + 1 + (s >> 2)) & 3) * 2);
    close();
  };
  auto Afs = [&](int xcd, int s0, int t, int h) {
    for (int s = h * 12; s < h * 12 + 12; ++s) put(xcd, s0 + s % 12, 1, PA_AWAY + ((s / 6 + 1 + s % 3) & 3) * 2 + t, 8 + s / 3);
    close();
  };
  auto Aow = [&](int xcd, int s0, int t, int c0, int cnt) {
    for (int s = 0; s < cnt; ++s) put(xcd, s0 + s, 0, PA_AWAY + ((c0 + s) & 3) * 2 + t, (c0 + s) / 4 * 2);
    if (c0 + cnt == 16) close();
  };
  auto E = [&](int xcd, int s0) { for (int s = 0; s < 12; ++s) early[id(xcd, s0 + s)] = true; };
#define SCHED 7
  auto Aos = [&](int xcd, int s0, int t, int h) {
    for (int s = h * 12; s < h * 12 + 12; ++s) put(xcd, s0 + s % 12, 1, PA_AWAY + (s & 3) * 2 + t, s / 4);
    close();
  };
  auto OA = [&](int xcd, int s0, int h) { for (int s = h * 8; s < h * 8 + 8; ++s) put(xcd, s0 + s % 8, 1, PA_OWNAWAY + (s & 1), s / 2); close(); };
  P(0, 0, 0, 0, 0); P(1, 0, 0, 0, 8); L(2, 0, 0, 8); L(3, 0, 1, 8); L(4, 0, 0, 0); L(5, 0, 1, 0);
  for (int x = 0; x < 6; ++x) E(x, 0);
  OA(6, 0, 0); OA(7, 0, 1);
  P(5, 12, 0, 1, 0); P(6, 12, 0, 1, 8);
  const int sw = SCHED == 7 ? 8 : 0;
  P(0, 12, 1, 2, 8 ^ sw); P(1, 12, 1, 2, 12 ^ sw); Afs(2, 12, 0, 0); Afs(3, 12, 0, 1); P(4, 12, 0, 2, sw); Aow(7, 12, 0, 0, 16);
  P(0, 0, 1, 3, 8); P(1, 0, 1, 3, 12); Afs(2, 0, 1, 0); Afs(3, 0, 1, 1);
  P(4, 0, 1, 3, 0); P(5, 0, 1, 3, 4); Aos(6, 0, 1, 0); Aos(7, 0, 1, 1);
  if (SCHED == 5) {
    Aow(7, 28, 1, 12, 4);
    for (int x = 0; x < 4; ++x) P(x, 12, 2, 4, 8 + 2 * x);
    P(5, 12, 1, 4, 0); P(6, 12, 1, 4, 4);
    for (int x = 0; x < 4; ++x) P(x, 12, 3, 5, 4 + x);
    P(0, 0, 2, 5, 0); P(1, 0, 2, 5, 2);
    int b = 8;
    for (int x = 2; x < 8; ++x) P(x, 0, 3, 5, b++);
    P(4, 12, 3, 5, b++); P(7, 12, 3, 5, b++);
  } else {
    for (int s = 0; s < 4; ++s) put(7, 28 + s, 1, PA_AWAY + s * 2 + 1, 6);
    for (int s = 0; s < 16; ++s) put(s < 12 ? 4 : 7, s < 12 ? 12 + s : 16 + s, 3, PA_AWAY + s / 4 * 2 + 1, 7, s & 1, (s >> 1) & 1);
    close();
    for (int x = 0; x < 4; ++x) P(x, 12, 1, 4, (8 + 4 * x) & 15);
    P(5, 12, 1, 5, 0); P(6, 12, 1, 5, 4);
#define MIX 0x76543210u
    for (int x = 0; x < 8; ++x) P(x, 0, 3, 5, 8 + ((MIX >> (4 * x)) & 7));
  }
#define CVT_W 56
  int nx = 0, nw = 0, c = 0;
  for (int pass = 0; pass < 3; ++pass) {
    for (int g = 0; g < NGEMM; ++g) {
      const int xcd = g & 7, s = g >> 3, ra = p.plan[g][1] & 63;
      if (early[g]) {
        if (xcd < 4 && s < HW) p.plan[g][0] = 0x9000u | (s * 4 + ((p.rank + xcd) & 3));
        if (xcd >= 4 && s * 2 + xcd - 4 < HX) p.plan[g][0] = 0x5000u | (s * 2 + xcd - 4);
        continue;
      }
      if (pass != (ra >= PA_OWNAWAY || (ra < 18 && ra % 6 == 1) ? 2 : ra > PA_AWAY && (ra & 1) ? 0 : 1)) continue;
      if (pass == 1 && c++ * CVT_W % 76 < CVT_W) p.plan[g][0] = 0x8000u;
      else p.plan[g][0] = 0x4000u | nx++;
    }
    p.counts = (pass ? p.counts << 8 : 0u) | nx;
  }
  for (int pass = 0, k = 0; pass < 2; ++pass)
    for (int g = 0; g < NGEMM; ++g)
      if ((p.plan[g][0] & 0x5000u) == 0x4000u && (g >= GSLOTS * 8 || (g & 7) >= 6) == !pass) p.plan[g][0] |= k++ << 16;
#define WORD 1
  for (int pass = 0; pass < 3; ++pass)
    for (int g = 0; g < NGEMM; ++g)
      if (p.plan[g][0] == 0x8000u && pass == (!WORD || g >= GSLOTS * 8 || (g & 7) >= 6 ? 0 : (g & 7) < 4 ? 1 : 2)) p.plan[g][0] |= 4 * HW + nw++;
  static_assert(HW <= 12 && HX <= 24, "");
  p.counts |= nw << 24;
}

}

extern "C" void kernel_launch(int rank, int nranks, ncclComm_t comm, hipStream_t stream,
                              void* const* d_in, const int* in_sizes, int n_in,
                              void* d_out, int out_size, const int* in_dtypes, int out_dtype,
                              void* const* d_out_all, void* const* d_ws_all, size_t ws_bytes) {
  (void)comm; (void)in_dtypes; (void)out_dtype; (void)d_out_all;
  if (nranks != NR || (unsigned)rank >= (unsigned)NR || n_in != 2 || in_sizes[0] != NSH * KDIM ||
      in_sizes[1] != KDIM * NSH || out_size != KDIM * NSH || ws_bytes < WS_NEED) {
    fprintf(stderr, "kernel_launch: unexpected problem shape\n");
    return;
  }
  Params p;
  p.x = (const float*)d_in[0];
  p.w = (const float*)d_in[1];
  p.rank = 0;
  for (int i = 0; i < NR; ++i) {
    const int r = (RANK_AT >> (4 * i)) & 7;
    p.ws[i] = (char*)d_ws_all[r];
    if (r == rank) p.rank = i;
  }
  p.mine = (char*)d_ws_all[rank];
  p.out = (float*)d_out;
  make_schedule(p);
  ag_gemm_silu<<<dim3(NWG), dim3(NTHREADS), 0, stream>>>(p);
  HIP_CHECK(hipGetLastError());
}
```

```cpp
#include <hip/hip_runtime.h>
#include <rccl/rccl.h>
#include <cstdio>
#include <cstdint>
#include <cstddef>

#define HIP_CHECK(expr) do { hipError_t _e = (expr); if (_e != hipSuccess) { \
    fprintf(stderr, "HIP error %s at %s:%d\n", hipGetErrorString(_e), __FILE__, __LINE__); \
    return; } } while (0)

namespace {

typedef _Float16 half8 __attribute__((ext_vector_type(8)));
typedef float f32x4 __attribute__((ext_vector_type(4)));
typedef unsigned u32x4 __attribute__((ext_vector_type(4)));
typedef unsigned u32x2 __attribute__((ext_vector_type(2)));
#define LAS __attribute__((address_space(3)))
#define GAS __attribute__((address_space(1)))

constexpr int NR = 8;
#define RANK_AT_V 0x75416320u
constexpr unsigned RANK_AT = RANK_AT_V;
constexpr int KDIM = 8192;
constexpr int NSH = 1024;
constexpr int KSTEPS = 128;
constexpr unsigned STEP_B = 16384;
constexpr size_t PANEL_B = (size_t)KSTEPS * STEP_B;
constexpr int PA_AWAY = 24, PA_OWNAWAY = 32, PB = 34, PB_FOREIGN = 42, NPANEL = 50;

constexpr unsigned BLK_B = 1664;
constexpr unsigned PSTEP_B = 1024 + 8 * BLK_B;
constexpr size_t PPANEL_B = (size_t)KSTEPS * PSTEP_B;

constexpr size_t OFF_CALL = 0;
constexpr size_t OFF_TMO = 4096;
constexpr size_t OFF_TICKET = 5120;
#define FCOPY 8
constexpr size_t OFF_FLAG = 1u << 18;
constexpr size_t OFF_RDY = 1u << 20;
constexpr int RDY_N = 1024, RDY_AWAY = 128, RDY_OUT = 384, RDY_HOP1 = 864, RDY_HOP2 = 896;
#define RELAY 32
#define FILL 0
#define HOPF 1
constexpr size_t OFF_DBG = 65536;
constexpr size_t OFF_OWN = 2u << 20;
constexpr size_t OFF_PACK = OFF_OWN + 32 * PANEL_B;
constexpr size_t OFF_RELAY = OFF_PACK + 32 * PPANEL_B;
constexpr size_t OFF_PULL = OFF_RELAY + PANEL_B;
#define HX 20
#define HW 6
constexpr int OUT_CHUNKS = 480;
#define CSTRIDE 8192
constexpr unsigned SUM_B = 3456, CHUNK_B = CSTRIDE;
constexpr size_t OFF_STG = OFF_PULL + 2 * NPANEL * PANEL_B;
constexpr size_t OFF_SPARE = OFF_STG + (size_t)4 * OUT_CHUNKS * CHUNK_B;
constexpr size_t WS_NEED = OFF_SPARE + (size_t)4 * OUT_CHUNKS * 2048;
static_assert(RELAY % 4 == 0 && RELAY <= 64, "");
static_assert(NPANEL <= 64 && OFF_FLAG + FCOPY * 65536 <= OFF_RDY && OFF_RDY + NR * RDY_N * 128 <= OFF_OWN, "");

constexpr int NWG = 256, NTHREADS = 512;
constexpr int GSLOTS = 24;
constexpr int NGEMM = GSLOTS * 8 + 8;
constexpr int PW = 8;
#define WIN 5
#define WIN1 6
#define WMASK 0
constexpr int NJOBS = 3;
constexpr unsigned long long DEADLINE = 100000000ull;

constexpr unsigned LDS_MB = 18 * 8192;
constexpr unsigned LDS_TMO = LDS_MB + NJOBS * 4 * 512;
constexpr unsigned LDS_LUT = LDS_TMO + NJOBS * 16;
constexpr unsigned LDS_FILL = LDS_LUT + 64;
constexpr unsigned LDS_CALL = LDS_FILL + 16;
constexpr unsigned LDS_EPI = LDS_CALL + 16;
constexpr unsigned LDS_BYTES = LDS_EPI + 4 * 512;
constexpr unsigned WAVE_LDS = LDS_MB / 8;

struct Params {
  const float* x;
  const float* w;
  char* mine;
  int rank;
  unsigned counts;
  float* out;
  char* ws[NR];
  u32x4 plan[NGEMM];
};

__device__ __forceinline__ int xcc_id() {
  return __builtin_amdgcn_s_getreg(20 | (3 << 11)) & 7;
}
#define STAMP(idx) do {} while (0)

#define EXP 0
#define EXPON(bit) ((EXP & (bit)) && call > 2)
__device__ __forceinline__ void store_sys(void* ptr, u32x4 v) {
  asm volatile("global_store_dwordx4 %0, %1, off sc0 sc1\n\ts_nop 1" :: "v"(ptr), "v"(v) : "memory");
}
__device__ __forceinline__ void wait_vm0() { asm volatile("s_waitcnt vmcnt(0)" ::: "memory"); }
__device__ __forceinline__ void wg_barrier() {
  asm volatile("" ::: "memory");
  __builtin_amdgcn_s_barrier();
  asm volatile("" ::: "memory");
}
#define OUT_NT " nt"
__device__ __forceinline__ void store_out(void* ptr, u32x4 v) {
  asm volatile("global_store_dwordx4 %0, %1, off sc0 sc1" OUT_NT "\n\ts_nop 1" :: "v"(ptr), "v"(v) : "memory");
}

__device__ __forceinline__ void set_local(char* ws, int copy, int step, unsigned v) {
  __hip_atomic_store((GAS unsigned*)(ws + OFF_FLAG) + copy * 16384 + (step >> 7) * 256 + (step & 7) * 32 + ((step & 127) >> 3), v,
                     __ATOMIC_RELAXED, __HIP_MEMORY_SCOPE_SYSTEM);
}
__device__ __forceinline__ void set_rdy(char* peer_ws, int rank, int idx, unsigned call, unsigned lines) {
  __hip_atomic_store((GAS unsigned*)(peer_ws + OFF_RDY) + (rank * RDY_N + idx) * 32, call << 6 | lines, __ATOMIC_RELAXED,
                     __HIP_MEMORY_SCOPE_SYSTEM);
}
__device__ __forceinline__ unsigned own_idx(unsigned call, int idx) { return (call & 1) * 16 + idx; }
__device__ __forceinline__ size_t pulled(unsigned call) { return OFF_PULL + (call & 1) * (NPANEL * PANEL_B); }
__device__ __forceinline__ int fresh_lane() {
  int l;
  asm volatile("v_mbcnt_lo_u32_b32 %0, -1, 0\n\tv_mbcnt_hi_u32_b32 %0, -1, %0" : "=v"(l));
  return l;
}
__device__ __forceinline__ void cbar() { asm volatile("" ::: "memory"); }
__device__ __forceinline__ unsigned wave_scan(unsigned v) {
  v += __builtin_amdgcn_update_dpp(0u, v, 0x111, 0xf, 0xf, false);
  v += __builtin_amdgcn_update_dpp(0u, v, 0x112, 0xf, 0xf, false);
  v += __builtin_amdgcn_update_dpp(0u, v, 0x114, 0xf, 0xf, false);
  v += __builtin_amdgcn_update_dpp(0u, v, 0x118, 0xf, 0xf, false);
  v += __builtin_amdgcn_update_dpp(0u, v, 0x142, 0xa, 0xf, false);
  v += __builtin_amdgcn_update_dpp(0u, v, 0x143, 0xc, 0xf, false);
  return v;
}

__device__ __forceinline__ unsigned codec_lut(int m, bool expand) {
  unsigned sel = 0x0c0c0c0cu;
  int n = 0;
  for (int i = 0; i < 4; ++i)
    if ((m >> i) & 1) {
      const int at = expand ? i : n, v = expand ? n : i;
      sel = (sel & ~(0xffu << (8 * at))) | ((unsigned)v << (8 * at));
      ++n;
    }
  return sel;
}

__device__ __forceinline__ unsigned lut_at(const LAS char* lut, unsigned E) {
  return *(const LAS unsigned*)(lut + __builtin_amdgcn_udot4(E, 0x08040201u, 0u, false));
}
__device__ __forceinline__ void lds_or(LAS unsigned* q, unsigned v) {
  __hip_atomic_fetch_or(q, v, __ATOMIC_RELAXED, __HIP_MEMORY_SCOPE_WORKGROUP);
}
struct Codes {
  unsigned pk[4], n[4], EM = 0, XM = 0;
  __device__ __forceinline__ void add(int k, const LAS char* lut, unsigned c, unsigned E, unsigned X) {
    const unsigned z = __builtin_amdgcn_perm(0u, c, lut_at(lut, E));
    pk[k] = __builtin_amdgcn_udot4(z, 0x00400801u, (z >> 24) << 9, false);
    n[k] = __builtin_popcount(E);
    EM |= (E >> 2) << k;
    XM |= X >> (7 - k);
  }
  __device__ __forceinline__ void put(LAS char* list, LAS char* patch, unsigned pre, const unsigned (&H)[4], unsigned base) {
    const unsigned long long W = (unsigned long long)(pk[0] | (pk[1] << (3 * n[0])))
        | ((unsigned long long)(pk[2] | (pk[3] << (3 * n[2]))) << (3 * (n[0] + n[1])));
    const unsigned o = 3 * pre, sh = o & 31u;
    LAS unsigned* q = (LAS unsigned*)(list + (o >> 5) * 4);
    const unsigned long long Ws = W << sh;
    lds_or(q, (unsigned)Ws);
    lds_or(q + 1, (unsigned)(Ws >> 32));
    lds_or(q + 2, ((unsigned)(W >> 32) >> 1) >> (31u - sh));
    cbar();
    const unsigned H0 = H[0], H1 = H[1], H2 = H[2], H3 = H[3];
    for (; XM; XM &= XM - 1u, patch -= 2) {
      const unsigned pos = __builtin_ctz(XM), k = pos & 3u, i8 = pos & 24u;
      const unsigned Hk = k & 2u ? (k & 1u ? H3 : H2) : (k & 1u ? H1 : H0);
      const unsigned kb = 0x01010101u << k;
      const unsigned rk = pre + __builtin_popcount(EM & ((kb - 0x01010101u) | (kb & ((1u << i8) - 1u))));
      const unsigned at = (unsigned)(uintptr_t)patch;
      lds_or((LAS unsigned*)(uintptr_t)(at & ~3u), ((rk << 5) | (((Hk >> (i8 + 2)) & 31u) ^ (base + 3u))) << ((at & 2u) * 8));
    }
    cbar();
  }
};

__device__ __forceinline__ void pack_block(u32x4 f0, u32x4 f1, LAS char* wl, const LAS char* lut, LAS char* lists,
                                           LAS unsigned* fill, char* step, int blk) {
  const int lane = threadIdx.x & 63;
  const unsigned h[8] = {f0[0], f0[1], f0[2], f0[3], f1[0], f1[1], f1[2], f1[3]};
  char* const dst = step + 1024 + blk * BLK_B;
  u32x4 D;
  unsigned H[4], t[4], sum = 0, sg = 0;
#pragma unroll
  for (int k = 0; k < 4; ++k) {
    D[k] = __builtin_amdgcn_perm(h[2 * k + 1], h[2 * k], 0x06040200u);
    H[k] = __builtin_amdgcn_perm(h[2 * k + 1], h[2 * k], 0x07050301u);
    sum = __builtin_amdgcn_udot4(H[k] & 0x7c7c7c7cu, 0x01010101u, sum, false);
    sg |= (H[k] & 0x80808080u) >> (7 - k);
  }
  store_sys(dst + lane * 16, D);
  const unsigned sgn = __builtin_amdgcn_update_dpp(0u, sg, 0xb1, 0xf, 0xf, false);
  if (!(lane & 1)) *(LAS unsigned*)(wl + 512 + lane * 2) = sg | (sgn << 4);
  const unsigned tot = __builtin_amdgcn_readlane(wave_scan(sum), 63);
  const unsigned base = min((tot + 256u) >> 12, 28u);
  Codes c;
  unsigned W = 0, n2 = 0;
#pragma unroll
  for (int k = 0; k < 4; ++k) {
    const unsigned g = ((H[k] >> 2) & 0x1f1f1f1fu) + (0x43u - base) * 0x01010101u;
    const unsigned x = g ^ 0x40404040u, y = (g + 0x07070707u) ^ 0x40404040u;
    const unsigned tt = __builtin_amdgcn_perm(0xff1c0804u, 0x001e1dffu, x | (((x & 0x78787878u) + 0x78787878u) & 0x80808080u));
    t[k] = (tt & 0x0c0c0c0cu) | (H[k] & 0x03030303u);
    const unsigned E = (tt >> 2) & 0x04040404u;
    W |= __builtin_amdgcn_udot4(__builtin_amdgcn_perm(0u, tt & 0x03030303u, lut_at(lut, E)), 0x40100401u, 0u, false) << (2 * n2);
    n2 += __builtin_popcount(E);
    c.add(k, lut, y & 0x07070707u, (tt >> 3) & 0x04040404u, ((y & 0x78787878u) + 0x78787878u) & tt & 0x80808080u);
  }
  *(LAS u32x2*)(wl + lane * 8) = (u32x2){t[0] | (t[1] << 4), t[2] | (t[3] << 4)};
  const unsigned mine = n2 | ((c.n[0] + c.n[1] + c.n[2] + c.n[3]) << 11) | (__builtin_popcount(c.XM) << 22);
  const unsigned incl = wave_scan(mine), pre = incl - mine;
  const unsigned all = __builtin_amdgcn_readlane(incl, 63), n4 = all >> 22, l2 = ((all & 2047u) + 15u) >> 4;
  const unsigned len = l2 + ((3u * ((all >> 11) & 2047u) + 16u * n4 + 31u) >> 5);
  unsigned head = 1u << 31, off = 0;
  cbar();
  if (l2 <= 32u && len < 64u && n4 < 64u) {
    asm volatile("" : "+v"(off));
    if (lane == 0) off = __hip_atomic_fetch_add(fill + off, len, __ATOMIC_RELAXED, __HIP_MEMORY_SCOPE_WORKGROUP);
    off = __builtin_amdgcn_readfirstlane(off);
    if (off + len < 256u) {
      head = base | (n4 << 5) | (l2 << 11) | (len << 17) | (off << 23);
      LAS char* const esc = lists + off * 4;
      const unsigned sh = (2u * pre) & 31u;
      LAS unsigned* q = (LAS unsigned*)esc + ((pre & 2047u) >> 4);
      lds_or(q, W << sh);
      lds_or(q + 1, (W >> 1) >> (31u - sh));
      c.put(esc + l2 * 4, esc + len * 4 - 2 - 2 * (pre >> 22), (pre >> 11) & 2047u, H, base);
    }
  }
  if (lane == 0) ((LAS unsigned*)lists)[blk] = head;
  cbar();
}
__device__ __forceinline__ void pack_flush(LAS char* wl, LAS char* lists, char* step, int blk) {
  const int lane = threadIdx.x & 63;
  if (lane < 48) {
    const unsigned line = blk * 128 + (lane - 40) * 16;
    LAS u32x4* const from = (LAS u32x4*)(lane < 40 ? wl + lane * 16 : lists + line);
    const u32x4 v = *from;
    *from = (u32x4){0u, 0u, 0u, 0u};
    store_sys(step + (lane < 40 ? 2048 + blk * BLK_B + lane * 16 : line), v);
  }
}
__device__ __forceinline__ void lds_barrier() {
  asm volatile("s_waitcnt lgkmcnt(0)\n\ts_barrier" ::: "memory");
}
__device__ __forceinline__ unsigned lines_used(LAS unsigned* fill) {
  const unsigned n = *fill;
  *fill = 8u;
  return min(n - 1u, 255u) >> 5;
}

constexpr unsigned SUM_BASE = 24;
__device__ __forceinline__ void pack_sums(int lane, const f32x4 (&a)[4], LAS char* wl, const LAS char* lut, char* dst, char* spare) {
  if (lane < 36) *(LAS unsigned*)(wl + 256 + lane * 4) = 0u;
  u32x4 P[3], E;
  unsigned H[4], t[4], sum = 0, any = 0;
#pragma unroll
  for (int k = 0; k < 4; ++k) {
    unsigned v[4];
#pragma unroll
    for (int i = 0; i < 4; ++i) v[i] = __builtin_rotateleft32(__builtin_bit_cast(u32x4, a[i])[k], 1);
#pragma unroll
    for (int i = 0; i < 3; ++i) P[i][k] = __builtin_amdgcn_perm(v[3], v[i], 0x04020100u + ((unsigned)i << 24));
    E[k] = __builtin_amdgcn_perm(v[1], v[0], 0x0c0c0703u) | __builtin_amdgcn_perm(v[3], v[2], 0x07030c0cu);
    sum = __builtin_amdgcn_udot4(E[k], 0x01010101u, sum, false);
  }
#pragma unroll
  for (int i = 0; i < 3; ++i) store_sys(dst + i * 1024 + lane * 16, P[i]);
  const unsigned tot = __builtin_amdgcn_readlane(wave_scan(sum), 63);
  const unsigned off = min(max((tot + 64u) >> 10, SUM_BASE) - SUM_BASE, 223u);
#pragma unroll
  for (int k = 0; k < 4; ++k) {
    const unsigned d = E[k] - off * 0x01010101u;
    any |= d;
    H[k] = d << 2;
  }
  Codes c;
#pragma unroll
  for (int k = 0; k < 4; ++k) {
    const unsigned g = ((H[k] & 0x7c7c7c7cu) | 0x80808080u) - SUM_BASE * 0x04040404u;
    const unsigned a7 = g & 0x7f7f7f7fu, hi = a7 + 0x70707070u, lo = a7 + 0x1c1c1c1cu;
    const unsigned o7 = (~g | hi) & 0x80808080u;
    t[k] = (g | (o7 >> 4) | (o7 >> 5)) & 0x0c0c0c0cu;
    const unsigned nb = ~g & 0x80808080u;
    c.add(k, lut, (g >> 2) & ((nb >> 4) - (nb >> 7)), t[k] & (t[k] >> 1), ((g & hi) | (~g & ~lo)) & 0x80808080u);
  }
  const unsigned mine = (c.n[0] + c.n[1] + c.n[2] + c.n[3]) | (__builtin_popcount(c.XM) << 16);
  const unsigned incl = wave_scan(mine), pre = incl - mine;
  const unsigned all = __builtin_amdgcn_readlane(incl, 63), n4 = all >> 16;
  const bool raw = __builtin_amdgcn_ballot_w64((any & 0xe0e0e0e0u) != 0u) != 0ull || 3u * (all & 0xffffu) + 16u * n4 > 1008u;
  cbar();
  if (!raw) c.put(wl + 256, wl + 380 - 2 * (pre >> 16), pre & 0xffffu, H, SUM_BASE);
  if (lane == 0) *(LAS unsigned short*)(wl + 382) = (unsigned short)(raw ? 0x8000u : (off & 31u) | (n4 << 5) | (off >> 5 << 11));
  *(LAS unsigned*)(wl + lane * 4) = (t[0] >> 2) | t[1] | (t[2] << 2) | (t[3] << 4);
  cbar();
  if (lane < 24) store_sys(dst + 3072 + lane * 16, *(const LAS u32x4*)(wl + lane * 16));
  if (raw) store_sys(spare + lane * 16, E);
  cbar();
}

__device__ __forceinline__ u32x2 read_codes(int lane, const LAS char* list) {
  const LAS unsigned* sp = (const LAS unsigned*)(list + ((3 * lane) & ~3));
  return (u32x2){sp[0], sp[1]};
}
__device__ __forceinline__ void expand_codes(int lane, u32x2 codes, unsigned base, unsigned first, unsigned keep,
                                             bool patch, unsigned ent, LAS char* ex) {
  __builtin_amdgcn_sched_barrier(0);
  const unsigned x = __builtin_amdgcn_alignbyte(codes[1], codes[0], 3 * lane);
  const unsigned b3 = (base + 3u) * 0x04040404u, b8 = (base + first) * 0x04040404u;
  const unsigned T0 = ((b8 + 0x0c080400u) ^ b3) & keep, T1 = ((b8 + 0x1c181410u) ^ b3) & 0x7c7c7c7cu;
  u32x2 X;
#pragma unroll
  for (int i = 0; i < 2; ++i) {
    const unsigned y = ((x >> (12 * i)) & 0x3fu) | ((i ? x >> 2 : x << 10) & 0x3f0000u);
    X[i] = __builtin_amdgcn_perm(T1, T0, (y | (y << 5)) & 0x07070707u);
  }
  *(LAS u32x2*)(ex + lane * 8) = X;
  cbar();
  if (patch) *(LAS unsigned char*)(ex + ((ent >> 5) & 511u)) = (unsigned char)((ent & 31u) << 2);
  cbar();
}
template <int N>
__device__ __forceinline__ void pick(const unsigned (&E)[N], const unsigned (&sel)[N], const LAS char* ex, unsigned pos,
                                     unsigned (&out)[N]) {
  unsigned at[N], w0[N], w1[N];
#pragma unroll
  for (int k = 0; k < N; ++k) { at[k] = pos; pos += __builtin_popcount(E[k]); }
  __builtin_amdgcn_sched_barrier(0);
#pragma unroll
  for (int k = 0; k < N; ++k) {
    const LAS unsigned* wp = (const LAS unsigned*)(ex + (at[k] & ~3u));
    w0[k] = wp[0];
    w1[k] = wp[1];
  }
  __builtin_amdgcn_sched_barrier(0);
#pragma unroll
  for (int k = 0; k < N; ++k) out[k] = __builtin_amdgcn_perm(0u, __builtin_amdgcn_alignbyte(w1[k], w0[k], at[k]), sel[k]);
}

__device__ __forceinline__ void unpack_block(int lane, const LAS char* step, int blk, unsigned hdr, LAS char* ex,
                                             const LAS char* lut, u32x4& f0, u32x4& f1) {
  const LAS char* const src = step + 1024 + blk * BLK_B;
  const LAS char* const esc = step + (hdr >> 23) * 4;
  const unsigned base = hdr & 31u;
  const bool patch = lane < ((hdr >> 5) & 63u);
  const u32x2 x3 = read_codes(lane, esc + ((hdr >> 11) & 63u) * 4);
  const unsigned ent = *(const LAS unsigned short*)(esc + ((hdr >> 17) & 63u) * 4 - (patch ? 2 + 2 * lane : 0));
  const unsigned x2 = *(const LAS unsigned short*)(esc + 2 * lane);
  const u32x2 N = *(const LAS u32x2*)(src + 1024 + lane * 8);
  unsigned sg = *(const LAS unsigned*)(src + 1536 + (lane >> 1) * 4);
  const u32x4 D = *(const LAS u32x4*)(src + lane * 16);
  expand_codes(lane, x3, base, 22u, 0x7c7c7c7cu, patch, ent, ex + 512);
  sg >>= (lane & 1) * 4;
  const unsigned E0 = N[0] & (N[0] >> 1) & 0x44444444u, E1 = N[1] & (N[1] >> 1) & 0x44444444u;
  unsigned c[2], F[2], sf[2], X[2], E[4], se[4], Y[4], h[8];
#pragma unroll
  for (int i = 0; i < 2; ++i) {
    const unsigned y = ((x2 >> (8 * i)) & 0xfu) | ((x2 << (12 - 8 * i)) & 0xf0000u);
    c[i] = (y | (y << 6)) & 0x03030303u;
    F[i] = (c[i] & (c[i] >> 1)) << 2;
    sf[i] = lut_at(lut, F[i]);
  }
#pragma unroll
  for (int k = 0; k < 4; ++k) {
    E[k] = ((k & 2 ? E1 : E0) >> (4 * (k & 1))) & 0x04040404u;
    se[k] = lut_at(lut, E[k]);
  }
  const unsigned mine = (__builtin_popcount(E0) + __builtin_popcount(E1)) | ((__builtin_popcount(F[0]) + __builtin_popcount(F[1])) << 16);
  const unsigned pre = wave_scan(mine) - mine;
  pick<2>(F, sf, ex + 512, pre >> 16, X);
  const unsigned T = ((base * 0x00040400u + 0x007c7800u) ^ ((base + 3u) * 0x00040400u)) & 0x007c7c00u;
  *(LAS u32x2*)(ex + lane * 8) = (u32x2){X[0] | __builtin_amdgcn_perm(0u, T, c[0]), X[1] | __builtin_amdgcn_perm(0u, T, c[1])};
  cbar();
  pick<4>(E, se, ex, pre & 0xffffu, Y);
#pragma unroll
  for (int k = 0; k < 4; ++k) {
    const unsigned t = ((N[k >> 1] >> (4 * (k & 1))) & 0x0f0f0f0fu) + base * 0x04040404u;
    const unsigned H = (t ^ Y[k]) | ((sg << (7 - k)) & 0x80808080u);
    h[2 * k] = __builtin_amdgcn_perm(H, D[k], 0x05010400u);
    h[2 * k + 1] = __builtin_amdgcn_perm(H, D[k], 0x07030602u);
  }
  cbar();
  f0 = (u32x4){h[0], h[1], h[2], h[3]};
  f1 = (u32x4){h[4], h[5], h[6], h[7]};
}

__device__ __forceinline__ void unpack_sums(int lane, const LAS char* blk, LAS char* ex, const LAS char* lut, const char* spare,
                                            f32x4 (&y)[4]) {
  const unsigned ent = *(const LAS unsigned short*)(blk + 3454 - 2 * lane);
  const u32x2 x3 = read_codes(lane, blk + 3328);
  const unsigned C = *(const LAS unsigned*)(blk + 3072 + lane * 4);
  u32x4 P[3], X;
#pragma unroll
  for (int i = 0; i < 3; ++i) P[i] = *(const LAS u32x4*)(blk + i * 1024 + lane * 16);
  const unsigned hdr = __builtin_amdgcn_readfirstlane(ent);
  if (hdr & 0x8000u) {
    asm volatile("global_load_dwordx4 %0, %1, off sc0 sc1\n\ts_waitcnt vmcnt(0)" : "=&v"(X) : "v"(spare + lane * 16) : "memory");
  } else {
    expand_codes(lane, x3, SUM_BASE, 24u, 0x7c7c7c00u, (unsigned)(lane - 1) < ((hdr >> 5) & 63u), ent, ex);
    unsigned t[4], E[4], se[4], Y[4];
#pragma unroll
    for (int k = 0; k < 4; ++k) {
      t[k] = (k ? C >> (2 * k - 2) : C << 2) & 0x0c0c0c0cu;
      E[k] = t[k] & (t[k] >> 1);
      se[k] = lut_at(lut, E[k]);
    }
    const unsigned mine = __builtin_popcount(C & (C >> 1) & 0x55555555u);
    pick<4>(E, se, ex, wave_scan(mine) - mine, Y);
    const unsigned off = ((hdr & 31u) | (hdr >> 11 << 5)) * 0x01010101u;
#pragma unroll
    for (int k = 0; k < 4; ++k) X[k] = (((t[k] + SUM_BASE * 0x04040404u) ^ Y[k]) >> 2) + off;
    cbar();
  }
#pragma unroll
  for (int k = 0; k < 4; ++k) {
    unsigned v[4];
#pragma unroll
    for (int i = 0; i < 3; ++i) v[i] = __builtin_amdgcn_perm(X[k], P[i][k], 0x04020100u + ((unsigned)i << 24));
    v[3] = __builtin_amdgcn_perm(P[1][k], P[0][k], 0x0c0c0703u) | __builtin_amdgcn_perm(X[k], P[2][k], 0x07030c0cu);
#pragma unroll
    for (int i = 0; i < 4; ++i) y[k][i] = __builtin_bit_cast(float, __builtin_rotateright32(v[i], 1));
  }
}

__device__ __forceinline__ float silu(float y) {
  const float c = fminf(fmaxf(y, -60.0f), 60.0f);
  return y * __builtin_amdgcn_rcpf(1.0f + __builtin_amdgcn_exp2f(c * -1.44269504088896341f));
}

__device__ __forceinline__ unsigned* call_word(const Params& p) { return (unsigned*)(p.mine + OFF_CALL) + blockIdx.x; }
template <int LOADS>
__device__ __forceinline__ unsigned begin(const Params& p, LAS char* smem, bool puller) {
  *(LAS u32x4*)(smem + LDS_MB + min(threadIdx.x * 16u, LDS_LUT - LDS_MB - 16u)) = (u32x4){0u, 0u, 0u, 0u};
  if (threadIdx.x < 20) ((LAS unsigned*)(smem + LDS_LUT))[threadIdx.x] = threadIdx.x < 16 ? codec_lut(threadIdx.x, puller) : 8u;
  asm volatile("s_waitcnt vmcnt(%0) lgkmcnt(0)" :: "n"(LOADS) : "memory");
  if (threadIdx.x == 0) STAMP(0);
  wg_barrier();
  return __builtin_amdgcn_readfirstlane(*(const LAS unsigned*)(smem + LDS_CALL)) + 1u;
}
static_assert(LDS_LUT - LDS_MB <= NTHREADS * 16 && (LDS_LUT - LDS_MB) % 16 == 0, "");

constexpr int XJ = (KSTEPS - HX) / 2;
static_assert(HX % 2 == 0, "");
__device__ __forceinline__ int x_unit(int u) {
  if (u < FILL) return 6 * KSTEPS + u;
  u -= FILL;
  constexpr int HF = HOPF ? RELAY : 0;
  if (u < HF) return 5 * KSTEPS + u;
  const int j = u < 7 * HF ? (u - HF) / 6 : u / 7, r = u < 7 * HF ? u - HF - j * 6 : u - j * 7;
  if (u < 7 * KSTEPS && (r >= 2 || j < XJ)) return r >= 2 ? (r - 1) * KSTEPS + j : HX + 2 * j + r;
  const int i = (u < 7 * KSTEPS ? 2 * j + r : u - 5 * KSTEPS) - 2 * XJ + FILL;
  return i < 2 * FILL ? 7 * KSTEPS + i - FILL : (6 + (i & 1)) * KSTEPS + (i >> 1);
}
#define XROUNDS 11
#define XLONG 5
#define INB 1
#define IN_BITS " nt"
__device__ __forceinline__ void load_in(const float* a, const float* b, unsigned dst) {
  unsigned keep;
  asm volatile("s_mov_b32 %0, m0\n\ts_mov_b32 m0, %3\n\ts_nop 0\n\tglobal_load_lds_dwordx4 %1, off" IN_BITS "\n\t"
               "s_add_u32 m0, m0, 1024\n\ts_nop 0\n\tglobal_load_lds_dwordx4 %2, off" IN_BITS "\n\ts_mov_b32 m0, %0"
               : "=&s"(keep) : "v"(a), "v"(b), "s"(dst) : "memory", "scc");
}
__device__ __forceinline__ void pace(unsigned t0, int at) {
  while ((int)((unsigned)__builtin_amdgcn_s_memrealtime() - t0) * 8 < at) __builtin_amdgcn_s_sleep(2);
}
#define XT 40
#define XO 16
#define WT 88
#define WO 24
#define LEAD 1200
template <int N>
__device__ __forceinline__ void wait_vm() { asm volatile("s_waitcnt vmcnt(%0)" :: "n"(N) : "memory"); }
__device__ __forceinline__ u32x4 cvt8(f32x4 a, f32x4 b) {
  half8 h;
  h[0] = (_Float16)a[0]; h[1] = (_Float16)a[1]; h[2] = (_Float16)a[2]; h[3] = (_Float16)a[3];
  h[4] = (_Float16)b[0]; h[5] = (_Float16)b[1]; h[6] = (_Float16)b[2]; h[7] = (_Float16)b[3];
  u32x4 r = __builtin_bit_cast(u32x4, h);
  asm volatile("" : "+v"(r));
  return r;
}
__device__ __forceinline__ char* member(const Params& p, int group, int i) {
  char *a = p.ws[group], *b = p.ws[group + 1], *c = p.ws[group + 2], *d = p.ws[group + 3];
  asm volatile("" : "+s"(a), "+s"(b), "+s"(c), "+s"(d));
  return i & 2 ? (i & 1 ? d : c) : (i & 1 ? b : a);
}
__device__ __forceinline__ unsigned convert_x(const Params& p, unsigned role, unsigned counts, int rank, LAS char* smem) {
  const int lane = threadIdx.x & 63, wave = __builtin_amdgcn_readfirstlane(threadIdx.x >> 6), t = threadIdx.x;
  const int n0 = counts & 255u, n1 = (counts >> 8) & 255u, n2 = (counts >> 16) & 255u;
  const int first = role & 255u, all = XROUNDS * n0, most = min(all + XLONG * n1, 8 * KSTEPS - HX);
  const int count = first < n2 ? 8 * KSTEPS - HX : first < n1 ? most : all;
  const bool push = role & 0x1000u;
  char* const ws = p.mine;
  LAS char* const wl = smem + wave * WAVE_LDS;
  const float* src0 = p.x + (size_t)((wave >> 2) * 64 + (wave & 1) * 32 + (lane >> 3)) * KDIM + (wave & 2) * 16 + ((lane ^ (lane >> 4)) & 7) * 4;
  const int odd8 = 8 * KDIM + (lane & 4 ? -16 : 16);
  auto fetch = [&](unsigned buf, int u) {
    const float* s = src0 + (size_t)(u >> 7) * 128 * KDIM + (u & 127) * 64;
    const unsigned d = (unsigned)(uintptr_t)wl + 2048 + buf;
    load_in(s, s + odd8, d);
    load_in(s + 16 * KDIM, s + 16 * KDIM + odd8, d + 2048);
  };
  u32x4 h0, h1;
  auto take = [&](unsigned buf) {
    const LAS f32x4* v = (const LAS f32x4*)(wl + 2048 + buf) + (lane & 15) * 8;
    const int i = (lane >> 4) ^ ((lane >> 1) & 7);
    const f32x4 a = v[i], b = v[i ^ 4], c = v[128 + i], d = v[128 + (i ^ 4)];
    h0 = cvt8(a, b);
    h1 = cvt8(c, d);
  };
  auto next = [=](int it) { return it + (int)((counts >> (it < all ? 0 : it < most ? 8 : 16)) & 255u); };
#define LATE 6
  const unsigned t0 = __builtin_amdgcn_s_memrealtime();
  if (!push && (LATE & 1)) pace(t0, ((int)(role >> 16) - XO) * XT);
  fetch(0, push ? first : x_unit(role >> 16));
  const unsigned call = begin<4>(p, smem, false);
  char* const in = member(p, rank & 4, t);
  if (push) {
    const unsigned at = first * STEP_B + wave * 2048 + lane * 16;
    char* b[4];
#pragma unroll
    for (int i = 0; i < 4; ++i)
      b[i] = at + (i == 3 ? ws + OFF_OWN + own_idx(call, 0) * PANEL_B
                          : member(p, rank & 4, rank + 1 + (i + first) % 3) + pulled(call) + (rank & 3) * 6 * PANEL_B);
    asm volatile("s_waitcnt vmcnt(0)" : "+v"(b[0]), "+v"(b[1]), "+v"(b[2]), "+v"(b[3]) :: "memory");
    take(0);
    if (wave == 0) STAMP(24);
#pragma unroll
    for (int i = 0; i < 4; ++i) {
      store_sys(b[i], h0);
      store_sys(b[i] + 1024, h1);
    }
    if (wave == 0) STAMP(25);
    wait_vm0();
    wg_barrier();
    if (wave == 0) STAMP(26);
    if (t < 4 * FCOPY) set_local(in, t >> 2, (rank & 3) * 6 * KSTEPS + first, call);
    return call;
  }
  char* const across = member(p, (rank & 4) ^ 4, t);
  LAS unsigned* const fill = (LAS unsigned*)(smem + LDS_FILL);
  unsigned odd = 0;
  auto ahead = [&](unsigned buf, int it) {
    if (it < count) { fetch(buf, x_unit(it)); wait_vm<4>(); } else wait_vm0();
  };
  wait_vm0();
  for (int it = first, it1 = next(it), it2; it < count; it = it1, it1 = it2, odd ^= 8192u) {
    const int u = x_unit(it == first ? role >> 16 : it);
    it2 = next(it1);
    take(odd);
    char* b = ws + OFF_OWN + (own_idx(call, 0) * KSTEPS + u) * STEP_B + wave * 2048 + lane * 16;
    store_sys(b, h0);
    store_sys(b + 1024, h1);
    char* pb = ws + OFF_PACK + (own_idx(call, 0) * KSTEPS + u) * PSTEP_B;
    pack_block(h0, h1, wl, smem + LDS_LUT, smem + LDS_MB, fill, pb, wave);
    lds_barrier();
    pack_flush(wl, smem + LDS_MB, pb, wave);
    unsigned lines = 0;
    if (wave == 0) lines = lines_used(fill);
    ahead(it == first ? 8192u : odd, it == first ? it1 : it2);
    lds_barrier();
    if (t < 4 + FCOPY) {
      const int tile = u >> 7, ks = u & 127;
      if (t >= 4) set_local(ws, t - 4, (tile < 6 ? (rank & 3) * 6 : PA_OWNAWAY - 6) * KSTEPS + u, call);
      else if (tile >= 6) set_rdy(across, rank, RDY_AWAY + u - 6 * KSTEPS, call, lines);
      else if (tile == 5 && ks < RELAY) { if (t == (ks & 3)) set_rdy(across, rank, RDY_HOP1 + (ks >> 2), call, lines); }
      else if (t != (rank & 3)) set_rdy(in, rank, u, call, lines);
    }
    if (it == first) {
      if (wave == 0) STAMP(27);
      ahead(0, it2);
    }
    if (it1 < count) pace(t0, (it1 - XO) * XT + LEAD);
  }
  return call;
}

__device__ __forceinline__ unsigned convert_w(const Params& p, unsigned role, unsigned counts, int rank, LAS char* smem) {
  const int lane = threadIdx.x & 63, wave = __builtin_amdgcn_readfirstlane(threadIdx.x >> 6), t = threadIdx.x;
  const int count = KSTEPS * 4, stride = counts >> 24, first = role & 255u;
  const bool push = role & 0x1000u;
  char* const ws = p.mine;
  LAS char* const wl = smem + wave * WAVE_LDS;
  const float* src0 = p.w + (size_t)((wave & 1) * 32 + (lane >> 4) * 4) * NSH + (wave >> 1) * 64 + (lane & 15) * 4;
  auto fetch = [&](unsigned buf, int u) {
    const float* s = src0 + (size_t)(u >> 2) * 64 * NSH + (u & 3) * 256;
#pragma unroll
    for (int j = 0; j < 8; j += 2) {
      const float* r = s + ((j & 3) + (j >> 2) * 16) * NSH;
      load_in(r, r + NSH, (unsigned)(uintptr_t)wl + 2048 + buf + j * 1024);
    }
  };
  u32x4 h[4];
  auto take = [&](unsigned buf) {
    f32x4 v[8];
#pragma unroll
    for (int j = 0; j < 8; ++j) v[j] = ((const LAS f32x4*)(wl + 2048 + buf))[j * 64 + lane];
#pragma unroll
    for (int f = 0; f < 4; ++f)
      h[f] = cvt8((f32x4){v[0][f], v[1][f], v[2][f], v[3][f]}, (f32x4){v[4][f], v[5][f], v[6][f], v[7][f]});
  };
  const bool late = push && (LATE & 4) && wave >= 4;
  const unsigned t0 = __builtin_amdgcn_s_memrealtime();
  if (!push && (LATE & 2)) pace(t0, (first - 4 * HW - WO) * WT);
  if (!late) fetch(0, first);
  const unsigned call = begin<8>(p, smem, false);
  if (late) fetch(0, first);
  char* const own = ws + OFF_OWN + own_idx(call, 8 + (wave >> 2)) * PANEL_B + (wave & 3) * 4096 + lane * 16;
  char* const across = member(p, (rank & 4) ^ 4, push ? first : t);
  if (push) {
    const int q = first & 3, ks = first >> 2;
    char* b = own + q * 2 * PANEL_B + ks * STEP_B;
    char* r = across + pulled(call) + (PB_FOREIGN + (rank & 3) * 2 + (wave >> 2)) * PANEL_B + ks * STEP_B + (wave & 3) * 4096 + lane * 16;
    asm volatile("s_waitcnt vmcnt(0)" : "+v"(b), "+v"(r) :: "memory");
    take(0);
    if (wave == 0) STAMP(28);
#pragma unroll
    for (int f = 0; f < 4; ++f) store_sys(r + f * 1024, h[f]);
#pragma unroll
    for (int f = 0; f < 4; ++f) store_sys(b + f * 1024, h[f]);
    if (wave == 0) STAMP(29);
    wait_vm0();
    wg_barrier();
    if (wave == 0) STAMP(30);
    if (t < 4 * FCOPY)
      set_local(t & 2 ? across : ws, t >> 2, ((t & 2 ? PB_FOREIGN + (rank & 3) * 2 : PB + q * 2) + (t & 1)) * KSTEPS + ks, call);
    return call;
  }
  LAS unsigned* const fill = (LAS unsigned*)(smem + LDS_FILL) + (wave >> 2);
  LAS char* const lists = smem + LDS_MB + (wave >> 2) * 2048;
  unsigned odd = 0;
  auto ahead = [&](unsigned buf, int u) {
    if (u < count) { fetch(buf, u); wait_vm<8>(); } else wait_vm0();
  };
  static_assert(FCOPY % 2 == 0, "");
  wait_vm0();
  for (int u = first; u < count; u += stride, odd ^= 8192u) {
    const int ks = u >> 2, q = u & 3, blk = (wave & 3) * 2;
    char* b = own + q * 2 * PANEL_B + ks * STEP_B;
    take(odd);
#pragma unroll
    for (int f = 0; f < 4; ++f) store_sys(b + f * 1024, h[f]);
    char* pb = ws + OFF_PACK + own_idx(call, 8 + q * 2 + (wave >> 2)) * PPANEL_B + ks * PSTEP_B;
    pack_block(h[0], h[1], wl, smem + LDS_LUT, lists, fill, pb, blk);
    pack_block(h[2], h[3], wl + 1024, smem + LDS_LUT, lists, fill, pb, blk + 1);
    lds_barrier();
    pack_flush(wl, lists, pb, blk);
    pack_flush(wl + 1024, lists, pb, blk + 1);
    unsigned lines = 0;
    if (wave == 0) lines = lines_used(fill) | lines_used(fill + 1) << 3;
    ahead(u == first ? 8192u : odd, u + (u == first ? stride : 2 * stride));
    lds_barrier();
    if (t < 2 * FCOPY) set_local(ws, t >> 1, (PB + q * 2 + (t & 1)) * KSTEPS + ks, call);
    else if (t == 2 * FCOPY + q) set_rdy(across, rank, ks, call, lines);
    if (u == first) {
      if (wave == 0) STAMP(31);
      ahead(0, u + 2 * stride);
    }
    if (u + stride < count) pace(t0, (u + stride - 4 * HW - WO) * WT + LEAD);
  }
  return call;
}

__device__ __forceinline__ int row_of(int r) { return ((RANK_AT >> (4 * r)) & 7) * NSH; }

#define ORDER_V 0x7648953210ull
constexpr unsigned long long BATCH_AT = ORDER_V;
constexpr unsigned long long batch_pos() {
  unsigned long long r = 0;
  for (int i = 0; i < 10; ++i) r |= (unsigned long long)i << (4 * ((BATCH_AT >> (4 * i)) & 15));
  return r;
}
constexpr unsigned long long BATCH_POS = batch_pos();
__device__ __forceinline__ int batch_row(int s, int o, int pos, int i) {
  const int bt = (BATCH_AT >> (4 * pos)) & 15;
  if (bt < 2) return row_of(s) + (bt * 3 + i) * 128;
  const int g = bt < 8 ? s : o;
  return row_of((g & 4) | ((g + 1 + i) & 3)) + (bt - 2) * 128;
}

__device__ __forceinline__ int unit_of(int it, int w) { return it * PW + ((w + it) & (PW - 1)); }

__device__ __forceinline__ unsigned lds_get(const LAS unsigned* c) {
  return __builtin_amdgcn_readfirstlane(__hip_atomic_load(c, __ATOMIC_RELAXED, __HIP_MEMORY_SCOPE_WORKGROUP));
}

constexpr int RING = 8;
constexpr unsigned PL_EX = RING * PSTEP_B;
constexpr int W_TICKET = 0, W_READY = 1, W_DEAD = 2, W_TURN = 3, W_SPARE = 4, W_DONE = 8, W_LANDED = 64, W_QUEUE = 128, W_LINES = 256;
constexpr unsigned Q_N = 128, Q_OUT = 1024, Q_LAST = 1u << 31;
static_assert(PL_EX + 8 * 1024 <= LDS_MB && W_QUEUE + Q_N <= W_LINES && W_LINES * 4 + 768 <= LDS_TMO - LDS_MB, "");

template <int N, int W>
__device__ __forceinline__ void stream_kib(const char* src, unsigned dst, unsigned& cnt, unsigned word) {
  unsigned keep;
#define SK(OFF) "s_waitcnt vmcnt(%5)\n\tglobal_load_lds_dwordx4 %2, off offset:" #OFF " sc0 sc1\n\t" \
                "v_add_u32 %1, 1, %1\n\tds_max_u32 %4, %1\n\t"
  if (N == 4)
    asm volatile("s_mov_b32 %0, m0\n\ts_mov_b32 m0, %3\n\ts_nop 0\n\t" SK(0) SK(1024) SK(2048) SK(3072) "s_mov_b32 m0, %0"
                 : "=&s"(keep), "+v"(cnt) : "v"(src), "s"(dst), "v"(word), "n"(W - 1) : "memory");
  else if (N == 2)
    asm volatile("s_mov_b32 %0, m0\n\ts_mov_b32 m0, %3\n\ts_nop 0\n\t" SK(0) SK(1024) "s_mov_b32 m0, %0"
                 : "=&s"(keep), "+v"(cnt) : "v"(src), "s"(dst), "v"(word), "n"(W - 1) : "memory");
  else
    asm volatile("s_mov_b32 %0, m0\n\ts_mov_b32 m0, %3\n\ts_nop 0\n\t" SK(0) "s_mov_b32 m0, %0"
                 : "=&s"(keep), "+v"(cnt) : "v"(src), "s"(dst), "v"(word), "n"(W - 1) : "memory");
}
template <int K, class F>
__device__ __forceinline__ bool stream_drain(unsigned cnt, unsigned word, F&& more) {
  if constexpr (K >= 0) {
    asm volatile("s_waitcnt vmcnt(%2)\n\tds_max_u32 %1, %0" :: "v"(cnt + 1), "v"(word), "n"(K) : "memory");
    return more() || stream_drain<K - 1>(cnt + 1, word, more);
  }
  return false;
}

__device__ __forceinline__ int zip(int u, int s, int d, int g) {
  const int i = u - (s - d), j = i - g;
  return (unsigned)i >= (unsigned)(2 * d + g) ? u : j < 0 ? s + i : (j & 1 ? s + g : s - d) + (j >> 1);
}
#define D2 12
#define G2 20
#define D3 0
#define G3 0
#define D4 40
#define G4 28
#define D5 52
#define G5 0
#define DA 40
#define GA 0
#define DH 48
template <bool CROSS>
__device__ __forceinline__ int in_order(int u) {
  constexpr int HOP = 4 * KSTEPS + RELAY / 4 - 2 * HW - DH;
  constexpr int AWAY = 2 * KSTEPS + RELAY / 4 - 2 * HW;
  if (CROSS && u < FILL) return AWAY + u;
  if (CROSS && u < AWAY + FILL) {
    u -= FILL;
    return !HOPF ? u : u < RELAY / 4 ? u + 2 * KSTEPS - 2 * HW : u - RELAY / 4;
  }
  if (CROSS) return u < HOP ? zip(u, 3 * KSTEPS + RELAY / 4 - 2 * HW, DA, GA) : u < HOP + RELAY / 4 * 3 ? u + DH : u - RELAY / 4 * 3;
  if (D2 && u < 2 * KSTEPS - HX + D2 + G2) return zip(u, 2 * KSTEPS - HX, D2, G2);
  if (D3 && u < 3 * KSTEPS - HX + D3 + G3) return zip(u, 3 * KSTEPS - HX, D3, G3);
  return u < 4 * KSTEPS - HX + D4 + G4 ? zip(u, 4 * KSTEPS - HX, D4, G4) : zip(u, 5 * KSTEPS - HX, D5, G5);
}
static_assert(FILL + DA <= KSTEPS && DA + GA + DH <= KSTEPS && D2 + G2 + D3 <= KSTEPS && D3 + G3 + D4 <= KSTEPS && D4 + G4 + D5 <= KSTEPS && D5 + G5 + RELAY <= KSTEPS, "");

struct Unit { unsigned src, raw, dst; int from, flag; };
template <bool CROSS>
__device__ __forceinline__ Unit input_unit(int u, unsigned call, int rank, int peer) {
  constexpr int H1 = RELAY / 4, RSTEP = 32 * KSTEPS;
  const unsigned par = own_idx(call, 0) * KSTEPS;
  Unit o;
  o.from = peer;
  u += CROSS ? 2 * HW : HX;
  if (!CROSS) {
    const int v = u < 5 * KSTEPS ? u : u + RELAY;
    o.raw = par + v; o.dst = (peer & 3) * 6 * KSTEPS + v; o.flag = v;
  } else if (u < 256) {
    const int v = (u & 1) * KSTEPS + (u >> 1);
    o.raw = par + (8 + (rank & 3) * 2) * KSTEPS + v; o.dst = (PB_FOREIGN + (peer & 3) * 2) * KSTEPS + v; o.flag = u >> 1;
  } else if (u < 256 + H1) {
    o.raw = par + 5 * KSTEPS + (u - 256) * 4 + (rank & 3); o.dst = ~((peer & 3) * 16 + u - 256); o.flag = RDY_HOP1 + u - 256;
  } else if (u < 512 + H1) {
    const int v = u - 256 - H1;
    o.raw = par + 6 * KSTEPS + v; o.dst = (PA_AWAY + (peer & 3) * 2) * KSTEPS + v; o.flag = RDY_AWAY + v;
  } else {
    const int v = u - 512 - H1, j = v / 3, s = (rank + 1 + v - j * 3) & 3, ks = 5 * KSTEPS + j * 4 + (peer & 3);
    o.from = (rank & 4) | s;
    o.raw = par + ks; o.dst = s * 6 * KSTEPS + ks; o.flag = RDY_HOP2 + s * 16 + j;
    o.src = RSTEP + s * 16 + j;
    return o;
  }
  o.src = o.raw;
  return o;
}

template <bool CROSS, int W>
__device__ __forceinline__ void stream(const Params& p, int lane, unsigned call, int rank, int peer, LAS char* smem,
                                       unsigned k, unsigned stop) {
  LAS unsigned* const sh = (LAS unsigned*)(smem + LDS_MB);
  const char* const pws = p.ws[peer];
  const unsigned lds0 = (unsigned)(uintptr_t)smem, word = (unsigned)(uintptr_t)(sh + W_LANDED + lane);
  unsigned cnt = WIN1 - W - 1 + k * 14;
  for (; k != stop; ++k) {
    unsigned ready, done, q;
    auto look = [&]() {
      const unsigned a = __hip_atomic_load(sh + W_READY, __ATOMIC_RELAXED, __HIP_MEMORY_SCOPE_WORKGROUP);
      cbar();
      const unsigned b = __hip_atomic_load(sh + W_DONE + (k & (RING - 1)), __ATOMIC_RELAXED, __HIP_MEMORY_SCOPE_WORKGROUP);
      const unsigned c = __hip_atomic_load(sh + W_QUEUE + (k & (Q_N - 1)), __ATOMIC_RELAXED, __HIP_MEMORY_SCOPE_WORKGROUP);
      cbar();
      ready = __builtin_amdgcn_readfirstlane(a);
      done = __builtin_amdgcn_readfirstlane(b);
      q = __builtin_amdgcn_readfirstlane(c);
      return k < (ready & ~Q_LAST) && done >= (k & ~(RING - 1u));
    };
    if (!look()) {
      if (!stream_drain<W - 1>(cnt, word, look)) {
        while (!look() && (k | Q_LAST) != ready) __builtin_amdgcn_s_sleep(1);
        if ((k | Q_LAST) == ready) break;
      }
    }
    const unsigned d = lds0 + (k & (RING - 1)) * PSTEP_B, u = q & 2047u;
    if (u < Q_OUT) {
      if ((u & 63) >= 56) STAMP(4 + (u >> 6));
      const char* s = pws + OFF_PACK + (size_t)input_unit<CROSS>(u, call, rank, peer).src * PSTEP_B + lane * 16;
      if ((unsigned)lane >> 3 <= ((q >> 11) & 7u)) stream_kib<1, W>(s, d, cnt, word);
      stream_kib<4, W>(s + 1024, d + 1024, cnt, word);
      stream_kib<4, W>(s + 5120, d + 5120, cnt, word);
      stream_kib<4, W>(s + 9216, d + 9216, cnt, word);
      stream_kib<1, W>(s + 13312, d + 13312, cnt, word);
      if (k == 0) STAMP(3);
    } else {
      if ((u - Q_OUT) % 48 >= 40) STAMP(20 + (u - Q_OUT) / 48);
      const char* s = pws + OFF_STG + (size_t)((rank & 3) * OUT_CHUNKS + u - Q_OUT) * CHUNK_B + lane * 16;
      stream_kib<4, W>(s, d, cnt, word);
      stream_kib<2, W>(s + 4096, d + 4096, cnt, word);
      if (lane < 48) stream_kib<1, W>(s + 6144, d + 6144, cnt, word);
    }
  }
}

template <bool CROSS>
__device__ __forceinline__ void pull(const Params& p, int w, bool far, int wave, unsigned call, int rank, int peer, LAS char* smem) {
  constexpr int NIN = CROSS ? 512 + RELAY - 2 * HW : 768 - RELAY - HX, NOUT = CROSS ? OUT_CHUNKS / PW : 0;
  const int lane = fresh_lane();
  char* const ws = p.mine;
  LAS unsigned* const sh = (LAS unsigned*)(smem + LDS_MB);
  float* const out = p.out + (size_t)((lane >> 4) * 4) * NSH + (peer & 3) * 256 + (lane & 15) * 4;
  auto where = [&](int c, int pc) {
    const int tile = c >> 3, bt = tile / 6, r = tile - bt * 6;
    return out + (size_t)(batch_row(peer, rank, bt, r >> 1) + ((c >> 2) & 1) * 64 + (c & 1) * 32 + (pc & 4) * 4 + (pc & 3)) * NSH
         + (r & 1) * 128 + ((c >> 1) & 1) * 64;
  };
#define PRIO 3
  if (wave >= 6) __builtin_amdgcn_s_setprio(PRIO);
  if (wave == 6) {
    const unsigned* const rdy = (const unsigned*)(ws + OFF_RDY) + peer * RDY_N * 32;
    unsigned* const tmo = (unsigned*)(ws + OFF_TMO);
    unsigned* const ticket = (unsigned*)(ws + OFF_TICKET) + peer * 32;
    unsigned long long t0 = __builtin_amdgcn_s_memrealtime();
    bool dead = false;
    unsigned k = 0, kib = 0, spin = 0;
    unsigned v;
    auto unset = [&](bool look, int fl) {
      v = ~0u;
      if (look && !dead)
        asm volatile("global_load_dword %0, %1, off sc0 sc1\n\ts_waitcnt vmcnt(0)" : "=v"(v) : "v"(rdy + fl * 32) : "memory");
      return __builtin_amdgcn_ballot_w64(v >> 6 < call);
    };
    auto idle = [&]() {
      if ((++spin & 63u) == 0u
          && (__builtin_amdgcn_readfirstlane(__hip_atomic_load(tmo, __ATOMIC_RELAXED, __HIP_MEMORY_SCOPE_AGENT)) == call
              || __builtin_amdgcn_s_memrealtime() - t0 > DEADLINE)) {
        if (lane == 0) {
          __hip_atomic_store(tmo, call, __ATOMIC_RELAXED, __HIP_MEMORY_SCOPE_AGENT);
          __hip_atomic_store(sh + W_DEAD, 1u, __ATOMIC_RELAXED, __HIP_MEMORY_SCOPE_WORKGROUP);
        }
        wait_vm0();
        dead = true;
      }
      __builtin_amdgcn_s_sleep(1);
    };
    auto publish = [&](unsigned v, unsigned near) {
      cbar();
      if (lane == 0) __hip_atomic_store(sh + W_READY, v, __ATOMIC_RELAXED, __HIP_MEMORY_SCOPE_WORKGROUP);
      while (lds_get(sh + W_LANDED) < near) __builtin_amdgcn_s_sleep(2);
      t0 = __builtin_amdgcn_s_memrealtime();
    };
    int known = EXPON(1) ? NIN : 0;
    LAS unsigned char* const lines = (LAS unsigned char*)(sh + W_LINES);
    auto look = [&](int from, int width) {
      const int i = min(from + lane, NIN - 1), o = in_order<CROSS>(i);
      const unsigned long long miss = unset(lane < width, input_unit<CROSS>(o, call, rank, peer).flag);
      known = from + (miss ? (int)__builtin_ctzll(miss) : width);
      if (lane < width) lines[i] = v >> 6 == call ? (v >> (CROSS && o < 256 - 2 * HW ? (o & 1) * 3 : 0)) & 7u : 7u;
    };
    for (int u = w; u < NIN;) {
      while (u >= known) { look(u, 16); if (u >= known) idle(); }
      if (k == 0) STAMP(30);
      if (k == 1) STAMP(31);
      if (k == 2) STAMP(19);
      if (lane == 0) sh[W_QUEUE + (k & (Q_N - 1))] = kib << 14 | lines[u] << 11 | in_order<CROSS>(u);
#define AHEAD 4
      publish(++k, max(kib, (unsigned)AHEAD) - AHEAD);
      kib += 14;
#define SR 3
      if (k < SR) { u += PW; continue; }
      unsigned tk = 0;
      if (lane == 0) tk = __hip_atomic_fetch_add(ticket, 1u, __ATOMIC_RELAXED, __HIP_MEMORY_SCOPE_AGENT);
      if (known < NIN && known - u < 32) look(known, 64);
      u = min(__builtin_amdgcn_readfirstlane(tk) - (call - 1u) * (NIN - (SR - 1) * PW) + SR * PW, (unsigned)NIN);
    }
    if (CROSS && lane == 0) __hip_atomic_store(sh + W_TURN, ~(k * 8u), __ATOMIC_RELAXED, __HIP_MEMORY_SCOPE_WORKGROUP);
    const unsigned long long all = (1ull << NOUT) - 1ull;
    for (unsigned long long left = all; left;) {
      const unsigned long long fresh = left & ~unset(!EXPON(17) && ((left >> lane) & 1), RDY_OUT + unit_of(lane, w));
      if (!fresh) { idle(); continue; }
      const unsigned pos = __builtin_amdgcn_mbcnt_hi(fresh >> 32, __builtin_amdgcn_mbcnt_lo(fresh, 0));
      if ((fresh >> lane) & 1) sh[W_QUEUE + ((k + pos) & (Q_N - 1))] = (kib + pos * 7) << 14 | (Q_OUT + unit_of(lane, w));
      k += __builtin_popcountll(fresh);
      kib += __builtin_popcountll(fresh) * 7;
      left &= ~fresh;
      if (left) publish(k, kib - 14);
    }
    publish(k | Q_LAST, 0);
    if (CROSS) {
      dead = __builtin_amdgcn_readfirstlane(__hip_atomic_load(tmo, __ATOMIC_RELAXED, __HIP_MEMORY_SCOPE_AGENT)) == call;
      wg_barrier();
      if (dead) store_sys(where(unit_of(0, w), 0), (u32x4){0x7fc00000u, 0x7fc00000u, 0x7fc00000u, 0x7fc00000u});
    }
    return;
  }
  if (wave == 7) {
#define KSW 0
    if (KSW) stream<CROSS, WIN1>(p, lane, call, rank, peer, smem, 0, KSW);
    stream<CROSS, WIN>(p, lane, call, rank, peer, smem, KSW, ~0u);
    STAMP(1);
    if (CROSS) wg_barrier();
    return;
  }
  static_assert(RING == 8 && Q_N == 128, "");
  const unsigned sh0 = (unsigned)(uintptr_t)sh, none = ~0u;
  const unsigned aq = sh0 + W_QUEUE * 4 + lane * 8, ah = (unsigned)(uintptr_t)smem + (lane >> 3) * PSTEP_B + (lane & 7) * 4;
  u32x4 st;
  u32x2 queue;
  unsigned count, heads;
  auto look = [&](int word) {
    unsigned r;
    unsigned long long keep;
    asm volatile("s_mov_b64 %5, exec\n\ts_mov_b64 exec, 1\n\tds_add_rtn_u32 %0, %6, %7\n\ts_mov_b64 exec, %5\n\t"
                 "ds_read_b128 %1, %8\n\tds_read_b64 %2, %9\n\tds_read_b32 %3, %8 offset:%11\n\tds_read_b32 %4, %10\n\t"
                 "s_waitcnt lgkmcnt(0)"
                 : "=&v"(r), "=&v"(st), "=&v"(queue), "=&v"(count), "=&v"(heads), "=&s"(keep)
                 : "v"(sh0 + word * 4), "v"(1u), "v"(sh0), "v"(aq), "v"(ah), "n"(W_LANDED * 4) : "memory");
    return __builtin_amdgcn_readfirstlane(r);
  };
  unsigned t, u, q;
  auto landed = [&](unsigned tk) {
    const unsigned ready = __builtin_amdgcn_readfirstlane(st[W_READY]);
    t = tk;
    if (CROSS) {
      const unsigned turn = ~__builtin_amdgcn_readfirstlane(st[W_TURN]);
      if (tk > turn) t = turn + (tk - turn) * 4u;
    }
    const unsigned q0 = __builtin_amdgcn_readlane(queue[0], (t >> 4) & 63), q1 = __builtin_amdgcn_readlane(queue[1], (t >> 4) & 63);
    q = t & 8 ? q1 : q0;
    const bool there = (t >> 3) < (ready & ~Q_LAST);
    u = there ? q & 2047u : none;
    const unsigned need = (q >> 14) + (!CROSS || u < Q_OUT ? ((t & 7) * 13 + 28) >> 3 : t & 4 ? 7 : 4) + WIN1 - 1;
    return there ? (unsigned)__builtin_amdgcn_readfirstlane(count) >= need : ready >= Q_LAST;
  };
  unsigned prev = none, pdst = 0, plines = 0;
  auto done = [&]() {
    const bool last = (look(prev != none ? W_DONE + ((prev >> 3) & (RING - 1)) : W_SPARE) & 7u) == 7u && prev != none;
    prev = none;
    if (!last) return;
    if (!CROSS && (pdst & 127) >= 112) STAMP(17 + ((~pdst >> 3) & 1));
    if (CROSS && (int)pdst >= PB_FOREIGN * KSTEPS && (pdst & 127) < HW + 8) STAMP(17 + ((pdst & 127) >= HW + 4));
    if (CROSS && (int)pdst < 0) STAMP(15);
    if (!CROSS || (int)pdst >= 0) { if (lane < FCOPY) set_local(ws, lane, pdst, call); }
    else if (lane < 3) set_rdy(p.ws[((peer & 4) | ((peer + 1 + lane) & 3)) & 7], rank, RDY_HOP2 + ~pdst, call, plines);
  };
  for (;;) {
    const unsigned tk = look(W_TICKET);
    if (!landed(tk)) {
      if (prev != none) wait_vm0();
      for (done(); !landed(tk); look(W_SPARE)) __builtin_amdgcn_s_sleep(1);
    }
    const LAS char* slot = smem + ((t >> 3) & (RING - 1)) * PSTEP_B;
    if (u >= Q_OUT) {
      if (prev != none) { wait_vm0(); done(); }
      if (!CROSS || u == none) break;
      const unsigned half = (t >> 2) & 1u;
      const bool dead = __builtin_amdgcn_readfirstlane(st[W_DEAD]);
      f32x4 y[4];
      unpack_sums(lane, slot + half * SUM_B, smem + PL_EX + wave * 1024, smem + LDS_LUT,
                  p.ws[peer] + OFF_SPARE + (size_t)(((rank & 3) * OUT_CHUNKS + u - Q_OUT) * 2 + half) * 1024, y);
      if (lane == 0) __hip_atomic_fetch_add(sh + W_DONE + ((t >> 3) & (RING - 1)), 4u, __ATOMIC_RELAXED, __HIP_MEMORY_SCOPE_WORKGROUP);
      float* const o = where(u - Q_OUT, t & 4);
#pragma unroll
      for (int r = 0; r < 4; ++r) {
        u32x4 v = __builtin_bit_cast(u32x4, (f32x4){silu(y[r][0]), silu(y[r][1]), silu(y[r][2]), silu(y[r][3])});
        if (dead) v = (u32x4){0x7fc00000u, 0x7fc00000u, 0x7fc00000u, 0x7fc00000u};
        store_out(o + r * NSH, v);
      }
      continue;
    }
    const Unit un = input_unit<CROSS>(u, call, rank, peer);
    u32x4 f0, f1;
    char* d;
    if (CROSS && RELAY && (int)un.dst < 0) {
      const int l1 = min(lane, 47);
      const LAS char* blk = slot + (t & 7) * (PSTEP_B / 8);
      f0 = *(const LAS u32x4*)(blk + lane * 16);
      f1 = *(const LAS u32x4*)(blk + 1024 + l1 * 16);
      d = ws + OFF_RELAY + (size_t)(~un.dst * 8 + (t & 7)) * (PSTEP_B / 8);
      store_sys(d + lane * 16, f0);
      store_sys(d + 1024 + l1 * 16, f1);
    } else {
      const unsigned hdr = __builtin_amdgcn_readlane(heads, t & 63);
      if ((int)hdr >= 0) {
        unpack_block(lane, slot, t & 7, hdr, smem + PL_EX + wave * 1024, smem + LDS_LUT, f0, f1);
      } else {
        const char* r = p.ws[un.from & 7] + OFF_OWN + (size_t)(un.raw * 8 + (t & 7)) * 2048 + lane * 16;
        asm volatile("global_load_dwordx4 %0, %2, off sc0 sc1\n\t"
                     "global_load_dwordx4 %1, %2, off offset:1024 sc0 sc1\n\ts_waitcnt vmcnt(0)"
                     : "=&v"(f0), "=&v"(f1) : "v"(r) : "memory");
      }
      d = ws + pulled(call) + (size_t)(un.dst * 8 + (t & 7)) * 2048 + lane * 16;
      store_sys(d, f0);
      store_sys(d + 1024, f1);
    }
    asm volatile("s_waitcnt vmcnt(2)" ::: "memory");
    done();
    prev = t;
    pdst = un.dst;
    plines = (q >> 11) & 7u;
    if (!landed(__builtin_amdgcn_readfirstlane(st[W_TICKET]))) { wait_vm0(); done(); }
  }
  if (CROSS) { wait_vm0(); wg_barrier(); }
}

struct Job { int pa, pb, ah, bh, tile; };
__device__ __forceinline__ void decode_job(unsigned code, int a, Job& o) {
  const int ra = code & 63;
  int bt, i;
  o.pb = PB + ((code >> 6) & 15);
  o.ah = (code >> 10) & 1;
  o.bh = (code >> 11) & 1;
  if (ra < 18) {
    i = ra / 6; bt = 2 + ra % 6; o.pa = ((a + 1 + i) & 3) * 6 + bt - 2;
  } else if (ra < 24) {
    o.pa = a * 6 + ra - 18; bt = (ra - 18) / 3; i = (ra - 18) % 3;
  } else {
    o.pa = ra; bt = 8 + (ra & 1); i = (((ra - PA_AWAY) >> 1) - ((o.pb - PB_FOREIGN) >> 1) - 1) & 3;
  }
  o.tile = (int)((BATCH_POS >> (4 * bt)) & 15) * 3 + i;
}

#define POLL_EVERY 1
#define POLL_CAD 2
#define POLL_SLEEP 2
#define GLDS_BITS " sc0 sc1"

template <int NB>
__device__ __forceinline__ unsigned next_stage(unsigned so) { return so == LDS_MB - (NB + 1) * 8192 ? 0u : so + (NB + 1) * 8192; }

template <bool TALL, int NB>
__device__ __forceinline__ void mfma_wave(const Params& p, LAS char* smem, const Job& jb, int job, int wave,
                                          int rank, unsigned call) {
  constexpr int MT = TALL ? 4 : NB, NT = TALL && NB == 2 ? 8 : 4, KH = TALL ? 1 : 2, STEPS = 2 * KSTEPS / KH;
  constexpr int NRA = MT * KH, NRB = NT * KH, NRD = NRA + NRB, NMM = MT * NT * KH;
  int lane = fresh_lane();
  const int wr = MT == 1 ? wave : wave >> 1, wc = MT == 1 ? 0 : wave & 1;
  f32x4 acc[MT][NT];
#pragma unroll
  for (int m = 0; m < MT; ++m)
#pragma unroll
    for (int n = 0; n < NT; ++n) acc[m][n] = (f32x4){0.f, 0.f, 0.f, 0.f};
  half8 fa[2][NRA], fb[2][NRB];
  const LAS char* la = smem + wr * MT * 1024 + lane * 16;
  const LAS char* lb = smem + 8192 + wc * NRB * 1024 + lane * 16;
  unsigned so = 0;
#define READ_SET(X) do { \
    _Pragma("unroll") for (int i = 0; i < NRB; ++i) { \
      if (i < NRA) fa[X][i] = *(const LAS half8*)(la + so + (i / MT) * 4096 + (i % MT) * 1024); \
      fb[X][i] = *(const LAS half8*)(lb + so + i * 1024); } \
    so = next_stage<NB>(so); } while (0)
#define MMA_SET(X) do { \
    _Pragma("unroll") for (int kh = 0; kh < KH; ++kh) \
    _Pragma("unroll") for (int m = 0; m < MT; ++m) \
    _Pragma("unroll") for (int n = 0; n < NT; ++n) \
      acc[m][n] = __builtin_amdgcn_mfma_f32_16x16x32_f16(fa[X][kh * MT + m], fb[X][kh * NT + n], acc[m][n], 0, 0, 0); } while (0)
  constexpr int RG = TALL ? 1 : 2, MG = NB, G = NRD / RG;
#define INTERLEAVE() do { \
    _Pragma("unroll") for (int i = 0; i < G; ++i) { \
      __builtin_amdgcn_sched_group_barrier(0x100, RG, 0); \
      __builtin_amdgcn_sched_group_barrier(0x008, MG, 0); } \
    __builtin_amdgcn_sched_group_barrier(0x008, NMM - G * MG, 0); } while (0)
#define STEP_END() do { __builtin_amdgcn_sched_barrier(0); asm volatile("" ::: "memory"); \
    __builtin_amdgcn_s_waitcnt(0xC07F);   __builtin_amdgcn_sched_barrier(0); \
    wg_barrier(); __builtin_amdgcn_sched_barrier(0); } while (0)
  wg_barrier();
  if (wave == 0) STAMP(4 + job * 4);
  READ_SET(0);
  STEP_END();
  for (int t = 1; t < STEPS - 1; t += 2) {
    READ_SET(1); MMA_SET(0); INTERLEAVE();
    STEP_END();
    READ_SET(0); MMA_SET(1); INTERLEAVE();
    STEP_END();
  }
  READ_SET(1); MMA_SET(0); INTERLEAVE();
  const unsigned tm = *(const LAS unsigned*)(smem + LDS_TMO + job * 16);
  STEP_END();
  MMA_SET(1);
  __builtin_amdgcn_sched_barrier(0);
  if (wave == 0) STAMP(5 + job * 4);
  lane = fresh_lane();
  asm volatile("" : "+s"(call));
  const bool dead = tm == call;

  const bool foreign = jb.pb >= PB_FOREIGN;
  const int ob = (jb.pb - PB_FOREIGN) >> 1;
  const int r16 = jb.ah * 4 + wr * MT;
  const int c64 = (jb.pb & 1) * 2 + jb.bh + wc * (NT / 4);
  const int chunk = (jb.tile * 2 + (c64 >> 1)) * 8 + (r16 & 4) + (c64 & 1) * 2 + ((r16 >> 1) & 1);
  const float qnan = __builtin_nanf("");
  if (foreign) {
    const unsigned blk = (ob * OUT_CHUNKS + chunk) * 2 + (r16 & 1);
#pragma unroll 1
    for (int i = 0; i < MT * NT / 4; ++i) {
      f32x4 v[4];
#pragma unroll
      for (int j = 0; j < MT * NT / 4; ++j)
        if (j == 0 || i == j) {
#pragma unroll
          for (int n = 0; n < 4; ++n) v[n] = acc[j % MT][j / MT * 4 + n];
          asm volatile("" : "+v"(v[0]), "+v"(v[1]), "+v"(v[2]), "+v"(v[3]));
        }
      if (dead) v[0] = v[1] = v[2] = v[3] = (f32x4){qnan, qnan, qnan, qnan};
      pack_sums(lane, v, smem + LDS_EPI + wave * 512, smem + LDS_LUT, p.mine + OFF_STG + (size_t)((blk + i) >> 1) * CHUNK_B + ((blk + i) & 1) * SUM_B,
                p.mine + OFF_SPARE + (size_t)(blk + i) * 1024);
    }
  } else {
    const int pa = jb.pa;
    const int row0 = pa < PA_AWAY ? row_of((rank & 4) + pa / 6) + (pa % 6) * 128
                   : row_of(pa < PA_OWNAWAY ? ((rank & 4) ^ 4) + ((pa - PA_AWAY) >> 1) : rank) + 768 + (pa & 1) * 128;
    float* o = p.out + (size_t)(row0 + r16 * 16 + (lane >> 4) * 4) * NSH + ((jb.pb - PB) >> 1) * 256 + c64 * 64 + (lane & 15) * 4;
#pragma unroll
    for (int h = 0; h < NT / 4; ++h)
#pragma unroll
      for (int m = 0; m < MT; ++m)
#pragma unroll
        for (int r = 0; r < 4; ++r) {
          f32x4 v = {silu(acc[m][h * 4][r]), silu(acc[m][h * 4 + 1][r]), silu(acc[m][h * 4 + 2][r]), silu(acc[m][h * 4 + 3][r])};
          if (dead) v = (f32x4){qnan, qnan, qnan, qnan};
          store_out(o + h * 64 + (m * 16 + r) * NSH, __builtin_bit_cast(u32x4, v));
        }
  }
  if (foreign) {
    wait_vm0();
    bool last = lane < (MT * NT + 7) / 8;
    if (MT == 1)
      last = last && __hip_atomic_fetch_add((LAS unsigned*)(smem + LDS_TMO) + job * 4 + 2 + (wave >> 1), 1u,
                                            __ATOMIC_RELAXED, __HIP_MEMORY_SCOPE_WORKGROUP) == 1u;
    if (last) set_rdy(p.ws[(((rank & 4) ^ 4) | ob) & 7], rank, RDY_OUT + chunk + lane, call, 0);
  }
  if (wave == 0) STAMP(6 + job * 4);
}

#define NEAR 4
template <int NB>
__device__ __forceinline__ void load_wave(const Params& p, LAS char* smem, const Job& jb, int job, unsigned code, int l,
                                          bool tall, int rank, unsigned call) {
  constexpr int DEPTH = NB == 2 ? 5 : 7;
  const int steps = tall ? 2 * KSTEPS : KSTEPS, ksh = tall ? 1 : 0;
  if (l > NB && l < 3) {
    for (int t = 0; t <= steps; ++t) wg_barrier();
    return;
  }
  const int lane = fresh_lane(), a = rank & 3;
  char* const ws = p.mine;
  const unsigned lds0 = (unsigned)(uintptr_t)smem;
  LAS unsigned* const rdyw = (LAS unsigned*)(smem + LDS_TMO) + job * 4 + 1;
  const unsigned gap = tall ? 8192 : 4096;
  auto panel_of = [&](int pc) { return pc == 0 ? jb.pa : tall ? jb.pb + pc - 1 : jb.pb; };
  auto source = [&](int pc) {
    const int panel = panel_of(pc);
    const int own = panel >= PB_FOREIGN ? -1 : panel >= PA_OWNAWAY ? panel - PA_OWNAWAY + 6
                  : (unsigned)(panel - a * 6) < 6u ? panel - a * 6 : -1;
    return ws + (own >= 0 ? OFF_OWN + own_idx(call, own) * PANEL_B : pulled(call) + (size_t)panel * PANEL_B)
         + (tall ? 0 : (pc == 0 ? jb.ah : jb.bh + pc - 1) * 8192);
  };
  int ready = EXPON(1) ? KSTEPS : 0;
  if (l == 3) {
    unsigned* const tmo = (unsigned*)(ws + OFF_TMO);
    const char* fsrc = ws + OFF_FLAG + (blockIdx.x & (FCOPY - 1)) * 65536 + panel_of(lane >> 5) * 1024 + ((lane >> 2) & 7) * 128 + (lane & 3) * 16;
    const unsigned mb = lds0 + LDS_MB + job * 2048;
    int ra = ready, rb = ready;
    auto poll = [&]() {
      if ((lane < 32 ? ra : rb) < KSTEPS) {
        unsigned keep;
        asm volatile("s_mov_b32 %0, m0\n\ts_mov_b32 m0, %2\n\ts_nop 0\n\t"
                     "global_load_lds_dwordx4 %1, off sc0 sc1\n\ts_mov_b32 m0, %0"
                     : "=&s"(keep) : "v"(fsrc), "s"(mb) : "memory");
        if (NB == 2 && tall && lane >= 32)
          asm volatile("s_mov_b32 %0, m0\n\ts_mov_b32 m0, %2\n\ts_nop 0\n\t"
                       "global_load_lds_dwordx4 %1, off offset:1024 sc0 sc1\n\ts_mov_b32 m0, %0"
                       : "=&s"(keep) : "v"(fsrc), "s"(mb - 512) : "memory");
      }
    };
    auto refresh = [&]() {
      u32x2 f, g, h;
      asm volatile("ds_read2_b32 %0, %3 offset1:8\n\tds_read2_b32 %1, %3 offset0:128 offset1:136\n\t"
                   "ds_read2_b32 %2, %4 offset1:8\n\ts_waitcnt lgkmcnt(0)"
                   : "=&v"(f), "=&v"(g), "=&v"(h)
                   : "v"(mb + (lane & 7) * 64 + (lane >> 3) * 4), "v"(mb + (NB == 2 && tall ? 1024 : 512) + (lane & 7) * 64 + (lane >> 3) * 4)
                   : "memory");
      const unsigned long long a0 = __builtin_amdgcn_ballot_w64(f[0] < call), a1 = __builtin_amdgcn_ballot_w64(f[1] < call);
      const unsigned long long b0 = __builtin_amdgcn_ballot_w64(min(g[0], h[0]) < call), b1 = __builtin_amdgcn_ballot_w64(min(g[1], h[1]) < call);
      ra = a0 ? __builtin_ctzll(a0) : a1 ? 64 + __builtin_ctzll(a1) : KSTEPS;
      rb = b0 ? __builtin_ctzll(b0) : b1 ? 64 + __builtin_ctzll(b1) : KSTEPS;
      ready = min(ra, rb);
      if (lane == 0) __hip_atomic_store(rdyw, (unsigned)ready, __ATOMIC_RELAXED, __HIP_MEMORY_SCOPE_WORKGROUP);
    };
    auto advance = [&]() {
      const int k = min(ready + lane, KSTEPS - 1);
      u32x2 f;
      unsigned h;
      asm volatile("ds_read2_b32 %0, %2 offset1:128\n\tds_read_b32 %1, %3\n\ts_waitcnt lgkmcnt(0)"
                   : "=&v"(f), "=&v"(h)
                   : "v"(mb + (k & 7) * 64 + (k >> 3) * 4), "v"(mb + (NB == 2 && tall ? 1024 : 512) + (k & 7) * 64 + (k >> 3) * 4)
                   : "memory");
      const unsigned long long m = __builtin_amdgcn_ballot_w64(min(min(f[0], f[1]), h) < call);
      ready = min(ready + (m ? (int)__builtin_ctzll(m) : 64), KSTEPS);
      __hip_atomic_store(rdyw, (unsigned)ready, __ATOMIC_RELAXED, __HIP_MEMORY_SCOPE_WORKGROUP);
    };
    for (int s = 0; s < steps; ++s) {
      if (s >= DEPTH) wg_barrier();
      const int ks = s >> ksh;
      if (ready < KSTEPS && ready - ks < NEAR) {
        advance();
        if (ks >= ready) {
          const unsigned long long t0 = __builtin_amdgcn_s_memrealtime();
          for (unsigned it = 1;; ++it) {
            if (it >= 32u || s == 0) { poll(); wait_vm0(); refresh(); }
            else { if ((it & (POLL_CAD - 1)) == 1u) poll(); advance(); }
            if (ks < ready) break;
            if ((it & 63u) == 0u) {
              if (__builtin_amdgcn_readfirstlane(__hip_atomic_load(tmo, __ATOMIC_RELAXED, __HIP_MEMORY_SCOPE_AGENT)) == call
                  || __builtin_amdgcn_s_memrealtime() - t0 > DEADLINE) {
                if (lane == 0) __hip_atomic_store(tmo, call, __ATOMIC_RELAXED, __HIP_MEMORY_SCOPE_AGENT);
                wait_vm0();
                ready = KSTEPS;
                if (lane == 0) __hip_atomic_store(rdyw, (unsigned)ready, __ATOMIC_RELAXED, __HIP_MEMORY_SCOPE_WORKGROUP);
                break;
              }
            }
            if (it >= 32u) __builtin_amdgcn_s_sleep(POLL_SLEEP); else __builtin_amdgcn_s_sleep(1);
          }
        } else if (!(s & (((POLL_EVERY) << ksh) - 1))) {
          poll();
        }
      }
    }
    if (lane == 0) {
      unsigned keep;
      asm volatile("s_mov_b32 %0, m0\n\ts_mov_b32 m0, %2\n\ts_nop 0\n\t"
                   "global_load_lds_dword %1, off sc1\n\ts_mov_b32 m0, %0"
                   : "=&s"(keep) : "v"(tmo), "s"(lds0 + LDS_TMO + job * 16) : "memory");
    }
    for (int t = 1; t < DEPTH; ++t) wg_barrier();
    wait_vm0();
    wg_barrier();
    wg_barrier();
    return;
  }
  const char* src = source(l) + lane * 16;
  const unsigned dst0 = lds0 + l * 8192;
  unsigned so = 0;
  for (int s = 0; s < steps; ++s) {
    if (s >= DEPTH) {
      asm volatile("s_waitcnt vmcnt(%0)" :: "n"(8 * (DEPTH - 1)) : "memory");
      wg_barrier();
    }
    const int ks = s >> ksh;
    if (ks >= ready)
      while ((ready = lds_get(rdyw)) <= ks) __builtin_amdgcn_s_sleep(1);
    const char* g = src + (size_t)ks * STEP_B + (s & ksh) * 4096;
    const unsigned d = dst0 + so;
    so = next_stage<NB>(so);
    unsigned keep;
#define GLDS4(R) "global_load_lds_dwordx4 " R ", off" GLDS_BITS "\n\t" \
                 "global_load_lds_dwordx4 " R ", off offset:1024" GLDS_BITS "\n\t" \
                 "global_load_lds_dwordx4 " R ", off offset:2048" GLDS_BITS "\n\t" \
                 "global_load_lds_dwordx4 " R ", off offset:3072" GLDS_BITS "\n\t"
    asm volatile("s_mov_b32 %0, m0\n\ts_mov_b32 m0, %3\n\ts_nop 0\n\t" GLDS4("%1")
                 "s_mov_b32 m0, %4\n\ts_nop 0\n\t" GLDS4("%2") "s_mov_b32 m0, %0"
                 : "=&s"(keep) : "v"(g), "v"(g + gap), "s"(d), "s"(d + 4096) : "memory");
  }
#define WB(N) "s_waitcnt vmcnt(" #N ")\n\ts_barrier\n\t"
  if (NB == 1) asm volatile(WB(48) WB(40) ::: "memory");
  asm volatile(WB(32) WB(24) WB(16) WB(8) WB(0) "s_barrier" ::: "memory");
}

__global__ void __launch_bounds__(NTHREADS, 2) ag_gemm_silu(const Params p) {
  __shared__ __attribute__((aligned(16))) char smem[LDS_BYTES];
  const int xcd = (xcc_id() - 4) & 7, slot = blockIdx.x >> 3;
  u32x4 plan;
  {
    unsigned l0, l1;
    asm volatile("s_load_dword %1, %3, 0\n\ts_load_dword %2, %3, 64\n\ts_load_dwordx4 %0, %3, %4\n\ts_waitcnt lgkmcnt(0)"
                 : "=&s"(plan), "=&s"(l0), "=&s"(l1)
                 : "s"(__builtin_amdgcn_kernarg_segment_ptr()),
                   "s"((unsigned)(offsetof(Params, plan) + 16 * (slot < GSLOTS ? slot * 8 + xcd : GSLOTS * 8 + slot - GSLOTS))) : "memory");
  }
  const int rank = p.rank & 7, a = rank & 3;
  const int wave = __builtin_amdgcn_readfirstlane(threadIdx.x >> 6);
  if (threadIdx.x == 0)
    asm volatile("s_mov_b32 m0, %1\n\ts_nop 0\n\tglobal_load_lds_dword %0, off sc1"
                 :: "v"(call_word(p)), "s"((unsigned)(uintptr_t)(LAS char*)smem + LDS_CALL) : "memory");
  const bool puller = slot >= GSLOTS && xcd < 7;
  unsigned call;
  if (!puller && (plan[0] & 0x4000u)) call = convert_x(p, plan[0], p.counts, rank, (LAS char*)smem);
  else if (!puller && (plan[0] & 0x8000u)) call = convert_w(p, plan[0], p.counts, rank, (LAS char*)smem);
  else call = begin<0>(p, (LAS char*)smem, puller);
  if (threadIdx.x == 0) __hip_atomic_store(call_word(p), call, __ATOMIC_RELAXED, __HIP_MEMORY_SCOPE_AGENT);
  __builtin_amdgcn_s_waitcnt(0x0F70);

  if (puller) {
    const int w = slot - GSLOTS;
    if (xcd < 4) pull<true>(p, w, w < (int)((WMASK >> (4 * xcd)) & 15), wave, call, rank, ((rank & 4) ^ 4) | ((a + xcd) & 3), (LAS char*)smem);
    else pull<false>(p, w, w < (int)((WMASK >> (4 * xcd)) & 15), wave, call, rank, (rank & 4) | ((a + xcd - 3) & 3), (LAS char*)smem);
    if (wave == 0) STAMP(2);
    return;
  }
  if (wave == 0) STAMP(1);

  for (int job = 0; job < NJOBS; ++job) {
    const unsigned code = job == 0 ? plan[1] : job == 1 ? plan[2] : plan[3];
    if (!(code & 0x8000u)) break;
    if (EXPON(4) && (job > 0 || slot >= 12)) break;
    if (EXPON(8) && job > 0) break;
    Job jb;
    decode_job(code, a, jb);
    const int kind = (code >> 12) & 3;
    if (kind == 3 && (code & 0x200) && wave == 0) STAMP(21 + (code & 63) / 6);
    if (wave >= 4) {
      if (kind & 1) load_wave<1>(p, (LAS char*)smem, jb, job, code, wave - 4, kind < 2, rank, call);
      else load_wave<2>(p, (LAS char*)smem, jb, job, code, wave - 4, kind < 2, rank, call);
    } else if (kind == 0) {
      mfma_wave<true, 2>(p, (LAS char*)smem, jb, job, wave, rank, call);
    } else if (kind == 1) {
      mfma_wave<true, 1>(p, (LAS char*)smem, jb, job, wave, rank, call);
    } else if (kind == 2) {
      mfma_wave<false, 2>(p, (LAS char*)smem, jb, job, wave, rank, call);
    } else {
      mfma_wave<false, 1>(p, (LAS char*)smem, jb, job, wave, rank, call);
    }
  }
}

void make_schedule(Params& p) {
  int n[NGEMM] = {};
  bool early[NGEMM] = {};
  for (int g = 0; g < NGEMM; ++g) p.plan[g] = (u32x4){0u, 0u, 0u, 0u};
  int team[32], nteam = 0;
  auto id = [](int xcd, int s) { return s < GSLOTS ? s * 8 + xcd : GSLOTS * 8 + s - GSLOTS; };
  auto put = [&](int xcd, int s, int kind, int ra, int b, int ah = 0, int bh = 0) {
    const int g = id(xcd, s);
    if (n[g] >= NJOBS || nteam >= 32) return;
    p.plan[g][++n[g]] = 0x8000 | (kind << 12) | (bh << 11) | (ah << 10) | (b << 6) | ra;
    team[nteam++] = g;
  };
  auto close = [&]() {
    for (int side = 0; side < 2; ++side)
      for (int i = 0; i < nteam; ++i) {
        const unsigned mask = side ? 0xbc0u : 0x43fu;
        int cnt = 0, idx = 0;
        for (int k = 0; k < nteam; ++k)
          if (((p.plan[team[i]][n[team[i]]] ^ p.plan[team[k]][n[team[k]]]) & mask) == 0) { idx += k < i; ++cnt; }
        p.plan[team[i]][n[team[i]]] |= (unsigned)((cnt - 1) | (idx << 3)) << (16 + 6 * side);
      }
    nteam = 0;
  };
  auto P = [&](int xcd, int s0, int kind, int t, int b0) {
    for (int s = 0; s < 12; ++s) {
      if (kind < 2) put(xcd, s0 + s, kind, s % 3 * 6 + t, b0 + s / 3 * (2 - kind));
      else if (kind == 2) put(xcd, s0 + s, kind, s % 3 * 6 + t, b0 + s / 6, (s / 3) & 1);
      else put(xcd, s0 + s, kind, s % 3 * 6 + t, b0, (s / 3) & 1, s / 6);
    }
    close();
  };
  auto L = [&](int xcd, int s0, int h, int b0) { for (int s = 0; s < 12; ++s) put(xcd, s0 + s, 0, 18 + h * 3 + s % 3, b0 + s / 3 * 2); close(); };
  auto Afw = [&](int xcd, int s0, int t) {
    for (int s = 0; s < 12; ++s) put(xcd, s0 + s, 0, PA_AWAY + (s & 3) * 2 + t, 8 + (((s & 3) + 1 + (s >> 2)) & 3) * 2);
    close();
  };
  auto Afs = [&](int xcd, int s0, int t, int h) {
    for (int s = h * 12; s < h * 12 + 12; ++s) put(xcd, s0 + s % 12, 1, PA_AWAY + ((s / 6 + 1 + s % 3) & 3) * 2 + t, 8 + s / 3);
    close();
  };
  auto Aow = [&](int xcd, int s0, int t, int c0, int cnt) {
    for (int s = 0; s < cnt; ++s) put(xcd, s0 + s, 0, PA_AWAY + ((c0 + s) & 3) * 2 + t, (c0 + s) / 4 * 2);
    if (c0 + cnt == 16) close();
  };
  auto E = [&](int xcd, int s0) { for (int s = 0; s < 12; ++s) early[id(xcd, s0 + s)] = true; };
#define SCHED 7
  auto Aos = [&](int xcd, int s0, int t, int h) {
    for (int s = h * 12; s < h * 12 + 12; ++s) put(xcd, s0 + s % 12, 1, PA_AWAY + (s & 3) * 2 + t, s / 4);
    close();
  };
  auto OA = [&](int xcd, int s0, int h) { for (int s = h * 8; s < h * 8 + 8; ++s) put(xcd, s0 + s % 8, 1, PA_OWNAWAY + (s & 1), s / 2); close(); };
  P(0, 0, 0, 0, 0); P(1, 0, 0, 0, 8); L(2, 0, 0, 8); L(3, 0, 1, 8); L(4, 0, 0, 0); L(5, 0, 1, 0);
  for (int x = 0; x < 6; ++x) E(x, 0);
  OA(6, 0, 0); OA(7, 0, 1);
  P(5, 12, 0, 1, 0); P(6, 12, 0, 1, 8);
  const int sw = SCHED == 7 ? 8 : 0;
  P(0, 12, 1, 2, 8 ^ sw); P(1, 12, 1, 2, 12 ^ sw); Afs(2, 12, 0, 0); Afs(3, 12, 0, 1); P(4, 12, 0, 2, sw); Aow(7, 12, 0, 0, 16);
  P(0, 0, 1, 3, 8); P(1, 0, 1, 3, 12); Afs(2, 0, 1, 0); Afs(3, 0, 1, 1);
  P(4, 0, 1, 3, 0); P(5, 0, 1, 3, 4); Aos(6, 0, 1, 0); Aos(7, 0, 1, 1);
  if (SCHED == 5) {
    Aow(7, 28, 1, 12, 4);
    for (int x = 0; x < 4; ++x) P(x, 12, 2, 4, 8 + 2 * x);
    P(5, 12, 1, 4, 0); P(6, 12, 1, 4, 4);
    for (int x = 0; x < 4; ++x) P(x, 12, 3, 5, 4 + x);
    P(0, 0, 2, 5, 0); P(1, 0, 2, 5, 2);
    int b = 8;
    for (int x = 2; x < 8; ++x) P(x, 0, 3, 5, b++);
    P(4, 12, 3, 5, b++); P(7, 12, 3, 5, b++);
  } else {
    for (int s = 0; s < 4; ++s) put(7, 28 + s, 1, PA_AWAY + s * 2 + 1, 6);
    for (int s = 0; s < 16; ++s) put(s < 12 ? 4 : 7, s < 12 ? 12 + s : 16 + s, 3, PA_AWAY + s / 4 * 2 + 1, 7, s & 1, (s >> 1) & 1);
    close();
    for (int x = 0; x < 4; ++x) P(x, 12, 1, 4, (8 + 4 * x) & 15);
    P(5, 12, 1, 5, 0); P(6, 12, 1, 5, 4);
#define MIX 0x76543210u
    for (int x = 0; x < 8; ++x) P(x, 0, 3, 5, 8 + ((MIX >> (4 * x)) & 7));
  }
#define CVT_W 56
  int nx = 0, nw = 0, c = 0;
  for (int pass = 0; pass < 3; ++pass) {
    for (int g = 0; g < NGEMM; ++g) {
      const int xcd = g & 7, s = g >> 3, ra = p.plan[g][1] & 63;
      if (early[g]) {
        if (xcd < 4 && s < HW) p.plan[g][0] = 0x9000u | (s * 4 + ((p.rank + xcd) & 3));
        if (xcd >= 4 && s * 2 + xcd - 4 < HX) p.plan[g][0] = 0x5000u | (s * 2 + xcd - 4);
        continue;
      }
      if (pass != (ra >= PA_OWNAWAY || (ra < 18 && ra % 6 == 1) ? 2 : ra > PA_AWAY && (ra & 1) ? 0 : 1)) continue;
      if (pass == 1 && c++ * CVT_W % 76 < CVT_W) p.plan[g][0] = 0x8000u;
      else p.plan[g][0] = 0x4000u | nx++;
    }
    p.counts = (pass ? p.counts << 8 : 0u) | nx;
  }
  for (int pass = 0, k = 0; pass < 2; ++pass)
    for (int g = 0; g < NGEMM; ++g)
      if ((p.plan[g][0] & 0x5000u) == 0x4000u && (g >= GSLOTS * 8 || (g & 7) >= 6) == !pass) p.plan[g][0] |= k++ << 16;
#define WORD 1
  for (int pass = 0; pass < 3; ++pass)
    for (int g = 0; g < NGEMM; ++g)
      if (p.plan[g][0] == 0x8000u && pass == (!WORD || g >= GSLOTS * 8 || (g & 7) >= 6 ? 0 : (g & 7) < 4 ? 1 : 2)) p.plan[g][0] |= 4 * HW + nw++;
  static_assert(HW <= 12 && HX <= 24, "");
  p.counts |= nw << 24;
}

}

extern "C" void kernel_launch(int rank, int nranks, ncclComm_t comm, hipStream_t stream,
                              void* const* d_in, const int* in_sizes, int n_in,
                              void* d_out, int out_size, const int* in_dtypes, int out_dtype,
                              void* const* d_out_all, void* const* d_ws_all, size_t ws_bytes) {
  (void)comm; (void)in_dtypes; (void)out_dtype; (void)d_out_all;
  if (nranks != NR || (unsigned)rank >= (unsigned)NR || n_in != 2 || in_sizes[0] != NSH * KDIM ||
      in_sizes[1] != KDIM * NSH || out_size != KDIM * NSH || ws_bytes < WS_NEED) {
    fprintf(stderr, "kernel_launch: unexpected problem shape\n");
    return;
  }
  Params p;
  p.x = (const float*)d_in[0];
  p.w = (const float*)d_in[1];
  p.rank = 0;
  for (int i = 0; i < NR; ++i) {
    const int r = (RANK_AT >> (4 * i)) & 7;
    p.ws[i] = (char*)d_ws_all[r];
    if (r == rank) p.rank = i;
  }
  p.mine = (char*)d_ws_all[rank];
  p.out = (float*)d_out;
  make_schedule(p);
  ag_gemm_silu<<<dim3(NWG), dim3(NTHREADS), 0, stream>>>(p);
  HIP_CHECK(hipGetLastError());
}
```

```cpp
#include <hip/hip_runtime.h>
#include <rccl/rccl.h>
#include <cstdio>
#include <cstdint>
#include <cstddef>

#define HIP_CHECK(expr) do { hipError_t _e = (expr); if (_e != hipSuccess) { \
    fprintf(stderr, "HIP error %s at %s:%d\n", hipGetErrorString(_e), __FILE__, __LINE__); \
    return; } } while (0)

namespace {

typedef _Float16 half8 __attribute__((ext_vector_type(8)));
typedef float f32x4 __attribute__((ext_vector_type(4)));
typedef unsigned u32x4 __attribute__((ext_vector_type(4)));
typedef unsigned u32x2 __attribute__((ext_vector_type(2)));
#define LAS __attribute__((address_space(3)))
#define GAS __attribute__((address_space(1)))

constexpr int NR = 8;
constexpr unsigned RANK_AT = 0x75416320u;
constexpr int KDIM = 8192;
constexpr int NSH = 1024;
constexpr int KSTEPS = 128;
constexpr unsigned STEP_B = 16384;
constexpr size_t PANEL_B = (size_t)KSTEPS * STEP_B;
constexpr int PA_AWAY = 24, PA_OWNAWAY = 32, PB = 34, PB_FOREIGN = 42, NPANEL = 50;

constexpr unsigned BLK_B = 1664;
constexpr unsigned PSTEP_B = 1024 + 8 * BLK_B;
constexpr size_t PPANEL_B = (size_t)KSTEPS * PSTEP_B;

constexpr size_t OFF_CALL = 0;
constexpr size_t OFF_TMO = 4096;
constexpr size_t OFF_TICKET = 5120;
constexpr int FCOPY = 8;
constexpr size_t OFF_FLAG = 1u << 18;
constexpr size_t OFF_RDY = 1u << 20;
constexpr int RDY_N = 1024, RDY_AWAY = 128, RDY_OUT = 384, RDY_HOP1 = 864, RDY_HOP2 = 896;
constexpr int RELAY = 32;
constexpr size_t OFF_OWN = 2u << 20;
constexpr size_t OFF_PACK = OFF_OWN + 32 * PANEL_B;
constexpr size_t OFF_RELAY = OFF_PACK + 32 * PPANEL_B;
constexpr size_t OFF_PULL = OFF_RELAY + PANEL_B;
constexpr int HX = 20;
constexpr int HW = 6;
constexpr int OUT_CHUNKS = 480;
constexpr unsigned SUM_B = 3456, CHUNK_B = 8192;
constexpr size_t OFF_STG = OFF_PULL + 2 * NPANEL * PANEL_B;
constexpr size_t OFF_SPARE = OFF_STG + (size_t)4 * OUT_CHUNKS * CHUNK_B;
constexpr size_t WS_NEED = OFF_SPARE + (size_t)4 * OUT_CHUNKS * 2048;
static_assert(RELAY % 4 == 0 && RELAY <= 64 && 64 * PSTEP_B <= PANEL_B, "");
static_assert(NPANEL <= 64 && OFF_FLAG + FCOPY * 65536 <= OFF_RDY && OFF_RDY + NR * RDY_N * 128 <= OFF_OWN, "");

constexpr int NWG = 256, NTHREADS = 512;
constexpr int GSLOTS = 24;
constexpr int NGEMM = GSLOTS * 8 + 8;
constexpr int PW = 8;
constexpr int WIN = 5;
constexpr int NJOBS = 3;
constexpr unsigned long long DEADLINE = 100000000ull;

constexpr unsigned LDS_MB = 18 * 8192;
constexpr unsigned LDS_TMO = LDS_MB + NJOBS * 4 * 512;
constexpr unsigned LDS_LUT = LDS_TMO + NJOBS * 16;
constexpr unsigned LDS_FILL = LDS_LUT + 64;
constexpr unsigned LDS_CALL = LDS_FILL + 16;
constexpr unsigned LDS_EPI = LDS_CALL + 16;
constexpr unsigned LDS_BYTES = LDS_EPI + 4 * 512;
constexpr unsigned WAVE_LDS = LDS_MB / 8;

struct Params {
  const float* x;
  const float* w;
  char* mine;
  int rank;
  unsigned counts;
  float* out;
  char* ws[NR];
  u32x4 plan[NGEMM];
};

__device__ __forceinline__ int xcc_id() {
  return __builtin_amdgcn_s_getreg(20 | (3 << 11)) & 7;
}

__device__ __forceinline__ void store_sys(void* ptr, u32x4 v) {
  asm volatile("global_store_dwordx4 %0, %1, off sc0 sc1\n\ts_nop 1" :: "v"(ptr), "v"(v) : "memory");
}
__device__ __forceinline__ void wait_vm0() { asm volatile("s_waitcnt vmcnt(0)" ::: "memory"); }
__device__ __forceinline__ void wg_barrier() {
  asm volatile("" ::: "memory");
  __builtin_amdgcn_s_barrier();
  asm volatile("" ::: "memory");
}
__device__ __forceinline__ void store_out(void* ptr, u32x4 v) {
  asm volatile("global_store_dwordx4 %0, %1, off sc0 sc1 nt\n\ts_nop 1" :: "v"(ptr), "v"(v) : "memory");
}

__device__ __forceinline__ void set_local(char* ws, int copy, int step, unsigned v) {
  __hip_atomic_store((GAS unsigned*)(ws + OFF_FLAG) + copy * 16384 + (step >> 7) * 256 + (step & 7) * 32 + ((step & 127) >> 3), v,
                     __ATOMIC_RELAXED, __HIP_MEMORY_SCOPE_SYSTEM);
}
__device__ __forceinline__ void set_rdy(char* peer_ws, int rank, int idx, unsigned call, unsigned lines) {
  __hip_atomic_store((GAS unsigned*)(peer_ws + OFF_RDY) + (rank * RDY_N + idx) * 32, call << 6 | lines, __ATOMIC_RELAXED,
                     __HIP_MEMORY_SCOPE_SYSTEM);
}
__device__ __forceinline__ unsigned own_idx(unsigned call, int idx) { return (call & 1) * 16 + idx; }
__device__ __forceinline__ size_t pulled(unsigned call) { return OFF_PULL + (call & 1) * (NPANEL * PANEL_B); }
__device__ __forceinline__ int fresh_lane() {
  int l;
  asm volatile("v_mbcnt_lo_u32_b32 %0, -1, 0\n\tv_mbcnt_hi_u32_b32 %0, -1, %0" : "=v"(l));
  return l;
}
__device__ __forceinline__ void cbar() { asm volatile("" ::: "memory"); }
__device__ __forceinline__ unsigned wave_scan(unsigned v) {
  v += __builtin_amdgcn_update_dpp(0u, v, 0x111, 0xf, 0xf, false);
  v += __builtin_amdgcn_update_dpp(0u, v, 0x112, 0xf, 0xf, false);
  v += __builtin_amdgcn_update_dpp(0u, v, 0x114, 0xf, 0xf, false);
  v += __builtin_amdgcn_update_dpp(0u, v, 0x118, 0xf, 0xf, false);
  v += __builtin_amdgcn_update_dpp(0u, v, 0x142, 0xa, 0xf, false);
  v += __builtin_amdgcn_update_dpp(0u, v, 0x143, 0xc, 0xf, false);
  asm("" : "+v"(v));
  return v;
}

__device__ __forceinline__ unsigned codec_lut(int m, bool expand) {
  unsigned sel = 0x0c0c0c0cu;
  int n = 0;
  for (int i = 0; i < 4; ++i)
    if ((m >> i) & 1) {
      const int at = expand ? i : n, v = expand ? n : i;
      sel = (sel & ~(0xffu << (8 * at))) | ((unsigned)v << (8 * at));
      ++n;
    }
  return sel;
}

__device__ __forceinline__ unsigned lut_at(const LAS char* lut, unsigned E) {
  return *(const LAS unsigned*)(uintptr_t)__builtin_amdgcn_udot4(E, 0x08040201u, (unsigned)(uintptr_t)lut, false);
}
__device__ __forceinline__ void lds_or(LAS unsigned* q, unsigned v) {
  __hip_atomic_fetch_or(q, v, __ATOMIC_RELAXED, __HIP_MEMORY_SCOPE_WORKGROUP);
}
struct Codes {
  unsigned pk[4], n[4], EM = 0, XM = 0;
  __device__ __forceinline__ void add(int k, const LAS char* lut, unsigned c, unsigned E, unsigned X) {
    const unsigned z = __builtin_amdgcn_perm(0u, c, lut_at(lut, E));
    pk[k] = __builtin_amdgcn_udot4(z, 0x00400801u, (z >> 24) << 9, false);
    n[k] = __builtin_popcount(E);
    EM |= (E >> 2) << k;
    XM |= X >> (7 - k);
  }
  template <int SH>
  __device__ __forceinline__ void put(LAS char* list, LAS char* patch, unsigned pre, const unsigned (&H)[4], unsigned base) {
    const unsigned long long W = (unsigned long long)(pk[0] | (pk[1] << (3 * n[0])))
        | ((unsigned long long)(pk[2] | (pk[3] << (3 * n[2]))) << (3 * (n[0] + n[1])));
    const unsigned o = 3 * pre, sh = o & 31u;
    LAS unsigned* q = (LAS unsigned*)(list + (o >> 5) * 4);
    const unsigned long long Ws = W << sh;
    lds_or(q, (unsigned)Ws);
    lds_or(q + 1, (unsigned)(Ws >> 32));
    lds_or(q + 2, ((unsigned)(W >> 32) >> 1) >> (31u - sh));
    cbar();
    const unsigned H0 = H[0], H1 = H[1], H2 = H[2], H3 = H[3];
    for (; XM; XM &= XM - 1u, patch -= 2) {
      const unsigned pos = __builtin_ctz(XM), k = pos & 3u, i8 = pos & 24u;
      const unsigned Hk = k & 2u ? (k & 1u ? H3 : H2) : (k & 1u ? H1 : H0);
      const unsigned kb = 0x01010101u << k;
      const unsigned rk = pre + __builtin_popcount(EM & ((kb - 0x01010101u) | (kb & ((1u << i8) - 1u))));
      const unsigned at = (unsigned)(uintptr_t)patch;
      lds_or((LAS unsigned*)(uintptr_t)(at & ~3u), ((rk << 5) | (((Hk >> (i8 + SH)) & 31u) ^ (base + 3u))) << ((at & 2u) * 8));
    }
    cbar();
  }
};

__device__ __forceinline__ void pack_block(u32x4 f0, u32x4 f1, LAS char* wl, const LAS char* lut, LAS char* lists,
                                           LAS unsigned* fill, char* step, int blk) {
  const int lane = threadIdx.x & 63;
  const unsigned h[8] = {f0[0], f0[1], f0[2], f0[3], f1[0], f1[1], f1[2], f1[3]};
  char* const dst = step + 1024 + blk * BLK_B;
  u32x4 D;
  unsigned H[4], t[4], sum = 0, sg = 0;
#pragma unroll
  for (int k = 0; k < 4; ++k) {
    D[k] = __builtin_amdgcn_perm(h[2 * k + 1], h[2 * k], 0x06040200u);
    H[k] = __builtin_amdgcn_perm(h[2 * k + 1], h[2 * k], 0x07050301u);
    sum = __builtin_amdgcn_udot4(H[k] & 0x7c7c7c7cu, 0x01010101u, sum, false);
    sg |= (H[k] & 0x80808080u) >> (7 - k);
  }
  store_sys(dst + lane * 16, D);
  const unsigned sgn = __builtin_amdgcn_update_dpp(0u, sg, 0xb1, 0xf, 0xf, false);
  if (!(lane & 1)) *(LAS unsigned*)(wl + 512 + lane * 2) = sg | (sgn << 4);
  const unsigned mean = __builtin_amdgcn_readlane(wave_scan(sum), 63) + 256u;
  const unsigned base = min(mean >> 12, 28u);
  const bool low = !(mean & 2048u);
  Codes c;
  unsigned W = 0, n2 = 0;
#pragma unroll
  for (int k = 0; k < 4; ++k) {
    const unsigned g = ((H[k] >> 2) & 0x1f1f1f1fu) + (0x43u - base) * 0x01010101u;
    const unsigned x = g ^ 0x40404040u, sel = x | (((x & 0x78787878u) + 0x78787878u) & 0x80808080u);
    const unsigned y = (g + 0x07070707u) ^ 0x40404040u ^ ~__builtin_amdgcn_perm(low ? 0xfff5ffffu : ~0u, ~0u, sel);
    const unsigned tt = __builtin_amdgcn_perm(low ? 0xffff0804u : 0xff1c0804u, low ? 0x001e1d1cu : 0x001e1dffu, sel);
    t[k] = (tt & 0x0c0c0c0cu) | (H[k] & 0x03030303u);
    const unsigned E = (tt >> 2) & 0x04040404u;
    W |= __builtin_amdgcn_udot4(__builtin_amdgcn_perm(0u, tt & 0x03030303u, lut_at(lut, E)), 0x40100401u, 0u, false) << (2 * n2);
    n2 += __builtin_popcount(E);
    c.add(k, lut, y & 0x07070707u, (tt >> 3) & 0x04040404u, ((y & 0x78787878u) + 0x78787878u) & tt & 0x80808080u);
  }
  *(LAS u32x2*)(wl + lane * 8) = (u32x2){t[0] | (t[1] << 4), t[2] | (t[3] << 4)};
  const unsigned mine = n2 | ((c.n[0] + c.n[1] + c.n[2] + c.n[3]) << 11) | (__builtin_popcount(c.XM) << 22);
  const unsigned incl = wave_scan(mine), pre = incl - mine;
  const unsigned all = __builtin_amdgcn_readlane(incl, 63), n4 = all >> 22, l2 = ((all & 2047u) + 15u) >> 4;
  const unsigned len = l2 + ((3u * ((all >> 11) & 2047u) + 16u * n4 + 31u) >> 5);
  unsigned head = 1u << 31, off = 0;
  cbar();
  if (l2 <= 32u && len < 64u && n4 < 32u) {
    asm volatile("" : "+v"(off));
    if (lane == 0) off = __hip_atomic_fetch_add(fill + off, len, __ATOMIC_RELAXED, __HIP_MEMORY_SCOPE_WORKGROUP);
    off = __builtin_amdgcn_readfirstlane(off);
    if (off + len < 256u) {
      head = base | (n4 << 5) | (low ? 1024u : 0u) | (l2 << 11) | (len << 17) | (off << 23);
      LAS char* const esc = lists + off * 4;
      const unsigned sh = (2u * pre) & 31u;
      LAS unsigned* q = (LAS unsigned*)esc + ((pre & 2047u) >> 4);
      lds_or(q, W << sh);
      lds_or(q + 1, (W >> 1) >> (31u - sh));
      c.put<2>(esc + l2 * 4, esc + len * 4 - 2 - 2 * (pre >> 22), (pre >> 11) & 2047u, H, base);
    }
  }
  if (lane == 0) ((LAS unsigned*)lists)[blk] = head;
  cbar();
}
__device__ __forceinline__ void pack_flush(LAS char* wl, LAS char* lists, char* step, int blk) {
  const int lane = threadIdx.x & 63;
  if (lane < 48) {
    const unsigned line = blk * 128 + (lane - 40) * 16;
    LAS u32x4* const from = (LAS u32x4*)(lane < 40 ? wl + lane * 16 : lists + line);
    const u32x4 v = *from;
    *from = (u32x4){0u, 0u, 0u, 0u};
    store_sys(step + (lane < 40 ? 2048 + blk * BLK_B + lane * 16 : line), v);
  }
}
__device__ __forceinline__ void lds_barrier() {
  asm volatile("s_waitcnt lgkmcnt(0)\n\ts_barrier" ::: "memory");
}
__device__ __forceinline__ unsigned lines_used(LAS unsigned* fill) {
  const unsigned n = *fill;
  *fill = 8u;
  return min(n - 1u, 255u) >> 5;
}

constexpr unsigned SUM_BASE = 24;
__device__ __forceinline__ void pack_sums(int lane, const u32x4 (&v)[4], unsigned& off, LAS char* wl, const LAS char* lut,
                                          char* dst, char* spare) {
  if (lane < 36) *(LAS unsigned*)(wl + 256 + lane * 4) = 0u;
  u32x4 P[3], E;
#pragma unroll
  for (int k = 0; k < 4; ++k) {
#pragma unroll
    for (int i = 0; i < 3; ++i) P[i][k] = __builtin_amdgcn_perm(v[3][k], v[i][k], 0x04020100u + ((unsigned)i << 24));
    E[k] = __builtin_amdgcn_perm(v[1][k], v[0][k], 0x0c0c0703u) | __builtin_amdgcn_perm(v[3][k], v[2][k], 0x07030c0cu);
  }
  asm volatile("global_store_dwordx4 %0, %1, off sc0 sc1\n\tglobal_store_dwordx4 %0, %2, off offset:1024 sc0 sc1\n\t"
               "global_store_dwordx4 %0, %3, off offset:2048 sc0 sc1\n\ts_nop 1"
               :: "v"(dst + lane * 16), "v"(P[0]), "v"(P[1]), "v"(P[2]) : "memory");
  if (off == ~0u) {
    unsigned sum = 0;
#pragma unroll
    for (int k = 0; k < 4; ++k) sum = __builtin_amdgcn_udot4(E[k], 0x01010101u, sum, false);
    off = min(max((__builtin_amdgcn_readlane(wave_scan(sum), 63) + 64u) >> 10, SUM_BASE) - SUM_BASE, 223u);
  }
  static_assert(SUM_BASE == 24, "");
  unsigned d[4], t[4];
  Codes c;
#pragma unroll
  for (int k = 0; k < 4; ++k) {
    d[k] = E[k] - off * 0x01010101u;
    t[k] = __builtin_amdgcn_perm(~0u, 0xf4a05000u, (d[k] + 0x68686868u) ^ 0x80808080u);
    c.add(k, lut, d[k] & t[k] & 0x07070707u, t[k] & 0x04040404u, 0u);
    c.XM |= __builtin_amdgcn_perm(0u, 0x0f0fu, (d[k] + 0x71717171u) ^ 0x80808080u) & (0x01010101u << k);
  }
  const unsigned mine = (c.n[0] + c.n[1] + c.n[2] + c.n[3]) | (__builtin_popcount(c.XM) << 16);
  const unsigned incl = wave_scan(mine), pre = incl - mine;
  const unsigned all = __builtin_amdgcn_readlane(incl, 63), n4 = all >> 16;
  const bool raw = __builtin_amdgcn_ballot_w64(((d[0] | d[1] | d[2] | d[3]) & 0xe0e0e0e0u) != 0u) != 0ull
                || 3u * (all & 0xffffu) + 16u * n4 > 1008u;
  cbar();
  if (!raw) c.put<0>(wl + 256, wl + 380 - 2 * (pre >> 16), pre & 0xffffu, d, SUM_BASE);
  if (lane == 0) *(LAS unsigned short*)(wl + 382) = (unsigned short)(raw ? 0x8000u : (off & 31u) | (n4 << 5) | (off >> 5 << 11));
  auto mix = [](unsigned m, unsigned x, unsigned y) { return (x & m) | (y & ~m); };
  *(LAS unsigned*)(wl + lane * 4) = mix(0xf0f0f0f0u, mix(0x30303030u, t[2], t[3]), mix(0x30303030u, t[0], t[1]) >> 4);
  cbar();
  if (lane < 24) store_sys(dst + 3072 + lane * 16, *(const LAS u32x4*)(wl + lane * 16));
  if (raw) store_sys(spare + lane * 16, E);
  cbar();
}

__device__ __forceinline__ u32x2 read_codes(int lane, const LAS char* list) {
  const LAS unsigned* sp = (const LAS unsigned*)(list + ((3 * lane) & ~3));
  return (u32x2){sp[0], sp[1]};
}
__device__ __forceinline__ void expand_codes(int lane, u32x2 codes, unsigned base, unsigned first, unsigned keep, unsigned keep1,
                                             bool patch, unsigned ent, LAS char* ex) {
  __builtin_amdgcn_sched_barrier(0);
  const unsigned x = __builtin_amdgcn_alignbyte(codes[1], codes[0], 3 * lane);
  const unsigned b3 = (base + 3u) * 0x04040404u, b8 = (base + first) * 0x04040404u;
  const unsigned T0 = ((b8 + 0x0c080400u) ^ b3) & keep, T1 = ((b8 + 0x1c181410u) ^ b3) & keep1;
  u32x2 X;
#pragma unroll
  for (int i = 0; i < 2; ++i) {
    const unsigned y = ((x >> (12 * i)) & 0x3fu) | ((i ? x >> 2 : x << 10) & 0x3f0000u);
    X[i] = __builtin_amdgcn_perm(T1, T0, (y | (y << 5)) & 0x07070707u);
  }
  *(LAS u32x2*)(ex + lane * 8) = X;
  cbar();
  if (patch) *(LAS unsigned char*)(ex + ((ent >> 5) & 511u)) = (unsigned char)((ent & 31u) << 2);
  cbar();
}
template <int N>
__device__ __forceinline__ void pick(const unsigned (&E)[N], const unsigned (&sel)[N], const LAS char* ex, unsigned pos,
                                     unsigned (&out)[N]) {
  unsigned at[N], w0[N], w1[N];
#pragma unroll
  for (int k = 0; k < N; ++k) { at[k] = pos; pos += __builtin_popcount(E[k]); }
  __builtin_amdgcn_sched_barrier(0);
#pragma unroll
  for (int k = 0; k < N; ++k) {
    const LAS unsigned* wp = (const LAS unsigned*)(ex + (at[k] & ~3u));
    w0[k] = wp[0];
    w1[k] = wp[1];
  }
  __builtin_amdgcn_sched_barrier(0);
#pragma unroll
  for (int k = 0; k < N; ++k) out[k] = __builtin_amdgcn_perm(0u, __builtin_amdgcn_alignbyte(w1[k], w0[k], at[k]), sel[k]);
}

__device__ __forceinline__ void unpack_block(int lane, const LAS char* step, int blk, unsigned hdr, LAS char* ex,
                                             const LAS char* lut, u32x4& f0, u32x4& f1) {
  const LAS char* const src = step + 1024 + blk * BLK_B;
  const LAS char* const esc = step + (hdr >> 23) * 4;
  const unsigned base = hdr & 31u;
  const bool patch = lane < ((hdr >> 5) & 31u), low = hdr & 1024u;
  const u32x2 x3 = read_codes(lane, esc + ((hdr >> 11) & 63u) * 4);
  const unsigned ent = *(const LAS unsigned short*)(esc + ((hdr >> 17) & 63u) * 4 - (patch ? 2 + 2 * lane : 0));
  const unsigned x2 = *(const LAS unsigned short*)(esc + 2 * lane);
  const u32x2 N = *(const LAS u32x2*)(src + 1024 + lane * 8);
  unsigned sg = *(const LAS unsigned*)(src + 1536 + (lane >> 1) * 4);
  const u32x4 D = *(const LAS u32x4*)(src + lane * 16);
  expand_codes(lane, x3, base, 22u, 0x7c7c7c7cu, low ? 0x007c7c7cu : 0x7c7c7c7cu, patch, ent, ex + 512);
  sg >>= (lane & 1) * 4;
  const unsigned E0 = N[0] & (N[0] >> 1) & 0x44444444u, E1 = N[1] & (N[1] >> 1) & 0x44444444u;
  unsigned c[2], F[2], sf[2], X[2], E[4], se[4], Y[4], h[8];
#pragma unroll
  for (int i = 0; i < 2; ++i) {
    const unsigned y = ((x2 >> (8 * i)) & 0xfu) | ((x2 << (12 - 8 * i)) & 0xf0000u);
    c[i] = (y | (y << 6)) & 0x03030303u;
    F[i] = (c[i] & (c[i] >> 1)) << 2;
    sf[i] = lut_at(lut, F[i]);
  }
#pragma unroll
  for (int k = 0; k < 4; ++k) {
    E[k] = ((k & 2 ? E1 : E0) >> (4 * (k & 1))) & 0x04040404u;
    se[k] = lut_at(lut, E[k]);
  }
  const unsigned mine = (__builtin_popcount(E0) + __builtin_popcount(E1)) | ((__builtin_popcount(F[0]) + __builtin_popcount(F[1])) << 16);
  const unsigned pre = wave_scan(mine) - mine;
  pick<2>(F, sf, ex + 512, pre >> 16, X);
  const unsigned T = ((base * 0x00040404u + 0x007c7874u) ^ ((base + 3u) * 0x00040404u)) & (low ? 0x007c7c7cu : 0x007c7c00u);
  *(LAS u32x2*)(ex + lane * 8) = (u32x2){X[0] | __builtin_amdgcn_perm(0u, T, c[0]), X[1] | __builtin_amdgcn_perm(0u, T, c[1])};
  cbar();
  pick<4>(E, se, ex, pre & 0xffffu, Y);
#pragma unroll
  for (int k = 0; k < 4; ++k) {
    const unsigned t = ((N[k >> 1] >> (4 * (k & 1))) & 0x0f0f0f0fu) + base * 0x04040404u;
    const unsigned H = (t ^ Y[k]) | ((sg << (7 - k)) & 0x80808080u);
    h[2 * k] = __builtin_amdgcn_perm(H, D[k], 0x05010400u);
    h[2 * k + 1] = __builtin_amdgcn_perm(H, D[k], 0x07030602u);
  }
  cbar();
  f0 = (u32x4){h[0], h[1], h[2], h[3]};
  f1 = (u32x4){h[4], h[5], h[6], h[7]};
}

__device__ __forceinline__ void unpack_sums(int lane, const LAS char* blk, LAS char* ex, const LAS char* lut, const char* spare,
                                            f32x4 (&y)[4]) {
  const unsigned ent = *(const LAS unsigned short*)(blk + 3454 - 2 * lane);
  const u32x2 x3 = read_codes(lane, blk + 3328);
  const unsigned C = *(const LAS unsigned*)(blk + 3072 + lane * 4);
  u32x4 P[3], X;
#pragma unroll
  for (int i = 0; i < 3; ++i) P[i] = *(const LAS u32x4*)(blk + i * 1024 + lane * 16);
  const unsigned hdr = __builtin_amdgcn_readfirstlane(ent);
  if (hdr & 0x8000u) {
    asm volatile("global_load_dwordx4 %0, %1, off sc0 sc1\n\ts_waitcnt vmcnt(0)" : "=&v"(X) : "v"(spare + lane * 16) : "memory");
  } else {
    expand_codes(lane, x3, SUM_BASE, 24u, 0x7c7c7c00u, 0x7c7c7c7cu, (unsigned)(lane - 1) < ((hdr >> 5) & 63u), ent, ex);
    unsigned t[4], E[4], se[4], Y[4];
#pragma unroll
    for (int k = 0; k < 4; ++k) {
      t[k] = (k ? C >> (2 * k - 2) : C << 2) & 0x0c0c0c0cu;
      E[k] = t[k] & (t[k] >> 1);
      se[k] = lut_at(lut, E[k]);
    }
    const unsigned mine = __builtin_popcount(C & (C >> 1) & 0x55555555u);
    pick<4>(E, se, ex, wave_scan(mine) - mine, Y);
    const unsigned off = ((hdr & 31u) | (hdr >> 11 << 5)) * 0x01010101u;
#pragma unroll
    for (int k = 0; k < 4; ++k) X[k] = (((t[k] + SUM_BASE * 0x04040404u) ^ Y[k]) >> 2) + off;
    cbar();
  }
#pragma unroll
  for (int k = 0; k < 4; ++k) {
    unsigned v[4];
#pragma unroll
    for (int i = 0; i < 3; ++i) v[i] = __builtin_amdgcn_perm(X[k], P[i][k], 0x04020100u + ((unsigned)i << 24));
    v[3] = __builtin_amdgcn_perm(P[1][k], P[0][k], 0x0c0c0703u) | __builtin_amdgcn_perm(X[k], P[2][k], 0x07030c0cu);
#pragma unroll
    for (int i = 0; i < 4; ++i) y[k][i] = __builtin_bit_cast(float, __builtin_rotateright32(v[i], 1));
  }
}

__device__ __forceinline__ f32x4 sigmoid(f32x4 y) {
  constexpr float K = 1.44269504088896341f;
  float lim = 60.0f * K;
  asm("" : "+s"(lim));
  f32x4 t = y * -K;
#pragma unroll
  for (int i = 0; i < 4; ++i) t[i] = __builtin_amdgcn_exp2f(__builtin_amdgcn_fmed3f(t[i], -lim, lim));
  t += 1.0f;
#pragma unroll
  for (int i = 0; i < 4; ++i) t[i] = __builtin_amdgcn_rcpf(t[i]);
  return t;
}

__device__ __forceinline__ unsigned* call_word(const Params& p) { return (unsigned*)(p.mine + OFF_CALL) + blockIdx.x; }
template <int LOADS>
__device__ __forceinline__ unsigned begin(const Params& p, LAS char* smem, bool puller) {
  *(LAS u32x4*)(smem + LDS_MB + min(threadIdx.x * 16u, LDS_LUT - LDS_MB - 16u)) = (u32x4){0u, 0u, 0u, 0u};
  if (threadIdx.x < 20) ((LAS unsigned*)(smem + LDS_LUT))[threadIdx.x] = threadIdx.x < 16 ? codec_lut(threadIdx.x, puller) : 8u;
  asm volatile("s_waitcnt vmcnt(%0) lgkmcnt(0)" :: "n"(LOADS) : "memory");
  wg_barrier();
  return __builtin_amdgcn_readfirstlane(*(const LAS unsigned*)(smem + LDS_CALL)) + 1u;
}
static_assert(LDS_LUT - LDS_MB <= NTHREADS * 16 && (LDS_LUT - LDS_MB) % 16 == 0, "");

constexpr int XJ = (KSTEPS - HX) / 2;
static_assert(HX % 2 == 0, "");
__device__ __forceinline__ int x_unit(int u) {
  if (u < RELAY) return 5 * KSTEPS + u;
  const int j = u < 7 * RELAY ? (u - RELAY) / 6 : u / 7, r = u < 7 * RELAY ? u - RELAY - j * 6 : u - j * 7;
  if (u < 7 * KSTEPS && (r >= 2 || j < XJ)) return r >= 2 ? (r - 1) * KSTEPS + j : HX + 2 * j + r;
  const int i = (u < 7 * KSTEPS ? 2 * j + r : u - 5 * KSTEPS) - 2 * XJ;
  return (6 + (i & 1)) * KSTEPS + (i >> 1);
}
constexpr int XROUNDS = 11;
constexpr int XLONG = 5;
__device__ __forceinline__ void load_in(const float* a, const float* b, unsigned dst) {
  unsigned keep;
  asm volatile("s_mov_b32 %0, m0\n\ts_mov_b32 m0, %3\n\ts_nop 0\n\tglobal_load_lds_dwordx4 %1, off nt\n\t"
               "s_add_u32 m0, m0, 1024\n\ts_nop 0\n\tglobal_load_lds_dwordx4 %2, off nt\n\ts_mov_b32 m0, %0"
               : "=&s"(keep) : "v"(a), "v"(b), "s"(dst) : "memory", "scc");
}
__device__ __forceinline__ void pace(unsigned t0, int at) {
  while ((int)((unsigned)__builtin_amdgcn_s_memrealtime() - t0) * 8 < at) __builtin_amdgcn_s_sleep(2);
}
constexpr int XT = 40, XO = 16, WT = 88, WO = 24, LEAD = 1200;
template <int N>
__device__ __forceinline__ void wait_vm() { asm volatile("s_waitcnt vmcnt(%0)" :: "n"(N) : "memory"); }
__device__ __forceinline__ u32x4 cvt8(f32x4 a, f32x4 b) {
  half8 h;
  h[0] = (_Float16)a[0]; h[1] = (_Float16)a[1]; h[2] = (_Float16)a[2]; h[3] = (_Float16)a[3];
  h[4] = (_Float16)b[0]; h[5] = (_Float16)b[1]; h[6] = (_Float16)b[2]; h[7] = (_Float16)b[3];
  u32x4 r = __builtin_bit_cast(u32x4, h);
  asm volatile("" : "+v"(r));
  return r;
}
__device__ __forceinline__ char* member(const Params& p, int group, int i) {
  char *a = p.ws[group], *b = p.ws[group + 1], *c = p.ws[group + 2], *d = p.ws[group + 3];
  asm volatile("" : "+s"(a), "+s"(b), "+s"(c), "+s"(d));
  return i & 2 ? (i & 1 ? d : c) : (i & 1 ? b : a);
}
__device__ __forceinline__ unsigned convert_x(const Params& p, unsigned role, unsigned counts, int rank, LAS char* smem) {
  const int lane = threadIdx.x & 63, wave = __builtin_amdgcn_readfirstlane(threadIdx.x >> 6), t = threadIdx.x;
  const int n0 = counts & 255u, n1 = (counts >> 8) & 255u, n2 = (counts >> 16) & 255u;
  const int first = role & 255u, all = XROUNDS * n0, most = min(all + XLONG * n1, 8 * KSTEPS - HX);
  const int count = first < n2 ? 8 * KSTEPS - HX : first < n1 ? most : all;
  const bool push = role & 0x1000u;
  char* const ws = p.mine;
  LAS char* const wl = smem + wave * WAVE_LDS;
  const float* src0 = p.x + (size_t)((wave >> 2) * 64 + (wave & 1) * 32 + (lane >> 3)) * KDIM + (wave & 2) * 16 + ((lane ^ (lane >> 4)) & 7) * 4;
  const int odd8 = 8 * KDIM + (lane & 4 ? -16 : 16);
  auto fetch = [&](unsigned buf, int u) {
    const float* s = src0 + (size_t)(u >> 7) * 128 * KDIM + (u & 127) * 64;
    const unsigned d = (unsigned)(uintptr_t)wl + 2048 + buf;
    load_in(s, s + odd8, d);
    load_in(s + 16 * KDIM, s + 16 * KDIM + odd8, d + 2048);
  };
  u32x4 h0, h1;
  auto take = [&](unsigned buf) {
    const LAS f32x4* v = (const LAS f32x4*)(wl + 2048 + buf) + (lane & 15) * 8;
    const int i = (lane >> 4) ^ ((lane >> 1) & 7);
    const f32x4 a = v[i], b = v[i ^ 4], c = v[128 + i], d = v[128 + (i ^ 4)];
    h0 = cvt8(a, b);
    h1 = cvt8(c, d);
  };
  auto next = [=](int it) { return it + (int)((counts >> (it < all ? 0 : it < most ? 8 : 16)) & 255u); };
  const unsigned t0 = __builtin_amdgcn_s_memrealtime();
  fetch(0, push ? first : x_unit(role >> 16));
  const unsigned call = begin<4>(p, smem, false);
  char* const in = member(p, rank & 4, t);
  if (push) {
    const unsigned at = first * STEP_B + wave * 2048 + lane * 16;
    char* b[4];
#pragma unroll
    for (int i = 0; i < 4; ++i)
      b[i] = at + (i == 3 ? ws + OFF_OWN + own_idx(call, 0) * PANEL_B
                          : member(p, rank & 4, rank + 1 + (i + first) % 3) + pulled(call) + (rank & 3) * 6 * PANEL_B);
    asm volatile("s_waitcnt vmcnt(0)" : "+v"(b[0]), "+v"(b[1]), "+v"(b[2]), "+v"(b[3]) :: "memory");
    take(0);
#pragma unroll
    for (int i = 0; i < 4; ++i) {
      store_sys(b[i], h0);
      store_sys(b[i] + 1024, h1);
    }
    wait_vm0();
    wg_barrier();
    if (t < 4 * FCOPY) set_local(in, t >> 2, (rank & 3) * 6 * KSTEPS + first, call);
    return call;
  }
  char* const across = member(p, (rank & 4) ^ 4, t);
  LAS unsigned* const fill = (LAS unsigned*)(smem + LDS_FILL);
  unsigned odd = 0;
  auto ahead = [&](unsigned buf, int it) {
    if (it < count) { fetch(buf, x_unit(it)); wait_vm<4>(); } else wait_vm0();
  };
  wait_vm0();
  for (int it = first, it1 = next(it), it2; it < count; it = it1, it1 = it2, odd ^= 8192u) {
    const int u = x_unit(it == first ? role >> 16 : it);
    it2 = next(it1);
    take(odd);
    char* b = ws + OFF_OWN + (own_idx(call, 0) * KSTEPS + u) * STEP_B + wave * 2048 + lane * 16;
    store_sys(b, h0);
    store_sys(b + 1024, h1);
    char* pb = ws + OFF_PACK + (own_idx(call, 0) * KSTEPS + u) * PSTEP_B;
    pack_block(h0, h1, wl, smem + LDS_LUT, smem + LDS_MB, fill, pb, wave);
    lds_barrier();
    pack_flush(wl, smem + LDS_MB, pb, wave);
    unsigned lines = 0;
    if (wave == 0) lines = lines_used(fill);
    ahead(it == first ? 8192u : odd, it == first ? it1 : it2);
    lds_barrier();
    if (t < 4 + FCOPY) {
      const int tile = u >> 7, ks = u & 127;
      if (t >= 4) set_local(ws, t - 4, (tile < 6 ? (rank & 3) * 6 : PA_OWNAWAY - 6) * KSTEPS + u, call);
      else if (tile >= 6) set_rdy(across, rank, RDY_AWAY + u - 6 * KSTEPS, call, lines);
      else if (tile == 5 && ks < RELAY) { if (t == (ks & 3)) set_rdy(across, rank, RDY_HOP1 + (ks >> 2), call, lines); }
      else if (t != (rank & 3)) set_rdy(in, rank, u, call, lines);
    }
    if (it == first) ahead(0, it2);
    if (it1 < count) pace(t0, (it1 - XO) * XT + LEAD);
  }
  return call;
}

__device__ __forceinline__ unsigned convert_w(const Params& p, unsigned role, unsigned counts, int rank, LAS char* smem) {
  const int lane = threadIdx.x & 63, wave = __builtin_amdgcn_readfirstlane(threadIdx.x >> 6), t = threadIdx.x;
  const int count = KSTEPS * 4, stride = counts >> 24, first = role & 255u;
  const bool push = role & 0x1000u;
  char* const ws = p.mine;
  LAS char* const wl = smem + wave * WAVE_LDS;
  const float* src0 = p.w + (size_t)((wave & 1) * 32 + (lane >> 4) * 4) * NSH + (wave >> 1) * 64 + (lane & 15) * 4;
  auto fetch = [&](unsigned buf, int u) {
    const float* s = src0 + (size_t)(u >> 2) * 64 * NSH + (u & 3) * 256;
#pragma unroll
    for (int j = 0; j < 8; j += 2) {
      const float* r = s + ((j & 3) + (j >> 2) * 16) * NSH;
      load_in(r, r + NSH, (unsigned)(uintptr_t)wl + 2048 + buf + j * 1024);
    }
  };
  u32x4 h[4];
  auto take = [&](unsigned buf) {
    f32x4 v[8];
#pragma unroll
    for (int j = 0; j < 8; ++j) v[j] = ((const LAS f32x4*)(wl + 2048 + buf))[j * 64 + lane];
#pragma unroll
    for (int f = 0; f < 4; ++f)
      h[f] = cvt8((f32x4){v[0][f], v[1][f], v[2][f], v[3][f]}, (f32x4){v[4][f], v[5][f], v[6][f], v[7][f]});
  };
  const bool late = push && wave >= 4;
  const unsigned t0 = __builtin_amdgcn_s_memrealtime();
  if (!push) pace(t0, (first - 4 * HW - WO) * WT);
  if (!late) fetch(0, first);
  const unsigned call = begin<8>(p, smem, false);
  if (late) fetch(0, first);
  char* const own = ws + OFF_OWN + own_idx(call, 8 + (wave >> 2)) * PANEL_B + (wave & 3) * 4096 + lane * 16;
  char* const across = member(p, (rank & 4) ^ 4, push ? first : t);
  if (push) {
    const int q = first & 3, ks = first >> 2;
    char* b = own + q * 2 * PANEL_B + ks * STEP_B;
    char* r = across + pulled(call) + (PB_FOREIGN + (rank & 3) * 2 + (wave >> 2)) * PANEL_B + ks * STEP_B + (wave & 3) * 4096 + lane * 16;
    asm volatile("s_waitcnt vmcnt(0)" : "+v"(b), "+v"(r) :: "memory");
    take(0);
#pragma unroll
    for (int f = 0; f < 4; ++f) store_sys(r + f * 1024, h[f]);
#pragma unroll
    for (int f = 0; f < 4; ++f) store_sys(b + f * 1024, h[f]);
    wait_vm0();
    wg_barrier();
    if (t < 4 * FCOPY)
      set_local(t & 2 ? across : ws, t >> 2, ((t & 2 ? PB_FOREIGN + (rank & 3) * 2 : PB + q * 2) + (t & 1)) * KSTEPS + ks, call);
    return call;
  }
  LAS unsigned* const fill = (LAS unsigned*)(smem + LDS_FILL) + (wave >> 2);
  LAS char* const lists = smem + LDS_MB + (wave >> 2) * 2048;
  unsigned odd = 0;
  auto ahead = [&](unsigned buf, int u) {
    if (u < count) { fetch(buf, u); wait_vm<8>(); } else wait_vm0();
  };
  static_assert(FCOPY % 2 == 0, "");
  wait_vm0();
  for (int u = first; u < count; u += stride, odd ^= 8192u) {
    const int ks = u >> 2, q = u & 3, blk = (wave & 3) * 2;
    char* b = own + q * 2 * PANEL_B + ks * STEP_B;
    take(odd);
#pragma unroll
    for (int f = 0; f < 4; ++f) store_sys(b + f * 1024, h[f]);
    char* pb = ws + OFF_PACK + own_idx(call, 8 + q * 2 + (wave >> 2)) * PPANEL_B + ks * PSTEP_B;
    pack_block(h[0], h[1], wl, smem + LDS_LUT, lists, fill, pb, blk);
    pack_block(h[2], h[3], wl + 1024, smem + LDS_LUT, lists, fill, pb, blk + 1);
    lds_barrier();
    pack_flush(wl, lists, pb, blk);
    pack_flush(wl + 1024, lists, pb, blk + 1);
    unsigned lines = 0;
    if (wave == 0) lines = lines_used(fill) | lines_used(fill + 1) << 3;
    ahead(u == first ? 8192u : odd, u + (u == first ? stride : 2 * stride));
    lds_barrier();
    if (t < 2 * FCOPY) set_local(ws, t >> 1, (PB + q * 2 + (t & 1)) * KSTEPS + ks, call);
    else if (t == 2 * FCOPY + q) set_rdy(across, rank, ks, call, lines);
    if (u == first) ahead(0, u + 2 * stride);
    if (u + stride < count) pace(t0, (u + stride - 4 * HW - WO) * WT + LEAD);
  }
  return call;
}

__device__ __forceinline__ int row_of(int r) { return ((RANK_AT >> (4 * r)) & 7) * NSH; }

constexpr unsigned long long BATCH_AT = 0x7648953210ull;
constexpr unsigned long long batch_pos() {
  unsigned long long r = 0;
  for (int i = 0; i < 10; ++i) r |= (unsigned long long)i << (4 * ((BATCH_AT >> (4 * i)) & 15));
  return r;
}
constexpr unsigned long long BATCH_POS = batch_pos();
__device__ __forceinline__ int batch_row(int s, int o, int pos, int i) {
  const int bt = (BATCH_AT >> (4 * pos)) & 15;
  if (bt < 2) return row_of(s) + (bt * 3 + i) * 128;
  const int g = bt < 8 ? s : o;
  return row_of((g & 4) | ((g + 1 + i) & 3)) + (bt - 2) * 128;
}

__device__ __forceinline__ int unit_of(int it, int w) { return it * PW + ((w + it) & (PW - 1)); }

__device__ __forceinline__ unsigned lds_get(const LAS unsigned* c) {
  return __builtin_amdgcn_readfirstlane(__hip_atomic_load(c, __ATOMIC_RELAXED, __HIP_MEMORY_SCOPE_WORKGROUP));
}

constexpr int RING = 8;
constexpr unsigned PL_EX = RING * PSTEP_B;
constexpr int W_TICKET = 0, W_READY = 1, W_DEAD = 2, W_TURN = 3, W_SPARE = 4, W_DONE = 8, W_LANDED = 64, W_QUEUE = 128, W_LINES = 256;
constexpr unsigned Q_N = 128, Q_OUT = 1024, Q_LAST = 1u << 31;
static_assert(PL_EX + 8 * 1024 <= LDS_MB && W_QUEUE + Q_N <= W_LINES && W_LINES * 4 + 768 <= LDS_TMO - LDS_MB, "");

template <int N>
__device__ __forceinline__ void stream_kib(const char* src, unsigned dst, unsigned& cnt, unsigned word) {
  unsigned keep;
#define SK(OFF) "s_waitcnt vmcnt(%5)\n\tglobal_load_lds_dwordx4 %2, off offset:" #OFF " sc0 sc1\n\t" \
                "v_add_u32 %1, 1, %1\n\tds_max_u32 %4, %1\n\t"
  if (N == 4)
    asm volatile("s_mov_b32 %0, m0\n\ts_mov_b32 m0, %3\n\ts_nop 0\n\t" SK(0) SK(1024) SK(2048) SK(3072) "s_mov_b32 m0, %0"
                 : "=&s"(keep), "+v"(cnt) : "v"(src), "s"(dst), "v"(word), "n"(WIN - 1) : "memory");
  else if (N == 2)
    asm volatile("s_mov_b32 %0, m0\n\ts_mov_b32 m0, %3\n\ts_nop 0\n\t" SK(0) SK(1024) "s_mov_b32 m0, %0"
                 : "=&s"(keep), "+v"(cnt) : "v"(src), "s"(dst), "v"(word), "n"(WIN - 1) : "memory");
  else
    asm volatile("s_mov_b32 %0, m0\n\ts_mov_b32 m0, %3\n\ts_nop 0\n\t" SK(0) "s_mov_b32 m0, %0"
                 : "=&s"(keep), "+v"(cnt) : "v"(src), "s"(dst), "v"(word), "n"(WIN - 1) : "memory");
}
template <int K, class F>
__device__ __forceinline__ bool stream_drain(unsigned cnt, unsigned word, F&& more) {
  if constexpr (K >= 0) {
    asm volatile("s_waitcnt vmcnt(%2)\n\tds_max_u32 %1, %0" :: "v"(cnt + 1), "v"(word), "n"(K) : "memory");
    return more() || stream_drain<K - 1>(cnt + 1, word, more);
  }
  return false;
}

__device__ __forceinline__ int zip(int u, int s, int d, int g) {
  const int i = u - (s - d), j = i - g;
  return (unsigned)i >= (unsigned)(2 * d + g) ? u : j < 0 ? s + i : (j & 1 ? s + g : s - d) + (j >> 1);
}
constexpr int D2 = 12, G2 = 20, D4 = 40, G4 = 28, D5 = 52, DA = 40;
constexpr int DH = 48;
template <bool CROSS>
__device__ __forceinline__ int in_order(int u) {
  constexpr int HOP = 4 * KSTEPS + RELAY / 4 - 2 * HW - DH;
  constexpr int AWAY = 2 * KSTEPS + RELAY / 4 - 2 * HW;
  if (CROSS && u < AWAY) return u < RELAY / 4 ? u + 2 * KSTEPS - 2 * HW : u - RELAY / 4;
  if (CROSS) return u < HOP ? zip(u, AWAY + KSTEPS, DA, 0) : u < HOP + RELAY / 4 * 3 ? u + DH : u - RELAY / 4 * 3;
  if (u < 2 * KSTEPS - HX + D2 + G2) return zip(u, 2 * KSTEPS - HX, D2, G2);
  return u < 4 * KSTEPS - HX + D4 + G4 ? zip(u, 4 * KSTEPS - HX, D4, G4) : zip(u, 5 * KSTEPS - HX, D5, 0);
}
static_assert(DA + DH <= KSTEPS && D2 + G2 <= KSTEPS && D4 + G4 + D5 <= KSTEPS && D5 + RELAY <= KSTEPS, "");

struct Unit { unsigned src, raw, dst; int from, flag; };
template <bool CROSS>
__device__ __forceinline__ Unit input_unit(int u, unsigned call, int rank, int peer) {
  constexpr int H1 = RELAY / 4, RSTEP = 32 * KSTEPS;
  const unsigned par = own_idx(call, 0) * KSTEPS;
  Unit o;
  o.from = peer;
  u += CROSS ? 2 * HW : HX;
  if (!CROSS) {
    const int v = u < 5 * KSTEPS ? u : u + RELAY;
    o.raw = par + v; o.dst = (peer & 3) * 6 * KSTEPS + v; o.flag = v;
  } else if (u < 256) {
    const int v = (u & 1) * KSTEPS + (u >> 1);
    o.raw = par + (8 + (rank & 3) * 2) * KSTEPS + v; o.dst = (PB_FOREIGN + (peer & 3) * 2) * KSTEPS + v; o.flag = u >> 1;
  } else if (u < 256 + H1) {
    o.raw = par + 5 * KSTEPS + (u - 256) * 4 + (rank & 3); o.dst = ~((peer & 3) * 16 + u - 256); o.flag = RDY_HOP1 + u - 256;
  } else if (u < 512 + H1) {
    const int v = u - 256 - H1;
    o.raw = par + 6 * KSTEPS + v; o.dst = (PA_AWAY + (peer & 3) * 2) * KSTEPS + v; o.flag = RDY_AWAY + v;
  } else {
    const int v = u - 512 - H1, j = v / 3, s = (rank + 1 + v - j * 3) & 3, ks = 5 * KSTEPS + j * 4 + (peer & 3);
    o.from = (rank & 4) | s;
    o.raw = par + ks; o.dst = s * 6 * KSTEPS + ks; o.flag = RDY_HOP2 + s * 16 + j;
    o.src = RSTEP + s * 16 + j;
    return o;
  }
  o.src = o.raw;
  return o;
}

template <bool CROSS>
__device__ __forceinline__ void stream(const Params& p, int lane, unsigned call, int rank, int peer, LAS char* smem) {
  LAS unsigned* const sh = (LAS unsigned*)(smem + LDS_MB);
  const char* const pws = p.ws[peer];
  const unsigned lds0 = (unsigned)(uintptr_t)smem, word = (unsigned)(uintptr_t)(sh + W_LANDED + lane);
  unsigned cnt = 0;
  for (unsigned k = 0;; ++k) {
    unsigned ready, done, q;
    auto look = [&]() {
      const unsigned a = __hip_atomic_load(sh + W_READY, __ATOMIC_RELAXED, __HIP_MEMORY_SCOPE_WORKGROUP);
      cbar();
      const unsigned b = __hip_atomic_load(sh + W_DONE + (k & (RING - 1)), __ATOMIC_RELAXED, __HIP_MEMORY_SCOPE_WORKGROUP);
      const unsigned c = __hip_atomic_load(sh + W_QUEUE + (k & (Q_N - 1)), __ATOMIC_RELAXED, __HIP_MEMORY_SCOPE_WORKGROUP);
      cbar();
      ready = __builtin_amdgcn_readfirstlane(a);
      done = __builtin_amdgcn_readfirstlane(b);
      q = __builtin_amdgcn_readfirstlane(c);
      return k < (ready & ~Q_LAST) && done >= (k & ~(RING - 1u));
    };
    if (!look() && !stream_drain<WIN - 1>(cnt, word, look)) {
      while (!look() && (k | Q_LAST) != ready) __builtin_amdgcn_s_sleep(1);
      if ((k | Q_LAST) == ready) break;
    }
    const unsigned d = lds0 + (k & (RING - 1)) * PSTEP_B, u = q & 2047u;
    if (u < Q_OUT) {
      const char* s = pws + OFF_PACK + (size_t)input_unit<CROSS>(u, call, rank, peer).src * PSTEP_B + lane * 16;
      if ((unsigned)lane >> 3 <= ((q >> 11) & 7u)) stream_kib<1>(s, d, cnt, word);
      stream_kib<4>(s + 1024, d + 1024, cnt, word);
      stream_kib<4>(s + 5120, d + 5120, cnt, word);
      stream_kib<4>(s + 9216, d + 9216, cnt, word);
      stream_kib<1>(s + 13312, d + 13312, cnt, word);
    } else {
      const char* s = pws + OFF_STG + (size_t)((rank & 3) * OUT_CHUNKS + u - Q_OUT) * CHUNK_B + lane * 16;
      stream_kib<4>(s, d, cnt, word);
      stream_kib<2>(s + 4096, d + 4096, cnt, word);
      if (lane < 48) stream_kib<1>(s + 6144, d + 6144, cnt, word);
    }
  }
}

template <bool CROSS>
__device__ __forceinline__ void pull(const Params& p, int w, int wave, unsigned call, int rank, int peer, LAS char* smem) {
  constexpr int NIN = CROSS ? 512 + RELAY - 2 * HW : 768 - RELAY - HX, NOUT = CROSS ? OUT_CHUNKS / PW : 0;
  constexpr int SR = 3;
  const int lane = fresh_lane();
  char* const ws = p.mine;
  LAS unsigned* const sh = (LAS unsigned*)(smem + LDS_MB);
  float* const out = p.out + (size_t)((lane >> 4) * 4) * NSH + (peer & 3) * 256 + (lane & 15) * 4;
  auto where = [&](int c, int pc) {
    const int tile = c >> 3, bt = tile / 6, r = tile - bt * 6;
    return out + (size_t)(batch_row(peer, rank, bt, r >> 1) + ((c >> 2) & 1) * 64 + (c & 1) * 32 + (pc & 4) * 4 + (pc & 3)) * NSH
         + (r & 1) * 128 + ((c >> 1) & 1) * 64;
  };
  if (wave >= 6) __builtin_amdgcn_s_setprio(3);
  if (wave == 6) {
    const unsigned* const rdy = (const unsigned*)(ws + OFF_RDY) + peer * RDY_N * 32;
    unsigned* const tmo = (unsigned*)(ws + OFF_TMO);
    unsigned* const ticket = (unsigned*)(ws + OFF_TICKET) + peer * 32;
    unsigned long long t0 = __builtin_amdgcn_s_memrealtime();
    bool dead = false;
    unsigned k = 0, kib = 0, spin = 0;
    unsigned v;
    auto unset = [&](bool look, int fl) {
      v = ~0u;
      if (look && !dead)
        asm volatile("global_load_dword %0, %1, off sc0 sc1\n\ts_waitcnt vmcnt(0)" : "=v"(v) : "v"(rdy + fl * 32) : "memory");
      return __builtin_amdgcn_ballot_w64(v >> 6 < call);
    };
    auto idle = [&]() {
      if ((++spin & 63u) == 0u
          && (__builtin_amdgcn_readfirstlane(__hip_atomic_load(tmo, __ATOMIC_RELAXED, __HIP_MEMORY_SCOPE_AGENT)) == call
              || __builtin_amdgcn_s_memrealtime() - t0 > DEADLINE)) {
        if (lane == 0) {
          __hip_atomic_store(tmo, call, __ATOMIC_RELAXED, __HIP_MEMORY_SCOPE_AGENT);
          __hip_atomic_store(sh + W_DEAD, 1u, __ATOMIC_RELAXED, __HIP_MEMORY_SCOPE_WORKGROUP);
        }
        wait_vm0();
        dead = true;
      }
      __builtin_amdgcn_s_sleep(1);
    };
    auto publish = [&](unsigned v, unsigned near) {
      cbar();
      if (lane == 0) __hip_atomic_store(sh + W_READY, v, __ATOMIC_RELAXED, __HIP_MEMORY_SCOPE_WORKGROUP);
      while (lds_get(sh + W_LANDED) < near) __builtin_amdgcn_s_sleep(2);
      t0 = __builtin_amdgcn_s_memrealtime();
    };
    int known = 0;
    LAS unsigned char* const lines = (LAS unsigned char*)(sh + W_LINES);
    auto look = [&](int from, int width) {
      const int i = min(from + lane, NIN - 1), o = in_order<CROSS>(i);
      const unsigned long long miss = unset(lane < width, input_unit<CROSS>(o, call, rank, peer).flag);
      known = from + (miss ? (int)__builtin_ctzll(miss) : width);
      if (lane < width) lines[i] = v >> 6 == call ? (v >> (CROSS && o < 256 - 2 * HW ? (o & 1) * 3 : 0)) & 7u : 7u;
    };
    for (int u = w; u < NIN;) {
      while (u >= known) { look(u, 16); if (u >= known) idle(); }
      if (lane == 0) sh[W_QUEUE + (k & (Q_N - 1))] = kib << 14 | lines[u] << 11 | in_order<CROSS>(u);
      publish(++k, max(kib, 4u) - 4u);
      kib += 14;
      if (k < SR) { u += PW; continue; }
      unsigned tk = 0;
      if (lane == 0) tk = __hip_atomic_fetch_add(ticket, 1u, __ATOMIC_RELAXED, __HIP_MEMORY_SCOPE_AGENT);
      if (known < NIN && known - u < 32) look(known, 64);
      u = min(__builtin_amdgcn_readfirstlane(tk) - (call - 1u) * (NIN - (SR - 1) * PW) + SR * PW, (unsigned)NIN);
    }
    if (CROSS && lane == 0) __hip_atomic_store(sh + W_TURN, ~(k * 8u), __ATOMIC_RELAXED, __HIP_MEMORY_SCOPE_WORKGROUP);
    const unsigned long long all = (1ull << NOUT) - 1ull;
    for (unsigned long long left = all; left;) {
      const unsigned long long fresh = left & ~unset((left >> lane) & 1, RDY_OUT + unit_of(lane, w));
      if (!fresh) { idle(); continue; }
      const unsigned pos = __builtin_amdgcn_mbcnt_hi(fresh >> 32, __builtin_amdgcn_mbcnt_lo(fresh, 0));
      if ((fresh >> lane) & 1) sh[W_QUEUE + ((k + pos) & (Q_N - 1))] = (kib + pos * 7) << 14 | (Q_OUT + unit_of(lane, w));
      k += __builtin_popcountll(fresh);
      kib += __builtin_popcountll(fresh) * 7;
      left &= ~fresh;
      if (left) publish(k, kib - 14);
    }
    publish(k | Q_LAST, 0);
    if (CROSS) {
      dead = __builtin_amdgcn_readfirstlane(__hip_atomic_load(tmo, __ATOMIC_RELAXED, __HIP_MEMORY_SCOPE_AGENT)) == call;
      wg_barrier();
      if (dead) store_sys(where(unit_of(0, w), 0), (u32x4){0x7fc00000u, 0x7fc00000u, 0x7fc00000u, 0x7fc00000u});
    }
    return;
  }
  if (wave == 7) {
    stream<CROSS>(p, lane, call, rank, peer, smem);
    if (CROSS) wg_barrier();
    return;
  }
  static_assert(RING == 8 && Q_N == 128, "");
  const unsigned sh0 = (unsigned)(uintptr_t)sh, none = ~0u;
  const unsigned aq = sh0 + W_QUEUE * 4 + lane * 8, ah = (unsigned)(uintptr_t)smem + (lane >> 3) * PSTEP_B + (lane & 7) * 4;
  u32x4 st;
  u32x2 queue;
  unsigned count, heads;
  auto look = [&](int word) {
    unsigned r;
    unsigned long long keep;
    asm volatile("s_mov_b64 %5, exec\n\ts_mov_b64 exec, 1\n\tds_add_rtn_u32 %0, %6, %7\n\ts_mov_b64 exec, %5\n\t"
                 "ds_read_b128 %1, %8\n\tds_read_b64 %2, %9\n\tds_read_b32 %3, %8 offset:%11\n\tds_read_b32 %4, %10\n\t"
                 "s_waitcnt lgkmcnt(0)"
                 : "=&v"(r), "=&v"(st), "=&v"(queue), "=&v"(count), "=&v"(heads), "=&s"(keep)
                 : "v"(sh0 + word * 4), "v"(1u), "v"(sh0), "v"(aq), "v"(ah), "n"(W_LANDED * 4) : "memory");
    return __builtin_amdgcn_readfirstlane(r);
  };
  unsigned t, u, q;
  auto landed = [&](unsigned tk) {
    const unsigned ready = __builtin_amdgcn_readfirstlane(st[W_READY]);
    t = tk;
    if (CROSS) {
      const unsigned turn = ~__builtin_amdgcn_readfirstlane(st[W_TURN]);
      if (tk > turn) t = turn + (tk - turn) * 4u;
    }
    const unsigned q0 = __builtin_amdgcn_readlane(queue[0], (t >> 4) & 63), q1 = __builtin_amdgcn_readlane(queue[1], (t >> 4) & 63);
    q = t & 8 ? q1 : q0;
    const bool there = (t >> 3) < (ready & ~Q_LAST);
    u = there ? q & 2047u : none;
    const unsigned need = (q >> 14) + (!CROSS || u < Q_OUT ? ((t & 7) * 13 + 28) >> 3 : t & 4 ? 7 : 4) + WIN;
    return there ? (unsigned)__builtin_amdgcn_readfirstlane(count) >= need : ready >= Q_LAST;
  };
  unsigned prev = none, pdst = 0, plines = 0;
  auto done = [&]() {
    const bool last = (look(prev != none ? W_DONE + ((prev >> 3) & (RING - 1)) : W_SPARE) & 7u) == 7u && prev != none;
    prev = none;
    if (!last) return;
    if (!CROSS || (int)pdst >= 0) { if (lane < FCOPY) set_local(ws, lane, pdst, call); }
    else if (lane < 3) set_rdy(p.ws[((peer & 4) | ((peer + 1 + lane) & 3)) & 7], rank, RDY_HOP2 + ~pdst, call, plines);
  };
  for (;;) {
    const unsigned tk = look(W_TICKET);
    if (!landed(tk)) {
      if (prev != none) wait_vm0();
      for (done(); !landed(tk); look(W_SPARE)) __builtin_amdgcn_s_sleep(1);
    }
    const LAS char* slot = smem + ((t >> 3) & (RING - 1)) * PSTEP_B;
    if (u >= Q_OUT) {
      if (prev != none) { wait_vm0(); done(); }
      if (!CROSS || u == none) break;
      const unsigned half = (t >> 2) & 1u;
      const bool dead = __builtin_amdgcn_readfirstlane(st[W_DEAD]);
      f32x4 y[4];
      unpack_sums(lane, slot + half * SUM_B, smem + PL_EX + wave * 1024, smem + LDS_LUT,
                  p.ws[peer] + OFF_SPARE + (size_t)(((rank & 3) * OUT_CHUNKS + u - Q_OUT) * 2 + half) * 1024, y);
      if (lane == 0) __hip_atomic_fetch_add(sh + W_DONE + ((t >> 3) & (RING - 1)), 4u, __ATOMIC_RELAXED, __HIP_MEMORY_SCOPE_WORKGROUP);
      float* const o = where(u - Q_OUT, t & 4);
#pragma unroll
      for (int r = 0; r < 4; ++r) {
        u32x4 v = __builtin_bit_cast(u32x4, y[r] * sigmoid(y[r]));
        if (dead) v = (u32x4){0x7fc00000u, 0x7fc00000u, 0x7fc00000u, 0x7fc00000u};
        store_out(o + r * NSH, v);
      }
      continue;
    }
    const Unit un = input_unit<CROSS>(u, call, rank, peer);
    u32x4 f0, f1;
    char* d;
    if (CROSS && (int)un.dst < 0) {
      const int l1 = min(lane, 47);
      const LAS char* blk = slot + (t & 7) * (PSTEP_B / 8);
      f0 = *(const LAS u32x4*)(blk + lane * 16);
      f1 = *(const LAS u32x4*)(blk + 1024 + l1 * 16);
      d = ws + OFF_RELAY + (size_t)(~un.dst * 8 + (t & 7)) * (PSTEP_B / 8);
      store_sys(d + lane * 16, f0);
      store_sys(d + 1024 + l1 * 16, f1);
    } else {
      const unsigned hdr = __builtin_amdgcn_readlane(heads, t & 63);
      if ((int)hdr >= 0) {
        unpack_block(lane, slot, t & 7, hdr, smem + PL_EX + wave * 1024, smem + LDS_LUT, f0, f1);
      } else {
        const char* r = p.ws[un.from & 7] + OFF_OWN + (size_t)(un.raw * 8 + (t & 7)) * 2048 + lane * 16;
        asm volatile("global_load_dwordx4 %0, %2, off sc0 sc1\n\t"
                     "global_load_dwordx4 %1, %2, off offset:1024 sc0 sc1\n\ts_waitcnt vmcnt(0)"
                     : "=&v"(f0), "=&v"(f1) : "v"(r) : "memory");
      }
      d = ws + pulled(call) + (size_t)(un.dst * 8 + (t & 7)) * 2048 + lane * 16;
      store_sys(d, f0);
      store_sys(d + 1024, f1);
    }
    asm volatile("s_waitcnt vmcnt(2)" ::: "memory");
    done();
    prev = t;
    pdst = un.dst;
    plines = (q >> 11) & 7u;
    if (!landed(__builtin_amdgcn_readfirstlane(st[W_TICKET]))) { wait_vm0(); done(); }
  }
  if (CROSS) { wait_vm0(); wg_barrier(); }
}

struct Job { int pa, pb, ah, bh, tile; };
__device__ __forceinline__ void decode_job(unsigned code, int a, Job& o) {
  const int ra = code & 63;
  int bt, i;
  o.pb = PB + ((code >> 6) & 15);
  o.ah = (code >> 10) & 1;
  o.bh = (code >> 11) & 1;
  if (ra < 18) {
    i = ra / 6; bt = 2 + ra % 6; o.pa = ((a + 1 + i) & 3) * 6 + bt - 2;
  } else if (ra < 24) {
    o.pa = a * 6 + ra - 18; bt = (ra - 18) / 3; i = (ra - 18) % 3;
  } else {
    o.pa = ra; bt = 8 + (ra & 1); i = (((ra - PA_AWAY) >> 1) - ((o.pb - PB_FOREIGN) >> 1) - 1) & 3;
  }
  o.tile = (int)((BATCH_POS >> (4 * bt)) & 15) * 3 + i;
}

template <int NB>
__device__ __forceinline__ unsigned next_stage(unsigned so) { return so == LDS_MB - (NB + 1) * 8192 ? 0u : so + (NB + 1) * 8192; }

template <bool TALL, int NB>
__device__ __forceinline__ void mfma_wave(const Params& p, LAS char* smem, const Job& jb, int job, int wave,
                                          int rank, unsigned call) {
  constexpr int MT = TALL ? 4 : 1, NT = 4 * NB, KH = TALL ? 1 : 2, STEPS = 2 * KSTEPS / KH;
  constexpr int NRA = MT * KH, NRB = NT * KH, NRD = NRA + NRB, NMM = MT * NT * KH;
  int lane = fresh_lane();
  const int wr = MT == 1 ? wave : wave >> 1, wc = MT == 1 ? 0 : wave & 1;
  f32x4 acc[MT][NT];
#pragma unroll
  for (int m = 0; m < MT; ++m)
#pragma unroll
    for (int n = 0; n < NT; ++n) acc[m][n] = (f32x4){0.f, 0.f, 0.f, 0.f};
  half8 fa[2][NRA], fb[2][NRB];
  const LAS char* la = smem + wr * MT * 1024 + lane * 16;
  const LAS char* lb = smem + 8192 + wc * NRB * 1024 + lane * 16;
  unsigned so = 0;
#define READ_SET(X) do { \
    _Pragma("unroll") for (int i = 0; i < NRB; ++i) { \
      if (i < NRA) fa[X][i] = *(const LAS half8*)(la + so + (i / MT) * 4096 + (i % MT) * 1024); \
      fb[X][i] = *(const LAS half8*)(lb + so + i * 1024); } \
    so = next_stage<NB>(so); } while (0)
#define MMA_SET(X) do { \
    _Pragma("unroll") for (int kh = 0; kh < KH; ++kh) \
    _Pragma("unroll") for (int m = 0; m < MT; ++m) \
    _Pragma("unroll") for (int n = 0; n < NT; ++n) \
      acc[m][n] = __builtin_amdgcn_mfma_f32_16x16x32_f16(fa[X][kh * MT + m], fb[X][kh * NT + n], acc[m][n], 0, 0, 0); } while (0)
  constexpr int RG = TALL ? 1 : 2, MG = NB, G = NRD / RG;
#define INTERLEAVE() do { \
    _Pragma("unroll") for (int i = 0; i < G; ++i) { \
      __builtin_amdgcn_sched_group_barrier(0x100, RG, 0); \
      __builtin_amdgcn_sched_group_barrier(0x008, MG, 0); } \
    __builtin_amdgcn_sched_group_barrier(0x008, NMM - G * MG, 0); } while (0)
#define STEP_END() do { __builtin_amdgcn_sched_barrier(0); asm volatile("" ::: "memory"); \
    __builtin_amdgcn_s_waitcnt(0xC07F);   __builtin_amdgcn_sched_barrier(0); \
    wg_barrier(); __builtin_amdgcn_sched_barrier(0); } while (0)
  wg_barrier();
  READ_SET(0);
  STEP_END();
  for (int t = 1; t < STEPS - 1; t += 2) {
    READ_SET(1); MMA_SET(0); INTERLEAVE();
    STEP_END();
    READ_SET(0); MMA_SET(1); INTERLEAVE();
    STEP_END();
  }
  READ_SET(1); MMA_SET(0); INTERLEAVE();
  const unsigned tm = *(const LAS unsigned*)(smem + LDS_TMO + job * 16);
  STEP_END();
  MMA_SET(1);
  __builtin_amdgcn_sched_barrier(0);
  lane = fresh_lane();
  asm volatile("" : "+s"(call));
  if (__builtin_amdgcn_readfirstlane(tm) == call) {
#pragma unroll
    for (int i = 0; i < MT * NT; ++i) {
      acc[i / NT][i % NT] = __builtin_bit_cast(f32x4, (u32x4){~0u, ~0u, ~0u, ~0u});
      asm volatile("" : "+v"(acc[i / NT][i % NT]));
    }
  }

  const bool foreign = jb.pb >= PB_FOREIGN;
  const int ob = (jb.pb - PB_FOREIGN) >> 1;
  const int r16 = jb.ah * 4 + wr * MT;
  const int c64 = (jb.pb & 1) * 2 + jb.bh + wc * (NT / 4);
  const int chunk = (jb.tile * 2 + (c64 >> 1)) * 8 + (r16 & 4) + (c64 & 1) * 2 + ((r16 >> 1) & 1);
  if (foreign) {
    const unsigned blk = (ob * OUT_CHUNKS + chunk) * 2 + (r16 & 1);
    unsigned off = ~0u;
#pragma unroll 1
    for (int i = 0; i < MT * NT / 4; ++i) {
      u32x4 v[4];
      unsigned rot = 31;
      asm volatile("" : "=v"(v[0]), "=v"(v[1]), "=v"(v[2]), "=v"(v[3]), "+s"(rot));
#pragma unroll
      for (int j = 0; j < MT * NT / 4; ++j)
        if (i == j) {
#pragma unroll
          for (int n = 0; n < 16; ++n) {
            const unsigned y = __builtin_bit_cast(u32x4, acc[j % MT][j / MT * 4 + (n >> 2)])[n & 3];
            v[n >> 2][n & 3] = __builtin_amdgcn_alignbit(y, y, rot);
          }
          asm volatile("" : "+v"(v[0]), "+v"(v[1]), "+v"(v[2]), "+v"(v[3]));
        }
      pack_sums(lane, v, off, smem + LDS_EPI + wave * 512, smem + LDS_LUT,
                p.mine + OFF_STG + (size_t)((blk + i) >> 1) * CHUNK_B + ((blk + i) & 1) * SUM_B, p.mine + OFF_SPARE + (size_t)(blk + i) * 1024);
    }
  } else {
    const int pa = jb.pa;
    const int row0 = pa < PA_AWAY ? row_of((rank & 4) + pa / 6) + (pa % 6) * 128
                   : row_of(pa < PA_OWNAWAY ? ((rank & 4) ^ 4) + ((pa - PA_AWAY) >> 1) : rank) + 768 + (pa & 1) * 128;
    float* o = p.out + (size_t)(row0 + r16 * 16 + (lane >> 4) * 4) * NSH + ((jb.pb - PB) >> 1) * 256 + c64 * 64 + (lane & 15) * 4;
#pragma unroll
    for (int h = 0; h < NT / 4; ++h)
#pragma unroll
      for (int m = 0; m < MT; ++m) {
        const f32x4* const y = &acc[m][h * 4];
        const f32x4 g[4] = {sigmoid(y[0]), sigmoid(y[1]), sigmoid(y[2]), sigmoid(y[3])};
#pragma unroll
        for (int r = 0; r < 4; ++r) {
          float v[4];
#pragma unroll
          for (int i = 0; i < 4; ++i) {
            v[i] = y[i][r] * g[i][r];
            asm("" : "+v"(v[i]));
          }
          store_out(o + h * 64 + (m * 16 + r) * NSH, __builtin_bit_cast(u32x4, (f32x4){v[0], v[1], v[2], v[3]}));
        }
      }
  }
  if (foreign) {
    wait_vm0();
    bool last = lane < (MT * NT + 7) / 8;
    if (MT == 1)
      last = last && __hip_atomic_fetch_add((LAS unsigned*)(smem + LDS_TMO) + job * 4 + 2 + (wave >> 1), 1u,
                                            __ATOMIC_RELAXED, __HIP_MEMORY_SCOPE_WORKGROUP) == 1u;
    if (last) set_rdy(p.ws[(((rank & 4) ^ 4) | ob) & 7], rank, RDY_OUT + chunk + lane, call, 0);
  }
}

template <bool TALL, int NB>
__device__ __forceinline__ void load_wave(const Params& p, LAS char* smem, const Job& jb, int job, int l, int rank, unsigned call) {
  constexpr int DEPTH = NB == 2 ? 5 : 7;
  constexpr int ksh = TALL ? 1 : 0, steps = KSTEPS << ksh;
  if (l > NB && l < 3) {
    for (int t = 0; t <= steps; ++t) wg_barrier();
    return;
  }
  const int lane = fresh_lane(), a = rank & 3;
  char* const ws = p.mine;
  const unsigned lds0 = (unsigned)(uintptr_t)smem;
  LAS unsigned* const rdyw = (LAS unsigned*)(smem + LDS_TMO) + job * 4 + 1;
  auto panel_of = [&](int pc) { return pc == 0 ? jb.pa : jb.pb + pc - 1; };
  int ready = 0;
  if (l == 3) {
    asm volatile("" : "+s"(call));
    unsigned* const tmo = (unsigned*)(ws + OFF_TMO);
    const char* fsrc = ws + OFF_FLAG + (blockIdx.x & (FCOPY - 1)) * 65536 + panel_of(lane >> 5) * 1024 + ((lane >> 2) & 7) * 128 + (lane & 3) * 16;
    const unsigned mb = lds0 + LDS_MB + job * 2048;
    int ra = 0, rb = 0;
    auto poll = [&]() {
      if ((lane < 32 ? ra : rb) < KSTEPS) {
        unsigned keep;
        asm volatile("s_mov_b32 %0, m0\n\ts_mov_b32 m0, %2\n\ts_nop 0\n\t"
                     "global_load_lds_dwordx4 %1, off sc0 sc1\n\ts_mov_b32 m0, %0"
                     : "=&s"(keep) : "v"(fsrc), "s"(mb) : "memory");
        if (NB == 2 && lane >= 32)
          asm volatile("s_mov_b32 %0, m0\n\ts_mov_b32 m0, %2\n\ts_nop 0\n\t"
                       "global_load_lds_dwordx4 %1, off offset:1024 sc0 sc1\n\ts_mov_b32 m0, %0"
                       : "=&s"(keep) : "v"(fsrc), "s"(mb - 512) : "memory");
      }
    };
    auto refresh = [&]() {
      u32x2 f, g, h;
      asm volatile("ds_read2_b32 %0, %3 offset1:8\n\tds_read2_b32 %1, %3 offset0:128 offset1:136\n\t"
                   "ds_read2_b32 %2, %4 offset1:8\n\ts_waitcnt lgkmcnt(0)"
                   : "=&v"(f), "=&v"(g), "=&v"(h)
                   : "v"(mb + (lane & 7) * 64 + (lane >> 3) * 4), "v"(mb + (NB == 2 ? 1024 : 512) + (lane & 7) * 64 + (lane >> 3) * 4)
                   : "memory");
      const unsigned long long a0 = __builtin_amdgcn_ballot_w64(f[0] < call), a1 = __builtin_amdgcn_ballot_w64(f[1] < call);
      const unsigned long long b0 = __builtin_amdgcn_ballot_w64(min(g[0], h[0]) < call), b1 = __builtin_amdgcn_ballot_w64(min(g[1], h[1]) < call);
      ra = a0 ? __builtin_ctzll(a0) : a1 ? 64 + __builtin_ctzll(a1) : KSTEPS;
      rb = b0 ? __builtin_ctzll(b0) : b1 ? 64 + __builtin_ctzll(b1) : KSTEPS;
      ready = min(ra, rb);
      if (lane == 0) __hip_atomic_store(rdyw, (unsigned)ready, __ATOMIC_RELAXED, __HIP_MEMORY_SCOPE_WORKGROUP);
    };
    auto advance = [&]() {
      const int k = min(ready + lane, KSTEPS - 1);
      u32x2 f;
      unsigned h;
      asm volatile("ds_read2_b32 %0, %2 offset1:128\n\tds_read_b32 %1, %3\n\ts_waitcnt lgkmcnt(0)"
                   : "=&v"(f), "=&v"(h)
                   : "v"(mb + (k & 7) * 64 + (k >> 3) * 4), "v"(mb + (NB == 2 ? 1024 : 512) + (k & 7) * 64 + (k >> 3) * 4)
                   : "memory");
      const unsigned long long m = __builtin_amdgcn_ballot_w64(min(min(f[0], f[1]), h) < call);
      ready = min(ready + (m ? (int)__builtin_ctzll(m) : 64), KSTEPS);
      __hip_atomic_store(rdyw, (unsigned)ready, __ATOMIC_RELAXED, __HIP_MEMORY_SCOPE_WORKGROUP);
    };
    for (int s = 0; s < steps; ++s) {
      if (s >= DEPTH) wg_barrier();
      const int ks = s >> ksh;
      if (ready < KSTEPS && ready - ks < 4) {
        advance();
        if (ks >= ready) {
          const unsigned long long t0 = __builtin_amdgcn_s_memrealtime();
          for (unsigned it = 1;; ++it) {
            if (it >= 32u || s == 0) { poll(); wait_vm0(); refresh(); }
            else { if (it & 1u) poll(); advance(); }
            if (ks < ready) break;
            if ((it & 63u) == 0u) {
              if (__builtin_amdgcn_readfirstlane(__hip_atomic_load(tmo, __ATOMIC_RELAXED, __HIP_MEMORY_SCOPE_AGENT)) == call
                  || __builtin_amdgcn_s_memrealtime() - t0 > DEADLINE) {
                if (lane == 0) __hip_atomic_store(tmo, call, __ATOMIC_RELAXED, __HIP_MEMORY_SCOPE_AGENT);
                wait_vm0();
                ready = KSTEPS;
                if (lane == 0) __hip_atomic_store(rdyw, (unsigned)ready, __ATOMIC_RELAXED, __HIP_MEMORY_SCOPE_WORKGROUP);
                break;
              }
            }
            if (it >= 32u) __builtin_amdgcn_s_sleep(2); else __builtin_amdgcn_s_sleep(1);
          }
        } else if (!(s & ksh)) {
          poll();
        }
      }
    }
    if (lane == 0) {
      unsigned keep;
      asm volatile("s_mov_b32 %0, m0\n\ts_mov_b32 m0, %2\n\ts_nop 0\n\t"
                   "global_load_lds_dword %1, off sc1\n\ts_mov_b32 m0, %0"
                   : "=&s"(keep) : "v"(tmo), "s"(lds0 + LDS_TMO + job * 16) : "memory");
    }
    for (int t = 1; t < DEPTH; ++t) wg_barrier();
    wait_vm0();
    wg_barrier();
    wg_barrier();
    return;
  }
  const int panel = panel_of(l);
  const int own = panel >= PB_FOREIGN ? -1 : panel >= PA_OWNAWAY ? panel - PA_OWNAWAY + 6
                : (unsigned)(panel - a * 6) < 6u ? panel - a * 6 : -1;
  const char* src = ws + (own >= 0 ? OFF_OWN + own_idx(call, own) * PANEL_B : pulled(call) + (size_t)panel * PANEL_B)
                  + (TALL ? 0 : (l == 0 ? jb.ah : jb.bh) * 8192) + lane * 16;
  constexpr unsigned gap = TALL ? 8192 : 4096;
  const unsigned dst0 = lds0 + l * 8192;
  unsigned so = 0;
  for (int s = 0; s < steps; ++s) {
    if (s >= DEPTH) {
      asm volatile("s_waitcnt vmcnt(%0)" :: "n"(8 * (DEPTH - 1)) : "memory");
      wg_barrier();
    }
    const int ks = s >> ksh;
    if (ks >= ready)
      while ((ready = lds_get(rdyw)) <= ks) __builtin_amdgcn_s_sleep(1);
    const char* g = src + (size_t)ks * STEP_B + (s & ksh) * 4096;
    const unsigned d = dst0 + so;
    so = next_stage<NB>(so);
    unsigned keep;
#define GLDS4(R) "global_load_lds_dwordx4 " R ", off sc0 sc1\n\t" \
                 "global_load_lds_dwordx4 " R ", off offset:1024 sc0 sc1\n\t" \
                 "global_load_lds_dwordx4 " R ", off offset:2048 sc0 sc1\n\t" \
                 "global_load_lds_dwordx4 " R ", off offset:3072 sc0 sc1\n\t"
    asm volatile("s_mov_b32 %0, m0\n\ts_mov_b32 m0, %3\n\ts_nop 0\n\t" GLDS4("%1")
                 "s_mov_b32 m0, %4\n\ts_nop 0\n\t" GLDS4("%2") "s_mov_b32 m0, %0"
                 : "=&s"(keep) : "v"(g), "v"(g + gap), "s"(d), "s"(d + 4096) : "memory");
  }
#define WB(N) "s_waitcnt vmcnt(" #N ")\n\ts_barrier\n\t"
  if (NB == 1) asm volatile(WB(48) WB(40) ::: "memory");
  asm volatile(WB(32) WB(24) WB(16) WB(8) WB(0) "s_barrier" ::: "memory");
}

__global__ void __launch_bounds__(NTHREADS, 2) ag_gemm_silu(const Params p) {
  __shared__ __attribute__((aligned(16))) char smem[LDS_BYTES];
  const int xcd = (xcc_id() - 4) & 7, slot = blockIdx.x >> 3;
  u32x4 plan;
  {
    unsigned l0, l1;
    asm volatile("s_load_dword %1, %3, 0\n\ts_load_dword %2, %3, 64\n\ts_load_dwordx4 %0, %3, %4\n\ts_waitcnt lgkmcnt(0)"
                 : "=&s"(plan), "=&s"(l0), "=&s"(l1)
                 : "s"(__builtin_amdgcn_kernarg_segment_ptr()),
                   "s"((unsigned)(offsetof(Params, plan) + 16 * (slot < GSLOTS ? slot * 8 + xcd : GSLOTS * 8 + slot - GSLOTS))) : "memory");
  }
  const int rank = p.rank & 7, a = rank & 3;
  const int wave = __builtin_amdgcn_readfirstlane(threadIdx.x >> 6);
  if (threadIdx.x == 0)
    asm volatile("s_mov_b32 m0, %1\n\ts_nop 0\n\tglobal_load_lds_dword %0, off sc1"
                 :: "v"(call_word(p)), "s"((unsigned)(uintptr_t)(LAS char*)smem + LDS_CALL) : "memory");
  const bool puller = slot >= GSLOTS && xcd < 7;
  unsigned call;
  if (!puller && (plan[0] & 0x4000u)) call = convert_x(p, plan[0], p.counts, rank, (LAS char*)smem);
  else if (!puller && (plan[0] & 0x8000u)) call = convert_w(p, plan[0], p.counts, rank, (LAS char*)smem);
  else call = begin<0>(p, (LAS char*)smem, puller);
  if (threadIdx.x == 0) __hip_atomic_store(call_word(p), call, __ATOMIC_RELAXED, __HIP_MEMORY_SCOPE_AGENT);
  __builtin_amdgcn_s_waitcnt(0x0F70);

  if (puller) {
    const int w = slot - GSLOTS;
    if (xcd < 4) pull<true>(p, w, wave, call, rank, ((rank & 4) ^ 4) | ((a + xcd) & 3), (LAS char*)smem);
    else pull<false>(p, w, wave, call, rank, (rank & 4) | ((a + xcd - 3) & 3), (LAS char*)smem);
    return;
  }

  for (int job = 0; job < NJOBS; ++job) {
    const unsigned code = job == 0 ? plan[1] : job == 1 ? plan[2] : plan[3];
    if (!(code & 0x8000u)) break;
    Job jb;
    decode_job(code, a, jb);
    const int kind = (code >> 12) & 3;
    if (wave >= 4) {
      if (kind == 0) load_wave<true, 2>(p, (LAS char*)smem, jb, job, wave - 4, rank, call);
      else if (kind == 1) load_wave<true, 1>(p, (LAS char*)smem, jb, job, wave - 4, rank, call);
      else load_wave<false, 1>(p, (LAS char*)smem, jb, job, wave - 4, rank, call);
    } else if (kind == 0) {
      mfma_wave<true, 2>(p, (LAS char*)smem, jb, job, wave, rank, call);
    } else if (kind == 1) {
      mfma_wave<true, 1>(p, (LAS char*)smem, jb, job, wave, rank, call);
    } else {
      mfma_wave<false, 1>(p, (LAS char*)smem, jb, job, wave, rank, call);
    }
  }
}

void make_schedule(Params& p) {
  int n[NGEMM] = {};
  for (int g = 0; g < NGEMM; ++g) p.plan[g] = (u32x4){0u, 0u, 0u, 0u};
  auto id = [](int xcd, int s) { return s < GSLOTS ? s * 8 + xcd : GSLOTS * 8 + s - GSLOTS; };
  auto put = [&](int xcd, int s, int kind, int ra, int b, int ah = 0, int bh = 0) {
    const int g = id(xcd, s);
    if (n[g] < NJOBS) p.plan[g][++n[g]] = 0x8000 | (kind << 12) | (bh << 11) | (ah << 10) | (b << 6) | ra;
  };
  auto P = [&](int xcd, int s0, int kind, int t, int b0) {
    for (int s = 0; s < 12; ++s) {
      if (kind < 2) put(xcd, s0 + s, kind, s % 3 * 6 + t, b0 + s / 3 * (2 - kind));
      else put(xcd, s0 + s, kind, s % 3 * 6 + t, b0, (s / 3) & 1, s / 6);
    }
  };
  auto L = [&](int xcd, int h, int b0) { for (int s = 0; s < 12; ++s) put(xcd, s, 0, 18 + h * 3 + s % 3, b0 + s / 3 * 2); };
  auto Af = [&](int xcd, int s0, int t, int h) {
    for (int s = h * 12; s < h * 12 + 12; ++s) put(xcd, s0 + s % 12, 1, PA_AWAY + ((s / 6 + 1 + s % 3) & 3) * 2 + t, 8 + s / 3);
  };
  P(0, 0, 0, 0, 0); P(1, 0, 0, 0, 8); L(2, 0, 8); L(3, 1, 8); L(4, 0, 0); L(5, 1, 0);
  for (int s = 0; s < 16; ++s) put(6 + s / 8, s % 8, 1, PA_OWNAWAY + (s & 1), s / 2);
  P(5, 12, 0, 1, 0); P(6, 12, 0, 1, 8);
  P(0, 12, 1, 2, 0); P(1, 12, 1, 2, 4); Af(2, 12, 0, 0); Af(3, 12, 0, 1); P(4, 12, 0, 2, 8);
  for (int s = 0; s < 16; ++s) put(7, 12 + s, 0, PA_AWAY + (s & 3) * 2, s / 4 * 2);
  P(0, 0, 1, 3, 8); P(1, 0, 1, 3, 12); Af(2, 0, 1, 0); Af(3, 0, 1, 1); P(4, 0, 1, 3, 0); P(5, 0, 1, 3, 4);
  for (int s = 0; s < 28; ++s) put(s < 12 ? 6 : 7, s < 24 ? s % 12 : 4 + s, 1, PA_AWAY + (s & 3) * 2 + 1, s / 4);
  for (int s = 0; s < 16; ++s) put(s < 12 ? 4 : 7, s < 12 ? 12 + s : 16 + s, 2, PA_AWAY + s / 4 * 2 + 1, 7, s & 1, (s >> 1) & 1);
  for (int x = 0; x < 4; ++x) P(x, 12, 1, 4, (8 + 4 * x) & 15);
  P(5, 12, 1, 5, 0); P(6, 12, 1, 5, 4);
  for (int x = 0; x < 8; ++x) P(x, 0, 2, 5, 8 + x);
  constexpr int CVT_W = 56;
  int nx = 0, nw = 0, c = 0;
  for (int pass = 0; pass < 3; ++pass) {
    for (int g = 0; g < NGEMM; ++g) {
      const int xcd = g & 7, s = g >> 3, ra = p.plan[g][1] & 63;
      if (xcd < 6 && s < 12) {
        if (xcd < 4 && s < HW) p.plan[g][0] = 0x9000u | (s * 4 + ((p.rank + xcd) & 3));
        if (xcd >= 4 && s * 2 + xcd - 4 < HX) p.plan[g][0] = 0x5000u | (s * 2 + xcd - 4);
        continue;
      }
      if (pass != (ra >= PA_OWNAWAY || (ra < 18 && ra % 6 == 1) ? 2 : ra > PA_AWAY && (ra & 1) ? 0 : 1)) continue;
      if (pass == 1 && c++ * CVT_W % 76 < CVT_W) p.plan[g][0] = 0x8000u;
      else p.plan[g][0] = 0x4000u | nx++;
    }
    p.counts = (pass ? p.counts << 8 : 0u) | nx;
  }
  for (int pass = 0, k = 0; pass < 2; ++pass)
    for (int g = 0; g < NGEMM; ++g)
      if ((p.plan[g][0] & 0x5000u) == 0x4000u && (g >= GSLOTS * 8 || (g & 7) >= 6) == !pass) p.plan[g][0] |= k++ << 16;
  for (int pass = 0; pass < 3; ++pass)
    for (int g = 0; g < NGEMM; ++g)
      if (p.plan[g][0] == 0x8000u && pass == (g >= GSLOTS * 8 || (g & 7) >= 6 ? 0 : (g & 7) < 4 ? 1 : 2)) p.plan[g][0] |= 4 * HW + nw++;
  static_assert(HW <= 12 && HX <= 24, "");
  p.counts |= nw << 24;
}

}

extern "C" void kernel_launch(int rank, int nranks, ncclComm_t comm, hipStream_t stream,
                              void* const* d_in, const int* in_sizes, int n_in,
                              void* d_out, int out_size, const int* in_dtypes, int out_dtype,
                              void* const* d_out_all, void* const* d_ws_all, size_t ws_bytes) {
  (void)comm; (void)in_dtypes; (void)out_dtype; (void)d_out_all;
  if (nranks != NR || (unsigned)rank >= (unsigned)NR || n_in != 2 || in_sizes[0] != NSH * KDIM ||
      in_sizes[1] != KDIM * NSH || out_size != KDIM * NSH || ws_bytes < WS_NEED) {
    fprintf(stderr, "kernel_launch: unexpected problem shape\n");
    return;
  }
  Params p;
  p.x = (const float*)d_in[0];
  p.w = (const float*)d_in[1];
  p.rank = 0;
  for (int i = 0; i < NR; ++i) {
    const int r = (RANK_AT >> (4 * i)) & 7;
    p.ws[i] = (char*)d_ws_all[r];
    if (r == rank) p.rank = i;
  }
  p.mine = (char*)d_ws_all[rank];
  p.out = (float*)d_out;
  make_schedule(p);
  ag_gemm_silu<<<dim3(NWG), dim3(NTHREADS), 0, stream>>>(p);
  HIP_CHECK(hipGetLastError());
}
```

```cpp
#include <hip/hip_runtime.h>
#include <rccl/rccl.h>
#include <cstdio>
#include <cstdint>
#include <cstddef>

#define HIP_CHECK(expr) do { hipError_t _e = (expr); if (_e != hipSuccess) { \
    fprintf(stderr, "HIP error %s at %s:%d\n", hipGetErrorString(_e), __FILE__, __LINE__); \
    return; } } while (0)

namespace {

typedef _Float16 half8 __attribute__((ext_vector_type(8)));
typedef float f32x4 __attribute__((ext_vector_type(4)));
typedef unsigned u32x4 __attribute__((ext_vector_type(4)));
typedef unsigned u32x2 __attribute__((ext_vector_type(2)));
#define LAS __attribute__((address_space(3)))
#define GAS __attribute__((address_space(1)))

constexpr int NR = 8;
constexpr unsigned RANK_AT = 0x75416320u;
constexpr int KDIM = 8192;
constexpr int NSH = 1024;
constexpr int KSTEPS = 128;
constexpr unsigned STEP_B = 16384;
constexpr size_t PANEL_B = (size_t)KSTEPS * STEP_B;
constexpr int PA_AWAY = 24, PA_OWNAWAY = 32, PB = 34, PB_FOREIGN = 42, NPANEL = 50;

constexpr unsigned BLK_B = 1664;
constexpr unsigned PSTEP_B = 1024 + 8 * BLK_B;
constexpr size_t PPANEL_B = (size_t)KSTEPS * PSTEP_B;

constexpr size_t OFF_CALL = 0;
constexpr size_t OFF_TMO = 4096;
constexpr size_t OFF_TICKET = 5120;
constexpr int FCOPY = 8;
constexpr size_t OFF_FLAG = 1u << 18;
constexpr size_t OFF_RDY = 1u << 20;
constexpr int RDY_N = 1024, RDY_AWAY = 128, RDY_OUT = 384, RDY_HOP1 = 864, RDY_HOP2 = 896;
constexpr int RELAY = 32;
constexpr size_t OFF_OWN = 2u << 20;
constexpr size_t OFF_PACK = OFF_OWN + 32 * PANEL_B;
constexpr size_t OFF_RELAY = OFF_PACK + 32 * PPANEL_B;
constexpr size_t OFF_PULL = OFF_RELAY + PANEL_B;
constexpr int HX = 20;
constexpr int HW = 6;
constexpr int OUT_CHUNKS = 480;
constexpr unsigned SUM_B = 3456, CHUNK_B = 8192;
constexpr size_t OFF_STG = OFF_PULL + 2 * NPANEL * PANEL_B;
constexpr size_t OFF_SPARE = OFF_STG + (size_t)4 * OUT_CHUNKS * CHUNK_B;
constexpr size_t WS_NEED = OFF_SPARE + (size_t)4 * OUT_CHUNKS * 2048;
static_assert(RELAY % 4 == 0 && RELAY <= 64 && 64 * PSTEP_B <= PANEL_B, "");
static_assert(NPANEL <= 64 && OFF_FLAG + FCOPY * 65536 <= OFF_RDY && OFF_RDY + NR * RDY_N * 128 <= OFF_OWN, "");

constexpr int NWG = 256, NTHREADS = 512;
constexpr int GSLOTS = 24;
constexpr int NGEMM = GSLOTS * 8 + 8;
constexpr int PW = 8;
constexpr int WIN = 5;
constexpr int NJOBS = 3;
constexpr unsigned long long DEADLINE = 100000000ull;

constexpr unsigned LDS_MB = 18 * 8192;
constexpr unsigned LDS_TMO = LDS_MB + NJOBS * 4 * 512;
constexpr unsigned LDS_LUT = LDS_TMO + NJOBS * 16;
constexpr unsigned LDS_FILL = LDS_LUT + 64;
constexpr unsigned LDS_CALL = LDS_FILL + 16;
constexpr unsigned LDS_EPI = LDS_CALL + 16;
constexpr unsigned LDS_BYTES = LDS_EPI + 4 * 512;
constexpr unsigned WAVE_LDS = LDS_MB / 8;

struct Params {
  const float* x;
  const float* w;
  char* mine;
  int rank;
  unsigned counts;
  float* out;
  char* ws[NR];
  u32x4 plan[NGEMM];
};

__device__ __forceinline__ int xcc_id() {
  return __builtin_amdgcn_s_getreg(20 | (3 << 11)) & 7;
}

__device__ __forceinline__ void store_sys(void* ptr, u32x4 v) {
  asm volatile("global_store_dwordx4 %0, %1, off sc0 sc1\n\ts_nop 1" :: "v"(ptr), "v"(v) : "memory");
}
__device__ __forceinline__ void wait_vm0() { asm volatile("s_waitcnt vmcnt(0)" ::: "memory"); }
__device__ __forceinline__ void wg_barrier() {
  asm volatile("" ::: "memory");
  __builtin_amdgcn_s_barrier();
  asm volatile("" ::: "memory");
}
__device__ __forceinline__ void store_out(void* ptr, u32x4 v) {
  asm volatile("global_store_dwordx4 %0, %1, off sc0 sc1 nt\n\ts_nop 1" :: "v"(ptr), "v"(v) : "memory");
}

__device__ __forceinline__ void set_local(char* ws, int copy, int step, unsigned v) {
  __hip_atomic_store((GAS unsigned*)(ws + OFF_FLAG) + copy * 16384 + (step >> 7) * 256 + (step & 7) * 32 + ((step & 127) >> 3), v,
                     __ATOMIC_RELAXED, __HIP_MEMORY_SCOPE_SYSTEM);
}
__device__ __forceinline__ void set_rdy(char* peer_ws, int rank, int idx, unsigned call, unsigned lines) {
  __hip_atomic_store((GAS unsigned*)(peer_ws + OFF_RDY) + (rank * RDY_N + idx) * 32, call << 6 | lines, __ATOMIC_RELAXED,
                     __HIP_MEMORY_SCOPE_SYSTEM);
}
__device__ __forceinline__ unsigned own_idx(unsigned call, int idx) { return (call & 1) * 16 + idx; }
__device__ __forceinline__ size_t pulled(unsigned call) { return OFF_PULL + (call & 1) * (NPANEL * PANEL_B); }
__device__ __forceinline__ int fresh_lane() {
  int l;
  asm volatile("v_mbcnt_lo_u32_b32 %0, -1, 0\n\tv_mbcnt_hi_u32_b32 %0, -1, %0" : "=v"(l));
  return l;
}
__device__ __forceinline__ void cbar() { asm volatile("" ::: "memory"); }
__device__ __forceinline__ unsigned wave_scan(unsigned v) {
  v += __builtin_amdgcn_update_dpp(0u, v, 0x111, 0xf, 0xf, false);
  v += __builtin_amdgcn_update_dpp(0u, v, 0x112, 0xf, 0xf, false);
  v += __builtin_amdgcn_update_dpp(0u, v, 0x114, 0xf, 0xf, false);
  v += __builtin_amdgcn_update_dpp(0u, v, 0x118, 0xf, 0xf, false);
  v += __builtin_amdgcn_update_dpp(0u, v, 0x142, 0xa, 0xf, false);
  v += __builtin_amdgcn_update_dpp(0u, v, 0x143, 0xc, 0xf, false);
  asm("" : "+v"(v));
  return v;
}

__device__ __forceinline__ unsigned codec_lut(int m, bool expand) {
  unsigned sel = 0x0c0c0c0cu;
  int n = 0;
  for (int i = 0; i < 4; ++i)
    if ((m >> i) & 1) {
      const int at = expand ? i : n, v = expand ? n : i;
      sel = (sel & ~(0xffu << (8 * at))) | ((unsigned)v << (8 * at));
      ++n;
    }
  return sel;
}

__device__ __forceinline__ unsigned lut_at(const LAS char* lut, unsigned E) {
  return *(const LAS unsigned*)(uintptr_t)__builtin_amdgcn_udot4(E, 0x08040201u, (unsigned)(uintptr_t)lut, false);
}
__device__ __forceinline__ void lds_or(LAS unsigned* q, unsigned v) {
  __hip_atomic_fetch_or(q, v, __ATOMIC_RELAXED, __HIP_MEMORY_SCOPE_WORKGROUP);
}
struct Codes {
  unsigned pk[4], n[4], EM = 0, XM = 0;
  __device__ __forceinline__ void add(int k, const LAS char* lut, unsigned c, unsigned E, unsigned X) {
    const unsigned z = __builtin_amdgcn_perm(0u, c, lut_at(lut, E));
    pk[k] = __builtin_amdgcn_udot4(z, 0x00400801u, (z >> 24) << 9, false);
    n[k] = __builtin_popcount(E);
    EM |= (E >> 2) << k;
    XM |= X >> (7 - k);
  }
  template <int SH>
  __device__ __forceinline__ void put(LAS char* list, LAS char* patch, unsigned pre, const unsigned (&H)[4], unsigned base) {
    const unsigned long long W = (unsigned long long)(pk[0] | (pk[1] << (3 * n[0])))
        | ((unsigned long long)(pk[2] | (pk[3] << (3 * n[2]))) << (3 * (n[0] + n[1])));
    const unsigned o = 3 * pre, sh = o & 31u;
    LAS unsigned* q = (LAS unsigned*)(list + (o >> 5) * 4);
    const unsigned long long Ws = W << sh;
    lds_or(q, (unsigned)Ws);
    lds_or(q + 1, (unsigned)(Ws >> 32));
    lds_or(q + 2, ((unsigned)(W >> 32) >> 1) >> (31u - sh));
    cbar();
    const unsigned H0 = H[0], H1 = H[1], H2 = H[2], H3 = H[3];
    for (; XM; XM &= XM - 1u, patch -= 2) {
      const unsigned pos = __builtin_ctz(XM), k = pos & 3u, i8 = pos & 24u;
      const unsigned Hk = k & 2u ? (k & 1u ? H3 : H2) : (k & 1u ? H1 : H0);
      const unsigned kb = 0x01010101u << k;
      const unsigned rk = pre + __builtin_popcount(EM & ((kb - 0x01010101u) | (kb & ((1u << i8) - 1u))));
      const unsigned at = (unsigned)(uintptr_t)patch;
      lds_or((LAS unsigned*)(uintptr_t)(at & ~3u), ((rk << 5) | (((Hk >> (i8 + SH)) & 31u) ^ (base + 3u))) << ((at & 2u) * 8));
    }
    cbar();
  }
};

__device__ __forceinline__ void pack_block(u32x4 f0, u32x4 f1, LAS char* wl, const LAS char* lut, LAS char* lists,
                                           LAS unsigned* fill, char* step, int blk) {
  const int lane = threadIdx.x & 63;
  const unsigned h[8] = {f0[0], f0[1], f0[2], f0[3], f1[0], f1[1], f1[2], f1[3]};
  char* const dst = step + 1024 + blk * BLK_B;
  u32x4 D;
  unsigned H[4], t[4], sum = 0, sg = 0;
#pragma unroll
  for (int k = 0; k < 4; ++k) {
    D[k] = __builtin_amdgcn_perm(h[2 * k + 1], h[2 * k], 0x06040200u);
    H[k] = __builtin_amdgcn_perm(h[2 * k + 1], h[2 * k], 0x07050301u);
    sum = __builtin_amdgcn_udot4(H[k] & 0x7c7c7c7cu, 0x01010101u, sum, false);
    sg |= (H[k] & 0x80808080u) >> (7 - k);
  }
  store_sys(dst + lane * 16, D);
  const unsigned sgn = __builtin_amdgcn_update_dpp(0u, sg, 0xb1, 0xf, 0xf, false);
  if (!(lane & 1)) *(LAS unsigned*)(wl + 512 + lane * 2) = sg | (sgn << 4);
  const unsigned mean = __builtin_amdgcn_readlane(wave_scan(sum), 63) + 256u;
  const unsigned base = min(mean >> 12, 28u);
  const bool low = !(mean & 2048u);
  Codes c;
  unsigned W = 0, n2 = 0;
#pragma unroll
  for (int k = 0; k < 4; ++k) {
    const unsigned g = ((H[k] >> 2) & 0x1f1f1f1fu) + (0x43u - base) * 0x01010101u;
    const unsigned x = g ^ 0x40404040u, sel = x | (((x & 0x78787878u) + 0x78787878u) & 0x80808080u);
    const unsigned y = (g + 0x07070707u) ^ 0x40404040u ^ ~__builtin_amdgcn_perm(low ? 0xfff5ffffu : ~0u, ~0u, sel);
    const unsigned tt = __builtin_amdgcn_perm(low ? 0xffff0804u : 0xff1c0804u, low ? 0x001e1d1cu : 0x001e1dffu, sel);
    t[k] = (tt & 0x0c0c0c0cu) | (H[k] & 0x03030303u);
    const unsigned E = (tt >> 2) & 0x04040404u;
    W |= __builtin_amdgcn_udot4(__builtin_amdgcn_perm(0u, tt & 0x03030303u, lut_at(lut, E)), 0x40100401u, 0u, false) << (2 * n2);
    n2 += __builtin_popcount(E);
    c.add(k, lut, y & 0x07070707u, (tt >> 3) & 0x04040404u, ((y & 0x78787878u) + 0x78787878u) & tt & 0x80808080u);
  }
  *(LAS u32x2*)(wl + lane * 8) = (u32x2){t[0] | (t[1] << 4), t[2] | (t[3] << 4)};
  const unsigned mine = n2 | ((c.n[0] + c.n[1] + c.n[2] + c.n[3]) << 11) | (__builtin_popcount(c.XM) << 22);
  const unsigned incl = wave_scan(mine), pre = incl - mine;
  const unsigned all = __builtin_amdgcn_readlane(incl, 63), n4 = all >> 22, l2 = ((all & 2047u) + 15u) >> 4;
  const unsigned len = l2 + ((3u * ((all >> 11) & 2047u) + 16u * n4 + 31u) >> 5);
  unsigned head = 1u << 31, off = 0;
  cbar();
  if (l2 <= 32u && len < 64u && n4 < 32u) {
    asm volatile("" : "+v"(off));
    if (lane == 0) off = __hip_atomic_fetch_add(fill + off, len, __ATOMIC_RELAXED, __HIP_MEMORY_SCOPE_WORKGROUP);
    off = __builtin_amdgcn_readfirstlane(off);
    if (off + len < 256u) {
      head = base | (n4 << 5) | (low ? 1024u : 0u) | (l2 << 11) | (len << 17) | (off << 23);
      LAS char* const esc = lists + off * 4;
      const unsigned sh = (2u * pre) & 31u;
      LAS unsigned* q = (LAS unsigned*)esc + ((pre & 2047u) >> 4);
      lds_or(q, W << sh);
      lds_or(q + 1, (W >> 1) >> (31u - sh));
      c.put<2>(esc + l2 * 4, esc + len * 4 - 2 - 2 * (pre >> 22), (pre >> 11) & 2047u, H, base);
    }
  }
  if (lane == 0) ((LAS unsigned*)lists)[blk] = head;
  cbar();
}
__device__ __forceinline__ void pack_flush(LAS char* wl, LAS char* lists, char* step, int blk) {
  const int lane = threadIdx.x & 63;
  if (lane < 48) {
    const unsigned line = blk * 128 + (lane - 40) * 16;
    LAS u32x4* const from = (LAS u32x4*)(lane < 40 ? wl + lane * 16 : lists + line);
    const u32x4 v = *from;
    *from = (u32x4){0u, 0u, 0u, 0u};
    store_sys(step + (lane < 40 ? 2048 + blk * BLK_B + lane * 16 : line), v);
  }
}
__device__ __forceinline__ void lds_barrier() {
  asm volatile("s_waitcnt lgkmcnt(0)\n\ts_barrier" ::: "memory");
}
__device__ __forceinline__ unsigned lines_used(LAS unsigned* fill) {
  const unsigned n = *fill;
  *fill = 8u;
  return min(n - 1u, 255u) >> 5;
}

constexpr unsigned SUM_BASE = 24;
__device__ __forceinline__ void pack_sums(int lane, const u32x4 (&v)[4], unsigned& off, LAS char* wl, const LAS char* lut,
                                          char* dst, char* spare) {
  if (lane < 36) *(LAS unsigned*)(wl + 256 + lane * 4) = 0u;
  u32x4 P[3], E;
#pragma unroll
  for (int k = 0; k < 4; ++k) {
#pragma unroll
    for (int i = 0; i < 3; ++i) P[i][k] = __builtin_amdgcn_perm(v[3][k], v[i][k], 0x04020100u + ((unsigned)i << 24));
    E[k] = __builtin_amdgcn_perm(v[1][k], v[0][k], 0x0c0c0703u) | __builtin_amdgcn_perm(v[3][k], v[2][k], 0x07030c0cu);
  }
  asm volatile("global_store_dwordx4 %0, %1, off sc0 sc1\n\tglobal_store_dwordx4 %0, %2, off offset:1024 sc0 sc1\n\t"
               "global_store_dwordx4 %0, %3, off offset:2048 sc0 sc1\n\ts_nop 1"
               :: "v"(dst + lane * 16), "v"(P[0]), "v"(P[1]), "v"(P[2]) : "memory");
  if (off == ~0u) {
    unsigned sum = 0;
#pragma unroll
    for (int k = 0; k < 4; ++k) sum = __builtin_amdgcn_udot4(E[k], 0x01010101u, sum, false);
    off = min(max((__builtin_amdgcn_readlane(wave_scan(sum), 63) + 64u) >> 10, SUM_BASE) - SUM_BASE, 223u);
  }
  static_assert(SUM_BASE == 24, "");
  unsigned d[4], t[4];
  Codes c;
#pragma unroll
  for (int k = 0; k < 4; ++k) {
    d[k] = E[k] - off * 0x01010101u;
    t[k] = __builtin_amdgcn_perm(~0u, 0xf4a05000u, (d[k] + 0x68686868u) ^ 0x80808080u);
    c.add(k, lut, d[k] & t[k] & 0x07070707u, t[k] & 0x04040404u, 0u);
    c.XM |= __builtin_amdgcn_perm(0u, 0x0f0fu, (d[k] + 0x71717171u) ^ 0x80808080u) & (0x01010101u << k);
  }
  const unsigned mine = (c.n[0] + c.n[1] + c.n[2] + c.n[3]) | (__builtin_popcount(c.XM) << 16);
  const unsigned incl = wave_scan(mine), pre = incl - mine;
  const unsigned all = __builtin_amdgcn_readlane(incl, 63), n4 = all >> 16;
  const bool raw = __builtin_amdgcn_ballot_w64(((d[0] | d[1] | d[2] | d[3]) & 0xe0e0e0e0u) != 0u) != 0ull
                || 3u * (all & 0xffffu) + 16u * n4 > 1008u;
  cbar();
  if (!raw) c.put<0>(wl + 256, wl + 380 - 2 * (pre >> 16), pre & 0xffffu, d, SUM_BASE);
  if (lane == 0) *(LAS unsigned short*)(wl + 382) = (unsigned short)(raw ? 0x8000u : (off & 31u) | (n4 << 5) | (off >> 5 << 11));
  auto mix = [](unsigned m, unsigned x, unsigned y) { return (x & m) | (y & ~m); };
  *(LAS unsigned*)(wl + lane * 4) = mix(0xf0f0f0f0u, mix(0x30303030u, t[2], t[3]), mix(0x30303030u, t[0], t[1]) >> 4);
  cbar();
  if (lane < 24) store_sys(dst + 3072 + lane * 16, *(const LAS u32x4*)(wl + lane * 16));
  if (raw) store_sys(spare + lane * 16, E);
  cbar();
}

__device__ __forceinline__ u32x2 read_codes(int lane, const LAS char* list) {
  const LAS unsigned* sp = (const LAS unsigned*)(list + ((3 * lane) & ~3));
  return (u32x2){sp[0], sp[1]};
}
__device__ __forceinline__ void expand_codes(int lane, u32x2 codes, unsigned base, unsigned first, unsigned keep, unsigned keep1,
                                             bool patch, unsigned ent, LAS char* ex) {
  __builtin_amdgcn_sched_barrier(0);
  const unsigned x = __builtin_amdgcn_alignbyte(codes[1], codes[0], 3 * lane);
  const unsigned b3 = (base + 3u) * 0x04040404u, b8 = (base + first) * 0x04040404u;
  const unsigned T0 = ((b8 + 0x0c080400u) ^ b3) & keep, T1 = ((b8 + 0x1c181410u) ^ b3) & keep1;
  u32x2 X;
#pragma unroll
  for (int i = 0; i < 2; ++i) {
    const unsigned y = ((x >> (12 * i)) & 0x3fu) | ((i ? x >> 2 : x << 10) & 0x3f0000u);
    X[i] = __builtin_amdgcn_perm(T1, T0, (y | (y << 5)) & 0x07070707u);
  }
  *(LAS u32x2*)(ex + lane * 8) = X;
  cbar();
  if (patch) *(LAS unsigned char*)(ex + ((ent >> 5) & 511u)) = (unsigned char)((ent & 31u) << 2);
  cbar();
}
template <int N>
__device__ __forceinline__ void pick(const unsigned (&E)[N], const unsigned (&sel)[N], const LAS char* ex, unsigned pos,
                                     unsigned (&out)[N]) {
  unsigned at[N], w0[N], w1[N];
#pragma unroll
  for (int k = 0; k < N; ++k) { at[k] = pos; pos += __builtin_popcount(E[k]); }
  __builtin_amdgcn_sched_barrier(0);
#pragma unroll
  for (int k = 0; k < N; ++k) {
    const LAS unsigned* wp = (const LAS unsigned*)(ex + (at[k] & ~3u));
    w0[k] = wp[0];
    w1[k] = wp[1];
  }
  __builtin_amdgcn_sched_barrier(0);
#pragma unroll
  for (int k = 0; k < N; ++k) out[k] = __builtin_amdgcn_perm(0u, __builtin_amdgcn_alignbyte(w1[k], w0[k], at[k]), sel[k]);
}

__device__ __forceinline__ void unpack_block(int lane, const LAS char* step, int blk, unsigned hdr, LAS char* ex,
                                             const LAS char* lut, u32x4& f0, u32x4& f1) {
  const LAS char* const src = step + 1024 + blk * BLK_B;
  const LAS char* const esc = step + (hdr >> 23) * 4;
  const unsigned base = hdr & 31u;
  const bool patch = lane < ((hdr >> 5) & 31u), low = hdr & 1024u;
  const u32x2 x3 = read_codes(lane, esc + ((hdr >> 11) & 63u) * 4);
  const unsigned ent = *(const LAS unsigned short*)(esc + ((hdr >> 17) & 63u) * 4 - (patch ? 2 + 2 * lane : 0));
  const unsigned x2 = *(const LAS unsigned short*)(esc + 2 * lane);
  const u32x2 N = *(const LAS u32x2*)(src + 1024 + lane * 8);
  unsigned sg = *(const LAS unsigned*)(src + 1536 + (lane >> 1) * 4);
  const u32x4 D = *(const LAS u32x4*)(src + lane * 16);
  expand_codes(lane, x3, base, 22u, 0x7c7c7c7cu, low ? 0x007c7c7cu : 0x7c7c7c7cu, patch, ent, ex + 512);
  sg >>= (lane & 1) * 4;
  const unsigned E0 = N[0] & (N[0] >> 1) & 0x44444444u, E1 = N[1] & (N[1] >> 1) & 0x44444444u;
  unsigned c[2], F[2], sf[2], X[2], E[4], se[4], Y[4], h[8];
#pragma unroll
  for (int i = 0; i < 2; ++i) {
    const unsigned y = ((x2 >> (8 * i)) & 0xfu) | ((x2 << (12 - 8 * i)) & 0xf0000u);
    c[i] = (y | (y << 6)) & 0x03030303u;
    F[i] = (c[i] & (c[i] >> 1)) << 2;
    sf[i] = lut_at(lut, F[i]);
  }
#pragma unroll
  for (int k = 0; k < 4; ++k) {
    E[k] = ((k & 2 ? E1 : E0) >> (4 * (k & 1))) & 0x04040404u;
    se[k] = lut_at(lut, E[k]);
  }
  const unsigned mine = (__builtin_popcount(E0) + __builtin_popcount(E1)) | ((__builtin_popcount(F[0]) + __builtin_popcount(F[1])) << 16);
  const unsigned pre = wave_scan(mine) - mine;
  pick<2>(F, sf, ex + 512, pre >> 16, X);
  const unsigned T = ((base * 0x00040404u + 0x007c7874u) ^ ((base + 3u) * 0x00040404u)) & (low ? 0x007c7c7cu : 0x007c7c00u);
  *(LAS u32x2*)(ex + lane * 8) = (u32x2){X[0] | __builtin_amdgcn_perm(0u, T, c[0]), X[1] | __builtin_amdgcn_perm(0u, T, c[1])};
  cbar();
  pick<4>(E, se, ex, pre & 0xffffu, Y);
#pragma unroll
  for (int k = 0; k < 4; ++k) {
    const unsigned t = ((N[k >> 1] >> (4 * (k & 1))) & 0x0f0f0f0fu) + base * 0x04040404u;
    const unsigned H = (t ^ Y[k]) | ((sg << (7 - k)) & 0x80808080u);
    h[2 * k] = __builtin_amdgcn_perm(H, D[k], 0x05010400u);
    h[2 * k + 1] = __builtin_amdgcn_perm(H, D[k], 0x07030602u);
  }
  cbar();
  f0 = (u32x4){h[0], h[1], h[2], h[3]};
  f1 = (u32x4){h[4], h[5], h[6], h[7]};
}

__device__ __forceinline__ void unpack_sums(int lane, const LAS char* blk, LAS char* ex, const LAS char* lut, const char* spare,
                                            f32x4 (&y)[4]) {
  const unsigned ent = *(const LAS unsigned short*)(blk + 3454 - 2 * lane);
  const u32x2 x3 = read_codes(lane, blk + 3328);
  const unsigned C = *(const LAS unsigned*)(blk + 3072 + lane * 4);
  u32x4 P[3], X;
#pragma unroll
  for (int i = 0; i < 3; ++i) P[i] = *(const LAS u32x4*)(blk + i * 1024 + lane * 16);
  const unsigned hdr = __builtin_amdgcn_readfirstlane(ent);
  if (hdr & 0x8000u) {
    asm volatile("global_load_dwordx4 %0, %1, off sc0 sc1\n\ts_waitcnt vmcnt(0)" : "=&v"(X) : "v"(spare + lane * 16) : "memory");
  } else {
    expand_codes(lane, x3, SUM_BASE, 24u, 0x7c7c7c00u, 0x7c7c7c7cu, (unsigned)(lane - 1) < ((hdr >> 5) & 63u), ent, ex);
    unsigned t[4], E[4], se[4], Y[4];
#pragma unroll
    for (int k = 0; k < 4; ++k) {
      t[k] = (k ? C >> (2 * k - 2) : C << 2) & 0x0c0c0c0cu;
      E[k] = t[k] & (t[k] >> 1);
      se[k] = lut_at(lut, E[k]);
    }
    const unsigned mine = __builtin_popcount(C & (C >> 1) & 0x55555555u);
    pick<4>(E, se, ex, wave_scan(mine) - mine, Y);
    const unsigned off = ((hdr & 31u) | (hdr >> 11 << 5)) * 0x01010101u;
#pragma unroll
    for (int k = 0; k < 4; ++k) X[k] = (((t[k] + SUM_BASE * 0x04040404u) ^ Y[k]) >> 2) + off;
    cbar();
  }
#pragma unroll
  for (int k = 0; k < 4; ++k) {
    unsigned v[4];
#pragma unroll
    for (int i = 0; i < 3; ++i) v[i] = __builtin_amdgcn_perm(X[k], P[i][k], 0x04020100u + ((unsigned)i << 24));
    v[3] = __builtin_amdgcn_perm(P[1][k], P[0][k], 0x0c0c0703u) | __builtin_amdgcn_perm(X[k], P[2][k], 0x07030c0cu);
#pragma unroll
    for (int i = 0; i < 4; ++i) y[k][i] = __builtin_bit_cast(float, __builtin_rotateright32(v[i], 1));
  }
}

__device__ __forceinline__ f32x4 sigmoid(f32x4 y) {
  constexpr float K = 1.44269504088896341f;
  float lim = 60.0f * K;
  asm("" : "+s"(lim));
  f32x4 t = y * -K;
#pragma unroll
  for (int i = 0; i < 4; ++i) t[i] = __builtin_amdgcn_exp2f(__builtin_amdgcn_fmed3f(t[i], -lim, lim));
  t += 1.0f;
#pragma unroll
  for (int i = 0; i < 4; ++i) t[i] = __builtin_amdgcn_rcpf(t[i]);
  return t;
}

__device__ __forceinline__ unsigned* call_word(const Params& p) { return (unsigned*)(p.mine + OFF_CALL) + blockIdx.x; }
template <int LOADS>
__device__ __forceinline__ unsigned begin(const Params& p, LAS char* smem, bool puller) {
  *(LAS u32x4*)(smem + LDS_MB + min(threadIdx.x * 16u, LDS_LUT - LDS_MB - 16u)) = (u32x4){0u, 0u, 0u, 0u};
  if (threadIdx.x < 20) ((LAS unsigned*)(smem + LDS_LUT))[threadIdx.x] = threadIdx.x < 16 ? codec_lut(threadIdx.x, puller) : 8u;
  asm volatile("s_waitcnt vmcnt(%0) lgkmcnt(0)" :: "n"(LOADS) : "memory");
  wg_barrier();
  return __builtin_amdgcn_readfirstlane(*(const LAS unsigned*)(smem + LDS_CALL)) + 1u;
}
static_assert(LDS_LUT - LDS_MB <= NTHREADS * 16 && (LDS_LUT - LDS_MB) % 16 == 0, "");

constexpr int XJ = (KSTEPS - HX) / 2;
static_assert(HX % 2 == 0, "");
__device__ __forceinline__ int x_unit(int u) {
  if (u < RELAY) return 5 * KSTEPS + u;
  const int j = u < 7 * RELAY ? (u - RELAY) / 6 : u / 7, r = u < 7 * RELAY ? u - RELAY - j * 6 : u - j * 7;
  if (u < 7 * KSTEPS && (r >= 2 || j < XJ)) return r >= 2 ? (r - 1) * KSTEPS + j : HX + 2 * j + r;
  const int i = (u < 7 * KSTEPS ? 2 * j + r : u - 5 * KSTEPS) - 2 * XJ;
  return (6 + (i & 1)) * KSTEPS + (i >> 1);
}
constexpr int XROUNDS = 11;
constexpr int XLONG = 5;
__device__ __forceinline__ void load_in(const float* a, const float* b, unsigned dst) {
  unsigned keep;
  asm volatile("s_mov_b32 %0, m0\n\ts_mov_b32 m0, %3\n\ts_nop 0\n\tglobal_load_lds_dwordx4 %1, off nt\n\t"
               "s_add_u32 m0, m0, 1024\n\ts_nop 0\n\tglobal_load_lds_dwordx4 %2, off nt\n\ts_mov_b32 m0, %0"
               : "=&s"(keep) : "v"(a), "v"(b), "s"(dst) : "memory", "scc");
}
__device__ __forceinline__ void pace(unsigned t0, int at) {
  while ((int)((unsigned)__builtin_amdgcn_s_memrealtime() - t0) * 8 < at) __builtin_amdgcn_s_sleep(2);
}
constexpr int XT = 40, XO = 16, WT = 88, WO = 24, LEAD = 1200;
template <int N>
__device__ __forceinline__ void wait_vm() { asm volatile("s_waitcnt vmcnt(%0)" :: "n"(N) : "memory"); }
__device__ __forceinline__ u32x4 cvt8(f32x4 a, f32x4 b) {
  half8 h;
  h[0] = (_Float16)a[0]; h[1] = (_Float16)a[1]; h[2] = (_Float16)a[2]; h[3] = (_Float16)a[3];
  h[4] = (_Float16)b[0]; h[5] = (_Float16)b[1]; h[6] = (_Float16)b[2]; h[7] = (_Float16)b[3];
  u32x4 r = __builtin_bit_cast(u32x4, h);
  asm volatile("" : "+v"(r));
  return r;
}
__device__ __forceinline__ char* member(const Params& p, int group, int i) {
  char *a = p.ws[group], *b = p.ws[group + 1], *c = p.ws[group + 2], *d = p.ws[group + 3];
  asm volatile("" : "+s"(a), "+s"(b), "+s"(c), "+s"(d));
  return i & 2 ? (i & 1 ? d : c) : (i & 1 ? b : a);
}
__device__ __forceinline__ unsigned convert_x(const Params& p, unsigned role, unsigned counts, int rank, LAS char* smem) {
  const int lane = threadIdx.x & 63, wave = __builtin_amdgcn_readfirstlane(threadIdx.x >> 6), t = threadIdx.x;
  const int n0 = counts & 255u, n1 = (counts >> 8) & 255u, n2 = (counts >> 16) & 255u;
  const int first = role & 255u, all = XROUNDS * n0, most = min(all + XLONG * n1, 8 * KSTEPS - HX);
  const int count = first < n2 ? 8 * KSTEPS - HX : first < n1 ? most : all;
  const bool push = role & 0x1000u;
  char* const ws = p.mine;
  LAS char* const wl = smem + wave * WAVE_LDS;
  const float* src0 = p.x + (size_t)((wave >> 2) * 64 + (wave & 1) * 32 + (lane >> 3)) * KDIM + (wave & 2) * 16 + ((lane ^ (lane >> 4)) & 7) * 4;
  const int odd8 = 8 * KDIM + (lane & 4 ? -16 : 16);
  auto fetch = [&](unsigned buf, int u) {
    const float* s = src0 + (size_t)(u >> 7) * 128 * KDIM + (u & 127) * 64;
    const unsigned d = (unsigned)(uintptr_t)wl + 2048 + buf;
    load_in(s, s + odd8, d);
    load_in(s + 16 * KDIM, s + 16 * KDIM + odd8, d + 2048);
  };
  u32x4 h0, h1;
  auto take = [&](unsigned buf) {
    const LAS f32x4* v = (const LAS f32x4*)(wl + 2048 + buf) + (lane & 15) * 8;
    const int i = (lane >> 4) ^ ((lane >> 1) & 7);
    const f32x4 a = v[i], b = v[i ^ 4], c = v[128 + i], d = v[128 + (i ^ 4)];
    h0 = cvt8(a, b);
    h1 = cvt8(c, d);
  };
  auto next = [=](int it) { return it + (int)((counts >> (it < all ? 0 : it < most ? 8 : 16)) & 255u); };
  const unsigned t0 = __builtin_amdgcn_s_memrealtime();
  fetch(0, push ? first : x_unit(role >> 16));
  const unsigned call = begin<4>(p, smem, false);
  char* const in = member(p, rank & 4, t);
  if (push) {
    const unsigned at = first * STEP_B + wave * 2048 + lane * 16;
    char* b[4];
#pragma unroll
    for (int i = 0; i < 4; ++i)
      b[i] = at + (i == 3 ? ws + OFF_OWN + own_idx(call, 0) * PANEL_B
                          : member(p, rank & 4, rank + 1 + (i + first) % 3) + pulled(call) + (rank & 3) * 6 * PANEL_B);
    asm volatile("s_waitcnt vmcnt(0)" : "+v"(b[0]), "+v"(b[1]), "+v"(b[2]), "+v"(b[3]) :: "memory");
    take(0);
#pragma unroll
    for (int i = 0; i < 4; ++i) {
      store_sys(b[i], h0);
      store_sys(b[i] + 1024, h1);
    }
    wait_vm0();
    wg_barrier();
    if (t < 4 * FCOPY) set_local(in, t >> 2, (rank & 3) * 6 * KSTEPS + first, call);
    return call;
  }
  char* const across = member(p, (rank & 4) ^ 4, t);
  LAS unsigned* const fill = (LAS unsigned*)(smem + LDS_FILL);
  unsigned odd = 0;
  auto ahead = [&](unsigned buf, int it) {
    if (it < count) { fetch(buf, x_unit(it)); wait_vm<4>(); } else wait_vm0();
  };
  wait_vm0();
  for (int it = first, it1 = next(it), it2; it < count; it = it1, it1 = it2, odd ^= 8192u) {
    const int u = x_unit(it == first ? role >> 16 : it);
    it2 = next(it1);
    take(odd);
    char* b = ws + OFF_OWN + (own_idx(call, 0) * KSTEPS + u) * STEP_B + wave * 2048 + lane * 16;
    store_sys(b, h0);
    store_sys(b + 1024, h1);
    char* pb = ws + OFF_PACK + (own_idx(call, 0) * KSTEPS + u) * PSTEP_B;
    pack_block(h0, h1, wl, smem + LDS_LUT, smem + LDS_MB, fill, pb, wave);
    lds_barrier();
    pack_flush(wl, smem + LDS_MB, pb, wave);
    unsigned lines = 0;
    if (wave == 0) lines = lines_used(fill);
    ahead(it == first ? 8192u : odd, it == first ? it1 : it2);
    lds_barrier();
    if (t < 4 + FCOPY) {
      const int tile = u >> 7, ks = u & 127;
      if (t >= 4) set_local(ws, t - 4, (tile < 6 ? (rank & 3) * 6 : PA_OWNAWAY - 6) * KSTEPS + u, call);
      else if (tile >= 6) set_rdy(across, rank, RDY_AWAY + u - 6 * KSTEPS, call, lines);
      else if (tile == 5 && ks < RELAY) { if (t == (ks & 3)) set_rdy(across, rank, RDY_HOP1 + (ks >> 2), call, lines); }
      else if (t != (rank & 3)) set_rdy(in, rank, u, call, lines);
    }
    if (it == first) ahead(0, it2);
    if (it1 < count) pace(t0, (it1 - XO) * XT + LEAD);
  }
  return call;
}

__device__ __forceinline__ unsigned convert_w(const Params& p, unsigned role, unsigned counts, int rank, LAS char* smem) {
  const int lane = threadIdx.x & 63, wave = __builtin_amdgcn_readfirstlane(threadIdx.x >> 6), t = threadIdx.x;
  const int count = KSTEPS * 4, stride = counts >> 24, first = role & 255u;
  const bool push = role & 0x1000u;
  char* const ws = p.mine;
  LAS char* const wl = smem + wave * WAVE_LDS;
  const float* src0 = p.w + (size_t)((wave & 1) * 32 + (lane >> 4) * 4) * NSH + (wave >> 1) * 64 + (lane & 15) * 4;
  auto fetch = [&](unsigned buf, int u) {
    const float* s = src0 + (size_t)(u >> 2) * 64 * NSH + (u & 3) * 256;
#pragma unroll
    for (int j = 0; j < 8; j += 2) {
      const float* r = s + ((j & 3) + (j >> 2) * 16) * NSH;
      load_in(r, r + NSH, (unsigned)(uintptr_t)wl + 2048 + buf + j * 1024);
    }
  };
  u32x4 h[4];
  auto take = [&](unsigned buf) {
    f32x4 v[8];
#pragma unroll
    for (int j = 0; j < 8; ++j) v[j] = ((const LAS f32x4*)(wl + 2048 + buf))[j * 64 + lane];
#pragma unroll
    for (int f = 0; f < 4; ++f)
      h[f] = cvt8((f32x4){v[0][f], v[1][f], v[2][f], v[3][f]}, (f32x4){v[4][f], v[5][f], v[6][f], v[7][f]});
  };
  const bool late = push && wave >= 4;
  const unsigned t0 = __builtin_amdgcn_s_memrealtime();
  if (!push) pace(t0, (first - 4 * HW - WO) * WT);
  if (!late) fetch(0, first);
  const unsigned call = begin<8>(p, smem, false);
  if (late) fetch(0, first);
  char* const own = ws + OFF_OWN + own_idx(call, 8 + (wave >> 2)) * PANEL_B + (wave & 3) * 4096 + lane * 16;
  char* const across = member(p, (rank & 4) ^ 4, push ? first : t);
  if (push) {
    const int q = first & 3, ks = first >> 2;
    char* b = own + q * 2 * PANEL_B + ks * STEP_B;
    char* r = across + pulled(call) + (PB_FOREIGN + (rank & 3) * 2 + (wave >> 2)) * PANEL_B + ks * STEP_B + (wave & 3) * 4096 + lane * 16;
    asm volatile("s_waitcnt vmcnt(0)" : "+v"(b), "+v"(r) :: "memory");
    take(0);
#pragma unroll
    for (int f = 0; f < 4; ++f) store_sys(r + f * 1024, h[f]);
#pragma unroll
    for (int f = 0; f < 4; ++f) store_sys(b + f * 1024, h[f]);
    wait_vm0();
    wg_barrier();
    if (t < 4 * FCOPY)
      set_local(t & 2 ? across : ws, t >> 2, ((t & 2 ? PB_FOREIGN + (rank & 3) * 2 : PB + q * 2) + (t & 1)) * KSTEPS + ks, call);
    return call;
  }
  LAS unsigned* const fill = (LAS unsigned*)(smem + LDS_FILL) + (wave >> 2);
  LAS char* const lists = smem + LDS_MB + (wave >> 2) * 2048;
  unsigned odd = 0;
  auto ahead = [&](unsigned buf, int u) {
    if (u < count) { fetch(buf, u); wait_vm<8>(); } else wait_vm0();
  };
  static_assert(FCOPY % 2 == 0, "");
  wait_vm0();
  for (int u = first; u < count; u += stride, odd ^= 8192u) {
    const int ks = u >> 2, q = u & 3, blk = (wave & 3) * 2;
    char* b = own + q * 2 * PANEL_B + ks * STEP_B;
    take(odd);
#pragma unroll
    for (int f = 0; f < 4; ++f) store_sys(b + f * 1024, h[f]);
    char* pb = ws + OFF_PACK + own_idx(call, 8 + q * 2 + (wave >> 2)) * PPANEL_B + ks * PSTEP_B;
    pack_block(h[0], h[1], wl, smem + LDS_LUT, lists, fill, pb, blk);
    pack_block(h[2], h[3], wl + 1024, smem + LDS_LUT, lists, fill, pb, blk + 1);
    lds_barrier();
    pack_flush(wl, lists, pb, blk);
    pack_flush(wl + 1024, lists, pb, blk + 1);
    unsigned lines = 0;
    if (wave == 0) lines = lines_used(fill) | lines_used(fill + 1) << 3;
    ahead(u == first ? 8192u : odd, u + (u == first ? stride : 2 * stride));
    lds_barrier();
    if (t < 2 * FCOPY) set_local(ws, t >> 1, (PB + q * 2 + (t & 1)) * KSTEPS + ks, call);
    else if (t == 2 * FCOPY + q) set_rdy(across, rank, ks, call, lines);
    if (u == first) ahead(0, u + 2 * stride);
    if (u + stride < count) pace(t0, (u + stride - 4 * HW - WO) * WT + LEAD);
  }
  return call;
}

__device__ __forceinline__ int row_of(int r) { return ((RANK_AT >> (4 * r)) & 7) * NSH; }

constexpr unsigned long long BATCH_AT = 0x7648953210ull;
constexpr unsigned long long batch_pos() {
  unsigned long long r = 0;
  for (int i = 0; i < 10; ++i) r |= (unsigned long long)i << (4 * ((BATCH_AT >> (4 * i)) & 15));
  return r;
}
constexpr unsigned long long BATCH_POS = batch_pos();
__device__ __forceinline__ int batch_row(int s, int o, int pos, int i) {
  const int bt = (BATCH_AT >> (4 * pos)) & 15;
  if (bt < 2) return row_of(s) + (bt * 3 + i) * 128;
  const int g = bt < 8 ? s : o;
  return row_of((g & 4) | ((g + 1 + i) & 3)) + (bt - 2) * 128;
}

__device__ __forceinline__ int unit_of(int it, int w) { return it * PW + ((w + it) & (PW - 1)); }

__device__ __forceinline__ unsigned lds_get(const LAS unsigned* c) {
  return __builtin_amdgcn_readfirstlane(__hip_atomic_load(c, __ATOMIC_RELAXED, __HIP_MEMORY_SCOPE_WORKGROUP));
}

constexpr int RING = 8;
constexpr unsigned PL_EX = RING * PSTEP_B;
constexpr int W_TICKET = 0, W_READY = 1, W_DEAD = 2, W_TURN = 3, W_SPARE = 4, W_DONE = 8, W_LANDED = 64, W_QUEUE = 128, W_SNAP = 256;
constexpr unsigned Q_N = 128, Q_OUT = 1024, Q_LAST = 1u << 31;
static_assert(PL_EX + 8 * 1024 <= LDS_MB && W_QUEUE + Q_N <= W_SNAP && (W_SNAP + 896) * 4 <= LDS_TMO - LDS_MB, "");

template <int N>
__device__ __forceinline__ void stream_kib(const char* src, unsigned dst, unsigned& cnt, unsigned word) {
  unsigned keep;
#define SK(OFF) "s_waitcnt vmcnt(%5)\n\tglobal_load_lds_dwordx4 %2, off offset:" #OFF " sc0 sc1\n\t" \
                "v_add_u32 %1, 1, %1\n\tds_max_u32 %4, %1\n\t"
  if (N == 4)
    asm volatile("s_mov_b32 %0, m0\n\ts_mov_b32 m0, %3\n\ts_nop 0\n\t" SK(0) SK(1024) SK(2048) SK(3072) "s_mov_b32 m0, %0"
                 : "=&s"(keep), "+v"(cnt) : "v"(src), "s"(dst), "v"(word), "n"(WIN - 1) : "memory");
  else if (N == 2)
    asm volatile("s_mov_b32 %0, m0\n\ts_mov_b32 m0, %3\n\ts_nop 0\n\t" SK(0) SK(1024) "s_mov_b32 m0, %0"
                 : "=&s"(keep), "+v"(cnt) : "v"(src), "s"(dst), "v"(word), "n"(WIN - 1) : "memory");
  else
    asm volatile("s_mov_b32 %0, m0\n\ts_mov_b32 m0, %3\n\ts_nop 0\n\t" SK(0) "s_mov_b32 m0, %0"
                 : "=&s"(keep), "+v"(cnt) : "v"(src), "s"(dst), "v"(word), "n"(WIN - 1) : "memory");
}
template <int K, class F>
__device__ __forceinline__ bool stream_drain(unsigned cnt, unsigned word, F&& more) {
  if constexpr (K >= 0) {
    asm volatile("s_waitcnt vmcnt(%2)\n\tds_max_u32 %1, %0" :: "v"(cnt + 1), "v"(word), "n"(K) : "memory");
    return more() || stream_drain<K - 1>(cnt + 1, word, more);
  }
  return false;
}

__device__ __forceinline__ int zip(int u, int s, int d, int g) {
  const int i = u - (s - d), j = i - g;
  return (unsigned)i >= (unsigned)(2 * d + g) ? u : j < 0 ? s + i : (j & 1 ? s + g : s - d) + (j >> 1);
}
constexpr int D2 = 12, G2 = 20, D4 = 40, G4 = 28, D5 = 52, DA = 40;
constexpr int DH = 48;
template <bool CROSS>
__device__ __forceinline__ int in_order(int u) {
  constexpr int HOP = 4 * KSTEPS + RELAY / 4 - 2 * HW - DH;
  constexpr int AWAY = 2 * KSTEPS + RELAY / 4 - 2 * HW;
  if (CROSS && u < AWAY) return u < RELAY / 4 ? u + 2 * KSTEPS - 2 * HW : u - RELAY / 4;
  if (CROSS) return u < HOP ? zip(u, AWAY + KSTEPS, DA, 0) : u < HOP + RELAY / 4 * 3 ? u + DH : u - RELAY / 4 * 3;
  if (u < 2 * KSTEPS - HX + D2 + G2) return zip(u, 2 * KSTEPS - HX, D2, G2);
  return u < 4 * KSTEPS - HX + D4 + G4 ? zip(u, 4 * KSTEPS - HX, D4, G4) : zip(u, 5 * KSTEPS - HX, D5, 0);
}
static_assert(DA + DH <= KSTEPS && D2 + G2 <= KSTEPS && D4 + G4 + D5 <= KSTEPS && D5 + RELAY <= KSTEPS, "");

struct Unit { unsigned src, raw, dst; int from, flag; };
template <bool CROSS>
__device__ __forceinline__ Unit input_unit(int u, unsigned call, int rank, int peer) {
  constexpr int H1 = RELAY / 4, RSTEP = 32 * KSTEPS;
  const unsigned par = own_idx(call, 0) * KSTEPS;
  Unit o;
  o.from = peer;
  u += CROSS ? 2 * HW : HX;
  if (!CROSS) {
    const int v = u < 5 * KSTEPS ? u : u + RELAY;
    o.raw = par + v; o.dst = (peer & 3) * 6 * KSTEPS + v; o.flag = v;
  } else if (u < 256) {
    const int v = (u & 1) * KSTEPS + (u >> 1);
    o.raw = par + (8 + (rank & 3) * 2) * KSTEPS + v; o.dst = (PB_FOREIGN + (peer & 3) * 2) * KSTEPS + v; o.flag = u >> 1;
  } else if (u < 256 + H1) {
    o.raw = par + 5 * KSTEPS + (u - 256) * 4 + (rank & 3); o.dst = ~((peer & 3) * 16 + u - 256); o.flag = RDY_HOP1 + u - 256;
  } else if (u < 512 + H1) {
    const int v = u - 256 - H1;
    o.raw = par + 6 * KSTEPS + v; o.dst = (PA_AWAY + (peer & 3) * 2) * KSTEPS + v; o.flag = RDY_AWAY + v;
  } else {
    const int v = u - 512 - H1, j = v / 3, s = (rank + 1 + v - j * 3) & 3, ks = 5 * KSTEPS + j * 4 + (peer & 3);
    o.from = (rank & 4) | s;
    o.raw = par + ks; o.dst = s * 6 * KSTEPS + ks; o.flag = RDY_HOP2 + s * 16 + j;
    o.src = RSTEP + s * 16 + j;
    return o;
  }
  o.src = o.raw;
  return o;
}

template <bool CROSS>
__device__ __forceinline__ void stream(const Params& p, int lane, unsigned call, int rank, int peer, LAS char* smem) {
  LAS unsigned* const sh = (LAS unsigned*)(smem + LDS_MB);
  const char* const pws = p.ws[peer];
  const unsigned lds0 = (unsigned)(uintptr_t)smem, word = (unsigned)(uintptr_t)(sh + W_LANDED + lane);
  unsigned cnt = 0;
  for (unsigned k = 0;; ++k) {
    unsigned ready, done, q;
    auto look = [&]() {
      const unsigned a = __hip_atomic_load(sh + W_READY, __ATOMIC_RELAXED, __HIP_MEMORY_SCOPE_WORKGROUP);
      cbar();
      const unsigned b = __hip_atomic_load(sh + W_DONE + (k & (RING - 1)), __ATOMIC_RELAXED, __HIP_MEMORY_SCOPE_WORKGROUP);
      const unsigned c = __hip_atomic_load(sh + W_QUEUE + (k & (Q_N - 1)), __ATOMIC_RELAXED, __HIP_MEMORY_SCOPE_WORKGROUP);
      cbar();
      ready = __builtin_amdgcn_readfirstlane(a);
      done = __builtin_amdgcn_readfirstlane(b);
      q = __builtin_amdgcn_readfirstlane(c);
      return k < (ready & ~Q_LAST) && done >= (k & ~(RING - 1u));
    };
    if (!look() && !stream_drain<WIN - 1>(cnt, word, look)) {
      while (!look() && (k | Q_LAST) != ready) __builtin_amdgcn_s_sleep(1);
      if ((k | Q_LAST) == ready) break;
    }
    const unsigned d = lds0 + (k & (RING - 1)) * PSTEP_B, u = q & 2047u;
    if (u < Q_OUT) {
      const char* s = pws + OFF_PACK + (size_t)input_unit<CROSS>(u, call, rank, peer).src * PSTEP_B + lane * 16;
      if ((unsigned)lane >> 3 <= ((q >> 11) & 7u)) stream_kib<1>(s, d, cnt, word);
      stream_kib<4>(s + 1024, d + 1024, cnt, word);
      stream_kib<4>(s + 5120, d + 5120, cnt, word);
      stream_kib<4>(s + 9216, d + 9216, cnt, word);
      stream_kib<1>(s + 13312, d + 13312, cnt, word);
    } else {
      const char* s = pws + OFF_STG + (size_t)((rank & 3) * OUT_CHUNKS + u - Q_OUT) * CHUNK_B + lane * 16;
      stream_kib<4>(s, d, cnt, word);
      stream_kib<2>(s + 4096, d + 4096, cnt, word);
      if (lane < 48) stream_kib<1>(s + 6144, d + 6144, cnt, word);
    }
  }
}

template <bool CROSS>
__device__ __forceinline__ void pull(const Params& p, int w, int wave, unsigned call, int rank, int peer, LAS char* smem) {
  constexpr int NIN = CROSS ? 512 + RELAY - 2 * HW : 768 - RELAY - HX, NOUT = CROSS ? OUT_CHUNKS / PW : 0;
  constexpr int SR = 3;
  const int lane = fresh_lane();
  char* const ws = p.mine;
  LAS unsigned* const sh = (LAS unsigned*)(smem + LDS_MB);
  float* const out = p.out + (size_t)((lane >> 4) * 4) * NSH + (peer & 3) * 256 + (lane & 15) * 4;
  auto where = [&](int c, int pc) {
    const int tile = c >> 3, bt = tile / 6, r = tile - bt * 6;
    return out + (size_t)(batch_row(peer, rank, bt, r >> 1) + ((c >> 2) & 1) * 64 + (c & 1) * 32 + (pc & 4) * 4 + (pc & 3)) * NSH
         + (r & 1) * 128 + ((c >> 1) & 1) * 64;
  };
  if (wave >= 6) __builtin_amdgcn_s_setprio(3);
  if (wave == 6) {
    const unsigned* const rdy = (const unsigned*)(ws + OFF_RDY) + peer * RDY_N * 32;
    unsigned* const tmo = (unsigned*)(ws + OFF_TMO);
    unsigned* const ticket = (unsigned*)(ws + OFF_TICKET) + peer * 32;
    unsigned long long t0 = __builtin_amdgcn_s_memrealtime();
    bool dead = false;
    unsigned k = 0, kib = 0, spin = 0;
    LAS unsigned* const snap = sh + W_SNAP;
    auto poll = [&](bool on, int fl, int at) {
      unsigned keep;
      if (on)
        asm volatile("s_mov_b32 %0, m0\n\ts_mov_b32 m0, %2\n\ts_nop 0\n\tglobal_load_lds_dword %1, off sc0 sc1\n\ts_mov_b32 m0, %0"
                     : "=&s"(keep) : "v"(rdy + fl * 32), "s"((unsigned)(uintptr_t)(snap + at)) : "memory");
    };
    auto unset = [&](int at) {
      const unsigned v = __hip_atomic_load(snap + at + lane, __ATOMIC_RELAXED, __HIP_MEMORY_SCOPE_WORKGROUP);
      return dead ? 0ull : __builtin_amdgcn_ballot_w64(v >> 6 < call);
    };
    auto idle = [&]() {
      if ((++spin & 63u) == 0u
          && (__builtin_amdgcn_readfirstlane(__hip_atomic_load(tmo, __ATOMIC_RELAXED, __HIP_MEMORY_SCOPE_AGENT)) == call
              || __builtin_amdgcn_s_memrealtime() - t0 > DEADLINE)) {
        if (lane == 0) {
          __hip_atomic_store(tmo, call, __ATOMIC_RELAXED, __HIP_MEMORY_SCOPE_AGENT);
          __hip_atomic_store(sh + W_DEAD, 1u, __ATOMIC_RELAXED, __HIP_MEMORY_SCOPE_WORKGROUP);
        }
        wait_vm0();
        dead = true;
      }
      __builtin_amdgcn_s_sleep(1);
    };
    auto publish = [&](unsigned v, unsigned near) {
      cbar();
      if (lane == 0) __hip_atomic_store(sh + W_READY, v, __ATOMIC_RELAXED, __HIP_MEMORY_SCOPE_WORKGROUP);
      while (lds_get(sh + W_LANDED) < near) __builtin_amdgcn_s_sleep(2);
      t0 = __builtin_amdgcn_s_memrealtime();
    };
    int known = 0;
    auto flag_of = [&](int from) { return input_unit<CROSS>(in_order<CROSS>(min(from + lane, NIN - 1)), call, rank, peer).flag; };
    auto see = [&](int from) {
      const unsigned long long miss = unset(from);
      known = from + (miss ? (int)__builtin_ctzll(miss) : 64);
    };
    for (int u = w; u < NIN;) {
      if (u >= known) {
        const int fl = flag_of(u);
        spin &= ~3u;
        for (see(u); u >= known; see(u)) { poll(lane < (spin & 3u ? 1 : 17), fl, u); idle(); }
      }
      const unsigned v = lds_get(snap + u), o = in_order<CROSS>(u);
      const unsigned lines = v >> 6 == call ? (v >> (CROSS && o < 256 - 2 * HW ? (o & 1) * 3 : 0)) & 7u : 7u;
      if (lane == 0) sh[W_QUEUE + (k & (Q_N - 1))] = kib << 14 | lines << 11 | o;
      publish(++k, max(kib, 4u) - 4u);
      kib += 14;
      if (k < SR) { u += PW; continue; }
      const bool more = known < NIN && known - u < 32;
      if (more) poll(true, flag_of(known), known);
      unsigned tk = 0;
      if (lane == 0) tk = __hip_atomic_fetch_add(ticket, 1u, __ATOMIC_RELAXED, __HIP_MEMORY_SCOPE_AGENT);
      if (more) { wait_vm0(); see(known); }
      u = min(__builtin_amdgcn_readfirstlane(tk) - (call - 1u) * (NIN - (SR - 1) * PW) + SR * PW, (unsigned)NIN);
    }
    if (CROSS && lane == 0) __hip_atomic_store(sh + W_TURN, ~(k * 8u), __ATOMIC_RELAXED, __HIP_MEMORY_SCOPE_WORKGROUP);
    const unsigned long long all = (1ull << NOUT) - 1ull;
    for (unsigned long long left = all; left;) {
      const unsigned long long fresh = left & ~unset(832);
      if (!fresh) {
        if (!(spin & 1u)) poll((left >> lane) & 1, RDY_OUT + unit_of(lane, w), 832);
        idle();
        continue;
      }
      spin = 0;
      const unsigned pos = __builtin_amdgcn_mbcnt_hi(fresh >> 32, __builtin_amdgcn_mbcnt_lo(fresh, 0));
      if ((fresh >> lane) & 1) sh[W_QUEUE + ((k + pos) & (Q_N - 1))] = (kib + pos * 7) << 14 | (Q_OUT + unit_of(lane, w));
      k += __builtin_popcountll(fresh);
      kib += __builtin_popcountll(fresh) * 7;
      left &= ~fresh;
      if (left) publish(k, kib - 14);
    }
    publish(k | Q_LAST, 0);
    if (CROSS) {
      dead = __builtin_amdgcn_readfirstlane(__hip_atomic_load(tmo, __ATOMIC_RELAXED, __HIP_MEMORY_SCOPE_AGENT)) == call;
      wg_barrier();
      if (dead) store_sys(where(unit_of(0, w), 0), (u32x4){0x7fc00000u, 0x7fc00000u, 0x7fc00000u, 0x7fc00000u});
    }
    return;
  }
  if (wave == 7) {
    stream<CROSS>(p, lane, call, rank, peer, smem);
    if (CROSS) wg_barrier();
    return;
  }
  static_assert(RING == 8 && Q_N == 128, "");
  const unsigned sh0 = (unsigned)(uintptr_t)sh, none = ~0u;
  const unsigned aq = sh0 + W_QUEUE * 4 + lane * 8, ah = (unsigned)(uintptr_t)smem + (lane >> 3) * PSTEP_B + (lane & 7) * 4;
  u32x4 st;
  u32x2 queue;
  unsigned count, heads;
  auto look = [&](int word) {
    unsigned r;
    unsigned long long keep;
    asm volatile("s_mov_b64 %5, exec\n\ts_mov_b64 exec, 1\n\tds_add_rtn_u32 %0, %6, %7\n\ts_mov_b64 exec, %5\n\t"
                 "ds_read_b128 %1, %8\n\tds_read_b64 %2, %9\n\tds_read_b32 %3, %8 offset:%11\n\tds_read_b32 %4, %10\n\t"
                 "s_waitcnt lgkmcnt(0)"
                 : "=&v"(r), "=&v"(st), "=&v"(queue), "=&v"(count), "=&v"(heads), "=&s"(keep)
                 : "v"(sh0 + word * 4), "v"(1u), "v"(sh0), "v"(aq), "v"(ah), "n"(W_LANDED * 4) : "memory");
    return __builtin_amdgcn_readfirstlane(r);
  };
  unsigned t, u, q;
  auto landed = [&](unsigned tk) {
    const unsigned ready = __builtin_amdgcn_readfirstlane(st[W_READY]);
    t = tk;
    if (CROSS) {
      const unsigned turn = ~__builtin_amdgcn_readfirstlane(st[W_TURN]);
      if (tk > turn) t = turn + (tk - turn) * 4u;
    }
    const unsigned q0 = __builtin_amdgcn_readlane(queue[0], (t >> 4) & 63), q1 = __builtin_amdgcn_readlane(queue[1], (t >> 4) & 63);
    q = t & 8 ? q1 : q0;
    const bool there = (t >> 3) < (ready & ~Q_LAST);
    u = there ? q & 2047u : none;
    const unsigned need = (q >> 14) + (!CROSS || u < Q_OUT ? ((t & 7) * 13 + 28) >> 3 : t & 4 ? 7 : 4) + WIN;
    return there ? (unsigned)__builtin_amdgcn_readfirstlane(count) >= need : ready >= Q_LAST;
  };
  unsigned prev = none, pdst = 0, plines = 0;
  auto done = [&]() {
    const bool last = (look(prev != none ? W_DONE + ((prev >> 3) & (RING - 1)) : W_SPARE) & 7u) == 7u && prev != none;
    prev = none;
    if (!last) return;
    if (!CROSS || (int)pdst >= 0) { if (lane < FCOPY) set_local(ws, lane, pdst, call); }
    else if (lane < 3) set_rdy(p.ws[((peer & 4) | ((peer + 1 + lane) & 3)) & 7], rank, RDY_HOP2 + ~pdst, call, plines);
  };
  for (;;) {
    const unsigned tk = look(W_TICKET);
    if (!landed(tk)) {
      if (prev != none) wait_vm0();
      for (done(); !landed(tk); look(W_SPARE)) __builtin_amdgcn_s_sleep(1);
    }
    const LAS char* slot = smem + ((t >> 3) & (RING - 1)) * PSTEP_B;
    if (u >= Q_OUT) {
      if (prev != none) { wait_vm0(); done(); }
      if (!CROSS || u == none) break;
      const unsigned half = (t >> 2) & 1u;
      const bool dead = __builtin_amdgcn_readfirstlane(st[W_DEAD]);
      f32x4 y[4];
      unpack_sums(lane, slot + half * SUM_B, smem + PL_EX + wave * 1024, smem + LDS_LUT,
                  p.ws[peer] + OFF_SPARE + (size_t)(((rank & 3) * OUT_CHUNKS + u - Q_OUT) * 2 + half) * 1024, y);
      if (lane == 0) __hip_atomic_fetch_add(sh + W_DONE + ((t >> 3) & (RING - 1)), 4u, __ATOMIC_RELAXED, __HIP_MEMORY_SCOPE_WORKGROUP);
      float* const o = where(u - Q_OUT, t & 4);
#pragma unroll
      for (int r = 0; r < 4; ++r) {
        u32x4 v = __builtin_bit_cast(u32x4, y[r] * sigmoid(y[r]));
        if (dead) v = (u32x4){0x7fc00000u, 0x7fc00000u, 0x7fc00000u, 0x7fc00000u};
        store_out(o + r * NSH, v);
      }
      continue;
    }
    const Unit un = input_unit<CROSS>(u, call, rank, peer);
    u32x4 f0, f1;
    char* d;
    if (CROSS && (int)un.dst < 0) {
      const int l1 = min(lane, 47);
      const LAS char* blk = slot + (t & 7) * (PSTEP_B / 8);
      f0 = *(const LAS u32x4*)(blk + lane * 16);
      f1 = *(const LAS u32x4*)(blk + 1024 + l1 * 16);
      d = ws + OFF_RELAY + (size_t)(~un.dst * 8 + (t & 7)) * (PSTEP_B / 8);
      store_sys(d + lane * 16, f0);
      store_sys(d + 1024 + l1 * 16, f1);
    } else {
      const unsigned hdr = __builtin_amdgcn_readlane(heads, t & 63);
      if ((int)hdr >= 0) {
        unpack_block(lane, slot, t & 7, hdr, smem + PL_EX + wave * 1024, smem + LDS_LUT, f0, f1);
      } else {
        const char* r = p.ws[un.from & 7] + OFF_OWN + (size_t)(un.raw * 8 + (t & 7)) * 2048 + lane * 16;
        asm volatile("global_load_dwordx4 %0, %2, off sc0 sc1\n\t"
                     "global_load_dwordx4 %1, %2, off offset:1024 sc0 sc1\n\ts_waitcnt vmcnt(0)"
                     : "=&v"(f0), "=&v"(f1) : "v"(r) : "memory");
      }
      d = ws + pulled(call) + (size_t)(un.dst * 8 + (t & 7)) * 2048 + lane * 16;
      store_sys(d, f0);
      store_sys(d + 1024, f1);
    }
    asm volatile("s_waitcnt vmcnt(2)" ::: "memory");
    done();
    prev = t;
    pdst = un.dst;
    plines = (q >> 11) & 7u;
    if (!landed(__builtin_amdgcn_readfirstlane(st[W_TICKET]))) { wait_vm0(); done(); }
  }
  if (CROSS) { wait_vm0(); wg_barrier(); }
}

struct Job { int pa, pb, ah, bh, tile; };
__device__ __forceinline__ void decode_job(unsigned code, int a, Job& o) {
  const int ra = code & 63;
  int bt, i;
  o.pb = PB + ((code >> 6) & 15);
  o.ah = (code >> 10) & 1;
  o.bh = (code >> 11) & 1;
  if (ra < 18) {
    i = ra / 6; bt = 2 + ra % 6; o.pa = ((a + 1 + i) & 3) * 6 + bt - 2;
  } else if (ra < 24) {
    o.pa = a * 6 + ra - 18; bt = (ra - 18) / 3; i = (ra - 18) % 3;
  } else {
    o.pa = ra; bt = 8 + (ra & 1); i = (((ra - PA_AWAY) >> 1) - ((o.pb - PB_FOREIGN) >> 1) - 1) & 3;
  }
  o.tile = (int)((BATCH_POS >> (4 * bt)) & 15) * 3 + i;
}

template <int NB>
__device__ __forceinline__ unsigned next_stage(unsigned so) { return so == LDS_MB - (NB + 1) * 8192 ? 0u : so + (NB + 1) * 8192; }

template <bool TALL, int NB>
__device__ __forceinline__ void mfma_wave(const Params& p, LAS char* smem, const Job& jb, int job, int wave,
                                          int rank, unsigned call) {
  constexpr int MT = TALL ? 4 : 1, NT = 4 * NB, KH = TALL ? 1 : 2, STEPS = 2 * KSTEPS / KH;
  constexpr int NRA = MT * KH, NRB = NT * KH, NRD = NRA + NRB, NMM = MT * NT * KH;
  int lane = fresh_lane();
  const int wr = MT == 1 ? wave : wave >> 1, wc = MT == 1 ? 0 : wave & 1;
  f32x4 acc[MT][NT];
#pragma unroll
  for (int m = 0; m < MT; ++m)
#pragma unroll
    for (int n = 0; n < NT; ++n) acc[m][n] = (f32x4){0.f, 0.f, 0.f, 0.f};
  half8 fa[2][NRA], fb[2][NRB];
  const LAS char* la = smem + wr * MT * 1024 + lane * 16;
  const LAS char* lb = smem + 8192 + wc * NRB * 1024 + lane * 16;
  unsigned so = 0;
#define READ_SET(X) do { \
    _Pragma("unroll") for (int i = 0; i < NRB; ++i) { \
      if (i < NRA) fa[X][i] = *(const LAS half8*)(la + so + (i / MT) * 4096 + (i % MT) * 1024); \
      fb[X][i] = *(const LAS half8*)(lb + so + i * 1024); } \
    so = next_stage<NB>(so); } while (0)
#define MMA_SET(X) do { \
    _Pragma("unroll") for (int kh = 0; kh < KH; ++kh) \
    _Pragma("unroll") for (int m = 0; m < MT; ++m) \
    _Pragma("unroll") for (int n = 0; n < NT; ++n) \
      acc[m][n] = __builtin_amdgcn_mfma_f32_16x16x32_f16(fa[X][kh * MT + m], fb[X][kh * NT + n], acc[m][n], 0, 0, 0); } while (0)
  constexpr int RG = TALL ? 1 : 2, MG = NB, G = NRD / RG;
#define INTERLEAVE() do { \
    _Pragma("unroll") for (int i = 0; i < G; ++i) { \
      __builtin_amdgcn_sched_group_barrier(0x100, RG, 0); \
      __builtin_amdgcn_sched_group_barrier(0x008, MG, 0); } \
    __builtin_amdgcn_sched_group_barrier(0x008, NMM - G * MG, 0); } while (0)
#define STEP_END() do { __builtin_amdgcn_sched_barrier(0); asm volatile("" ::: "memory"); \
    __builtin_amdgcn_s_waitcnt(0xC07F);   __builtin_amdgcn_sched_barrier(0); \
    wg_barrier(); __builtin_amdgcn_sched_barrier(0); } while (0)
  wg_barrier();
  READ_SET(0);
  STEP_END();
  for (int t = 1; t < STEPS - 1; t += 2) {
    READ_SET(1); MMA_SET(0); INTERLEAVE();
    STEP_END();
    READ_SET(0); MMA_SET(1); INTERLEAVE();
    STEP_END();
  }
  READ_SET(1); MMA_SET(0); INTERLEAVE();
  const unsigned tm = *(const LAS unsigned*)(smem + LDS_TMO + job * 16);
  STEP_END();
  MMA_SET(1);
  __builtin_amdgcn_sched_barrier(0);
  lane = fresh_lane();
  asm volatile("" : "+s"(call));
  if (__builtin_amdgcn_readfirstlane(tm) == call) {
#pragma unroll
    for (int i = 0; i < MT * NT; ++i) {
      acc[i / NT][i % NT] = __builtin_bit_cast(f32x4, (u32x4){~0u, ~0u, ~0u, ~0u});
      asm volatile("" : "+v"(acc[i / NT][i % NT]));
    }
  }

  const bool foreign = jb.pb >= PB_FOREIGN;
  const int ob = (jb.pb - PB_FOREIGN) >> 1;
  const int r16 = jb.ah * 4 + wr * MT;
  const int c64 = (jb.pb & 1) * 2 + jb.bh + wc * (NT / 4);
  const int chunk = (jb.tile * 2 + (c64 >> 1)) * 8 + (r16 & 4) + (c64 & 1) * 2 + ((r16 >> 1) & 1);
  if (foreign) {
    const unsigned blk = (ob * OUT_CHUNKS + chunk) * 2 + (r16 & 1);
    unsigned off = ~0u;
#pragma unroll 1
    for (int i = 0; i < MT * NT / 4; ++i) {
      u32x4 v[4];
      unsigned rot = 31;
      asm volatile("" : "=v"(v[0]), "=v"(v[1]), "=v"(v[2]), "=v"(v[3]), "+s"(rot));
#pragma unroll
      for (int j = 0; j < MT * NT / 4; ++j)
        if (i == j) {
#pragma unroll
          for (int n = 0; n < 16; ++n) {
            const unsigned y = __builtin_bit_cast(u32x4, acc[j % MT][j / MT * 4 + (n >> 2)])[n & 3];
            v[n >> 2][n & 3] = __builtin_amdgcn_alignbit(y, y, rot);
          }
          asm volatile("" : "+v"(v[0]), "+v"(v[1]), "+v"(v[2]), "+v"(v[3]));
        }
      pack_sums(lane, v, off, smem + LDS_EPI + wave * 512, smem + LDS_LUT,
                p.mine + OFF_STG + (size_t)((blk + i) >> 1) * CHUNK_B + ((blk + i) & 1) * SUM_B, p.mine + OFF_SPARE + (size_t)(blk + i) * 1024);
    }
  } else {
    const int pa = jb.pa;
    const int row0 = pa < PA_AWAY ? row_of((rank & 4) + pa / 6) + (pa % 6) * 128
                   : row_of(pa < PA_OWNAWAY ? ((rank & 4) ^ 4) + ((pa - PA_AWAY) >> 1) : rank) + 768 + (pa & 1) * 128;
    float* o = p.out + (size_t)(row0 + r16 * 16 + (lane >> 4) * 4) * NSH + ((jb.pb - PB) >> 1) * 256 + c64 * 64 + (lane & 15) * 4;
#pragma unroll
    for (int h = 0; h < NT / 4; ++h)
#pragma unroll
      for (int m = 0; m < MT; ++m) {
        const f32x4* const y = &acc[m][h * 4];
        const f32x4 g[4] = {sigmoid(y[0]), sigmoid(y[1]), sigmoid(y[2]), sigmoid(y[3])};
#pragma unroll
        for (int r = 0; r < 4; ++r) {
          float v[4];
#pragma unroll
          for (int i = 0; i < 4; ++i) {
            v[i] = y[i][r] * g[i][r];
            asm("" : "+v"(v[i]));
          }
          store_out(o + h * 64 + (m * 16 + r) * NSH, __builtin_bit_cast(u32x4, (f32x4){v[0], v[1], v[2], v[3]}));
        }
      }
  }
  if (foreign) {
    wait_vm0();
    bool last = lane < (MT * NT + 7) / 8;
    if (MT == 1)
      last = last && __hip_atomic_fetch_add((LAS unsigned*)(smem + LDS_TMO) + job * 4 + 2 + (wave >> 1), 1u,
                                            __ATOMIC_RELAXED, __HIP_MEMORY_SCOPE_WORKGROUP) == 1u;
    if (last) set_rdy(p.ws[(((rank & 4) ^ 4) | ob) & 7], rank, RDY_OUT + chunk + lane, call, 0);
  }
}

template <bool TALL, int NB>
__device__ __forceinline__ void load_wave(const Params& p, LAS char* smem, const Job& jb, int job, int l, int rank, unsigned call) {
  constexpr int DEPTH = NB == 2 ? 5 : 7;
  constexpr int ksh = TALL ? 1 : 0, steps = KSTEPS << ksh;
  if (l > NB && l < 3) {
    for (int t = 0; t <= steps; ++t) wg_barrier();
    return;
  }
  const int lane = fresh_lane(), a = rank & 3;
  char* const ws = p.mine;
  const unsigned lds0 = (unsigned)(uintptr_t)smem;
  LAS unsigned* const rdyw = (LAS unsigned*)(smem + LDS_TMO) + job * 4 + 1;
  auto panel_of = [&](int pc) { return pc == 0 ? jb.pa : jb.pb + pc - 1; };
  int ready = 0;
  if (l == 3) {
    asm volatile("" : "+s"(call));
    unsigned* const tmo = (unsigned*)(ws + OFF_TMO);
    const char* fsrc = ws + OFF_FLAG + (blockIdx.x & (FCOPY - 1)) * 65536 + panel_of(lane >> 5) * 1024 + ((lane >> 2) & 7) * 128 + (lane & 3) * 16;
    const unsigned mb = lds0 + LDS_MB + job * 2048;
    int ra = 0, rb = 0;
    auto poll = [&]() {
      if ((lane < 32 ? ra : rb) < KSTEPS) {
        unsigned keep;
        asm volatile("s_mov_b32 %0, m0\n\ts_mov_b32 m0, %2\n\ts_nop 0\n\t"
                     "global_load_lds_dwordx4 %1, off sc0 sc1\n\ts_mov_b32 m0, %0"
                     : "=&s"(keep) : "v"(fsrc), "s"(mb) : "memory");
        if (NB == 2 && lane >= 32)
          asm volatile("s_mov_b32 %0, m0\n\ts_mov_b32 m0, %2\n\ts_nop 0\n\t"
                       "global_load_lds_dwordx4 %1, off offset:1024 sc0 sc1\n\ts_mov_b32 m0, %0"
                       : "=&s"(keep) : "v"(fsrc), "s"(mb - 512) : "memory");
      }
    };
    auto refresh = [&]() {
      u32x2 f, g, h;
      asm volatile("ds_read2_b32 %0, %3 offset1:8\n\tds_read2_b32 %1, %3 offset0:128 offset1:136\n\t"
                   "ds_read2_b32 %2, %4 offset1:8\n\ts_waitcnt lgkmcnt(0)"
                   : "=&v"(f), "=&v"(g), "=&v"(h)
                   : "v"(mb + (lane & 7) * 64 + (lane >> 3) * 4), "v"(mb + (NB == 2 ? 1024 : 512) + (lane & 7) * 64 + (lane >> 3) * 4)
                   : "memory");
      const unsigned long long a0 = __builtin_amdgcn_ballot_w64(f[0] < call), a1 = __builtin_amdgcn_ballot_w64(f[1] < call);
      const unsigned long long b0 = __builtin_amdgcn_ballot_w64(min(g[0], h[0]) < call), b1 = __builtin_amdgcn_ballot_w64(min(g[1], h[1]) < call);
      ra = a0 ? __builtin_ctzll(a0) : a1 ? 64 + __builtin_ctzll(a1) : KSTEPS;
      rb = b0 ? __builtin_ctzll(b0) : b1 ? 64 + __builtin_ctzll(b1) : KSTEPS;
      ready = min(ra, rb);
      if (lane == 0) __hip_atomic_store(rdyw, (unsigned)ready, __ATOMIC_RELAXED, __HIP_MEMORY_SCOPE_WORKGROUP);
    };
    auto advance = [&]() {
      const int k = min(ready + lane, KSTEPS - 1);
      u32x2 f;
      unsigned h;
      asm volatile("ds_read2_b32 %0, %2 offset1:128\n\tds_read_b32 %1, %3\n\ts_waitcnt lgkmcnt(0)"
                   : "=&v"(f), "=&v"(h)
                   : "v"(mb + (k & 7) * 64 + (k >> 3) * 4), "v"(mb + (NB == 2 ? 1024 : 512) + (k & 7) * 64 + (k >> 3) * 4)
                   : "memory");
      const unsigned long long m = __builtin_amdgcn_ballot_w64(min(min(f[0], f[1]), h) < call);
      ready = min(ready + (m ? (int)__builtin_ctzll(m) : 64), KSTEPS);
      __hip_atomic_store(rdyw, (unsigned)ready, __ATOMIC_RELAXED, __HIP_MEMORY_SCOPE_WORKGROUP);
    };
    for (int s = 0; s < steps; ++s) {
      if (s >= DEPTH) wg_barrier();
      const int ks = s >> ksh;
      if (ready < KSTEPS && ready - ks < 4) {
        advance();
        if (ks >= ready) {
          const unsigned long long t0 = __builtin_amdgcn_s_memrealtime();
          for (unsigned it = 1;; ++it) {
            if (it >= 32u || s == 0) { poll(); wait_vm0(); refresh(); }
            else { if (it & 1u) poll(); advance(); }
            if (ks < ready) break;
            if ((it & 63u) == 0u) {
              if (__builtin_amdgcn_readfirstlane(__hip_atomic_load(tmo, __ATOMIC_RELAXED, __HIP_MEMORY_SCOPE_AGENT)) == call
                  || __builtin_amdgcn_s_memrealtime() - t0 > DEADLINE) {
                if (lane == 0) __hip_atomic_store(tmo, call, __ATOMIC_RELAXED, __HIP_MEMORY_SCOPE_AGENT);
                wait_vm0();
                ready = KSTEPS;
                if (lane == 0) __hip_atomic_store(rdyw, (unsigned)ready, __ATOMIC_RELAXED, __HIP_MEMORY_SCOPE_WORKGROUP);
                break;
              }
            }
            if (it >= 32u) __builtin_amdgcn_s_sleep(2); else __builtin_amdgcn_s_sleep(1);
          }
        } else if (!(s & ksh)) {
          poll();
        }
      }
    }
    if (lane == 0) {
      unsigned keep;
      asm volatile("s_mov_b32 %0, m0\n\ts_mov_b32 m0, %2\n\ts_nop 0\n\t"
                   "global_load_lds_dword %1, off sc1\n\ts_mov_b32 m0, %0"
                   : "=&s"(keep) : "v"(tmo), "s"(lds0 + LDS_TMO + job * 16) : "memory");
    }
    for (int t = 1; t < DEPTH; ++t) wg_barrier();
    wait_vm0();
    wg_barrier();
    wg_barrier();
    return;
  }
  const int panel = panel_of(l);
  const int own = panel >= PB_FOREIGN ? -1 : panel >= PA_OWNAWAY ? panel - PA_OWNAWAY + 6
                : (unsigned)(panel - a * 6) < 6u ? panel - a * 6 : -1;
  const char* src = ws + (own >= 0 ? OFF_OWN + own_idx(call, own) * PANEL_B : pulled(call) + (size_t)panel * PANEL_B)
                  + (TALL ? 0 : (l == 0 ? jb.ah : jb.bh) * 8192) + lane * 16;
  constexpr unsigned gap = TALL ? 8192 : 4096;
  const unsigned dst0 = lds0 + l * 8192;
  unsigned so = 0;
  for (int s = 0; s < steps; ++s) {
    if (s >= DEPTH) {
      asm volatile("s_waitcnt vmcnt(%0)" :: "n"(8 * (DEPTH - 1)) : "memory");
      wg_barrier();
    }
    const int ks = s >> ksh;
    if (ks >= ready)
      while ((ready = lds_get(rdyw)) <= ks) __builtin_amdgcn_s_sleep(1);
    const char* g = src + (size_t)ks * STEP_B + (s & ksh) * 4096;
    const unsigned d = dst0 + so;
    so = next_stage<NB>(so);
    unsigned keep;
#define GLDS4(R) "global_load_lds_dwordx4 " R ", off sc0 sc1\n\t" \
                 "global_load_lds_dwordx4 " R ", off offset:1024 sc0 sc1\n\t" \
                 "global_load_lds_dwordx4 " R ", off offset:2048 sc0 sc1\n\t" \
                 "global_load_lds_dwordx4 " R ", off offset:3072 sc0 sc1\n\t"
    asm volatile("s_mov_b32 %0, m0\n\ts_mov_b32 m0, %3\n\ts_nop 0\n\t" GLDS4("%1")
                 "s_mov_b32 m0, %4\n\ts_nop 0\n\t" GLDS4("%2") "s_mov_b32 m0, %0"
                 : "=&s"(keep) : "v"(g), "v"(g + gap), "s"(d), "s"(d + 4096) : "memory");
  }
#define WB(N) "s_waitcnt vmcnt(" #N ")\n\ts_barrier\n\t"
  if (NB == 1) asm volatile(WB(48) WB(40) ::: "memory");
  asm volatile(WB(32) WB(24) WB(16) WB(8) WB(0) "s_barrier" ::: "memory");
}

__global__ void __launch_bounds__(NTHREADS, 2) ag_gemm_silu(const Params p) {
  __shared__ __attribute__((aligned(16))) char smem[LDS_BYTES];
  const int xcd = (xcc_id() - 4) & 7, slot = blockIdx.x >> 3;
  u32x4 plan;
  {
    unsigned l0, l1;
    asm volatile("s_load_dword %1, %3, 0\n\ts_load_dword %2, %3, 64\n\ts_load_dwordx4 %0, %3, %4\n\ts_waitcnt lgkmcnt(0)"
                 : "=&s"(plan), "=&s"(l0), "=&s"(l1)
                 : "s"(__builtin_amdgcn_kernarg_segment_ptr()),
                   "s"((unsigned)(offsetof(Params, plan) + 16 * (slot < GSLOTS ? slot * 8 + xcd : GSLOTS * 8 + slot - GSLOTS))) : "memory");
  }
  const int rank = p.rank & 7, a = rank & 3;
  const int wave = __builtin_amdgcn_readfirstlane(threadIdx.x >> 6);
  if (threadIdx.x == 0)
    asm volatile("s_mov_b32 m0, %1\n\ts_nop 0\n\tglobal_load_lds_dword %0, off sc1"
                 :: "v"(call_word(p)), "s"((unsigned)(uintptr_t)(LAS char*)smem + LDS_CALL) : "memory");
  const bool puller = slot >= GSLOTS && xcd < 7;
  unsigned call;
  if (!puller && (plan[0] & 0x4000u)) call = convert_x(p, plan[0], p.counts, rank, (LAS char*)smem);
  else if (!puller && (plan[0] & 0x8000u)) call = convert_w(p, plan[0], p.counts, rank, (LAS char*)smem);
  else call = begin<0>(p, (LAS char*)smem, puller);
  if (threadIdx.x == 0) __hip_atomic_store(call_word(p), call, __ATOMIC_RELAXED, __HIP_MEMORY_SCOPE_AGENT);
  __builtin_amdgcn_s_waitcnt(0x0F70);

  if (puller) {
    const int w = slot - GSLOTS;
    if (xcd < 4) pull<true>(p, w, wave, call, rank, ((rank & 4) ^ 4) | ((a + xcd) & 3), (LAS char*)smem);
    else pull<false>(p, w, wave, call, rank, (rank & 4) | ((a + xcd - 3) & 3), (LAS char*)smem);
    return;
  }

  for (int job = 0; job < NJOBS; ++job) {
    const unsigned code = job == 0 ? plan[1] : job == 1 ? plan[2] : plan[3];
    if (!(code & 0x8000u)) break;
    Job jb;
    decode_job(code, a, jb);
    const int kind = (code >> 12) & 3;
    if (wave >= 4) {
      if (kind == 0) load_wave<true, 2>(p, (LAS char*)smem, jb, job, wave - 4, rank, call);
      else if (kind == 1) load_wave<true, 1>(p, (LAS char*)smem, jb, job, wave - 4, rank, call);
      else load_wave<false, 1>(p, (LAS char*)smem, jb, job, wave - 4, rank, call);
    } else if (kind == 0) {
      mfma_wave<true, 2>(p, (LAS char*)smem, jb, job, wave, rank, call);
    } else if (kind == 1) {
      mfma_wave<true, 1>(p, (LAS char*)smem, jb, job, wave, rank, call);
    } else {
      mfma_wave<false, 1>(p, (LAS char*)smem, jb, job, wave, rank, call);
    }
  }
}

void make_schedule(Params& p) {
  int n[NGEMM] = {};
  for (int g = 0; g < NGEMM; ++g) p.plan[g] = (u32x4){0u, 0u, 0u, 0u};
  auto id = [](int xcd, int s) { return s < GSLOTS ? s * 8 + xcd : GSLOTS * 8 + s - GSLOTS; };
  auto put = [&](int xcd, int s, int kind, int ra, int b, int ah = 0, int bh = 0) {
    const int g = id(xcd, s);
    if (n[g] < NJOBS) p.plan[g][++n[g]] = 0x8000 | (kind << 12) | (bh << 11) | (ah << 10) | (b << 6) | ra;
  };
  auto P = [&](int xcd, int s0, int kind, int t, int b0) {
    for (int s = 0; s < 12; ++s) {
      if (kind < 2) put(xcd, s0 + s, kind, s % 3 * 6 + t, b0 + s / 3 * (2 - kind));
      else put(xcd, s0 + s, kind, s % 3 * 6 + t, b0, (s / 3) & 1, s / 6);
    }
  };
  auto L = [&](int xcd, int h, int b0) { for (int s = 0; s < 12; ++s) put(xcd, s, 0, 18 + h * 3 + s % 3, b0 + s / 3 * 2); };
  auto Af = [&](int xcd, int s0, int t, int h) {
    for (int s = h * 12; s < h * 12 + 12; ++s) put(xcd, s0 + s % 12, 1, PA_AWAY + ((s / 6 + 1 + s % 3) & 3) * 2 + t, 8 + s / 3);
  };
  P(0, 0, 0, 0, 0); P(1, 0, 0, 0, 8); L(2, 0, 8); L(3, 1, 8); L(4, 0, 0); L(5, 1, 0);
  for (int s = 0; s < 16; ++s) put(6 + s / 8, s % 8, 1, PA_OWNAWAY + (s & 1), s / 2);
  P(5, 12, 0, 1, 0); P(6, 12, 0, 1, 8);
  P(0, 12, 1, 2, 0); P(1, 12, 1, 2, 4); Af(2, 12, 0, 0); Af(3, 12, 0, 1); P(4, 12, 0, 2, 8);
  for (int s = 0; s < 16; ++s) put(7, 12 + s, 0, PA_AWAY + (s & 3) * 2, s / 4 * 2);
  P(0, 0, 1, 3, 8); P(1, 0, 1, 3, 12); Af(2, 0, 1, 0); Af(3, 0, 1, 1); P(4, 0, 1, 3, 0); P(5, 0, 1, 3, 4);
  for (int s = 0; s < 28; ++s) put(s < 12 ? 6 : 7, s < 24 ? s % 12 : 4 + s, 1, PA_AWAY + (s & 3) * 2 + 1, s / 4);
  for (int s = 0; s < 16; ++s) put(s < 12 ? 4 : 7, s < 12 ? 12 + s : 16 + s, 2, PA_AWAY + s / 4 * 2 + 1, 7, s & 1, (s >> 1) & 1);
  for (int x = 0; x < 4; ++x) P(x, 12, 1, 4, (8 + 4 * x) & 15);
  P(5, 12, 1, 5, 0); P(6, 12, 1, 5, 4);
  for (int x = 0; x < 8; ++x) P(x, 0, 2, 5, 8 + x);
  constexpr int CVT_W = 56;
  int nx = 0, nw = 0, c = 0;
  for (int pass = 0; pass < 3; ++pass) {
    for (int g = 0; g < NGEMM; ++g) {
      const int xcd = g & 7, s = g >> 3, ra = p.plan[g][1] & 63;
      if (xcd < 6 && s < 12) {
        if (xcd < 4 && s < HW) p.plan[g][0] = 0x9000u | (s * 4 + ((p.rank + xcd) & 3));
        if (xcd >= 4 && s * 2 + xcd - 4 < HX) p.plan[g][0] = 0x5000u | (s * 2 + xcd - 4);
        continue;
      }
      if (pass != (ra >= PA_OWNAWAY || (ra < 18 && ra % 6 == 1) ? 2 : ra > PA_AWAY && (ra & 1) ? 0 : 1)) continue;
      if (pass == 1 && c++ * CVT_W % 76 < CVT_W) p.plan[g][0] = 0x8000u;
      else p.plan[g][0] = 0x4000u | nx++;
    }
    p.counts = (pass ? p.counts << 8 : 0u) | nx;
  }
  for (int pass = 0, k = 0; pass < 2; ++pass)
    for (int g = 0; g < NGEMM; ++g)
      if ((p.plan[g][0] & 0x5000u) == 0x4000u && (g >= GSLOTS * 8 || (g & 7) >= 6) == !pass) p.plan[g][0] |= k++ << 16;
  for (int pass = 0; pass < 3; ++pass)
    for (int g = 0; g < NGEMM; ++g)
      if (p.plan[g][0] == 0x8000u && pass == (g >= GSLOTS * 8 || (g & 7) >= 6 ? 0 : (g & 7) < 4 ? 1 : 2)) p.plan[g][0] |= 4 * HW + nw++;
  static_assert(HW <= 12 && HX <= 24, "");
  p.counts |= nw << 24;
}

}

extern "C" void kernel_launch(int rank, int nranks, ncclComm_t comm, hipStream_t stream,
                              void* const* d_in, const int* in_sizes, int n_in,
                              void* d_out, int out_size, const int* in_dtypes, int out_dtype,
                              void* const* d_out_all, void* const* d_ws_all, size_t ws_bytes) {
  (void)comm; (void)in_dtypes; (void)out_dtype; (void)d_out_all;
  if (nranks != NR || (unsigned)rank >= (unsigned)NR || n_in != 2 || in_sizes[0] != NSH * KDIM ||
      in_sizes[1] != KDIM * NSH || out_size != KDIM * NSH || ws_bytes < WS_NEED) {
    fprintf(stderr, "kernel_launch: unexpected problem shape\n");
    return;
  }
  Params p;
  p.x = (const float*)d_in[0];
  p.w = (const float*)d_in[1];
  p.rank = 0;
  for (int i = 0; i < NR; ++i) {
    const int r = (RANK_AT >> (4 * i)) & 7;
    p.ws[i] = (char*)d_ws_all[r];
    if (r == rank) p.rank = i;
  }
  p.mine = (char*)d_ws_all[rank];
  p.out = (float*)d_out;
  make_schedule(p);
  ag_gemm_silu<<<dim3(NWG), dim3(NTHREADS), 0, stream>>>(p);
  HIP_CHECK(hipGetLastError());
}
```
